```python
import numpy as np

kernel_name = "gpu_ag_gemm_m4096_k4096_n4096_bf16_silu"


ABSMAX_RTOL = 0.008

COMPUTE_DTYPE = "bfloat16"
EPILOGUE = "silu"
IN_QUANT = None
OUT_QUANT = None
IN_DTYPES = {"x": "bfloat16", "w_mat": "bfloat16"}
WORLD_SIZE = 8

def _to_bf16(x):
    u = np.ascontiguousarray(x, dtype=np.float32).view(np.uint32)
    return ((u + 0x7FFF + ((u >> 16) & 1)) & 0xFFFF0000).view(np.float32)

def setup_inputs():

    x = _to_bf16(np.random.randn(4096, 4096).astype(np.float32))
    w_mat = _to_bf16((np.random.randn(4096, 4096) / np.sqrt(4096)).astype(np.float32))
    return {"x": x, "w_mat": w_mat}

def _epilogue(y):
    return (y / (1.0 + np.exp(-np.clip(y, -60.0, 60.0)))).astype(np.float32)

def reference(x, w_mat):
    y = (x @ w_mat).astype(np.float32)
    return _epilogue(y)
```

```cpp
#include <hip/hip_runtime.h>
#include <rccl/rccl.h>
#include <cstdio>
#include <cstdint>
#include <algorithm>
#include <cstdlib>

typedef unsigned u32;
typedef unsigned long long u64;
typedef unsigned u32x4 __attribute__((ext_vector_type(4)));
typedef unsigned u32x2 __attribute__((ext_vector_type(2)));
typedef unsigned short u16x2 __attribute__((ext_vector_type(2)));
typedef __bf16 bf16x2 __attribute__((ext_vector_type(2)));
typedef __bf16 bf16x4 __attribute__((ext_vector_type(4)));
typedef __bf16 bf16x8 __attribute__((ext_vector_type(8)));
typedef float f32x2 __attribute__((ext_vector_type(2)));
typedef float f32x16 __attribute__((ext_vector_type(16)));
#define LAS __attribute__((address_space(3)))

#define P_NA 11
#define P_LATE 4
#define P_W 12
#define P_WPW 2
#define P_LEAD 4.0
#define P_THR 0
#define P_SFLIP 4
#define P_RHO 0.3
#define P_SIGMA 0.8

constexpr int NR = 8;
constexpr int KDIM = 4096, NLOC = 512;
constexpr int NSTEP = 32;
constexpr int NA = P_NA, NB = 16 - NA, NLATE = P_LATE, NEARLY = NA - NLATE;
constexpr int NSTREAM = 112;
constexpr int NUNIT = NSTREAM * NSTEP;
constexpr size_t SLOT = 8192;
constexpr size_t PAR_BYTES = (size_t)NUNIT * SLOT;
constexpr size_t META_OFF = 2 * PAR_BYTES;
constexpr size_t META_PAR = (size_t)NUNIT * 32;
constexpr size_t DONE_OFF = META_OFF + 2 * META_PAR;
constexpr size_t CNT_OFF = DONE_OFF + 512;
constexpr size_t TMO_OFF = CNT_OFF + 256 * 16 * 4;
constexpr size_t DBG_OFF = TMO_OFF + 128;
constexpr size_t WS_NEED = DBG_OFF + 256 * 16 * 64 * 8;
constexpr int N_INTRA = NA * NSTEP, N_INTER = NB * NSTEP + 4 * NSTEP;
constexpr int N_ORD = N_INTRA > N_INTER ? N_INTRA : N_INTER;
constexpr int RT_BIG = 13, RT_SMALL = 6;
constexpr int NBIG = RT_BIG * 8, NSMALL = RT_SMALL * 16;
constexpr int NGEMM = NBIG + NSMALL;
constexpr int NFLAG = RT_BIG + 2 * RT_SMALL;
constexpr int NSENDW = NR * P_W;
constexpr int NSENDWG = ((NSENDW + P_WPW - 1) / P_WPW + 7) / 8 * 8;
constexpr int NUNPWG = 8, NUNP = NUNPWG * 8;
constexpr int GRID = NGEMM + NSENDWG + NUNPWG;
static_assert(GRID <= 256, "one workgroup per CU, all resident");
constexpr int NTHREAD = 512;
constexpr int EPI_OFF = 0;
constexpr int LDS_BYTES = 131072;
constexpr u64 DEADLINE_TICKS = 50000000ull;
constexpr int ERS = 272;

struct Params {
  const char* x;
  const char* w;
  char* out[NR];
  char* ws[NR];
  int rank;
  int pad;
  u32 ord[3][N_ORD / 2];
};

typedef const __attribute__((address_space(4))) Params* KP;

#define STAMP(i) do {} while (0)
#define FSTAMP(t, i) do {} while (0)
#define WG_BARRIER() do { asm volatile("" ::: "memory"); __builtin_amdgcn_s_waitcnt(0xC07F); \
    asm volatile("s_waitcnt lgkmcnt(0)\n\ts_barrier" ::: "memory"); } while (0)

__device__ __forceinline__ u32 draw_call(char* myws, int wv) {
  u32 v = 0;
  if ((threadIdx.x & 63) == 0)
    v = __hip_atomic_fetch_add((u32*)(myws + CNT_OFF) + blockIdx.x * 16 + wv, 1u, __ATOMIC_RELAXED, __HIP_MEMORY_SCOPE_AGENT);
  return __builtin_amdgcn_readfirstlane(v) + 1u;
}
__device__ __forceinline__ u32 call_tag(u32 call) { return (call % 2047u + 1u) << 21; }

__device__ __forceinline__ bool spin_fail(char* myws, u32 call, u64 t0) {
  u32* tmo = (u32*)(myws + TMO_OFF);
  u32 t = __builtin_amdgcn_readfirstlane(__hip_atomic_load(tmo, __ATOMIC_RELAXED, __HIP_MEMORY_SCOPE_AGENT));
  if (t == call || __builtin_amdgcn_s_memrealtime() - t0 > DEADLINE_TICKS) {
    __hip_atomic_store(tmo, call, __ATOMIC_RELAXED, __HIP_MEMORY_SCOPE_AGENT);
    return true;
  }
  return false;
}

struct UnitDesc { const char* src; u32 s0; bool isw; u32 u_rem, u_loc; };

#define ST16(voff, data, sbase, imm) \
  asm volatile("global_store_dwordx4 %0, %1, %2 offset:" #imm " sc0 sc1\n\ts_nop 1" :: "v"(voff), "v"(data), "s"(sbase) : "memory")
#define ST4(voff, data, sbase) \
  asm volatile("global_store_dword %0, %1, %2 sc0 sc1" :: "v"(voff), "v"(data), "s"(sbase) : "memory")

__device__ __forceinline__ void sender(KP p, char* smem, char* myws, int wv, int lane) {
  const int gw = (int)(blockIdx.x - NGEMM) + wv * NSENDWG;
  if (gw >= NSENDW) return;
  const int rank = p->rank;
  const int link = gw & 7, w0 = gw >> 3;
  const u32 dest = (u32)(rank + link) & 7u;
  const bool self = dest == (u32)rank;
  const bool intra = (dest >> 2) == ((u32)rank >> 2);
  const int nent = intra ? N_INTRA : N_INTER;
  STAMP(0);

  const u32 lx = (lane & 31) * (KDIM * 2) + (lane >> 5) * 16;
  const u32 lw = (8 * ((lane >> 4) & 1) + 4 * (lane >> 5) + ((lane >> 1) & 3)) * (NLOC * 2) + (2 * ((lane >> 3) & 1) + (lane & 1)) * 16;
  const u32 l16 = lane * 16, l4 = lane * 4;

  const u32 lxw = lx ^ lw;
  const int kind = self ? 0 : intra ? 1 : 2;
  const u32 jj = (((u32)rank - dest) & 3) - 1;
  const u32 cE = self ? 0 : NA + 4 * NB + jj * NEARLY;
  const u32 cL = self ? 0 : intra ? 64 - 3 * NLATE + jj * NLATE - NEARLY : (rank & 3) * NB;
  const u32 cWr = 96 + (rank & 3) * 4 - 16, cWl = 64 + (dest & 3) * 4 - 16;
  const char* const wsrc = p->w + (dest & 3) * 256 - (size_t)16 * 32 * (NLOC * 2);
  const char* const xsrc = p->x;

  auto desc = [=](int e) {
    UnitDesc d;
    if (e >= nent) e = w0;
    const u32 code = (p->ord[kind][e >> 1] >> (16 * (e & 1))) & 0xffffu;
    const u32 sl = code >> 5, t = code & 31;
    d.isw = sl >= 16;
    const u64 aw = (u64)wsrc + (u64)((t * 128 + sl * 32) * (NLOC * 2)), ax = (u64)xsrc + (u64)(sl * 32 * (KDIM * 2) + t * 256);
    d.src = (const char*)(d.isw ? aw : ax);
    d.s0 = d.isw ? 16 * NLOC * 2 : 32;
    const u32 mw = d.isw ? 0xffffffffu : 0u, me = sl < (u32)NEARLY ? 0xffffffffu : 0u;
    d.u_rem = (sl + cL + ((cWr - cL) & mw) + ((cE - cL) & me)) * NSTEP + t;
    d.u_loc = (sl + cWl) * NSTEP + t;
    return d;
  };
  auto load_unit = [=](u32x4 (&d)[8], const UnitDesc& u) {
    const u32 vo = lx ^ (lxw & (u.isw ? 0xffffffffu : 0u));
    const char* b1 = u.src + u.s0;
    asm volatile("global_load_dwordx4 %0, %8, %9\n\tglobal_load_dwordx4 %1, %8, %10\n\t"
                 "global_load_dwordx4 %2, %8, %9 offset:64\n\tglobal_load_dwordx4 %3, %8, %10 offset:64\n\t"
                 "global_load_dwordx4 %4, %8, %9 offset:128\n\tglobal_load_dwordx4 %5, %8, %10 offset:128\n\t"
                 "global_load_dwordx4 %6, %8, %9 offset:192\n\tglobal_load_dwordx4 %7, %8, %10 offset:192"
                 : "=&v"(d[0]), "=&v"(d[1]), "=&v"(d[2]), "=&v"(d[3]), "=&v"(d[4]), "=&v"(d[5]), "=&v"(d[6]), "=&v"(d[7])
                 : "v"(vo), "s"(u.src), "s"(b1) : "memory");
  };

  u32x4 d0[8], d1[8];
  UnitDesc cur = desc(w0), nxt = desc(w0 + P_W);
  load_unit(d0, cur);
  load_unit(d1, nxt);
  const u32 call = draw_call(myws, wv);
  asm volatile("s_waitcnt vmcnt(0)" ::: "memory");
  const u32 tag = call_tag(call);
  char* const rws = p->ws[dest];
  const size_t par = call & 1;
  char* const rslot = rws + par * PAR_BYTES + 4096;
  char* const lslot = myws + par * PAR_BYTES + 4096;
  char* const rmeta = rws + META_OFF + par * META_PAR;
  char* const lmeta = myws + META_OFF + par * META_PAR;

  u32 pmeta = 0;  UnitDesc prev = cur;  bool have_prev = false;
  auto publish_prev = [&]() {
    if (lane < 8) {
      char* const rm = rmeta + (size_t)prev.u_rem * 32;
      ST4(l4, pmeta, rm);
      if (prev.isw) { char* const lm = lmeta + (size_t)prev.u_loc * 32; ST4(l4, pmeta, lm); }
    }
  };
  auto one = [&](u32x4 (&d)[8], int e) {
#pragma unroll
    for (int j = 0; j < 8; ++j) asm volatile("" : "+v"(d[j]));
    const UnitDesc nn = desc(e + 2 * P_W);
    if (cur.isw) {
      LAS char* const q = (LAS char*)smem + wv * 8192;
#pragma unroll
      for (int j = 0; j < 8; ++j) *(LAS u32x4*)(q + j * 1024 + l16) = d[j];
      asm volatile("" ::: "memory");
#pragma unroll
      for (int j = 0; j < 8; ++j) {
        const bf16x4 lo = __builtin_amdgcn_ds_read_tr16_b64_v4bf16((LAS bf16x4*)(q + j * 1024 + lane * 8));
        const bf16x4 hi = __builtin_amdgcn_ds_read_tr16_b64_v4bf16((LAS bf16x4*)(q + j * 1024 + 512 + lane * 8));
        d[j] = __builtin_bit_cast(u32x4, __builtin_shufflevector(lo, hi, 0, 1, 2, 3, 4, 5, 6, 7));
      }
      char* const lb = lslot + (size_t)cur.u_loc * SLOT;
      ST16(l16, d[0], lb, -4096); ST16(l16, d[1], lb, -3072); ST16(l16, d[2], lb, -2048); ST16(l16, d[3], lb, -1024);
      ST16(l16, d[4], lb, 0);     ST16(l16, d[5], lb, 1024);  ST16(l16, d[6], lb, 2048);  ST16(l16, d[7], lb, 3072);
    }
    u32 L[16], H[16], T[16];
    u16x2 m1 = {0, 0}, m2 = {0, 0}, m3 = {0, 0}, m4 = {0, 0};
#pragma unroll
    for (int i = 0; i < 16; ++i) {
      const u32 a = d[i >> 1][2 * (i & 1)], b = d[i >> 1][2 * (i & 1) + 1];
      L[i] = __builtin_amdgcn_perm(b, a, 0x06040200u);
      H[i] = __builtin_amdgcn_perm(b, a, 0x07050301u);
    }
#pragma unroll
    for (int i = 0; i < 16; ++i) { asm volatile("" : "+v"(L[i])); asm volatile("" : "+v"(H[i])); }
    load_unit(d, nn);
#pragma unroll
    for (int i = 0; i < 16; i += 2) {
      T[i] = H[i] | 0x80808080u;
      T[i + 1] = H[i + 1] | 0x80808080u;
      m1 = __builtin_elementwise_max(m1, __builtin_bit_cast(u16x2, T[i]));
      m2 = __builtin_elementwise_max(m2, __builtin_bit_cast(u16x2, T[i] << 8));
      m3 = __builtin_elementwise_max(m3, __builtin_bit_cast(u16x2, T[i + 1]));
      m4 = __builtin_elementwise_max(m4, __builtin_bit_cast(u16x2, T[i + 1] << 8));
    }
    m1 = __builtin_elementwise_max(__builtin_elementwise_max(m1, m2), __builtin_elementwise_max(m3, m4));
    u32 mx = (u32)(m1.x > m1.y ? m1.x : m1.y);
    mx = max(mx, (u32)__builtin_amdgcn_update_dpp(0, mx, 0x111, 0xf, 0xf, true));
    mx = max(mx, (u32)__builtin_amdgcn_update_dpp(0, mx, 0x112, 0xf, 0xf, true));
    mx = max(mx, (u32)__builtin_amdgcn_update_dpp(0, mx, 0x114, 0xf, 0xf, true));
    mx = max(mx, (u32)__builtin_amdgcn_update_dpp(0, mx, 0x118, 0xf, 0xf, true));
    const u32 top = max(max((u32)__builtin_amdgcn_readlane(mx, 15), (u32)__builtin_amdgcn_readlane(mx, 31)),
                        max((u32)__builtin_amdgcn_readlane(mx, 47), (u32)__builtin_amdgcn_readlane(mx, 63)));
    const u32 h7 = (top >> 8) & 0x7f;
    const u32 base = h7 > 7 ? h7 - 7 : 0;
    const u32 base4 = base * 0x01010101u;
    u32 ok = 0xffffffffu;
    u32 N[8];
#pragma unroll
    for (int j = 0; j < 8; ++j) {
      const u32 ta = T[2 * j] - base4, tb = T[2 * j + 1] - base4;
      ok &= ta & tb;
      const u32 ca = (ta & 0x07070707u) | (H[2 * j] & 0x80808080u);
      const u32 cb = (tb & 0x07070707u) | (H[2 * j + 1] & 0x80808080u);
      N[j] = ca | __builtin_amdgcn_alignbit(cb, cb, 28);
    }
    u32 meta = 0;
    int cnt = 2;
    bool first = true;
    if (__any((~ok & 0x80808080u) != 0)) {
#pragma unroll
      for (int i = 0; i < 16; ++i) {
        if (i == 8) first = true;
        const u32 m = ~((H[i] | 0x80808080u) - base4) & 0x80808080u;
        u64 bal = __ballot(m != 0);
        while (bal != 0 && cnt <= 7) {
          const int ln = __builtin_ctzll(bal);
          bal &= bal - 1;
          u32 mm = __builtin_amdgcn_readlane(m, ln);
          const u32 hh = __builtin_amdgcn_readlane(H[i], ln);
          while (mm != 0) {
            const int by = __builtin_ctz(mm) >> 3;
            mm &= mm - 1;
            if (!first) ++cnt;
            const int slot = first ? 1 + (i >> 3) : cnt;
            first = false;
            const u32 hb = (hh >> (8 * by)) & 0x7f;
            const u32 ent = (u32)(i << 17 | ln << 11 | by << 9) | ((((hb - base) & 7) + base) ^ hb);
            if (lane == slot) meta = ent;
          }
        }
      }
    }
    const bool raw = cnt > 7;
    if (lane == 0) meta = base | (raw ? 0x400u : cnt > 2 ? 0x200u : 0u);
    meta |= tag;
    u32x4 s[6];
#pragma unroll
    for (int j = 0; j < 4; ++j) s[j] = u32x4{L[4 * j], L[4 * j + 1], L[4 * j + 2], L[4 * j + 3]};
    s[4] = u32x4{N[0], N[1], N[2], N[3]};
    s[5] = u32x4{N[4], N[5], N[6], N[7]};
    char* const rb = rslot + (size_t)cur.u_rem * SLOT;
    if (__builtin_expect(raw, 0)) {
      s[4] = u32x4{H[0], H[1], H[2], H[3]};
      s[5] = u32x4{H[4], H[5], H[6], H[7]};
      const u32x4 s6 = {H[8], H[9], H[10], H[11]}, s7 = {H[12], H[13], H[14], H[15]};
      ST16(l16, s6, rb, 2048); ST16(l16, s7, rb, 3072);
    }
    ST16(l16, s[0], rb, -4096); ST16(l16, s[1], rb, -3072); ST16(l16, s[2], rb, -2048);
    ST16(l16, s[3], rb, -1024); ST16(l16, s[4], rb, 0);     ST16(l16, s[5], rb, 1024);
    asm volatile("s_waitcnt vmcnt(14)" ::: "memory");
    if (have_prev) publish_prev();
    pmeta = meta; prev = cur; have_prev = true; cur = nxt; nxt = nn;
  };
  for (int e = w0; e < nent; e += 2 * P_W) {
    one(d0, e);
    if (e + P_W >= nent) break;
    one(d1, e + P_W);
  }
  asm volatile("s_waitcnt vmcnt(0)" ::: "memory");
#pragma unroll
  for (int j = 0; j < 8; ++j) { asm volatile("" : "+v"(d0[j])); asm volatile("" : "+v"(d1[j])); }
  publish_prev();
  STAMP(1);
  if (gw == 0) {
    const u64 t0 = __builtin_amdgcn_s_memrealtime();
    const u64* f = (const u64*)(myws + DONE_OFF) + lane;
    const bool n0 = ((2 * lane) & 31) < NFLAG, n1 = ((2 * lane + 1) & 31) < NFLAG;
    for (u32 n = 1;; ++n) {
      const u64 v = __hip_atomic_load(f, __ATOMIC_RELAXED, __HIP_MEMORY_SCOPE_SYSTEM);
      if (__all((!n0 || (u32)v >= call) && (!n1 || (u32)(v >> 32) >= call))) break;
      if ((n & 31) == 0 && spin_fail(myws, call, t0)) {
        if (lane == 0) *(unsigned short*)(p->out[rank] + (size_t)rank * 512 * (NLOC * 2)) = 0x7fc0;
        break;
      }
    }
    STAMP(2);
  }
}

typedef unsigned u32x8 __attribute__((ext_vector_type(8)));
typedef unsigned u32x16 __attribute__((ext_vector_type(16)));
#define CAT16(a, b) __builtin_shufflevector(a, b, 0, 1, 2, 3, 4, 5, 6, 7, 8, 9, 10, 11, 12, 13, 14, 15)

__device__ __forceinline__ u32x4 join(u32 l0, u32 l1, u32 h0, u32 h1) {
  return u32x4{__builtin_amdgcn_perm(h0, l0, 0x05010400u), __builtin_amdgcn_perm(h0, l0, 0x07030602u),
               __builtin_amdgcn_perm(h1, l1, 0x05010400u), __builtin_amdgcn_perm(h1, l1, 0x07030602u)};
}
__device__ __forceinline__ u32x8 expand(const u32x4& n, u32 base4) {
  u32x8 h;
#pragma unroll
  for (int j = 0; j < 4; ++j) { h[2 * j] = n[j]; h[2 * j + 1] = __builtin_amdgcn_alignbit(n[j], n[j], 4); }
  __builtin_amdgcn_sched_barrier(0);
  h &= 0x87878787u;
  __builtin_amdgcn_sched_barrier(0);
  h += base4;
  __builtin_amdgcn_sched_barrier(0);
  return h;
}
__device__ __forceinline__ u32x8 raw_high(const char* hi, u32 l16) {
  u32x4 a, b;
  asm volatile("global_load_dwordx4 %0, %2, %3 sc0 sc1\n\t"
               "global_load_dwordx4 %1, %2, %3 offset:1024 sc0 sc1\n\ts_waitcnt vmcnt(0)"
               : "=&v"(a), "=&v"(b) : "v"(l16), "s"(hi) : "memory");
  return __builtin_shufflevector(a, b, 0, 1, 2, 3, 4, 5, 6, 7);
}

__device__ __forceinline__ float silu(float y) {
  const float z = __builtin_amdgcn_fmed3f(y, -60.0f, 60.0f);
  const float e = __builtin_amdgcn_exp2f(z * -1.44269504088896341f);
  return y * __builtin_amdgcn_rcpf(1.0f + e);
}

template <int N> struct Pay { u32x4 v[N]; };
__device__ __forceinline__ void pay_load(Pay<6>& P, u32 l16, u32, const char* sb) {
  asm volatile("global_load_dwordx4 %4, %6, %7 offset:0 sc0 sc1\n\t"
               "global_load_dwordx4 %0, %6, %7 offset:-4096 sc0 sc1\n\t"
               "global_load_dwordx4 %1, %6, %7 offset:-3072 sc0 sc1\n\t"
               "global_load_dwordx4 %5, %6, %7 offset:1024 sc0 sc1\n\t"
               "global_load_dwordx4 %2, %6, %7 offset:-2048 sc0 sc1\n\t"
               "global_load_dwordx4 %3, %6, %7 offset:-1024 sc0 sc1"
               : "=&v"(P.v[0]), "=&v"(P.v[1]), "=&v"(P.v[2]), "=&v"(P.v[3]), "=&v"(P.v[4]), "=&v"(P.v[5]) : "v"(l16), "s"(sb) : "memory");
}
__device__ __forceinline__ void pay_load(Pay<3>& P, u32 vlo, u32 vcode, const char* sb) {
  asm volatile("global_load_dwordx4 %2, %4, %5 offset:0 sc0 sc1\n\t"
               "global_load_dwordx4 %0, %3, %5 offset:-4096 sc0 sc1\n\t"
               "global_load_dwordx4 %1, %3, %5 offset:-3072 sc0 sc1"
               : "=&v"(P.v[0]), "=&v"(P.v[1]), "=&v"(P.v[2]) : "v"(vlo), "v"(vcode), "s"(sb) : "memory");
}
template <int C> __device__ __forceinline__ void pay_wait(Pay<6>& P) {
  asm volatile("s_waitcnt vmcnt(%6)" : "+v"(P.v[0]), "+v"(P.v[1]), "+v"(P.v[2]), "+v"(P.v[3]), "+v"(P.v[4]), "+v"(P.v[5]) : "n"(C) : "memory");
}
template <int C> __device__ __forceinline__ void pay_wait(Pay<3>& P) {
  asm volatile("s_waitcnt vmcnt(%3)" : "+v"(P.v[0]), "+v"(P.v[1]), "+v"(P.v[2]) : "n"(C) : "memory");
}

struct Stream {
  const char* sb; const char* mb; char* myws;
  u32 l16, vlo, vcode, hf, tag, call; u64 t0;
  u32x4 W; int rdy;
  __device__ __forceinline__ void init(char* ws, u32 call_, u32 sid, u32 hf_, int lane) {
    const size_t par = call_ & 1;
    myws = ws; call = call_; hf = hf_; tag = call_tag(call_);
    sb = ws + par * PAR_BYTES + (size_t)sid * NSTEP * SLOT + 4096;
    mb = ws + META_OFF + par * META_PAR + (size_t)sid * NSTEP * 32;
    l16 = lane * 16; vlo = l16 + hf * 2048; vcode = l16 + hf * 1024;
    t0 = __builtin_amdgcn_s_memrealtime();
    W = u32x4{0u, 0u, 0u, 0u}; rdy = 0;
  }
  __device__ __forceinline__ void poll() {
    const u64 mask = 0xffull << (rdy < 28 ? 2 * rdy : 56);
    asm volatile("s_mov_b64 exec, %3\n\tglobal_load_dwordx4 %0, %1, %2 sc0 sc1\n\ts_mov_b64 exec, -1"
                 : "+v"(W) : "v"(l16), "s"(mb), "s"(mask) : "memory");
  }
  __device__ __forceinline__ void look() {
    asm volatile("" : "+v"(W));
    const u32 x = (W[0] ^ tag) | (W[1] ^ tag) | (W[2] ^ tag) | (W[3] ^ tag);
    const u64 bad = __ballot(x >= (1u << 21));
    rdy = bad ? __builtin_ctzll(bad) >> 1 : NSTEP;
  }
  __device__ __forceinline__ void spin(int t) {
    for (u32 n = 1; rdy <= t; ++n) {
      poll();
      asm volatile("s_waitcnt vmcnt(0)" ::: "memory");
      look();
      if ((n & 31) == 0 && spin_fail(myws, call, t0)) rdy = NSTEP;
    }
  }
  template <class PT> __device__ __forceinline__ void fetch(PT& P, int t) { poll(); pay_load(P, vlo, vcode, sb + (size_t)t * SLOT); }
  __device__ __forceinline__ u32 header(int t) { asm volatile("" : "+v"(W)); return __builtin_amdgcn_readlane(W[0], 2 * t); }
  static __device__ __forceinline__ void fix(u32x16& v, u32 e, u32 flip, int lane) {
    const u32 idx = ((e >> 17) & 15) ^ flip;
    v[idx] ^= lane == (int)((e >> 11) & 63) ? (e & 0xff) << ((e >> 6) & 24) : 0u;
  }
  __device__ __forceinline__ void patch_rest(u32x16& v, int t, u32 flip, int lane) {
    fix(v, __builtin_amdgcn_readlane(W[3], 2 * t), flip, lane);
#pragma unroll
    for (int k = 0; k < 4; ++k) fix(v, __builtin_amdgcn_readlane(W[k], 2 * t + 1), flip, lane);
  }
};

__device__ __forceinline__ void unpacker(char* myws, int wv, int lane) {
  const int r = (int)(blockIdx.x - NGEMM - NSENDWG) + wv * NUNPWG;
  const u32 call = draw_call(myws, wv);
  Stream S;
  S.init(myws, call, 96 + (r & 15), 0, lane);
  const u32 l4 = lane * 4;
  for (int t = r >> 4; t < NSTEP; t += NUNP / 16) {
    S.spin(t);
    Pay<6> P;
    pay_load(P, S.l16, 0, S.sb + (size_t)t * SLOT);
    pay_wait<0>(P);
    const u32 hdr = S.header(t), base4 = (hdr & 0x7f) * 0x01010101u;
    u32x16 h = CAT16(expand(P.v[4], base4), expand(P.v[5], base4));
    Stream::fix(h, __builtin_amdgcn_readlane(S.W[1], 2 * t), 0, lane);
    Stream::fix(h, __builtin_amdgcn_readlane(S.W[2], 2 * t), 0, lane);
    if (hdr & 0x200u) S.patch_rest(h, t, 0, lane);
    if (hdr & 0x400u) h = CAT16(raw_high(S.sb + (size_t)t * SLOT, S.l16), raw_high(S.sb + (size_t)t * SLOT + 2048, S.l16));
    u32x4 o[8];
#pragma unroll
    for (int j = 0; j < 8; ++j) o[j] = join(P.v[j >> 1][2 * (j & 1)], P.v[j >> 1][2 * (j & 1) + 1], h[2 * j], h[2 * j + 1]);
    const char* const ob = S.sb + (size_t)t * SLOT - (size_t)16 * NSTEP * SLOT;
    ST16(S.l16, o[0], ob, -4096); ST16(S.l16, o[1], ob, -3072); ST16(S.l16, o[2], ob, -2048); ST16(S.l16, o[3], ob, -1024);
    ST16(S.l16, o[4], ob, 0);     ST16(S.l16, o[5], ob, 1024);  ST16(S.l16, o[6], ob, 2048);  ST16(S.l16, o[7], ob, 3072);
    asm volatile("s_waitcnt vmcnt(0)" ::: "memory");
    const char* const mo = S.mb + t * 32 - 16 * NSTEP * 32;
    if (lane < 8) ST4(l4, S.tag, mo);
    STAMP(1 + t);
  }
}

template <bool BIG>
__device__ __forceinline__ void gemm_tile(KP p, char* smem, char* myws, int wv, int lane, u32 sid0, u32 tn, u32 ch, u32 fl) {
  const int rank = p->rank;
  const u32 call = draw_call(myws, wv);
  STAMP(0);
  Stream S;

  if (wv >= 4) {
    constexpr int NL = BIG ? 8 : 4;
    const u32 un = wv - 4;
    S.init(myws, call, 64 + 4 * tn + un, 0, lane);
    const u32 lds0 = (u32)(size_t)(LAS char*)smem + un * 8192 + (BIG ? 0 : ch * 4096);
    const char* const g0 = S.sb - 4096 + (BIG ? 0 : ch * 4096);
    auto dma4 = [&](const char* g, u32 l) {
      u32 keep;
      asm volatile("s_mov_b32 %0, m0\n\ts_mov_b32 m0, %3\n\ts_nop 0\n\t"
                   "global_load_lds_dwordx4 %1, %2 sc0 sc1\n\tglobal_load_lds_dwordx4 %1, %2 offset:1024 sc0 sc1\n\t"
                   "global_load_lds_dwordx4 %1, %2 offset:2048 sc0 sc1\n\tglobal_load_lds_dwordx4 %1, %2 offset:3072 sc0 sc1\n\t"
                   "s_mov_b32 m0, %0" : "=&s"(keep) : "v"(S.l16), "s"(g), "s"(l) : "memory");
    };
    auto issue = [&](int t) {
      S.look();
      S.spin(t);
      S.poll();
      const char* g = g0 + (size_t)t * SLOT;
      const u32 l = lds0 + (t & 3) * 32768;
      dma4(g, l);
      if (BIG) dma4(g + 4096, l + 4096);
    };
    issue(0); issue(1); issue(2);
    for (int t = 0; t < NSTEP - 3; ++t) {
      asm volatile("s_waitcnt vmcnt(%0)" :: "n"(2 * (NL + 1)) : "memory");
      WG_BARRIER();
      STAMP(1 + t);
      issue(t + 3);
    }
    asm volatile("s_waitcnt vmcnt(%0)" :: "n"(2 * (NL + 1)) : "memory");
    WG_BARRIER();
    STAMP(30);
    asm volatile("s_waitcnt vmcnt(%0)" :: "n"(NL + 1) : "memory");
    WG_BARRIER();
    STAMP(31);
    asm volatile("s_waitcnt vmcnt(0)" ::: "memory");
    WG_BARRIER();
    STAMP(32);
    asm volatile("" : "+v"(S.W));
    if constexpr (!BIG) WG_BARRIER();
  } else {
    constexpr int NC = BIG ? 4 : 2;
    constexpr int G = 4 * NC;
    constexpr int HS = BIG ? 2 : 1;
    constexpr int NH = NSTEP * HS;
    const u32 kh = BIG ? 0 : wv & 1;
    S.init(myws, call, sid0 + (BIG ? wv : wv >> 1), kh, lane);
    f32x16 acc[NC] = {};
    u32 boff = (BIG ? 0 : kh * 16384 + ch * 4096) + lane * 16;
    bf16x8 wf[8];
    u32x4 X[2][4];
    Pay<3> P[4];
    u32 h[8];
    u32 hdr = 0, base4 = 0;
    auto rd = [&](int f) {
      const int s = f / NC, n = f % NC;
      wf[f & 7] = *(const LAS bf16x8*)((LAS char*)smem + boff + (s >> 1) * 8192 + (s & 1) * 1024 + n * 2048);
    };
    auto load_half = [&](Pay<3>& Q, int u, int hs) {
      const char* b = S.sb + (size_t)(u / HS) * SLOT;
      if (hs == 0) pay_load(Q, S.vlo, S.vcode, b);
      else asm volatile("global_load_dwordx4 %2, %3, %4 offset:1024 sc0 sc1\n\t"
                        "global_load_dwordx4 %0, %3, %4 offset:-2048 sc0 sc1\n\t"
                        "global_load_dwordx4 %1, %3, %4 offset:-1024 sc0 sc1"
                        : "=&v"(Q.v[0]), "=&v"(Q.v[1]), "=&v"(Q.v[2]) : "v"(S.l16), "s"(b) : "memory");
    };
    u32 fe = 0, fm = 0;
    auto piece = [&](int k, int v, int us, int waitn, bool fetch) {
      Pay<3>& Q = P[us];
      const int t = v / HS;
      const u32 hv = BIG ? us & 1 : kh;
      if (k == 0) {
        if (waitn == 3) pay_wait<BIG ? 10 : 12>(Q); else if (waitn == 2) pay_wait<BIG ? 7 : 8>(Q); else pay_wait<BIG ? 3 : 4>(Q);
        hdr = S.header(t);
        base4 = (hdr & 0x7f) * 0x01010101u;
        const u32 e1 = __builtin_amdgcn_readlane(S.W[BIG ? 1 + (us & 1) : 1], 2 * t);
        fe = BIG ? e1 : (kh ? __builtin_amdgcn_readlane(S.W[2], 2 * t) : e1);
      } else if (k < 6) {
#pragma unroll
        for (int j = 0; j < 4; ++j) {
          const int i = k < 4 ? 2 * j + (k & 1) : 4 * (k - 4) + j;
          if (k == 1) h[i] = __builtin_amdgcn_alignbit(Q.v[2][j], Q.v[2][j], 4);
          else if (k == 2) h[i] = Q.v[2][j] & 0x87878787u;
          else if (k == 3) h[i] &= 0x87878787u;
          else h[i] += base4;
          asm volatile("" : "+v"(h[i]));
        }
      } else if (k == 6) {
        fm = lane == (int)((fe >> 11) & 63) ? (fe & 0xff) << ((fe >> 6) & 24) : 0u;
        asm volatile("" : "+v"(fm));
      } else if (k == 7) {
        u32x8 g8 = {h[0], h[1], h[2], h[3], h[4], h[5], h[6], h[7]};
        asm volatile("s_set_gpr_idx_on %1, gpr_idx(SRC1,DST)\n\tv_xor_b32 v8, %2, v8\n\ts_set_gpr_idx_off"
                     : "+{v[8:15]}"(g8) : "s"((fe >> 17) & 7), "v"(fm));
        if (__builtin_expect(hdr & 0x600u, 0)) {
          if (hdr & 0x200u) {
            u32x16 g = __builtin_shufflevector(g8, g8, 0, 1, 2, 3, 4, 5, 6, 7, -1, -1, -1, -1, -1, -1, -1, -1);
            S.patch_rest(g, t, hv * 8, lane);
            g8 = __builtin_shufflevector(g, g, 0, 1, 2, 3, 4, 5, 6, 7);
          } else g8 = raw_high(S.sb + (size_t)t * SLOT + hv * 2048, S.l16);
          asm volatile("" : "+{v[8:15]}"(g8));
        }
#pragma unroll
        for (int j = 0; j < 8; ++j) h[j] = g8[j];
      } else if (k < 12) {
        const int c = k - 8;
        X[us & 1][c] = join(Q.v[c >> 1][2 * (c & 1)], Q.v[c >> 1][2 * (c & 1) + 1], h[2 * c], h[2 * c + 1]);
        asm volatile("" : "+v"(X[us & 1][c]));
      } else if (fetch) {
        const int vn = v + 4 < NH ? v + 4 : NH - HS + (us % HS);
        if (us % HS == 0) {
          if (k == 12) S.look();
          else if (k == 13) { S.spin(vn / HS); S.poll(); }
        }
        if (k == 14) load_half(Q, vn, us % HS);
      }
    };
    auto half_step = [&](int u, int us, int waitn, bool decode, bool fetch, bool more) {
      const bool last = us % HS == HS - 1;
#pragma unroll
      for (int q = 0; q < G; ++q) {
        if (last && q == G - 8 && more) { FSTAMP(u / HS, 1); WG_BARRIER(); FSTAMP(u / HS, 2); boff = (boff + 32768) & 131071; }
        else if (q % 4 == 0 && (more || !last || q < G - 4)) __builtin_amdgcn_s_waitcnt(0xC47F);
        __builtin_amdgcn_sched_barrier(0);
        acc[q % NC] = __builtin_amdgcn_mfma_f32_32x32x16_bf16(wf[q & 7], __builtin_bit_cast(bf16x8, X[us & 1][q / NC]), acc[q % NC], 0, 0, 0);
        __builtin_amdgcn_sched_barrier(0);
        const int f = (us % HS) * G + q + 8;
        if (f < HS * G) rd(f);
        else if (more) rd(f - HS * G);
        if (decode) {
          if (BIG) piece(q, u + 1, (us + 1) & 3, waitn, fetch);
          else { piece(2 * q, u + 1, (us + 1) & 3, waitn, fetch); piece(2 * q + 1, u + 1, (us + 1) & 3, waitn, fetch); }
        }
      }
    };
#pragma unroll
    for (int d = 0; d < 4; ++d) { if (d % HS == 0) { S.look(); S.spin(d / HS); S.poll(); } load_half(P[d], d, d % HS); }
#pragma unroll
    for (int k = 0; k < 15; ++k) piece(k, 0, 0, 3, true);
    WG_BARRIER();
#pragma unroll
    for (int f = 0; f < 8; ++f) rd(f);
    for (int ub = 0; ub < NH - 4; ub += 4) {
      FSTAMP(ub / HS, 0);
#pragma unroll
      for (int us = 0; us < 4; ++us) half_step(ub + us, us, 3, true, true, true);
      FSTAMP(ub / HS, 3);
    }
    half_step(NH - 4, 0, 3, true, false, true);
    half_step(NH - 3, 1, 2, true, false, true);
    half_step(NH - 2, 2, 1, true, false, !BIG);
    half_step(NH - 1, 3, 0, false, false, false);
    pay_wait<0>(P[0]);
    asm volatile("" : "+v"(S.W));
    STAMP(1);
    auto put = [&](const f32x16& a16, int row, int col0) {
#pragma unroll
      for (int g = 0; g < 4; ++g) {
        const f32x2 a = {silu(a16[4 * g]), silu(a16[4 * g + 1])};
        const f32x2 b = {silu(a16[4 * g + 2]), silu(a16[4 * g + 3])};
        const u32x2 v = {__builtin_bit_cast(u32, __builtin_convertvector(a, bf16x2)), __builtin_bit_cast(u32, __builtin_convertvector(b, bf16x2))};
        *(u32x2*)(smem + EPI_OFF + row * ERS + (col0 + 8 * g + 4 * (lane >> 5)) * 2) = v;
      }
    };
    if constexpr (BIG) {
#pragma unroll
      for (int n = 0; n < NC; ++n) put(acc[n], wv * 32 + (lane & 31), n * 32);
    } else {
      typedef float f32x4 __attribute__((ext_vector_type(4)));
      char* const xch = smem + EPI_OFF + 64 * ERS + lane * 16;
      const f32x16 give = kh ? acc[0] : acc[1];
      f32x16 keep = kh ? acc[1] : acc[0];
#pragma unroll
      for (int g = 0; g < 4; ++g) *(f32x4*)(xch + wv * 4096 + g * 1024) = f32x4{give[4 * g], give[4 * g + 1], give[4 * g + 2], give[4 * g + 3]};
      WG_BARRIER();
#pragma unroll
      for (int g = 0; g < 4; ++g) {
        const f32x4 o = *(const f32x4*)(xch + (wv ^ 1) * 4096 + g * 1024);
#pragma unroll
        for (int i = 0; i < 4; ++i) keep[4 * g + i] += o[i];
      }
      put(keep, (wv >> 1) * 32 + (lane & 31), kh * 32);
    }
  }
  constexpr int RW = BIG ? 16 : 8, LPR = BIG ? 16 : 8, RPI = 64 / LPR, NI = RW / RPI;
  const u32 sid = sid0 + wv * RW / 32;
  const u32 g0 = (u32)rank & 4u, o0 = g0 ^ 4u;
  u32 src, bi;
  if (sid < (u32)NA) { src = rank; bi = sid; }
  else if (sid < (u32)(NA + 4 * NB)) { src = o0 + (sid - NA) / NB; bi = NA + (sid - NA) % NB; }
  else if (sid < (u32)(64 - 3 * NLATE)) { const u32 q = sid - (NA + 4 * NB); src = g0 + ((rank + 1 + q / NEARLY) & 3); bi = q % NEARLY; }
  else { const u32 q = sid - (64 - 3 * NLATE); src = g0 + ((rank + 1 + q / NLATE) & 3); bi = NEARLY + q % NLATE; }
  const u32 grow = src * 512 + bi * 32 + (wv * RW) % 32;
  const bool remote = tn >= 4;
  const u32 dst = (remote ? o0 + (tn - 4) : (u32)rank) & 7u;
  const u32 colb = (remote ? (rank & 3) : tn) * 256 + ch * 128;
  char* const ob = p->out[dst] + (size_t)grow * (NLOC * 2) + colb;
  u32* const flag = (u32*)(p->ws[dst] + DONE_OFF) + (rank & 3) * 32 + fl;
  const u32 vo = (lane / LPR) * (NLOC * 2) + (lane % LPR) * 16;
  WG_BARRIER();
  STAMP(40);
  const u32 tmo = __hip_atomic_load((u32*)(myws + TMO_OFF), __ATOMIC_RELAXED, __HIP_MEMORY_SCOPE_AGENT);
  u32x4 v[NI];
#pragma unroll
  for (int q = 0; q < NI; ++q) {
    v[q] = *(const u32x4*)(smem + EPI_OFF + (wv * RW + q * RPI + lane / LPR) * ERS + (lane % LPR) * 16);
    if (tmo == call) v[q] = u32x4{0x7fc07fc0u, 0x7fc07fc0u, 0x7fc07fc0u, 0x7fc07fc0u};
  }
#pragma unroll
  for (int q = 0; q < NI; ++q) {
    char* const b = ob + q * RPI * (NLOC * 2);
    asm volatile("global_store_dwordx4 %0, %1, %2 sc0 sc1\n\ts_nop 1" :: "v"(vo), "v"(v[q]), "s"(b) : "memory");
    if (P_THR) asm volatile("s_waitcnt vmcnt(0)" ::: "memory");
  }
  if (remote) {
    asm volatile("s_waitcnt vmcnt(0)" ::: "memory");
    WG_BARRIER();
    if (threadIdx.x == 0) __hip_atomic_store(flag, call, __ATOMIC_RELAXED, __HIP_MEMORY_SCOPE_SYSTEM);
  }
  STAMP(41);
}

__host__ __device__ inline void tile_of_block(u32 rank, u32 b, u32& sid0, u32& tn, u32& ch, u32& fl) {
  const u32 peer = (rank + b + (b < (u32)NBIG ? 0 : P_SFLIP)) & 7;
  tn = (peer & 3) + (((peer ^ rank) & 4) ? 4 : 0);
  if (b < (u32)NBIG) {
    const u32 rt = b >> 3;
    sid0 = rt * 4; ch = 0; fl = rt;
  } else {
    const u32 i = (b - NBIG) >> 3;
    sid0 = RT_BIG * 4 + (i >> 1) * 2; ch = i & 1; fl = RT_BIG + i;
  }
}

__global__ void __launch_bounds__(NTHREAD, 2) ag_gemm_silu(const Params) {
  KP p = (KP)__builtin_amdgcn_kernarg_segment_ptr();
  __shared__ __attribute__((aligned(16))) char smem[LDS_BYTES];
  const int lane = threadIdx.x & 63;
  const int wv = __builtin_amdgcn_readfirstlane(threadIdx.x >> 6);
  char* const myws = p->ws[p->rank];
  if (blockIdx.x >= NGEMM) {
    if (blockIdx.x >= NGEMM + NSENDWG) unpacker(myws, wv, lane);
    else if (wv < P_WPW) sender(p, smem, myws, wv, lane);
    return;
  }
  u32 sid0, tn, ch, fl;
  tile_of_block(p->rank, blockIdx.x, sid0, tn, ch, fl);
  if (blockIdx.x < NBIG) gemm_tile<true>(p, smem, myws, wv, lane, sid0, tn, ch, fl);
  else gemm_tile<false>(p, smem, myws, wv, lane, sid0, tn, ch, fl);
}

static void build_orders(Params& p) {
  struct E { double tau; unsigned short code; };
  E e[N_ORD];
  double rho = P_RHO, sigma = P_SIGMA, lead = P_LEAD;
  for (int kind = 0; kind < 3; ++kind) {
    int n = 0;
    for (int t = 0; t < NSTEP; ++t) {
      if (kind < 2) {
        for (int b = 0; b < NA; ++b) e[n++] = {kind == 0 || b < NEARLY ? (t + 0.5) : (t + 0.5) / rho, (unsigned short)(b << 5 | t)};
      } else {
        for (int i = 0; i < 4; ++i) e[n++] = {(t + 0.5) * sigma - lead, (unsigned short)((16 + i) << 5 | t)};
        for (int b = NA; b < 16; ++b) e[n++] = {t + 0.5, (unsigned short)(b << 5 | t)};
      }
    }
    std::stable_sort(e, e + n, [](const E& a, const E& b) { return a.tau < b.tau; });
    for (int i = 0; i < N_ORD; i += 2) p.ord[kind][i >> 1] = i < n ? e[i].code | (u32)e[i + 1].code << 16 : 0u;
  }
}

extern "C" void kernel_launch(int rank, int nranks, ncclComm_t comm, hipStream_t stream,
                              void* const* d_in, const int* in_sizes, int n_in,
                              void* d_out, int out_size, const int* in_dtypes, int out_dtype,
                              void* const* d_out_all, void* const* d_ws_all, size_t ws_bytes) {
  (void)comm; (void)in_dtypes; (void)out_dtype; (void)d_out;
  if (nranks != NR || (unsigned)rank >= (unsigned)NR || n_in != 2 || ws_bytes < WS_NEED ||
      in_sizes[0] != 512 * KDIM || in_sizes[1] != KDIM * NLOC || out_size != 4096 * NLOC) {
    fprintf(stderr, "unexpected problem shape\n");
    return;
  }
  Params p;
  p.x = (const char*)d_in[0];
  p.w = (const char*)d_in[1];
  for (int i = 0; i < NR; ++i) { p.out[i] = (char*)d_out_all[i]; p.ws[i] = (char*)d_ws_all[i]; }
  p.rank = rank;
  p.pad = 0;
  build_orders(p);
  hipLaunchKernelGGL(ag_gemm_silu, dim3(GRID), dim3(NTHREAD), 0, stream, p);
  hipError_t err = hipGetLastError();
  if (err != hipSuccess) fprintf(stderr, "launch failed: %s\n", hipGetErrorString(err));
}
```

```cpp
#include <hip/hip_runtime.h>
#include <rccl/rccl.h>
#include <cstdio>
#include <cstdint>
#include <cstdlib>
#include <algorithm>

typedef unsigned u32;
typedef unsigned long long u64;
typedef unsigned u32x2 __attribute__((ext_vector_type(2)));
typedef unsigned u32x4 __attribute__((ext_vector_type(4)));
typedef unsigned u32x8 __attribute__((ext_vector_type(8)));
typedef unsigned u32x16 __attribute__((ext_vector_type(16)));
typedef unsigned short u16x2 __attribute__((ext_vector_type(2)));
typedef __bf16 bf16x2 __attribute__((ext_vector_type(2)));
typedef __bf16 bf16x4 __attribute__((ext_vector_type(4)));
typedef __bf16 bf16x8 __attribute__((ext_vector_type(8)));
typedef float f32x2 __attribute__((ext_vector_type(2)));
typedef float f32x4 __attribute__((ext_vector_type(4)));
typedef float f32x16 __attribute__((ext_vector_type(16)));
#define LAS __attribute__((address_space(3)))

#define P_NA 11
#define P_NCWG 4
#define P_WIN 3
#define P_WIN0 6
#define P_SIGMA 0.1
#define P_OVL 0.0
#define P_PP 10
#define P_PARK 10
#define P_REL 2

constexpr int NR = 8;
constexpr int KDIM = 4096, NLOC = 512;
constexpr int NSTEP = 32;
constexpr int NA = P_NA, NB = 16 - NA;
constexpr int N1 = 3, N3 = 4, N2 = NA - N1 - N3;
constexpr int S_1 = NA, S_2 = S_1 + 3 * N1, S_INTER = S_2 + 3 * N2, S_3 = 64 - 3 * N3, S_WIN = 64;
constexpr int NS_PAR = 80, S_IMG = 2 * NS_PAR, S_PARK = S_IMG + 32, NSTREAM = S_PARK + 4;
constexpr size_t SLOT = 8192;
constexpr size_t META_OFF = (size_t)NSTREAM * NSTEP * SLOT;
constexpr size_t DONE_OFF = META_OFF + (size_t)NSTREAM * NSTEP * 32;
constexpr size_t REL_OFF = DONE_OFF + 1024;
constexpr size_t CNT_OFF = REL_OFF + 1024;
constexpr size_t TMO_OFF = CNT_OFF + 256 * 16 * 4;
constexpr size_t DBG_OFF = TMO_OFF + 128;
constexpr size_t WS_NEED = DBG_OFF + 256 * 32 * 64 * 8;
constexpr int N_INTRA = NA * NSTEP, N_INTER = NB * NSTEP + 4 * NSTEP;
constexpr int N_ORD = N_INTRA > N_INTER ? N_INTRA : N_INTER;
constexpr int RT_MED = 13;
constexpr int NGEMM = RT_MED * 16 + 16;
constexpr int NFLAG = 2 * RT_MED + 12 + P_PARK;
constexpr int NCWG = P_NCWG;
constexpr int GRID = NGEMM + 8 * NCWG;
static_assert(GRID <= 256, "one workgroup per CU, all resident");
constexpr int NTHREAD = 768;
constexpr int SCR_OFF = 65536;
constexpr int LDS_BYTES = SCR_OFF + 2 * 128 * 272;
constexpr u64 DEADLINE_TICKS = 50000000ull;
constexpr int ERS = 272;

struct Params {
  const char* x;
  const char* w;
  char* out[NR];
  char* ws[NR];
  int rank;
  alignas(4) unsigned short ord[2][N_ORD];
};
typedef const __attribute__((address_space(4))) Params* KP;

#define STAMP(i) do {} while (0)
#define WG_BARRIER() do { asm volatile("" ::: "memory"); __builtin_amdgcn_s_waitcnt(0xC07F); \
    asm volatile("s_barrier" ::: "memory"); } while (0)
#define ST16(voff, data, sbase, imm) \
  asm volatile("global_store_dwordx4 %0, %1, %2 offset:" #imm " sc0 sc1\n\ts_nop 1" :: "v"(voff), "v"(data), "s"(sbase) : "memory")
#define ST4(voff, data, sbase) \
  asm volatile("global_store_dword %0, %1, %2 sc0 sc1" :: "v"(voff), "v"(data), "s"(sbase) : "memory")
#define ST_SIX(voff, s, b) do { ST16(voff, s[0], b, -4096); ST16(voff, s[1], b, -3072); ST16(voff, s[2], b, -2048); \
    ST16(voff, s[3], b, -1024); ST16(voff, s[4], b, 0); ST16(voff, s[5], b, 1024); } while (0)
#define CAT16(a, b) __builtin_shufflevector(a, b, 0, 1, 2, 3, 4, 5, 6, 7, 8, 9, 10, 11, 12, 13, 14, 15)

__device__ __forceinline__ u32 draw_call(char* myws, int wv) {
  u32 v = 0;
  if ((threadIdx.x & 63) == 0)
    v = __hip_atomic_fetch_add((u32*)(myws + CNT_OFF) + blockIdx.x * 16 + wv, 1u, __ATOMIC_RELAXED, __HIP_MEMORY_SCOPE_AGENT);
  return __builtin_amdgcn_readfirstlane(v) + 1u;
}
__device__ __forceinline__ u32 call_tag(u32 call) { return (call % 2047u + 1u) << 21; }

__device__ __forceinline__ bool spin_fail(char* myws, u32 call, u64 t0) {
  u32* tmo = (u32*)(myws + TMO_OFF);
  u32 t = __builtin_amdgcn_readfirstlane(__hip_atomic_load(tmo, __ATOMIC_RELAXED, __HIP_MEMORY_SCOPE_AGENT));
  if (t == call || __builtin_amdgcn_s_memrealtime() - t0 > DEADLINE_TICKS) {
    __hip_atomic_store(tmo, call, __ATOMIC_RELAXED, __HIP_MEMORY_SCOPE_AGENT);
    return true;
  }
  return false;
}

__device__ __forceinline__ u32 dest_stream(u32 sl, u32 rank, u32 dest, u32 par) {
  const u32 j = ((rank - dest) & 3) - 1;
  const u32 s = sl >= 16 ? S_WIN + (rank & 3) * 4 + (sl - 16) : sl >= (u32)NA ? S_INTER + (rank & 3) * NB + (sl - NA)
              : sl >= (u32)(N1 + N2) ? S_3 + j * N3 + (sl - N1 - N2) : sl >= (u32)N1 ? S_2 + j * N2 + (sl - N1) : S_1 + j * N1 + sl;
  return par * NS_PAR + s;
}

__device__ __forceinline__ u32 global_row(u32 rank, u32 sid) {
  const u32 g0 = rank & 4u, o0 = g0 ^ 4u;
  u32 src, bi;
  if (sid < (u32)S_1) { src = rank; bi = sid; }
  else if (sid < (u32)S_2) { const u32 q = sid - S_1; src = g0 + ((rank + 1 + q / N1) & 3); bi = q % N1; }
  else if (sid < (u32)S_INTER) { const u32 q = sid - S_2; src = g0 + ((rank + 1 + q / N2) & 3); bi = N1 + q % N2; }
  else if (sid < (u32)S_3) { const u32 q = sid - S_INTER; src = o0 + q / NB; bi = NA + q % NB; }
  else { const u32 q = sid - S_3; src = g0 + ((rank + 1 + q / N3) & 3); bi = N1 + N2 + q % N3; }
  return (src * 512 + bi * 32) & 4095u;
}

__device__ __forceinline__ void load_raw(KP p, u32x4 (&d)[8], int lane, u32 sl, u32 q, u32 t) {
  const bool isw = sl >= 16;
  const u32 rs = isw ? NLOC * 2 : KDIM * 2;
  const u32 vo = (lane >> 4) * rs + (lane & 15) * 16;
  const u64 aw = (u64)p->w + (u64)((t * 128 + (sl & 3) * 32) * (NLOC * 2) + q * 256), ax = (u64)p->x + (u64)(sl * 32 * (KDIM * 2) + t * 256);
  const u64 b0 = isw ? aw : ax;
#pragma unroll
  for (int j = 0; j < 8; ++j)
    asm volatile("global_load_dwordx4 %0, %1, %2" : "=&v"(d[j]) : "v"(vo), "s"(b0 + (u64)(j * 4 * rs)) : "memory");
}
#define WAIT_RAW(d) asm volatile("s_waitcnt vmcnt(0)" : "+v"(d[0]), "+v"(d[1]), "+v"(d[2]), "+v"(d[3]), "+v"(d[4]), "+v"(d[5]), "+v"(d[6]), "+v"(d[7]) :: "memory")
__device__ __forceinline__ void to_fragments(u32x4 (&d)[8], bool isw, char* scr, int lane) {
  LAS char* const sc = (LAS char*)scr;
  auto off = [](u32 row, u32 ch) { return 256u * row + 16u * (ch ^ (((row & 3) << 2) | ((row >> 2) & 3))); };
#pragma unroll
  for (int j = 0; j < 8; ++j) *(LAS u32x4*)(sc + off(4 * j + (lane >> 4), lane & 15)) = d[j];
  asm volatile("" ::: "memory");
  if (isw) {
    const u32 r0 = 8 * (lane >> 5) + ((lane & 15) >> 2), c0 = 2 * ((lane >> 4) & 1) + ((lane & 3) >> 1);
#pragma unroll
    for (int j = 0; j < 8; ++j) {
      const bf16x4 lo = __builtin_amdgcn_ds_read_tr16_b64_v4bf16((LAS bf16x4*)(sc + off(16 * (j & 1) + r0, 4 * (j >> 1) + c0) + 8 * (lane & 1)));
      const bf16x4 hi = __builtin_amdgcn_ds_read_tr16_b64_v4bf16((LAS bf16x4*)(sc + off(16 * (j & 1) + r0 + 4, 4 * (j >> 1) + c0) + 8 * (lane & 1)));
      d[j] = __builtin_bit_cast(u32x4, __builtin_shufflevector(lo, hi, 0, 1, 2, 3, 4, 5, 6, 7));
    }
  } else {
#pragma unroll
    for (int j = 0; j < 8; ++j) d[j] = *(const LAS u32x4*)(sc + off(lane & 31, 2 * j + (lane >> 5)));
  }
  asm volatile("" ::: "memory");
}
__device__ __forceinline__ void low_slabs(const u32x4 (&d)[8], u32x4 (&s)[8]) {
#pragma unroll
  for (int i = 0; i < 16; ++i) s[i >> 2][i & 3] = __builtin_amdgcn_perm(d[i >> 1][2 * (i & 1) + 1], d[i >> 1][2 * (i & 1)], 0x06040200u);
}
__device__ __forceinline__ bool encode(const u32x4 (&d)[8], u32x4 (&s)[8], u32& meta, int lane) {
  u32 H[16], T[16];
  u16x2 m1 = {0, 0}, m2 = {0, 0}, m3 = {0, 0}, m4 = {0, 0};
#pragma unroll
  for (int i = 0; i < 16; ++i) {
    const u32 a = d[i >> 1][2 * (i & 1)], b = d[i >> 1][2 * (i & 1) + 1];
    H[i] = __builtin_amdgcn_perm(b, a, 0x07050301u);
  }
#pragma unroll
  for (int i = 0; i < 16; i += 2) {
    T[i] = H[i] | 0x80808080u;
    T[i + 1] = H[i + 1] | 0x80808080u;
    m1 = __builtin_elementwise_max(m1, __builtin_bit_cast(u16x2, T[i]));
    m2 = __builtin_elementwise_max(m2, __builtin_bit_cast(u16x2, T[i] << 8));
    m3 = __builtin_elementwise_max(m3, __builtin_bit_cast(u16x2, T[i + 1]));
    m4 = __builtin_elementwise_max(m4, __builtin_bit_cast(u16x2, T[i + 1] << 8));
  }
  m1 = __builtin_elementwise_max(__builtin_elementwise_max(m1, m2), __builtin_elementwise_max(m3, m4));
  u32 mx = (u32)(m1.x > m1.y ? m1.x : m1.y);
  mx = max(mx, (u32)__builtin_amdgcn_update_dpp(0, mx, 0x111, 0xf, 0xf, true));
  mx = max(mx, (u32)__builtin_amdgcn_update_dpp(0, mx, 0x112, 0xf, 0xf, true));
  mx = max(mx, (u32)__builtin_amdgcn_update_dpp(0, mx, 0x114, 0xf, 0xf, true));
  mx = max(mx, (u32)__builtin_amdgcn_update_dpp(0, mx, 0x118, 0xf, 0xf, true));
  const u32 top = max(max((u32)__builtin_amdgcn_readlane(mx, 15), (u32)__builtin_amdgcn_readlane(mx, 31)),
                      max((u32)__builtin_amdgcn_readlane(mx, 47), (u32)__builtin_amdgcn_readlane(mx, 63)));
  const u32 h7 = (top >> 8) & 0x7f;
  const u32 base = __builtin_amdgcn_readfirstlane(h7 > 7 ? h7 - 7 : 0);
  const u32 base4 = base * 0x01010101u;
  u32 ok = 0xffffffffu;
  u32 N[8];
#pragma unroll
  for (int j = 0; j < 8; ++j) {
    const u32 ta = T[2 * j] - base4, tb = T[2 * j + 1] - base4;
    ok &= ta & tb;
    const u32 ca = (ta & 0x07070707u) | (H[2 * j] & 0x80808080u);
    const u32 cb = (tb & 0x07070707u) | (H[2 * j + 1] & 0x80808080u);
    N[j] = ca | __builtin_amdgcn_alignbit(cb, cb, 28);
  }
  meta = 0;
  int c0 = 0, c1 = 0;
  if (__any((~ok & 0x80808080u) != 0)) {
    u32 M0 = 0, M1 = 0;
#pragma unroll
    for (int i = 0; i < 8; ++i) {
      M0 |= ~((T[i] - base4) >> i) & (0x80808080u >> i);
      M1 |= ~((T[8 + i] - base4) >> i) & (0x80808080u >> i);
    }
    u64 bal = __ballot((M0 | M1) != 0);
    while (bal != 0 && c0 + c1 <= 7) {
      const int ln = __builtin_ctzll(bal);
      bal &= bal - 1;
      u64 mm = (u64)(u32)__builtin_amdgcn_readlane(M0, ln) | (u64)(u32)__builtin_amdgcn_readlane(M1, ln) << 32;
      while (mm != 0) {
        const u32 bit = __builtin_ctzll(mm), hf = bit >> 5, by = (bit >> 3) & 3, i = hf * 8 + 7 - (bit & 7);
        mm &= mm - 1;
        c0 += hf ^ 1; c1 += hf;
        u32 hw = 0;
#pragma unroll
        for (int z = 0; z < 16; ++z) hw |= H[z] & (u32)__builtin_amdgcn_sbfe(1 << i, z, 1);
        const u32 hb = ((u32)__builtin_amdgcn_readlane(hw, ln) >> (8 * by)) & 0x7f;
        const u32 ent = (i << 17 | (u32)ln << 11 | by << 9) | ((((hb - base) & 7) + base) ^ hb);
        if (lane == (hf ? 8 - c1 : c0)) meta = ent;
      }
    }
  }
  const bool raw = c0 + c1 > 7;
  if (lane == 0) meta = base | (raw ? 0x400u : (u32)(c0 << 11 | c1 << 14));
  s[4] = raw ? u32x4{H[0], H[1], H[2], H[3]} : u32x4{N[0], N[1], N[2], N[3]};
  s[5] = raw ? u32x4{H[4], H[5], H[6], H[7]} : u32x4{N[4], N[5], N[6], N[7]};
  s[6] = u32x4{H[8], H[9], H[10], H[11]};
  s[7] = u32x4{H[12], H[13], H[14], H[15]};
#pragma unroll
  for (int j = 0; j < 8; ++j) asm volatile("" : "+v"(s[j]));
  return raw;
}

template <int N> struct Pay { u32x4 v[N]; };
__device__ __forceinline__ void pay_load(Pay<6>& P, u32 l16, const char* sb) {
  asm volatile("global_load_dwordx4 %0, %6, %7 offset:-4096 sc0 sc1\n\t"
               "global_load_dwordx4 %1, %6, %7 offset:-3072 sc0 sc1\n\t"
               "global_load_dwordx4 %2, %6, %7 offset:-2048 sc0 sc1\n\t"
               "global_load_dwordx4 %3, %6, %7 offset:-1024 sc0 sc1\n\t"
               "global_load_dwordx4 %4, %6, %7 offset:0 sc0 sc1\n\t"
               "global_load_dwordx4 %5, %6, %7 offset:1024 sc0 sc1"
               : "=&v"(P.v[0]), "=&v"(P.v[1]), "=&v"(P.v[2]), "=&v"(P.v[3]), "=&v"(P.v[4]), "=&v"(P.v[5]) : "v"(l16), "s"(sb) : "memory");
}
__device__ __forceinline__ void pay_load(Pay<3>& P, u32 vlo, u32 vcode, const char* sb) {
  asm volatile("global_load_dwordx4 %2, %4, %5 offset:0 sc0 sc1\n\t"
               "global_load_dwordx4 %0, %3, %5 offset:-4096 sc0 sc1\n\t"
               "global_load_dwordx4 %1, %3, %5 offset:-3072 sc0 sc1"
               : "=&v"(P.v[0]), "=&v"(P.v[1]), "=&v"(P.v[2]) : "v"(vlo), "v"(vcode), "s"(sb) : "memory");
}
template <int C> __device__ __forceinline__ void pay_wait(Pay<6>& P) {
  asm volatile("s_waitcnt vmcnt(%6)" : "+v"(P.v[0]), "+v"(P.v[1]), "+v"(P.v[2]), "+v"(P.v[3]), "+v"(P.v[4]), "+v"(P.v[5]) : "n"(C) : "memory");
}
template <int C> __device__ __forceinline__ void pay_wait(Pay<3>& P) {
  asm volatile("s_waitcnt vmcnt(%3)" : "+v"(P.v[0]), "+v"(P.v[1]), "+v"(P.v[2]) : "n"(C) : "memory");
}
__device__ __forceinline__ u32x8 load_two(const char* b, u32 l16) {
  u32x4 x, y;
  asm volatile("global_load_dwordx4 %0, %2, %3 sc0 sc1\n\t"
               "global_load_dwordx4 %1, %2, %3 offset:1024 sc0 sc1\n\ts_waitcnt vmcnt(0)"
               : "=&v"(x), "=&v"(y) : "v"(l16), "s"(b) : "memory");
  return __builtin_shufflevector(x, y, 0, 1, 2, 3, 4, 5, 6, 7);
}

__device__ __forceinline__ void sender(KP p, char* smem, char* myws, int wv, int lane) {
  [[maybe_unused]] const int sto = 0;
  const u32 rank = p->rank;
  const u32 cb = blockIdx.x - NGEMM;
  const u32 link = cb & 7, cw = cb >> 3;
  const u32 l16 = lane * 16;
  STAMP(0);
  if (link != 0) {
    const u32 dest = (rank + link) & 7u;
    const bool intra = !((dest ^ rank) & 4);
    const int n = ((intra ? N_INTRA : N_INTER) - (int)cw + NCWG - 1) / NCWG;
    auto code_of = [=](int k) {
      const u32 e = cw + (u32)(k < n ? k : 0) * NCWG;
      const u32 word = ((const __attribute__((address_space(4))) u32*)p->ord[intra ? 0 : 1])[e >> 1];
      return (word >> (16 * (e & 1))) & 0xffffu;
    };
    LAS int* const done = (LAS int*)(smem + 12 * 8192);
    if (threadIdx.x == 0) *done = 0;
    u32 ticket = 0;
    if (lane == 0) ticket = __hip_atomic_fetch_add((u32*)(myws + CNT_OFF) + blockIdx.x * 16 + wv, 1u, __ATOMIC_RELAXED, __HIP_MEMORY_SCOPE_AGENT);
    u32x4 d[8];
    u32 code = code_of(wv);
    if (wv >= 4) { WG_BARRIER(); if (wv >= 8) __builtin_amdgcn_s_sleep(16); }
    load_raw(p, d, lane, code >> 5, dest & 3, code & 31);
    if (wv < 4) WG_BARRIER();
    const u32 call = __builtin_amdgcn_readfirstlane(ticket) + 1u;
    const u32 tag = call_tag(call), par = call & 1;
    char* const rws = p->ws[dest];
    if (wv < 4) __builtin_amdgcn_s_setprio(2); else if (wv < 8) __builtin_amdgcn_s_setprio(1);
    int k = wv;
    for (; k < n; k += 12) {
      WAIT_RAW(d);
      if (k < 12) STAMP(20);
      to_fragments(d, code >= 512, smem + wv * 8192, lane);
      const u32 ud = dest_stream(code >> 5, rank, dest, par) * NSTEP + (code & 31);
      char* const b = rws + (size_t)ud * SLOT + 4096;
      u32x4 s[8];
      low_slabs(d, s);
      if (k < 4) { ST16(l16, s[0], b, -4096); ST16(l16, s[1], b, -3072); ST16(l16, s[2], b, -2048); ST16(l16, s[3], b, -1024); }
      u32 meta;
      const bool raw = encode(d, s, meta, lane);
      meta |= tag;
      code = code_of(k + 12);
      if (k < 12) STAMP(21);
      __builtin_amdgcn_s_setprio(0);
      while (__hip_atomic_load(done, __ATOMIC_RELAXED, __HIP_MEMORY_SCOPE_WORKGROUP) < k - (k < P_WIN0 ? P_WIN0 : P_WIN - 1)) __builtin_amdgcn_s_sleep(1);
      if (k >= 4) { ST16(l16, s[0], b, -4096); ST16(l16, s[1], b, -3072); ST16(l16, s[2], b, -2048); ST16(l16, s[3], b, -1024); }
      ST16(l16, s[4], b, 0); ST16(l16, s[5], b, 1024);
      if (raw) { ST16(l16, s[6], b, 2048); ST16(l16, s[7], b, 3072); }
      if (P_PARK == 0 && k == n - 1 - P_REL && lane == 0)
        __hip_atomic_store((u32*)(myws + REL_OFF + link * 128) + cw, call, __ATOMIC_RELAXED, __HIP_MEMORY_SCOPE_AGENT);
      asm volatile("s_waitcnt vmcnt(0)" ::: "memory");
      char* const m = rws + META_OFF + (size_t)ud * 32;
      if (lane < 8) ST4(lane * 4, meta, m);
      if (lane == 0) __hip_atomic_fetch_add(done, 1, __ATOMIC_RELAXED, __HIP_MEMORY_SCOPE_WORKGROUP);
      load_raw(p, d, lane, code >> 5, dest & 3, code & 31);
      STAMP(3 + k / 12);
    }
    WAIT_RAW(d);
    STAMP(1);
    const u32 y = cw + (u32)(k - n) * NCWG;
    if (!intra && y < 2u * P_PARK) {
      const u32 us = (S_PARK + (dest & 3)) * NSTEP + y;
      const u64 t0 = __builtin_amdgcn_s_memrealtime();
      const u32* const hd = (const u32*)(myws + META_OFF + (size_t)us * 32);
      for (u32 it = 1; __builtin_amdgcn_readfirstlane(__hip_atomic_load(hd, __ATOMIC_RELAXED, __HIP_MEMORY_SCOPE_SYSTEM)) != tag; ++it) {
        if ((it & 31) == 0 && spin_fail(myws, call, t0)) break;
        __builtin_amdgcn_s_sleep(4);
      }
      const char* const sp = myws + (size_t)us * SLOT + 4096;
      asm volatile("global_load_dwordx4 %0, %8, %9 offset:-4096 sc0 sc1\n\tglobal_load_dwordx4 %1, %8, %9 offset:-3072 sc0 sc1\n\t"
                   "global_load_dwordx4 %2, %8, %9 offset:-2048 sc0 sc1\n\tglobal_load_dwordx4 %3, %8, %9 offset:-1024 sc0 sc1\n\t"
                   "global_load_dwordx4 %4, %8, %9 offset:0 sc0 sc1\n\tglobal_load_dwordx4 %5, %8, %9 offset:1024 sc0 sc1\n\t"
                   "global_load_dwordx4 %6, %8, %9 offset:2048 sc0 sc1\n\tglobal_load_dwordx4 %7, %8, %9 offset:3072 sc0 sc1\n\ts_waitcnt vmcnt(0)"
                   : "=&v"(d[0]), "=&v"(d[1]), "=&v"(d[2]), "=&v"(d[3]), "=&v"(d[4]), "=&v"(d[5]), "=&v"(d[6]), "=&v"(d[7])
                   : "v"(l16), "s"(sp) : "memory");
      if (__hip_atomic_load((u32*)(myws + TMO_OFF), __ATOMIC_RELAXED, __HIP_MEMORY_SCOPE_AGENT) == call) d[0] = u32x4{0x7fc07fc0u, 0x7fc07fc0u, 0x7fc07fc0u, 0x7fc07fc0u};
      const u32 tile = y >> 1, sid = (tile >> 1) * 4 + (y & 1) * 2;
      char* const ob = p->out[dest] + (rank & 3) * 256 + (tile & 1) * 128;
      const u32 vo = (lane >> 3) * (NLOC * 2) + (lane & 7) * 16;
      while (__hip_atomic_load(done, __ATOMIC_RELAXED, __HIP_MEMORY_SCOPE_WORKGROUP) < k - (P_WIN - 1)) __builtin_amdgcn_s_sleep(1);
#pragma unroll
      for (int g = 0; g < 8; ++g) {
        char* const b = ob + (size_t)(global_row(rank, sid + (g >> 2)) + (g & 3) * 8) * (NLOC * 2);
        ST16(vo, d[g], b, 0);
      }
      if (y + P_REL * NCWG >= 2u * P_PARK && y + P_REL * NCWG < 2u * P_PARK + NCWG && lane == 0)
        __hip_atomic_store((u32*)(myws + REL_OFF + link * 128) + cw, call, __ATOMIC_RELAXED, __HIP_MEMORY_SCOPE_AGENT);
      asm volatile("s_waitcnt vmcnt(0)" ::: "memory");
      char* const fl = rws + DONE_OFF + ((rank & 3) * 64 + ((y & 1) ? 2 * RT_MED + 12 + tile : tile)) * 4;
      if (lane == 0) { ST4(0u, call, fl); __hip_atomic_fetch_add(done, 1, __ATOMIC_RELAXED, __HIP_MEMORY_SCOPE_WORKGROUP); }
      STAMP(2);
    }
  } else if (cb == 0 && wv == 0) {
    const u32 call = draw_call(myws, wv);
    const u64 t0 = __builtin_amdgcn_s_memrealtime();
    const char* const f = myws + DONE_OFF;
    const int f0 = (4 * lane) & 63;
    for (u32 n = 1;; ++n) {
      u32x4 v;
      asm volatile("global_load_dwordx4 %0, %1, %2 sc0 sc1\n\ts_waitcnt vmcnt(0)" : "=&v"(v) : "v"(l16), "s"(f) : "memory");
      if (__all((f0 >= NFLAG || v[0] >= call) && (f0 + 1 >= NFLAG || v[1] >= call) && (f0 + 2 >= NFLAG || v[2] >= call) && (f0 + 3 >= NFLAG || v[3] >= call))) break;
      if ((n & 31) == 0 && spin_fail(myws, call, t0)) {
        if (lane == 0) *(unsigned short*)(p->out[rank] + (size_t)rank * 512 * (NLOC * 2)) = 0x7fc0;
        break;
      }
    }
    STAMP(2);
  }
}

__device__ __forceinline__ u32x4 join(u32 l0, u32 l1, u32 h0, u32 h1) {
  return u32x4{__builtin_amdgcn_perm(h0, l0, 0x05010400u), __builtin_amdgcn_perm(h0, l0, 0x07030602u),
               __builtin_amdgcn_perm(h1, l1, 0x05010400u), __builtin_amdgcn_perm(h1, l1, 0x07030602u)};
}
__device__ __forceinline__ void fix_word(u32 (&h)[8], u32 e) {
  asm volatile("s_lshl_b64 exec, 1, %8\n\ts_getpc_b64 vcc\n\ts_add_u32 vcc_lo, vcc_lo, %9\n\ts_addc_u32 vcc_hi, vcc_hi, 0\n\ts_setpc_b64 vcc\n\t"
               "v_xor_b32_e32 %0, %10, %0\n\ts_branch 1f\n\tv_xor_b32_e32 %1, %10, %1\n\ts_branch 1f\n\t"
               "v_xor_b32_e32 %2, %10, %2\n\ts_branch 1f\n\tv_xor_b32_e32 %3, %10, %3\n\ts_branch 1f\n\t"
               "v_xor_b32_e32 %4, %10, %4\n\ts_branch 1f\n\tv_xor_b32_e32 %5, %10, %5\n\ts_branch 1f\n\t"
               "v_xor_b32_e32 %6, %10, %6\n\ts_branch 1f\n\tv_xor_b32_e32 %7, %10, %7\n"
               "1:\n\ts_mov_b64 exec, -1"
               : "+v"(h[0]), "+v"(h[1]), "+v"(h[2]), "+v"(h[3]), "+v"(h[4]), "+v"(h[5]), "+v"(h[6]), "+v"(h[7])
               : "s"(e >> 11), "s"(((e >> 14) & 56) + 12), "s"((e & 0xff) << ((e >> 6) & 24)) : "vcc", "scc");
}

struct Stream {
  const char* sb; const char* mb; char* myws;
  u32 l16, tag, call; u64 t0;
  u32x4 W; int rdy;
  __device__ __forceinline__ void init(char* ws, u32 call_, u32 sid, int lane) {
    myws = ws; call = call_; tag = call_tag(call_);
    sb = ws + (size_t)sid * NSTEP * SLOT + 4096;
    mb = ws + META_OFF + (size_t)sid * NSTEP * 32;
    l16 = lane * 16;
    t0 = __builtin_amdgcn_s_memrealtime();
    W = u32x4{0u, 0u, 0u, 0u}; rdy = 0;
  }
  __device__ __forceinline__ void poll() {
    const u64 mask = 0xffffull << (rdy < 24 ? 2 * rdy : 48);
    asm volatile("s_mov_b64 exec, %3\n\tglobal_load_dwordx4 %0, %1, %2 sc0 sc1\n\ts_mov_b64 exec, -1"
                 : "+v"(W) : "v"(l16), "s"(mb), "s"(mask) : "memory");
  }
  __device__ __forceinline__ void look() {
    asm volatile("" : "+v"(W));
    const u32 x = (W[0] ^ tag) | (W[1] ^ tag) | (W[2] ^ tag) | (W[3] ^ tag);
    const u64 bad = __ballot(x >= (1u << 21));
    rdy = bad ? __builtin_ctzll(bad) >> 1 : NSTEP;
  }
  __device__ __forceinline__ void spin(int t) {
    for (u32 n = 1; rdy <= t; ++n) {
      poll();
      asm volatile("s_waitcnt vmcnt(0)" ::: "memory");
      look();
      if (rdy == 0) __builtin_amdgcn_s_sleep(8);
      if ((n & 31) == 0 && spin_fail(myws, call, t0)) rdy = NSTEP;
    }
  }
  __device__ __forceinline__ u32 header(int t) { asm volatile("" : "+v"(W)); return __builtin_amdgcn_readlane(W[0], 2 * t); }
  __device__ __forceinline__ void repair(u32 (&h)[8], u32 hdr, int t, u32 hf) {
#pragma nounroll
    for (u32 n = (hdr >> (11 + 3 * hf)) & 7, k = hf ? 7 : 1; n != 0; --n, k += hf ? -1 : 1) {
      const int sl = 2 * t + (k >> 2);
      const u32 e0 = __builtin_amdgcn_readlane(W[0], sl), e1 = __builtin_amdgcn_readlane(W[1], sl);
      const u32 e2 = __builtin_amdgcn_readlane(W[2], sl), e3 = __builtin_amdgcn_readlane(W[3], sl);
      fix_word(h, (k & 2) ? ((k & 1) ? e3 : e2) : ((k & 1) ? e1 : e0));
    }
    if (__builtin_expect(hdr & 0x400u, 0)) {
      const u32x8 g = load_two(sb + (size_t)t * SLOT + hf * 2048, l16);
#pragma unroll
      for (int j = 0; j < 8; ++j) asm volatile("v_mov_b32 %0, %1" : "+v"(h[j]) : "v"(g[j]));
    }
  }
};

__device__ __forceinline__ void unpack_unit(char* myws, [[maybe_unused]] int wv, int lane, u32 call, u32 r, int t) {
  [[maybe_unused]] const int sto = 0;
  Stream S;
  S.init(myws, call, (call & 1) * NS_PAR + S_WIN + r, lane);
  const u32 l4 = lane * 4;
  const u32 uo = (S_IMG + 16 + r) * NSTEP;
  {
    for (u32 n = 1;; ++n) {
      const u64 mask = 3ull << (2 * t);
      asm volatile("s_mov_b64 exec, %3\n\tglobal_load_dwordx4 %0, %1, %2 sc0 sc1\n\ts_mov_b64 exec, -1\n\ts_waitcnt vmcnt(0)"
                   : "+v"(S.W) : "v"(S.l16), "s"(S.mb), "s"(mask) : "memory");
      const u32 x = (S.W[0] ^ S.tag) | (S.W[1] ^ S.tag) | (S.W[2] ^ S.tag) | (S.W[3] ^ S.tag);
      if (!(__ballot(x >= (1u << 21)) & mask)) break;
      if ((n & 31) == 0 && spin_fail(myws, call, S.t0)) break;
      __builtin_amdgcn_s_sleep(4);
    }
    Pay<6> P;
    pay_load(P, S.l16, S.sb + (size_t)t * SLOT);
    pay_wait<0>(P);
    const u32 hdr = S.header(t), base4 = (hdr & 0x7f) * 0x01010101u;
    u32x4 o[8];
#pragma unroll
    for (int hf = 0; hf < 2; ++hf) {
      u32 g[8];
#pragma unroll
      for (int j = 0; j < 4; ++j) {
        const u32 n = P.v[4 + hf][j];
        g[2 * j] = (n & 0x87878787u) + base4;
        g[2 * j + 1] = (__builtin_amdgcn_alignbit(n, n, 4) & 0x87878787u) + base4;
      }
      if (hdr & (0x400u | 0x3800u << (3 * hf))) S.repair(g, hdr, t, hf);
#pragma unroll
      for (int j = 0; j < 4; ++j)
        o[4 * hf + j] = join(P.v[2 * hf + (j >> 1)][2 * (j & 1)], P.v[2 * hf + (j >> 1)][2 * (j & 1) + 1], g[2 * j], g[2 * j + 1]);
    }
    char* const ob = myws + (size_t)(uo + t) * SLOT + 4096;
    ST_SIX(S.l16, o, ob); ST16(S.l16, o[6], ob, 2048); ST16(S.l16, o[7], ob, 3072);
    asm volatile("s_waitcnt vmcnt(0)" ::: "memory");
    char* const mo = myws + META_OFF + (size_t)(uo + t) * 32;
    if (lane < 8) ST4(l4, S.tag, mo);
    STAMP(50);
  }
}

__device__ __forceinline__ float silu(float y) {
  const float z = __builtin_amdgcn_fmed3f(y, -60.0f, 60.0f);
  const float e = __builtin_amdgcn_exp2f(z * -1.44269504088896341f);
  return y * __builtin_amdgcn_rcpf(1.0f + e);
}

__device__ __forceinline__ void gemm_tile(KP p, char* smem, char* myws, int wv, int lane, u32 call, u32 nb, u32 sid0, u32 tn, u32 ch, u32 fl, int hold,
                                          [[maybe_unused]] int sto) {
  const u32 rank = p->rank;
  STAMP(0);
  const u32 o0 = (rank & 4u) ^ 4u;
  const bool remote = tn >= 4;
  const u32 dst = (remote ? o0 + (tn - 4) : rank) & 7u;
  char* const ob = p->out[dst] + (remote ? (rank & 3) : tn) * 256 + ch * 128;
  u32* flag = (u32*)(p->ws[dst] + DONE_OFF) + (rank & 3) * 64 + fl;
  auto grow = [&](u32 rg) { return global_row(rank, sid0 + (rg >> 2)) + (rg & 3) * 8; };
  char* b0 = ob + (size_t)grow(wv & 7) * (NLOC * 2);
  char* b1 = ob + (size_t)grow((wv & 7) + 8) * (NLOC * 2);
  if (hold == 2) {
    const u32 us = (S_PARK + (tn & 3)) * NSTEP + 2 * fl;
    b0 = myws + (size_t)us * SLOT + (wv & 7) * 1024; b1 = b0 + SLOT;
    flag = (u32*)(myws + META_OFF + (size_t)us * 32);
  }
  asm volatile("" : "+s"(b0), "+s"(b1), "+s"(flag));
  Stream S;
  const u32 m = wv >> 1, kh = wv & 1;
  if (wv >= 8) {
    const u32 un = wv - 8;
    S.init(myws, call, S_IMG + 4 * tn + un, lane);
    const u32 lds0 = (u32)(size_t)(LAS char*)smem + un * 4096;
    const char* const g0 = S.sb - 4096 + ch * 4096;
    auto issue = [&](int t) {
      S.look();
      S.spin(t);
      S.poll();
      const char* g = g0 + (size_t)t * SLOT;
      const u32 l = lds0 + (t & 3) * 16384;
      u32 keep;
      asm volatile("s_mov_b32 %0, m0\n\ts_mov_b32 m0, %3\n\ts_nop 0\n\t"
                   "global_load_lds_dwordx4 %1, %2 sc0 sc1\n\tglobal_load_lds_dwordx4 %1, %2 offset:1024 sc0 sc1\n\t"
                   "global_load_lds_dwordx4 %1, %2 offset:2048 sc0 sc1\n\tglobal_load_lds_dwordx4 %1, %2 offset:3072 sc0 sc1\n\t"
                   "s_mov_b32 m0, %0" : "=&s"(keep) : "v"(S.l16), "s"(g), "s"(l) : "memory");
    };
    issue(0); issue(1); issue(2);
    for (int t = 0; t < NSTEP - 3; ++t) {
      asm volatile("s_waitcnt vmcnt(10)" ::: "memory");
      WG_BARRIER();
      STAMP(1 + t);
      issue(t + 3);
    }
    asm volatile("s_waitcnt vmcnt(10)" ::: "memory");
    WG_BARRIER();
    STAMP(30);
    asm volatile("s_waitcnt vmcnt(5)" ::: "memory");
    WG_BARRIER();
    STAMP(31);
    asm volatile("s_waitcnt vmcnt(0)" : "+v"(S.W) :: "memory");
    WG_BARRIER();
    STAMP(32);
    if (hold == 1 && wv == 8) {
      const u32* const r = (const u32*)(myws + REL_OFF + (blockIdx.x & 7) * 128) + (lane < NCWG ? lane : 0);
      for (u32 n = 1; !__all(__hip_atomic_load(r, __ATOMIC_RELAXED, __HIP_MEMORY_SCOPE_AGENT) >= call); ++n) {
        if ((n & 31) == 0 && spin_fail(myws, call, S.t0)) break;
        __builtin_amdgcn_s_sleep(4);
      }
    }
  } else if (m >= nb) {
    for (int t = 0; t < NSTEP; ++t) WG_BARRIER();
  } else {
    S.init(myws, call, (call & 1) * NS_PAR + sid0 + m, lane);
    const u32 vlo = S.l16 + kh * 2048, vcode = S.l16 + kh * 1024;
    const u32 rmask = 0x400u | 0x3800u << (3 * kh);
    f32x16 acc[2] = {};
    const LAS char* const bb = (LAS char*)smem + kh * 8192 + lane * 16;
    bf16x8 wf[4];
    u32x4 X[2][4];
    Pay<3> P[4];
    u32 h[8];
    u32 hdr = 0, base4 = 0;
    auto rd = [&](int stage, int f) {
      wf[f & 3] = *(const LAS bf16x8*)(bb + stage * 16384 + (f >> 2) * 4096 + ((f >> 1) & 1) * 1024 + (f & 1) * 2048);
    };
    auto piece = [&](int k, int t, int ts, int waitn, bool fetch) {
      Pay<3>& Q = P[ts];
      if (k == 0) {
        if (waitn == 3) pay_wait<9>(Q); else if (waitn == 2) pay_wait<6>(Q); else pay_wait<3>(Q);
        hdr = S.header(t);
        base4 = (hdr & 0x7f) * 0x01010101u;
      } else if (k < 6) {
#pragma unroll
        for (int j = 0; j < 4; ++j) {
          const int i = k < 4 ? 2 * j + (k & 1) : 4 * (k - 4) + j;
          if (k == 1) h[i] = __builtin_amdgcn_alignbit(Q.v[2][j], Q.v[2][j], 4);
          else if (k == 2) h[i] = Q.v[2][j] & 0x87878787u;
          else if (k == 3) h[i] &= 0x87878787u;
          else h[i] += base4;
          asm volatile("" : "+v"(h[i]));
        }
      } else if (k == 6) {
        if (hdr & rmask) S.repair(h, hdr, t, kh);
      } else if (k == 7) {
      } else if (k < 12) {
        const int c = k - 8;
        X[ts & 1][c] = join(Q.v[c >> 1][2 * (c & 1)], Q.v[c >> 1][2 * (c & 1) + 1], h[2 * c], h[2 * c + 1]);
        asm volatile("" : "+v"(X[ts & 1][c]));
      } else if (fetch) {
        const int tn4 = t + 4 < NSTEP ? t + 4 : NSTEP - 1;
        if (k == 12) { if (S.rdy < NSTEP && S.rdy <= tn4 + 2) { S.look(); S.spin(tn4); S.poll(); } }
        else if (k == 13) pay_load(Q, vlo, vcode, S.sb + (size_t)tn4 * SLOT);
      }
    };
    auto step = [&](int t, int ts, int waitn, bool fetch, bool more) {
#pragma unroll
      for (int q = 0; q < 8; ++q) {
        if (q == 4 && more) WG_BARRIER();
        else if (q % 2 == 0 && (more || q < 6)) __builtin_amdgcn_s_waitcnt(0xC27F);
        __builtin_amdgcn_sched_barrier(0);
        acc[q & 1] = __builtin_amdgcn_mfma_f32_32x32x16_bf16(wf[q & 3], __builtin_bit_cast(bf16x8, X[ts & 1][q >> 1]), acc[q & 1], 0, 0, 0);
        __builtin_amdgcn_sched_barrier(0);
        if (q < 4) rd(ts, q + 4);
        else if (more) rd((ts + 1) & 3, q - 4);
        if (more) {
          piece(2 * q, t + 1, (ts + 1) & 3, waitn, fetch);
          piece(2 * q + 1, t + 1, (ts + 1) & 3, waitn, fetch);
        }
      }
    };
#pragma unroll
    for (int d = 0; d < 4; ++d) { S.look(); S.spin(d); S.poll(); pay_load(P[d], vlo, vcode, S.sb + (size_t)d * SLOT); }
#pragma unroll
    for (int k = 0; k < 15; ++k) piece(k, 0, 0, 3, true);
    WG_BARRIER();
#pragma unroll
    for (int f = 0; f < 4; ++f) { rd(0, f); __builtin_amdgcn_sched_barrier(0); }
    for (int tb = 0; tb < NSTEP - 4; tb += 4) {
#pragma unroll
      for (int ts = 0; ts < 4; ++ts) step(tb + ts, ts, 3, true, true);
    }
    step(NSTEP - 4, 0, 3, false, true);
    step(NSTEP - 3, 1, 2, false, true);
    step(NSTEP - 2, 2, 1, false, true);
    step(NSTEP - 1, 3, 0, false, false);
    STAMP(1);
    char* const xq = smem + SCR_OFF + (kh * 128 + m * 32 + (lane & 31)) * ERS + (lane >> 5) * 16;
#pragma unroll
    for (int n = 0; n < 2; ++n)
#pragma unroll
      for (int g = 0; g < 4; ++g)
        *(f32x4*)(xq + n * 128 + g * 32) = f32x4{acc[n][4 * g], acc[n][4 * g + 1], acc[n][4 * g + 2], acc[n][4 * g + 3]};
    pay_wait<0>(P[0]);
    asm volatile("" : "+v"(S.W));
  }
  const u32 tmo = __hip_atomic_load((u32*)(myws + TMO_OFF), __ATOMIC_RELAXED, __HIP_MEMORY_SCOPE_AGENT);
  WG_BARRIER();
  STAMP(40);
  if (wv < 8) {
    const u32 vo = hold == 2 ? lane * 16 : (lane >> 3) * (NLOC * 2) + (lane & 7) * 16;
    auto get = [&](u32 rg) {
      const char* const q = smem + SCR_OFF + (rg * 8 + (lane >> 3)) * ERS + (lane & 7) * 32;
      const f32x4 a = *(const f32x4*)q + *(const f32x4*)(q + 128 * ERS), c = *(const f32x4*)(q + 16) + *(const f32x4*)(q + 128 * ERS + 16);
      auto pk = [](float lo, float hi) { const f32x2 v = {silu(lo), silu(hi)}; return __builtin_bit_cast(u32, __builtin_convertvector(v, bf16x2)); };
      u32x4 v = {pk(a[0], a[1]), pk(a[2], a[3]), pk(c[0], c[1]), pk(c[2], c[3])};
      if (tmo == call) v = u32x4{0x7fc07fc0u, 0x7fc07fc0u, 0x7fc07fc0u, 0x7fc07fc0u};
      return v;
    };
    const u32x4 v0 = get(wv);
    ST16(vo, v0, b0, 0);
    if (nb > 2) { const u32x4 v1 = get(wv + 8); ST16(vo, v1, b1, 0); }
  }
  if (remote) {
    asm volatile("s_waitcnt vmcnt(0)" ::: "memory");
    WG_BARRIER();
    if (threadIdx.x < (hold == 2 ? 2 : 1)) ST4(threadIdx.x * 32, hold == 2 ? call_tag(call) : call, flag);
  }
  STAMP(41);
}

__host__ __device__ inline bool tile_of(u32 rank, u32 b, int k, u32& nb, u32& sid0, u32& tn, u32& ch, u32& fl, int& hold) {
  const u32 peer = (rank + b) & 7, i = b >> 3;
  const bool home = !((peer ^ rank) & 4), spare = i >= 2 * RT_MED;
  hold = 0;
  if (k == 0 && !spare) {
    nb = 4; tn = (peer & 3) + (home ? 0 : 4); sid0 = (i >> 1) * 4; ch = i & 1; fl = i;
    hold = home || sid0 >= (u32)S_INTER ? 0 : i < (u32)P_PARK ? 2 : i >= 10 ? 1 : 0;
    return true;
  }
  const u32 j = spare ? i - 2 * RT_MED + 10 : i;
  nb = 2; tn = (peer & 3) + (home ? 4 : 0); sid0 = RT_MED * 4 + (j >> 1) * 2; ch = j & 1; fl = 2 * RT_MED + j;
  return spare ? k == 0 : i < 10;
}

constexpr u32 NHOME = (NA + 16) * NSTEP;
__host__ __device__ inline u32 pack_index(u32 b, u32 wv) {
  constexpr u32 NP1 = 80;
  return b < NP1 ? (wv < 2 ? b + NP1 * wv : NHOME) : 2 * NP1 + (b - NP1) + (NGEMM - NP1) * wv;
}

__host__ __device__ inline u32 unpack_index(u32 b, u32 wv) {
  constexpr u32 B0 = S_INTER / 4 * 16, NW = RT_MED * 16 - B0;
  return b >= B0 && b < B0 + NW ? (11 - wv) * NW + (b - B0) : 512u;
}

__global__ void __launch_bounds__(NTHREAD, 3) ag_gemm_silu(const Params) {
  KP p = (KP)__builtin_amdgcn_kernarg_segment_ptr();
  __shared__ __attribute__((aligned(16))) char smem[LDS_BYTES];
  const int lane = threadIdx.x & 63;
  const int wv = __builtin_amdgcn_readfirstlane(threadIdx.x >> 6);
  const u32 rank = p->rank;
  char* const myws = p->ws[rank];
  if (blockIdx.x >= NGEMM) { sender(p, smem, myws, wv, lane); return; }
  u32 call;
  const u32 u = pack_index(blockIdx.x, wv);
  if (u < NHOME) {
    const u32 t = u / (NA + 16), c = u % (NA + 16);
    const bool isw = c >= (u32)NA;
    const u32 cw = c - NA, l16 = lane * 16;
    const u64 t0 = __builtin_amdgcn_s_memrealtime();
    while (__builtin_amdgcn_s_memrealtime() - t0 < t * P_PP) __builtin_amdgcn_s_sleep(2);
    u32x4 d[8], s[8];
    load_raw(p, d, lane, isw ? 16 + (cw & 3) : c, (cw >> 2) & 3, t);
    call = draw_call(myws, wv);
    WAIT_RAW(d);
    u32 meta = 0;
    bool eight = true;
    to_fragments(d, isw, smem + SCR_OFF + wv * 8192, lane);
    if (isw) {
#pragma unroll
      for (int j = 0; j < 8; ++j) s[j] = d[j];
    } else { low_slabs(d, s); eight = encode(d, s, meta, lane); }
    const u32 ud = (isw ? S_IMG + cw : (call & 1) * NS_PAR + c) * NSTEP + t;
    char* const b = myws + (size_t)ud * SLOT + 4096;
    ST_SIX(l16, s, b);
    if (eight) { ST16(l16, s[6], b, 2048); ST16(l16, s[7], b, 3072); }
    asm volatile("s_waitcnt vmcnt(0)" ::: "memory");
    char* const m = myws + META_OFF + (size_t)ud * 32;
    if (lane < 8) ST4(lane * 4, meta | call_tag(call), m);
    { [[maybe_unused]] const int sto = 0; STAMP(51); }
  } else call = draw_call(myws, wv);
  const u32 v = unpack_index(blockIdx.x, wv);
  if (v < 512) unpack_unit(myws, wv, lane, call, v & 15, v >> 4);
#pragma nounroll
  for (int k = 0; k < 2; ++k) {
    u32 nb, sid0, tn, ch, fl; int hold;
    if (!tile_of(rank, blockIdx.x, k, nb, sid0, tn, ch, fl, hold)) break;
    gemm_tile(p, smem, myws, wv, lane, call, nb, sid0, tn, ch, fl, hold, 16 * k);
  }
}

static void build_orders(Params& p) {
  struct E { double tau; unsigned short code; };
  E e[N_ORD];
  double sigma = P_SIGMA, ovl = P_OVL;
  for (int kind = 0; kind < 2; ++kind) {
    int n = 0;
    for (int t = 0; t < NSTEP; ++t) {
      if (kind == 0) {
        for (int b = 0; b < NA; ++b) e[n++] = {t + (b < N1 ? 0 : b < N1 + N2 ? 1 : 2) * (NSTEP - ovl), (unsigned short)(b << 5 | t)};
      } else {
        for (int i = 0; i < 4; ++i) e[n++] = {t * sigma, (unsigned short)((16 + i) << 5 | t)};
        for (int b = NA; b < 16; ++b) e[n++] = {t + 0.5, (unsigned short)(b << 5 | t)};
      }
    }
    std::stable_sort(e, e + n, [](const E& a, const E& b) { return a.tau < b.tau; });
    for (int i = 0; i < N_ORD; ++i) p.ord[kind][i] = i < n ? e[i].code : 0;
  }
}

extern "C" void kernel_launch(int rank, int nranks, ncclComm_t comm, hipStream_t stream,
                              void* const* d_in, const int* in_sizes, int n_in,
                              void* d_out, int out_size, const int* in_dtypes, int out_dtype,
                              void* const* d_out_all, void* const* d_ws_all, size_t ws_bytes) {
  (void)comm; (void)in_dtypes; (void)out_dtype; (void)d_out;
  if (nranks != NR || (unsigned)rank >= (unsigned)NR || n_in != 2 || ws_bytes < WS_NEED ||
      in_sizes[0] != 512 * KDIM || in_sizes[1] != KDIM * NLOC || out_size != 4096 * NLOC) {
    fprintf(stderr, "unexpected problem shape\n");
    return;
  }
  Params p = {};
  p.x = (const char*)d_in[0];
  p.w = (const char*)d_in[1];
  for (int i = 0; i < NR; ++i) { p.out[i] = (char*)d_out_all[i]; p.ws[i] = (char*)d_ws_all[i]; }
  p.rank = rank;
  build_orders(p);
  hipLaunchKernelGGL(ag_gemm_silu, dim3(GRID), dim3(NTHREAD), 0, stream, p);
  hipError_t err = hipGetLastError();
  if (err != hipSuccess) fprintf(stderr, "launch failed: %s\n", hipGetErrorString(err));
}
```

```cpp
#include <hip/hip_runtime.h>
#include <rccl/rccl.h>
#include <cstdio>
#include <cstdint>
#include <cstdlib>
#include <algorithm>

typedef unsigned u32;
typedef unsigned long long u64;
typedef unsigned u32x2 __attribute__((ext_vector_type(2)));
typedef unsigned u32x4 __attribute__((ext_vector_type(4)));
typedef unsigned u32x8 __attribute__((ext_vector_type(8)));
typedef unsigned u32x16 __attribute__((ext_vector_type(16)));
typedef unsigned short u16x2 __attribute__((ext_vector_type(2)));
typedef __bf16 bf16x2 __attribute__((ext_vector_type(2)));
typedef __bf16 bf16x4 __attribute__((ext_vector_type(4)));
typedef __bf16 bf16x8 __attribute__((ext_vector_type(8)));
typedef float f32x2 __attribute__((ext_vector_type(2)));
typedef float f32x4 __attribute__((ext_vector_type(4)));
typedef float f32x16 __attribute__((ext_vector_type(16)));
#define LAS __attribute__((address_space(3)))

#define P_NA 11
#define P_NCWG 4
#define P_WIN 3
#define P_WIN0 6
#define P_WINE 0
#define P_WIN1 3
#define P_DIST 3
#define P_SIGMA 0.1
#define P_OVL 0.0
#define P_OVL2 0.0
#define P_LEAD 0.0
#define P_ANT ""
#define P_PP 10
#define P_PARK 10
#define P_REL 2

constexpr int NR = 8;
constexpr int KDIM = 4096, NLOC = 512;
constexpr int NSTEP = 32;
constexpr int NA = P_NA, NB = 16 - NA;
constexpr int N1 = 3, N3 = 4, N2 = NA - N1 - N3;
constexpr int S_1 = NA, S_2 = S_1 + 3 * N1, S_INTER = S_2 + 3 * N2, S_3 = 64 - 3 * N3, S_WIN = 64;
constexpr int NS_PAR = 80, S_IMG = 2 * NS_PAR, S_PARK = S_IMG + 32, NSTREAM = S_PARK + 4;
constexpr size_t SLOT = 8192;
constexpr size_t META_OFF = (size_t)NSTREAM * NSTEP * SLOT;
constexpr size_t DONE_OFF = META_OFF + (size_t)NSTREAM * NSTEP * 32;
constexpr size_t REL_OFF = DONE_OFF + 1024;
constexpr size_t CNT_OFF = REL_OFF + 1024;
constexpr size_t TMO_OFF = CNT_OFF + 256 * 16 * 4;
constexpr size_t DBG_OFF = TMO_OFF + 128;
constexpr size_t WS_NEED = DBG_OFF + 256 * 32 * 64 * 8;
constexpr int N_INTRA = NA * NSTEP, N_INTER = NB * NSTEP + 4 * NSTEP;
constexpr int N_ORD = N_INTRA > N_INTER ? N_INTRA : N_INTER;
constexpr int RT_MED = 13;
constexpr int NGEMM = RT_MED * 16 + 16;
constexpr int NFLAG = 2 * RT_MED + 12 + P_PARK;
constexpr int NCWG = P_NCWG;
constexpr int GRID = NGEMM + 8 * NCWG;
static_assert(GRID <= 256, "one workgroup per CU, all resident");
constexpr int NTHREAD = 768;
constexpr int SCR_OFF = 65536;
constexpr int LDS_BYTES = SCR_OFF + 2 * 128 * 272;
constexpr u64 DEADLINE_TICKS = 50000000ull;
constexpr int ERS = 272;

#define P_PHYS 0x75416320u
__host__ __device__ inline u32 phys_rank(u32 l) { return (P_PHYS >> (4 * l)) & 7u; }
__host__ __device__ inline u32 peer_of_xcd(u32 rank, u32 x) { return (((rank ^ x) & 4u) | ((rank + x) & 3u)) & 7u; }

struct Params {
  const char* x;
  const char* w;
  char* myws;
  char* out[NR];
  char* ws[NR];
  int rank;
  alignas(4) unsigned short ord[2][N_ORD];
};
typedef const __attribute__((address_space(4))) Params* KP;

#define STAMP(i) do {} while (0)
#define WG_BARRIER() do { asm volatile("" ::: "memory"); __builtin_amdgcn_s_waitcnt(0xC07F); \
    asm volatile("s_barrier" ::: "memory"); } while (0)
#define ST16(voff, data, sbase, imm) \
  asm volatile("global_store_dwordx4 %0, %1, %2 offset:" #imm " sc0 sc1\n\ts_nop 1" :: "v"(voff), "v"(data), "s"(sbase) : "memory")
#define ST4(voff, data, sbase) \
  asm volatile("global_store_dword %0, %1, %2 sc0 sc1" :: "v"(voff), "v"(data), "s"(sbase) : "memory")
#define ST_SIX(voff, s, b) do { ST16(voff, s[0], b, -4096); ST16(voff, s[1], b, -3072); ST16(voff, s[2], b, -2048); \
    ST16(voff, s[3], b, -1024); ST16(voff, s[4], b, 0); ST16(voff, s[5], b, 1024); } while (0)
#define CAT16(a, b) __builtin_shufflevector(a, b, 0, 1, 2, 3, 4, 5, 6, 7, 8, 9, 10, 11, 12, 13, 14, 15)

__device__ __forceinline__ u32 draw_call(char* myws, int wv) {
  u32 v = 0;
  if ((threadIdx.x & 63) == 0)
    v = __hip_atomic_fetch_add((u32*)(myws + CNT_OFF) + blockIdx.x * 16 + wv, 1u, __ATOMIC_RELAXED, __HIP_MEMORY_SCOPE_AGENT);
  return __builtin_amdgcn_readfirstlane(v) + 1u;
}
__device__ __forceinline__ u32 call_tag(u32 call) { return (call % 2047u + 1u) << 21; }

__device__ __forceinline__ bool spin_fail(char* myws, u32 call, u64 t0) {
  u32* tmo = (u32*)(myws + TMO_OFF);
  u32 t = __builtin_amdgcn_readfirstlane(__hip_atomic_load(tmo, __ATOMIC_RELAXED, __HIP_MEMORY_SCOPE_AGENT));
  if (t == call || __builtin_amdgcn_s_memrealtime() - t0 > DEADLINE_TICKS) {
    __hip_atomic_store(tmo, call, __ATOMIC_RELAXED, __HIP_MEMORY_SCOPE_AGENT);
    return true;
  }
  return false;
}

__device__ __forceinline__ u32 dest_stream(u32 sl, u32 rank, u32 dest, u32 par) {
  const u32 j = ((rank - dest) & 3) - 1;
  const u32 s = sl >= 16 ? S_WIN + (rank & 3) * 4 + (sl - 16) : sl >= (u32)NA ? S_INTER + (rank & 3) * NB + (sl - NA)
              : sl >= (u32)(N1 + N2) ? S_3 + j * N3 + (sl - N1 - N2) : sl >= (u32)N1 ? S_2 + j * N2 + (sl - N1) : S_1 + j * N1 + sl;
  return par * NS_PAR + s;
}

__device__ __forceinline__ u32 global_row(u32 rank, u32 sid) {
  const u32 g0 = rank & 4u, o0 = g0 ^ 4u;
  u32 src, bi;
  if (sid < (u32)S_1) { src = rank; bi = sid; }
  else if (sid < (u32)S_2) { const u32 q = sid - S_1; src = g0 + ((rank + 1 + q / N1) & 3); bi = q % N1; }
  else if (sid < (u32)S_INTER) { const u32 q = sid - S_2; src = g0 + ((rank + 1 + q / N2) & 3); bi = N1 + q % N2; }
  else if (sid < (u32)S_3) { const u32 q = sid - S_INTER; src = o0 + q / NB; bi = NA + q % NB; }
  else { const u32 q = sid - S_3; src = g0 + ((rank + 1 + q / N3) & 3); bi = N1 + N2 + q % N3; }
  return (phys_rank(src & 7u) * 512 + bi * 32) & 4095u;
}

__device__ __forceinline__ void load_raw(KP p, u32x4 (&d)[8], int lane, u32 sl, u32 q, u32 t) {
  const bool isw = sl >= 16;
  const u32 rs = isw ? NLOC * 2 : KDIM * 2;
  const u32 vo = (lane >> 4) * rs + (lane & 15) * 16;
  const u64 aw = (u64)p->w + (u64)((t * 128 + (sl & 3) * 32) * (NLOC * 2) + q * 256), ax = (u64)p->x + (u64)(sl * 32 * (KDIM * 2) + t * 256);
  const u64 b0 = isw ? aw : ax;
#pragma unroll
  for (int j = 0; j < 8; ++j)
    asm volatile("global_load_dwordx4 %0, %1, %2" : "=&v"(d[j]) : "v"(vo), "s"(b0 + (u64)(j * 4 * rs)) : "memory");
}
#define WAIT_RAW(d) asm volatile("s_waitcnt vmcnt(0)" : "+v"(d[0]), "+v"(d[1]), "+v"(d[2]), "+v"(d[3]), "+v"(d[4]), "+v"(d[5]), "+v"(d[6]), "+v"(d[7]) :: "memory")
__device__ __forceinline__ void to_fragments(u32x4 (&d)[8], bool isw, char* scr, int lane) {
  LAS char* const sc = (LAS char*)scr;
  auto off = [](u32 row, u32 ch) { return 256u * row + 16u * (ch ^ (((row & 3) << 2) | ((row >> 2) & 3))); };
#pragma unroll
  for (int j = 0; j < 8; ++j) *(LAS u32x4*)(sc + off(4 * j + (lane >> 4), lane & 15)) = d[j];
  asm volatile("" ::: "memory");
  if (isw) {
    const u32 r0 = 8 * (lane >> 5) + ((lane & 15) >> 2), c0 = 2 * ((lane >> 4) & 1) + ((lane & 3) >> 1);
#pragma unroll
    for (int j = 0; j < 8; ++j) {
      const bf16x4 lo = __builtin_amdgcn_ds_read_tr16_b64_v4bf16((LAS bf16x4*)(sc + off(16 * (j & 1) + r0, 4 * (j >> 1) + c0) + 8 * (lane & 1)));
      const bf16x4 hi = __builtin_amdgcn_ds_read_tr16_b64_v4bf16((LAS bf16x4*)(sc + off(16 * (j & 1) + r0 + 4, 4 * (j >> 1) + c0) + 8 * (lane & 1)));
      d[j] = __builtin_bit_cast(u32x4, __builtin_shufflevector(lo, hi, 0, 1, 2, 3, 4, 5, 6, 7));
    }
  } else {
#pragma unroll
    for (int j = 0; j < 8; ++j) d[j] = *(const LAS u32x4*)(sc + off(lane & 31, 2 * j + (lane >> 5)));
  }
  asm volatile("" ::: "memory");
}
__device__ __forceinline__ void low_slabs(const u32x4 (&d)[8], u32x4 (&s)[8]) {
#pragma unroll
  for (int i = 0; i < 16; ++i) s[i >> 2][i & 3] = __builtin_amdgcn_perm(d[i >> 1][2 * (i & 1) + 1], d[i >> 1][2 * (i & 1)], 0x06040200u);
}
__device__ __forceinline__ bool encode(const u32x4 (&d)[8], u32x4 (&s)[8], u32& meta, int lane) {
  u32 H[16], T[16];
  u16x2 m1 = {0, 0}, m2 = {0, 0}, m3 = {0, 0}, m4 = {0, 0};
#pragma unroll
  for (int i = 0; i < 16; ++i) {
    const u32 a = d[i >> 1][2 * (i & 1)], b = d[i >> 1][2 * (i & 1) + 1];
    H[i] = __builtin_amdgcn_perm(b, a, 0x07050301u);
  }
#pragma unroll
  for (int i = 0; i < 16; i += 2) {
    T[i] = H[i] | 0x80808080u;
    T[i + 1] = H[i + 1] | 0x80808080u;
    m1 = __builtin_elementwise_max(m1, __builtin_bit_cast(u16x2, T[i]));
    m2 = __builtin_elementwise_max(m2, __builtin_bit_cast(u16x2, T[i] << 8));
    m3 = __builtin_elementwise_max(m3, __builtin_bit_cast(u16x2, T[i + 1]));
    m4 = __builtin_elementwise_max(m4, __builtin_bit_cast(u16x2, T[i + 1] << 8));
  }
  m1 = __builtin_elementwise_max(__builtin_elementwise_max(m1, m2), __builtin_elementwise_max(m3, m4));
  u32 mx = (u32)(m1.x > m1.y ? m1.x : m1.y);
  mx = max(mx, (u32)__builtin_amdgcn_update_dpp(0, mx, 0x111, 0xf, 0xf, true));
  mx = max(mx, (u32)__builtin_amdgcn_update_dpp(0, mx, 0x112, 0xf, 0xf, true));
  mx = max(mx, (u32)__builtin_amdgcn_update_dpp(0, mx, 0x114, 0xf, 0xf, true));
  mx = max(mx, (u32)__builtin_amdgcn_update_dpp(0, mx, 0x118, 0xf, 0xf, true));
  const u32 top = max(max((u32)__builtin_amdgcn_readlane(mx, 15), (u32)__builtin_amdgcn_readlane(mx, 31)),
                      max((u32)__builtin_amdgcn_readlane(mx, 47), (u32)__builtin_amdgcn_readlane(mx, 63)));
  const u32 h7 = (top >> 8) & 0x7f;
  const u32 base = __builtin_amdgcn_readfirstlane(h7 > 7 ? h7 - 7 : 0);
  const u32 base4 = base * 0x01010101u;
  u32 ok = 0xffffffffu;
  u32 N[8];
#pragma unroll
  for (int j = 0; j < 8; ++j) {
    const u32 ta = T[2 * j] - base4, tb = T[2 * j + 1] - base4;
    ok &= ta & tb;
    const u32 ca = (ta & 0x07070707u) | (H[2 * j] & 0x80808080u);
    const u32 cb = (tb & 0x07070707u) | (H[2 * j + 1] & 0x80808080u);
    N[j] = ca | __builtin_amdgcn_alignbit(cb, cb, 28);
  }
  meta = 0;
  int c0 = 0, c1 = 0;
  if (__any((~ok & 0x80808080u) != 0)) {
    u32 M0 = 0, M1 = 0;
#pragma unroll
    for (int i = 0; i < 8; ++i) {
      M0 |= ~((T[i] - base4) >> i) & (0x80808080u >> i);
      M1 |= ~((T[8 + i] - base4) >> i) & (0x80808080u >> i);
    }
    u64 bal = __ballot((M0 | M1) != 0);
    while (bal != 0 && c0 + c1 <= 7) {
      const int ln = __builtin_ctzll(bal);
      bal &= bal - 1;
      u64 mm = (u64)(u32)__builtin_amdgcn_readlane(M0, ln) | (u64)(u32)__builtin_amdgcn_readlane(M1, ln) << 32;
      while (mm != 0) {
        const u32 bit = __builtin_ctzll(mm), hf = bit >> 5, by = (bit >> 3) & 3, i = hf * 8 + 7 - (bit & 7);
        mm &= mm - 1;
        c0 += hf ^ 1; c1 += hf;
        u32 hw = 0;
#pragma unroll
        for (int z = 0; z < 16; ++z) hw |= H[z] & (u32)__builtin_amdgcn_sbfe(1 << i, z, 1);
        const u32 hb = ((u32)__builtin_amdgcn_readlane(hw, ln) >> (8 * by)) & 0x7f;
        const u32 ent = (i << 17 | (u32)ln << 11 | by << 9) | ((((hb - base) & 7) + base) ^ hb);
        if (lane == (hf ? 8 - c1 : c0)) meta = ent;
      }
    }
  }
  const bool raw = c0 + c1 > 7;
  if (lane == 0) meta = base | (raw ? 0x400u : (u32)(c0 << 11 | c1 << 14));
  s[4] = raw ? u32x4{H[0], H[1], H[2], H[3]} : u32x4{N[0], N[1], N[2], N[3]};
  s[5] = raw ? u32x4{H[4], H[5], H[6], H[7]} : u32x4{N[4], N[5], N[6], N[7]};
  s[6] = u32x4{H[8], H[9], H[10], H[11]};
  s[7] = u32x4{H[12], H[13], H[14], H[15]};
#pragma unroll
  for (int j = 0; j < 8; ++j) asm volatile("" : "+v"(s[j]));
  return raw;
}

template <int N> struct Pay { u32x4 v[N]; };
__device__ __forceinline__ void pay_load(Pay<6>& P, u32 l16, const char* sb) {
  asm volatile("global_load_dwordx4 %0, %6, %7 offset:-4096 sc0 sc1\n\t"
               "global_load_dwordx4 %1, %6, %7 offset:-3072 sc0 sc1\n\t"
               "global_load_dwordx4 %2, %6, %7 offset:-2048 sc0 sc1\n\t"
               "global_load_dwordx4 %3, %6, %7 offset:-1024 sc0 sc1\n\t"
               "global_load_dwordx4 %4, %6, %7 offset:0 sc0 sc1\n\t"
               "global_load_dwordx4 %5, %6, %7 offset:1024 sc0 sc1"
               : "=&v"(P.v[0]), "=&v"(P.v[1]), "=&v"(P.v[2]), "=&v"(P.v[3]), "=&v"(P.v[4]), "=&v"(P.v[5]) : "v"(l16), "s"(sb) : "memory");
}
__device__ __forceinline__ void pay_load(Pay<3>& P, u32 vlo, u32 vcode, const char* sb) {
  asm volatile("global_load_dwordx4 %2, %4, %5 offset:0 sc0 sc1" P_ANT "\n\t"
               "global_load_dwordx4 %0, %3, %5 offset:-4096 sc0 sc1" P_ANT "\n\t"
               "global_load_dwordx4 %1, %3, %5 offset:-3072 sc0 sc1" P_ANT
               : "=&v"(P.v[0]), "=&v"(P.v[1]), "=&v"(P.v[2]) : "v"(vlo), "v"(vcode), "s"(sb) : "memory");
}
template <int C> __device__ __forceinline__ void pay_wait(Pay<6>& P) {
  asm volatile("s_waitcnt vmcnt(%6)" : "+v"(P.v[0]), "+v"(P.v[1]), "+v"(P.v[2]), "+v"(P.v[3]), "+v"(P.v[4]), "+v"(P.v[5]) : "n"(C) : "memory");
}
template <int C> __device__ __forceinline__ void pay_wait(Pay<3>& P) {
  asm volatile("s_waitcnt vmcnt(%3)" : "+v"(P.v[0]), "+v"(P.v[1]), "+v"(P.v[2]) : "n"(C) : "memory");
}
__device__ __forceinline__ u32x8 load_two(const char* b, u32 l16) {
  u32x4 x, y;
  asm volatile("global_load_dwordx4 %0, %2, %3 sc0 sc1\n\t"
               "global_load_dwordx4 %1, %2, %3 offset:1024 sc0 sc1\n\ts_waitcnt vmcnt(0)"
               : "=&v"(x), "=&v"(y) : "v"(l16), "s"(b) : "memory");
  return __builtin_shufflevector(x, y, 0, 1, 2, 3, 4, 5, 6, 7);
}

__device__ __forceinline__ void sender(KP p, char* smem, char* myws, int wv, int lane) {
  [[maybe_unused]] const int sto = 0;
  const u32 rank = p->rank;
  const u32 cb = blockIdx.x - NGEMM;
  const u32 link = cb & 7, cw = cb >> 3;
  const u32 l16 = lane * 16;
  STAMP(0);
  if (link != 0) {
    const u32 dest = peer_of_xcd(rank, link);
    const bool intra = link < 4;
    const int n = ((intra ? N_INTRA : N_INTER) - (int)cw + NCWG - 1) / NCWG;
    auto code_of = [=](int k) {
      const u32 e = cw + (u32)(k < n ? k : 0) * NCWG;
      const u32 word = ((const __attribute__((address_space(4))) u32*)p->ord[intra ? 0 : 1])[e >> 1];
      return (word >> (16 * (e & 1))) & 0xffffu;
    };
    LAS int* const done = (LAS int*)(smem + 12 * 8192);
    if (threadIdx.x == 0) *done = 0;
    u32 ticket = 0;
    if (lane == 0) ticket = __hip_atomic_fetch_add((u32*)(myws + CNT_OFF) + blockIdx.x * 16 + wv, 1u, __ATOMIC_RELAXED, __HIP_MEMORY_SCOPE_AGENT);
    u32x4 d[8];
    u32 code = code_of(wv);
    if (wv >= 4) { WG_BARRIER(); if (wv >= 8) __builtin_amdgcn_s_sleep(16); }
    load_raw(p, d, lane, code >> 5, dest & 3, code & 31);
    if (wv < 4) WG_BARRIER();
    const u32 call = __builtin_amdgcn_readfirstlane(ticket) + 1u;
    const u32 tag = call_tag(call), par = call & 1;
    char* const rws = p->ws[dest];
    if (wv < 4) __builtin_amdgcn_s_setprio(2); else if (wv < 8) __builtin_amdgcn_s_setprio(1);
    int k = wv;
    for (; k < n; k += 12) {
      WAIT_RAW(d);
      if (k < 12) STAMP(20);
      to_fragments(d, code >= 512, smem + wv * 8192, lane);
      const u32 ud = dest_stream(code >> 5, rank, dest, par) * NSTEP + (code & 31);
      char* const b = rws + (size_t)ud * SLOT + 4096;
      u32x4 s[8];
      low_slabs(d, s);
      if (k < 4) { ST16(l16, s[0], b, -4096); ST16(l16, s[1], b, -3072); ST16(l16, s[2], b, -2048); ST16(l16, s[3], b, -1024); }
      u32 meta;
      const bool raw = encode(d, s, meta, lane);
      meta |= tag;
      code = code_of(k + 12);
      if (k < 12) STAMP(21);
      __builtin_amdgcn_s_setprio(0);
      const int need = k < P_WIN0 ? 0 : min(k, intra ? n - P_WINE : n) - ((k < 24 ? P_WIN1 : P_WIN) - 1);
      while (__hip_atomic_load(done, __ATOMIC_RELAXED, __HIP_MEMORY_SCOPE_WORKGROUP) < need) __builtin_amdgcn_s_sleep(1);
      if (k < 12) STAMP(22);
      if (k >= 4) { ST16(l16, s[0], b, -4096); ST16(l16, s[1], b, -3072); ST16(l16, s[2], b, -2048); ST16(l16, s[3], b, -1024); }
      ST16(l16, s[4], b, 0); ST16(l16, s[5], b, 1024);
      if (raw) { ST16(l16, s[6], b, 2048); ST16(l16, s[7], b, 3072); }
      if (P_PARK == 0 && k == n - 1 - P_REL && lane == 0)
        __hip_atomic_store((u32*)(myws + REL_OFF + link * 128) + cw, call, __ATOMIC_RELAXED, __HIP_MEMORY_SCOPE_AGENT);
      asm volatile("s_waitcnt vmcnt(0)" ::: "memory");
      char* const m = rws + META_OFF + (size_t)ud * 32;
      if (lane < 8) ST4(lane * 4, meta, m);
      if (lane == 0) __hip_atomic_fetch_add(done, 1, __ATOMIC_RELAXED, __HIP_MEMORY_SCOPE_WORKGROUP);
      load_raw(p, d, lane, code >> 5, dest & 3, code & 31);
      STAMP(3 + k / 12);
    }
    WAIT_RAW(d);
    STAMP(1);
    const u32 y = cw + (u32)(k - n) * NCWG;
    if (!intra && y < 2u * P_PARK) {
      const u32 us = (S_PARK + (dest & 3)) * NSTEP + y;
      const u64 t0 = __builtin_amdgcn_s_memrealtime();
      const u32* const hd = (const u32*)(myws + META_OFF + (size_t)us * 32);
      for (u32 it = 1; __builtin_amdgcn_readfirstlane(__hip_atomic_load(hd, __ATOMIC_RELAXED, __HIP_MEMORY_SCOPE_SYSTEM)) != tag; ++it) {
        if ((it & 31) == 0 && spin_fail(myws, call, t0)) break;
        __builtin_amdgcn_s_sleep(4);
      }
      const char* const sp = myws + (size_t)us * SLOT + 4096;
      asm volatile("global_load_dwordx4 %0, %8, %9 offset:-4096 sc0 sc1\n\tglobal_load_dwordx4 %1, %8, %9 offset:-3072 sc0 sc1\n\t"
                   "global_load_dwordx4 %2, %8, %9 offset:-2048 sc0 sc1\n\tglobal_load_dwordx4 %3, %8, %9 offset:-1024 sc0 sc1\n\t"
                   "global_load_dwordx4 %4, %8, %9 offset:0 sc0 sc1\n\tglobal_load_dwordx4 %5, %8, %9 offset:1024 sc0 sc1\n\t"
                   "global_load_dwordx4 %6, %8, %9 offset:2048 sc0 sc1\n\tglobal_load_dwordx4 %7, %8, %9 offset:3072 sc0 sc1\n\ts_waitcnt vmcnt(0)"
                   : "=&v"(d[0]), "=&v"(d[1]), "=&v"(d[2]), "=&v"(d[3]), "=&v"(d[4]), "=&v"(d[5]), "=&v"(d[6]), "=&v"(d[7])
                   : "v"(l16), "s"(sp) : "memory");
      if (__hip_atomic_load((u32*)(myws + TMO_OFF), __ATOMIC_RELAXED, __HIP_MEMORY_SCOPE_AGENT) == call) d[0] = u32x4{0x7fc07fc0u, 0x7fc07fc0u, 0x7fc07fc0u, 0x7fc07fc0u};
      const u32 tile = y >> 1, sid = (tile >> 1) * 4 + (y & 1) * 2;
      char* const ob = p->out[dest] + (rank & 3) * 256 + (tile & 1) * 128;
      const u32 vo = (lane >> 3) * (NLOC * 2) + (lane & 7) * 16;
      while (__hip_atomic_load(done, __ATOMIC_RELAXED, __HIP_MEMORY_SCOPE_WORKGROUP) < k - (P_WIN - 1)) __builtin_amdgcn_s_sleep(1);
#pragma unroll
      for (int g = 0; g < 8; ++g) {
        char* const b = ob + (size_t)(global_row(rank, sid + (g >> 2)) + (g & 3) * 8) * (NLOC * 2);
        ST16(vo, d[g], b, 0);
      }
      if (y + P_REL * NCWG >= 2u * P_PARK && y + P_REL * NCWG < 2u * P_PARK + NCWG && lane == 0)
        __hip_atomic_store((u32*)(myws + REL_OFF + link * 128) + cw, call, __ATOMIC_RELAXED, __HIP_MEMORY_SCOPE_AGENT);
      asm volatile("s_waitcnt vmcnt(0)" ::: "memory");
      char* const fl = rws + DONE_OFF + ((rank & 3) * 64 + ((y & 1) ? 2 * RT_MED + 12 + tile : tile)) * 4;
      if (lane == 0) { ST4(0u, call, fl); __hip_atomic_fetch_add(done, 1, __ATOMIC_RELAXED, __HIP_MEMORY_SCOPE_WORKGROUP); }
      STAMP(2);
    }
  } else if (cb == 0 && wv == 0) {
    const u32 call = draw_call(myws, wv);
    const u64 t0 = __builtin_amdgcn_s_memrealtime();
    const char* const f = myws + DONE_OFF;
    const int f0 = (4 * lane) & 63;
    for (u32 n = 1;; ++n) {
      u32x4 v;
      asm volatile("global_load_dwordx4 %0, %1, %2 sc0 sc1\n\ts_waitcnt vmcnt(0)" : "=&v"(v) : "v"(l16), "s"(f) : "memory");
      if (__all((f0 >= NFLAG || v[0] >= call) && (f0 + 1 >= NFLAG || v[1] >= call) && (f0 + 2 >= NFLAG || v[2] >= call) && (f0 + 3 >= NFLAG || v[3] >= call))) break;
      if ((n & 31) == 0 && spin_fail(myws, call, t0)) {
        if (lane == 0) *(unsigned short*)(p->out[rank] + (size_t)phys_rank(rank) * 512 * (NLOC * 2)) = 0x7fc0;
        break;
      }
    }
    STAMP(2);
  }
}

__device__ __forceinline__ u32x4 join(u32 l0, u32 l1, u32 h0, u32 h1) {
  return u32x4{__builtin_amdgcn_perm(h0, l0, 0x05010400u), __builtin_amdgcn_perm(h0, l0, 0x07030602u),
               __builtin_amdgcn_perm(h1, l1, 0x05010400u), __builtin_amdgcn_perm(h1, l1, 0x07030602u)};
}
__device__ __forceinline__ void fix_word(u32 (&h)[8], u32 e) {
  asm volatile("s_lshl_b64 exec, 1, %8\n\ts_getpc_b64 vcc\n\ts_add_u32 vcc_lo, vcc_lo, %9\n\ts_addc_u32 vcc_hi, vcc_hi, 0\n\ts_setpc_b64 vcc\n\t"
               "v_xor_b32_e32 %0, %10, %0\n\ts_branch 1f\n\tv_xor_b32_e32 %1, %10, %1\n\ts_branch 1f\n\t"
               "v_xor_b32_e32 %2, %10, %2\n\ts_branch 1f\n\tv_xor_b32_e32 %3, %10, %3\n\ts_branch 1f\n\t"
               "v_xor_b32_e32 %4, %10, %4\n\ts_branch 1f\n\tv_xor_b32_e32 %5, %10, %5\n\ts_branch 1f\n\t"
               "v_xor_b32_e32 %6, %10, %6\n\ts_branch 1f\n\tv_xor_b32_e32 %7, %10, %7\n"
               "1:\n\ts_mov_b64 exec, -1"
               : "+v"(h[0]), "+v"(h[1]), "+v"(h[2]), "+v"(h[3]), "+v"(h[4]), "+v"(h[5]), "+v"(h[6]), "+v"(h[7])
               : "s"(e >> 11), "s"(((e >> 14) & 56) + 12), "s"((e & 0xff) << ((e >> 6) & 24)) : "vcc", "scc");
}

struct Stream {
  const char* sb; const char* mb; char* myws;
  u32 l16, tag, call; u64 t0;
  u32x4 W; int rdy;
  __device__ __forceinline__ void init(char* ws, u32 call_, u32 sid, int lane) {
    myws = ws; call = call_; tag = call_tag(call_);
    sb = ws + (size_t)sid * NSTEP * SLOT + 4096;
    mb = ws + META_OFF + (size_t)sid * NSTEP * 32;
    l16 = lane * 16;
    t0 = __builtin_amdgcn_s_memrealtime();
    W = u32x4{0u, 0u, 0u, 0u}; rdy = 0;
  }
  __device__ __forceinline__ void poll() {
    const u64 mask = 0xffffull << (rdy < 24 ? 2 * rdy : 48);
    asm volatile("s_mov_b64 exec, %3\n\tglobal_load_dwordx4 %0, %1, %2 sc0 sc1\n\ts_mov_b64 exec, -1"
                 : "+v"(W) : "v"(l16), "s"(mb), "s"(mask) : "memory");
  }
  __device__ __forceinline__ void look() {
    asm volatile("" : "+v"(W));
    const u32 x = (W[0] ^ tag) | (W[1] ^ tag) | (W[2] ^ tag) | (W[3] ^ tag);
    const u64 bad = __ballot(x >= (1u << 21));
    rdy = bad ? __builtin_ctzll(bad) >> 1 : NSTEP;
  }
  __device__ __forceinline__ void spin(int t) {
    for (u32 n = 1; rdy <= t; ++n) {
      poll();
      asm volatile("s_waitcnt vmcnt(0)" ::: "memory");
      look();
      if (rdy == 0) __builtin_amdgcn_s_sleep(8);
      if ((n & 31) == 0 && spin_fail(myws, call, t0)) rdy = NSTEP;
    }
  }
  __device__ __forceinline__ u32 header(int t) { asm volatile("" : "+v"(W)); return __builtin_amdgcn_readlane(W[0], 2 * t); }
  __device__ __forceinline__ void repair(u32 (&h)[8], u32 hdr, int t, u32 hf) {
#pragma nounroll
    for (u32 n = (hdr >> (11 + 3 * hf)) & 7, k = hf ? 7 : 1; n != 0; --n, k += hf ? -1 : 1) {
      const int sl = 2 * t + (k >> 2);
      const u32 e0 = __builtin_amdgcn_readlane(W[0], sl), e1 = __builtin_amdgcn_readlane(W[1], sl);
      const u32 e2 = __builtin_amdgcn_readlane(W[2], sl), e3 = __builtin_amdgcn_readlane(W[3], sl);
      fix_word(h, (k & 2) ? ((k & 1) ? e3 : e2) : ((k & 1) ? e1 : e0));
    }
    if (__builtin_expect(hdr & 0x400u, 0)) {
      const u32x8 g = load_two(sb + (size_t)t * SLOT + hf * 2048, l16);
#pragma unroll
      for (int j = 0; j < 8; ++j) asm volatile("v_mov_b32 %0, %1" : "+v"(h[j]) : "v"(g[j]));
    }
  }
};

__device__ __forceinline__ void unpack_unit(char* myws, [[maybe_unused]] int wv, int lane, u32 call, u32 r, int t) {
  [[maybe_unused]] const int sto = 0;
  Stream S;
  S.init(myws, call, (call & 1) * NS_PAR + S_WIN + r, lane);
  const u32 l4 = lane * 4;
  const u32 uo = (S_IMG + 16 + r) * NSTEP;
  {
    for (u32 n = 1;; ++n) {
      const u64 mask = 3ull << (2 * t);
      asm volatile("s_mov_b64 exec, %3\n\tglobal_load_dwordx4 %0, %1, %2 sc0 sc1\n\ts_mov_b64 exec, -1\n\ts_waitcnt vmcnt(0)"
                   : "+v"(S.W) : "v"(S.l16), "s"(S.mb), "s"(mask) : "memory");
      const u32 x = (S.W[0] ^ S.tag) | (S.W[1] ^ S.tag) | (S.W[2] ^ S.tag) | (S.W[3] ^ S.tag);
      if (!(__ballot(x >= (1u << 21)) & mask)) break;
      if ((n & 31) == 0 && spin_fail(myws, call, S.t0)) break;
      __builtin_amdgcn_s_sleep(4);
    }
    Pay<6> P;
    pay_load(P, S.l16, S.sb + (size_t)t * SLOT);
    pay_wait<0>(P);
    const u32 hdr = S.header(t), base4 = (hdr & 0x7f) * 0x01010101u;
    u32x4 o[8];
#pragma unroll
    for (int hf = 0; hf < 2; ++hf) {
      u32 g[8];
#pragma unroll
      for (int j = 0; j < 4; ++j) {
        const u32 n = P.v[4 + hf][j];
        g[2 * j] = (n & 0x87878787u) + base4;
        g[2 * j + 1] = (__builtin_amdgcn_alignbit(n, n, 4) & 0x87878787u) + base4;
      }
      if (hdr & (0x400u | 0x3800u << (3 * hf))) S.repair(g, hdr, t, hf);
#pragma unroll
      for (int j = 0; j < 4; ++j)
        o[4 * hf + j] = join(P.v[2 * hf + (j >> 1)][2 * (j & 1)], P.v[2 * hf + (j >> 1)][2 * (j & 1) + 1], g[2 * j], g[2 * j + 1]);
    }
    char* const ob = myws + (size_t)(uo + t) * SLOT + 4096;
    ST_SIX(S.l16, o, ob); ST16(S.l16, o[6], ob, 2048); ST16(S.l16, o[7], ob, 3072);
    asm volatile("s_waitcnt vmcnt(0)" ::: "memory");
    char* const mo = myws + META_OFF + (size_t)(uo + t) * 32;
    if (lane < 8) ST4(l4, S.tag, mo);
    STAMP(50);
  }
}

__device__ __forceinline__ float silu(float y) {
  const float z = __builtin_amdgcn_fmed3f(y, -60.0f, 60.0f);
  const float e = __builtin_amdgcn_exp2f(z * -1.44269504088896341f);
  return y * __builtin_amdgcn_rcpf(1.0f + e);
}

__device__ __forceinline__ void gemm_tile(KP p, char* smem, char* myws, int wv, int lane, u32 call, u32 nb, u32 sid0, u32 tn, u32 ch, u32 fl, int hold,
                                          [[maybe_unused]] int sto) {
  const u32 rank = p->rank;
  STAMP(0);
  const u32 o0 = (rank & 4u) ^ 4u;
  const bool remote = tn >= 4;
  const u32 dst = (remote ? o0 + (tn - 4) : rank) & 7u;
  char* const ob = p->out[dst] + (remote ? (rank & 3) : tn) * 256 + ch * 128;
  u32* flag = (u32*)(p->ws[dst] + DONE_OFF) + (rank & 3) * 64 + fl;
  auto grow = [&](u32 rg) { return global_row(rank, sid0 + (rg >> 2)) + (rg & 3) * 8; };
  char* b0 = ob + (size_t)grow(wv & 7) * (NLOC * 2);
  char* b1 = ob + (size_t)grow((wv & 7) + 8) * (NLOC * 2);
  if (hold == 2) {
    const u32 us = (S_PARK + (tn & 3)) * NSTEP + 2 * fl;
    b0 = myws + (size_t)us * SLOT + (wv & 7) * 1024; b1 = b0 + SLOT;
    flag = (u32*)(myws + META_OFF + (size_t)us * 32);
  }
  asm volatile("" : "+s"(b0), "+s"(b1), "+s"(flag));
  Stream S;
  const u32 m = wv >> 1, kh = wv & 1;
  if (wv >= 8) {
    const u32 un = wv - 8;
    S.init(myws, call, S_IMG + 4 * tn + un, lane);
    const u32 lds0 = (u32)(size_t)(LAS char*)smem + un * 4096;
    const char* const g0 = S.sb - 4096 + ch * 4096;
    auto issue = [&](int t) {
      S.look();
      S.spin(t);
      S.poll();
      const char* g = g0 + (size_t)t * SLOT;
      const u32 l = lds0 + (t & 3) * 16384;
      u32 keep;
      asm volatile("s_mov_b32 %0, m0\n\ts_mov_b32 m0, %3\n\ts_nop 0\n\t"
                   "global_load_lds_dwordx4 %1, %2 sc0 sc1\n\tglobal_load_lds_dwordx4 %1, %2 offset:1024 sc0 sc1\n\t"
                   "global_load_lds_dwordx4 %1, %2 offset:2048 sc0 sc1\n\tglobal_load_lds_dwordx4 %1, %2 offset:3072 sc0 sc1\n\t"
                   "s_mov_b32 m0, %0" : "=&s"(keep) : "v"(S.l16), "s"(g), "s"(l) : "memory");
    };
    issue(0); issue(1); issue(2);
    for (int t = 0; t < NSTEP - 3; ++t) {
      asm volatile("s_waitcnt vmcnt(10)" ::: "memory");
      WG_BARRIER();
      STAMP(1 + t);
      issue(t + 3);
    }
    asm volatile("s_waitcnt vmcnt(10)" ::: "memory");
    WG_BARRIER();
    STAMP(30);
    asm volatile("s_waitcnt vmcnt(5)" ::: "memory");
    WG_BARRIER();
    STAMP(31);
    asm volatile("s_waitcnt vmcnt(0)" : "+v"(S.W) :: "memory");
    WG_BARRIER();
    STAMP(32);
    if (hold == 1 && wv == 8) {
      const u32* const r = (const u32*)(myws + REL_OFF + (blockIdx.x & 7) * 128) + (lane < NCWG ? lane : 0);
      for (u32 n = 1; !__all(__hip_atomic_load(r, __ATOMIC_RELAXED, __HIP_MEMORY_SCOPE_AGENT) >= call); ++n) {
        if ((n & 31) == 0 && spin_fail(myws, call, S.t0)) break;
        __builtin_amdgcn_s_sleep(4);
      }
    }
  } else if (m >= nb) {
    for (int t = 0; t < NSTEP; ++t) WG_BARRIER();
  } else {
    S.init(myws, call, (call & 1) * NS_PAR + sid0 + m, lane);
    const u32 vlo = S.l16 + kh * 2048, vcode = S.l16 + kh * 1024;
    const u32 rmask = 0x400u | 0x3800u << (3 * kh);
    f32x16 acc[2] = {};
    const LAS char* const bb = (LAS char*)smem + kh * 8192 + lane * 16;
    bf16x8 wf[4];
    u32x4 X[2][4];
    Pay<3> P[4];
    u32 h[8];
    u32 hdr = 0, base4 = 0;
    auto rd = [&](int stage, int f) {
      wf[f & 3] = *(const LAS bf16x8*)(bb + stage * 16384 + (f >> 2) * 4096 + ((f >> 1) & 1) * 1024 + (f & 1) * 2048);
    };
    auto piece = [&](int k, int t, int ts, int waitn, bool fetch) {
      Pay<3>& Q = P[ts];
      if (k == 0) {
        if (waitn == 3) pay_wait<9>(Q); else if (waitn == 2) pay_wait<6>(Q); else if (waitn == 1) pay_wait<3>(Q); else pay_wait<0>(Q);
        hdr = S.header(t);
        base4 = (hdr & 0x7f) * 0x01010101u;
      } else if (k < 6) {
#pragma unroll
        for (int j = 0; j < 4; ++j) {
          const int i = k < 4 ? 2 * j + (k & 1) : 4 * (k - 4) + j;
          if (k == 1) h[i] = __builtin_amdgcn_alignbit(Q.v[2][j], Q.v[2][j], 4);
          else if (k == 2) h[i] = Q.v[2][j] & 0x87878787u;
          else if (k == 3) h[i] &= 0x87878787u;
          else h[i] += base4;
          asm volatile("" : "+v"(h[i]));
        }
      } else if (k == 6) {
        if (hdr & rmask) S.repair(h, hdr, t, kh);
      } else if (k == 7) {
      } else if (k < 12) {
        const int c = k - 8;
        X[ts & 1][c] = join(Q.v[c >> 1][2 * (c & 1)], Q.v[c >> 1][2 * (c & 1) + 1], h[2 * c], h[2 * c + 1]);
        asm volatile("" : "+v"(X[ts & 1][c]));
      } else if (fetch) {
        const int tn4 = t + P_DIST < NSTEP ? t + P_DIST : NSTEP - 1;
        if (k == 12) { if (S.rdy < NSTEP && S.rdy <= tn4 + 2) { S.look(); S.spin(tn4); S.poll(); } }
        else if (k == 13) pay_load(P[(ts + P_DIST) & 3], vlo, vcode, S.sb + (size_t)tn4 * SLOT);
      }
    };
    auto step = [&](int t, int ts, int waitn, bool fetch, bool more) {
#pragma unroll
      for (int q = 0; q < 8; ++q) {
        if (q == 4 && more) WG_BARRIER();
        else if (q % 2 == 0 && (more || q < 6)) __builtin_amdgcn_s_waitcnt(0xC27F);
        __builtin_amdgcn_sched_barrier(0);
        acc[q & 1] = __builtin_amdgcn_mfma_f32_32x32x16_bf16(wf[q & 3], __builtin_bit_cast(bf16x8, X[ts & 1][q >> 1]), acc[q & 1], 0, 0, 0);
        __builtin_amdgcn_sched_barrier(0);
        if (q < 4) rd(ts, q + 4);
        else if (more) rd((ts + 1) & 3, q - 4);
        if (more) {
          piece(2 * q, t + 1, (ts + 1) & 3, waitn, fetch);
          piece(2 * q + 1, t + 1, (ts + 1) & 3, waitn, fetch);
        }
      }
    };
#pragma unroll
    for (int d = 0; d < P_DIST; ++d) { S.look(); S.spin(d); S.poll(); pay_load(P[d], vlo, vcode, S.sb + (size_t)d * SLOT); }
#pragma unroll
    for (int k = 0; k < 15; ++k) piece(k, 0, 0, P_DIST - 1, true);
    WG_BARRIER();
#pragma unroll
    for (int f = 0; f < 4; ++f) { rd(0, f); __builtin_amdgcn_sched_barrier(0); }
    for (int tb = 0; tb < NSTEP - 4; tb += 4) {
#pragma unroll
      for (int ts = 0; ts < 4; ++ts) step(tb + ts, ts, P_DIST - 1, true, true);
    }
    step(NSTEP - 4, 0, P_DIST - 1, false, true);
    step(NSTEP - 3, 1, P_DIST - 2, false, true);
    step(NSTEP - 2, 2, P_DIST - 3, false, true);
    step(NSTEP - 1, 3, 0, false, false);
    STAMP(1);
    char* const xq = smem + SCR_OFF + (kh * 128 + m * 32 + (lane & 31)) * ERS + (lane >> 5) * 16;
#pragma unroll
    for (int n = 0; n < 2; ++n)
#pragma unroll
      for (int g = 0; g < 4; ++g)
        *(f32x4*)(xq + n * 128 + g * 32) = f32x4{acc[n][4 * g], acc[n][4 * g + 1], acc[n][4 * g + 2], acc[n][4 * g + 3]};
    if (P_DIST == 4) pay_wait<0>(P[0]);
    asm volatile("s_waitcnt vmcnt(0)" : "+v"(S.W) :: "memory");
  }
  const u32 tmo = __hip_atomic_load((u32*)(myws + TMO_OFF), __ATOMIC_RELAXED, __HIP_MEMORY_SCOPE_AGENT);
  WG_BARRIER();
  STAMP(40);
  if (wv < 8) {
    const u32 vo = hold == 2 ? lane * 16 : (lane >> 3) * (NLOC * 2) + (lane & 7) * 16;
    auto get = [&](u32 rg) {
      const char* const q = smem + SCR_OFF + (rg * 8 + (lane >> 3)) * ERS + (lane & 7) * 32;
      const f32x4 a = *(const f32x4*)q + *(const f32x4*)(q + 128 * ERS), c = *(const f32x4*)(q + 16) + *(const f32x4*)(q + 128 * ERS + 16);
      auto pk = [](float lo, float hi) { const f32x2 v = {silu(lo), silu(hi)}; return __builtin_bit_cast(u32, __builtin_convertvector(v, bf16x2)); };
      u32x4 v = {pk(a[0], a[1]), pk(a[2], a[3]), pk(c[0], c[1]), pk(c[2], c[3])};
      if (tmo == call) v = u32x4{0x7fc07fc0u, 0x7fc07fc0u, 0x7fc07fc0u, 0x7fc07fc0u};
      return v;
    };
    const u32x4 v0 = get(wv);
    ST16(vo, v0, b0, 0);
    if (nb > 2) { const u32x4 v1 = get(wv + 8); ST16(vo, v1, b1, 0); }
  }
  if (remote) {
    asm volatile("s_waitcnt vmcnt(0)" ::: "memory");
    WG_BARRIER();
    if (threadIdx.x < (hold == 2 ? 2 : 1)) ST4(threadIdx.x * 32, hold == 2 ? call_tag(call) : call, flag);
  }
  STAMP(41);
}

__host__ __device__ inline bool tile_of(u32 rank, u32 b, int k, u32& nb, u32& sid0, u32& tn, u32& ch, u32& fl, int& hold) {
  const u32 peer = peer_of_xcd(rank, b & 7), i = b >> 3;
  const bool home = !((peer ^ rank) & 4), spare = i >= 2 * RT_MED;
  hold = 0;
  if (k == 0 && !spare) {
    nb = 4; tn = (peer & 3) + (home ? 0 : 4); sid0 = (i >> 1) * 4; ch = i & 1; fl = i;
    hold = home || sid0 >= (u32)S_INTER ? 0 : i < (u32)P_PARK ? 2 : i >= 10 ? 1 : 0;
    return true;
  }
  const u32 j = spare ? i - 2 * RT_MED + 10 : i;
  nb = 2; tn = (peer & 3) + (home ? 4 : 0); sid0 = RT_MED * 4 + (j >> 1) * 2; ch = j & 1; fl = 2 * RT_MED + j;
  return spare ? k == 0 : i < 10;
}

constexpr u32 NHOME = (NA + 16) * NSTEP;
__host__ __device__ inline u32 pack_index(u32 b, u32 wv) {
  constexpr u32 NP1 = 80;
  return b < NP1 ? (wv < 2 ? b + NP1 * wv : NHOME) : 2 * NP1 + (b - NP1) + (NGEMM - NP1) * wv;
}

constexpr u32 UB0 = RT_MED * 16, UNW = 16;
__host__ __device__ inline u32 unpack_index(u32 b, u32 wv) {
  return b >= UB0 && b < UB0 + UNW ? (11 - wv) * UNW + (b - UB0) : 512u;
}

__global__ void __launch_bounds__(NTHREAD, 3) ag_gemm_silu(const Params) {
  KP p = (KP)__builtin_amdgcn_kernarg_segment_ptr();
  __shared__ __attribute__((aligned(16))) char smem[LDS_BYTES];
  const int lane = threadIdx.x & 63;
  const int wv = __builtin_amdgcn_readfirstlane(threadIdx.x >> 6);
  const u32 rank = p->rank;
  char* const myws = p->myws;
  if (blockIdx.x >= NGEMM) { sender(p, smem, myws, wv, lane); return; }
  u32 call;
  const u32 u = pack_index(blockIdx.x, wv);
  if (u < NHOME) {
    const u32 t = u / (NA + 16), c = u % (NA + 16);
    const bool isw = c >= (u32)NA;
    const u32 cw = c - NA, l16 = lane * 16;
    const u64 t0 = __builtin_amdgcn_s_memrealtime();
    while (__builtin_amdgcn_s_memrealtime() - t0 < t * P_PP) __builtin_amdgcn_s_sleep(2);
    u32x4 d[8], s[8];
    load_raw(p, d, lane, isw ? 16 + (cw & 3) : c, (cw >> 2) & 3, t);
    call = draw_call(myws, wv);
    WAIT_RAW(d);
    u32 meta = 0;
    bool eight = true;
    to_fragments(d, isw, smem + SCR_OFF + wv * 8192, lane);
    if (isw) {
#pragma unroll
      for (int j = 0; j < 8; ++j) s[j] = d[j];
    } else { low_slabs(d, s); eight = encode(d, s, meta, lane); }
    const u32 ud = (isw ? S_IMG + cw : (call & 1) * NS_PAR + c) * NSTEP + t;
    char* const b = myws + (size_t)ud * SLOT + 4096;
    ST_SIX(l16, s, b);
    if (eight) { ST16(l16, s[6], b, 2048); ST16(l16, s[7], b, 3072); }
    asm volatile("s_waitcnt vmcnt(0)" ::: "memory");
    char* const m = myws + META_OFF + (size_t)ud * 32;
    if (lane < 8) ST4(lane * 4, meta | call_tag(call), m);
    { [[maybe_unused]] const int sto = 0; STAMP(51); }
  } else call = draw_call(myws, wv);
#pragma nounroll
  for (u32 v = unpack_index(blockIdx.x, wv); v < 512; v += 12 * UNW) unpack_unit(myws, wv, lane, call, v & 15, v >> 4);
#pragma nounroll
  for (int k = 0; k < 2; ++k) {
    u32 nb, sid0, tn, ch, fl; int hold;
    if (!tile_of(rank, blockIdx.x, k, nb, sid0, tn, ch, fl, hold)) break;
    gemm_tile(p, smem, myws, wv, lane, call, nb, sid0, tn, ch, fl, hold, 16 * k);
  }
}

static void build_orders(Params& p) {
  struct E { double tau; unsigned short code; };
  E e[N_ORD];
  double sigma = P_SIGMA, ovl = P_OVL, ovl2 = P_OVL2, lead = P_LEAD;
  for (int kind = 0; kind < 2; ++kind) {
    int n = 0;
    for (int t = 0; t < NSTEP; ++t) {
      if (kind == 0) {
        for (int b = 0; b < NA; ++b) e[n++] = {t + (b < N1 ? 0 : b < N1 + N2 ? NSTEP - ovl : 2 * NSTEP - ovl - ovl2), (unsigned short)(b << 5 | t)};
      } else {
        for (int i = 0; i < 4; ++i) e[n++] = {t * sigma - lead, (unsigned short)((16 + i) << 5 | t)};
        for (int b = NA; b < 16; ++b) e[n++] = {t + 0.5, (unsigned short)(b << 5 | t)};
      }
    }
    std::stable_sort(e, e + n, [](const E& a, const E& b) { return a.tau < b.tau; });
    for (int i = 0; i < N_ORD; ++i) p.ord[kind][i] = i < n ? e[i].code : 0;
  }
}

extern "C" void kernel_launch(int rank, int nranks, ncclComm_t comm, hipStream_t stream,
                              void* const* d_in, const int* in_sizes, int n_in,
                              void* d_out, int out_size, const int* in_dtypes, int out_dtype,
                              void* const* d_out_all, void* const* d_ws_all, size_t ws_bytes) {
  (void)comm; (void)in_dtypes; (void)out_dtype; (void)d_out;
  if (nranks != NR || (unsigned)rank >= (unsigned)NR || n_in != 2 || ws_bytes < WS_NEED ||
      in_sizes[0] != 512 * KDIM || in_sizes[1] != KDIM * NLOC || out_size != 4096 * NLOC) {
    fprintf(stderr, "unexpected problem shape\n");
    return;
  }
  Params p = {};
  p.x = (const char*)d_in[0];
  p.w = (const char*)d_in[1];
  for (int i = 0; i < NR; ++i) {
    p.out[i] = (char*)d_out_all[phys_rank(i)]; p.ws[i] = (char*)d_ws_all[phys_rank(i)];
    if ((int)phys_rank(i) == rank) p.rank = i;
  }
  p.myws = (char*)d_ws_all[rank];
  build_orders(p);
  hipLaunchKernelGGL(ag_gemm_silu, dim3(GRID), dim3(NTHREAD), 0, stream, p);
  hipError_t err = hipGetLastError();
  if (err != hipSuccess) fprintf(stderr, "launch failed: %s\n", hipGetErrorString(err));
}
```

```cpp
#include <hip/hip_runtime.h>
#include <rccl/rccl.h>
#include <cstdio>
#include <cstdint>
#include <cstdlib>
#include <algorithm>

typedef unsigned u32;
typedef unsigned long long u64;
typedef unsigned u32x2 __attribute__((ext_vector_type(2)));
typedef unsigned u32x4 __attribute__((ext_vector_type(4)));
typedef unsigned u32x8 __attribute__((ext_vector_type(8)));
typedef unsigned u32x16 __attribute__((ext_vector_type(16)));
typedef unsigned short u16x2 __attribute__((ext_vector_type(2)));
typedef __bf16 bf16x2 __attribute__((ext_vector_type(2)));
typedef __bf16 bf16x4 __attribute__((ext_vector_type(4)));
typedef __bf16 bf16x8 __attribute__((ext_vector_type(8)));
typedef float f32x2 __attribute__((ext_vector_type(2)));
typedef float f32x4 __attribute__((ext_vector_type(4)));
typedef float f32x16 __attribute__((ext_vector_type(16)));
#define LAS __attribute__((address_space(3)))

#define P_NA 11
#define P_NCWG 4
#define P_WIN 3
#define P_WIN0 4
#define P_TAPER 8
#define P_WIN1 5
#define P_DIST 3
#define P_BD 7
#define P_ALOOK 3
#define P_LPRIO 2
#define P_MPAD 0
#define P_UROT(t) (5 * (t))
#define P_STG 0
#define P_SIGMA 0.1
#define P_OVL 0.0
#define P_OVL2 0.0
#define P_LEAD 0.0
#define P_PP 10
#define P_PARK 10
#define P_RLY 2
#define P_S0 16
#define P_S1 16
#define P_S2 4
#define P_UT 100
#define P_REL 2

constexpr int NR = 8;
constexpr int KDIM = 4096, NLOC = 512;
constexpr int NSTEP = 32;
constexpr int NA = P_NA, NB = 16 - NA;
constexpr int N1 = 3, N3 = 4, N2 = NA - N1 - N3;
constexpr int S_1 = NA, S_2 = S_1 + 3 * N1, S_INTER = S_2 + 3 * N2, S_3 = 64 - 3 * N3, S_WIN = 64, S_RLY = 80, S_OWN2 = 81;
constexpr int RB = N1 + N2 - 1, RT0 = NSTEP - 4 * P_RLY;
constexpr int NS_PAR = S_OWN2 + NB, S_IMG = 2 * NS_PAR, S_PARK = S_IMG + 32, NSTREAM = S_PARK + 4;
constexpr size_t SLOT = 8192;
constexpr size_t META_OFF = (size_t)NSTREAM * NSTEP * SLOT;
constexpr u32 MPAD = P_MPAD;
constexpr size_t DONE_OFF = META_OFF + (size_t)NSTREAM * (NSTEP * 32 + MPAD);
constexpr size_t REL_OFF = DONE_OFF + 1024;
constexpr size_t CNT_OFF = REL_OFF + 1024;
constexpr size_t TMO_OFF = CNT_OFF + 256 * 128;
constexpr size_t DBG_OFF = TMO_OFF + 128;
constexpr size_t WS_NEED = DBG_OFF + 256 * 32 * 64 * 8;
constexpr int N_INTRA = NA * NSTEP - 4 * P_RLY, N_INTER = NB * NSTEP + 4 * NSTEP + 4 * P_RLY;
constexpr int N_ORD = N_INTRA > N_INTER ? N_INTRA : N_INTER;
constexpr int RT_MED = 13;
constexpr int NGEMM = RT_MED * 16 + 16;
constexpr int NFLAG = 2 * RT_MED + 12 + P_PARK;
constexpr int NCWG = P_NCWG;
constexpr int GRID = NGEMM + 8 * NCWG;
static_assert(P_BD <= 7 && P_RLY <= 8 && N_ORD % 2 == 0 && NB > 4 && NB <= 8 && S_INTER % 4 == 0, "tile map");
static_assert(GRID <= 256, "one workgroup per CU, all resident");
constexpr int NTHREAD = 768;
constexpr int SCR_OFF = 0;
constexpr int HDR_OFF = 131072;
constexpr int LDS_BYTES = HDR_OFF + 4096;
constexpr u64 DEADLINE_TICKS = 50000000ull;
constexpr int ERS = 272;

#define P_PHYS 0x75416320u
__host__ __device__ inline u32 phys_rank(u32 l) { return (P_PHYS >> (4 * l)) & 7u; }
__host__ __device__ inline u32 peer_of_xcd(u32 rank, u32 x) { return (((rank ^ x) & 4u) | ((rank + x) & 3u)) & 7u; }

struct Params {
  const char* x;
  const char* w;
  char* myws;
  char* out[NR];
  char* ws[NR];
  int rank;
  u32 row[NS_PAR + 4];
  u32 tile[NGEMM][2];
  alignas(4) unsigned short ord[2][N_ORD];
};
static_assert(sizeof(Params) <= 4096, "kernel argument");
typedef const __attribute__((address_space(4))) Params* KP;

#define STAMP(i) do {} while (0)
#define WG_BARRIER() do { asm volatile("" ::: "memory"); __builtin_amdgcn_s_waitcnt(0xC07F); \
    asm volatile("s_barrier" ::: "memory"); } while (0)
#define ST16(voff, data, sbase, imm) \
  asm volatile("global_store_dwordx4 %0, %1, %2 offset:" #imm " sc0 sc1\n\ts_nop 1" :: "v"(voff), "v"(data), "s"(sbase) : "memory")
#define ST4(voff, data, sbase) \
  asm volatile("global_store_dword %0, %1, %2 sc0 sc1" :: "v"(voff), "v"(data), "s"(sbase) : "memory")
#define ST_SIX(voff, s, b) do { ST16(voff, s[0], b, -4096); ST16(voff, s[1], b, -3072); ST16(voff, s[2], b, -2048); \
    ST16(voff, s[3], b, -1024); ST16(voff, s[4], b, 0); ST16(voff, s[5], b, 1024); } while (0)
#define CAT16(a, b) __builtin_shufflevector(a, b, 0, 1, 2, 3, 4, 5, 6, 7, 8, 9, 10, 11, 12, 13, 14, 15)

__device__ __forceinline__ void count_read(u32& v, char* myws) {
  const char* const c = myws + CNT_OFF + blockIdx.x * 128;
  asm volatile("global_load_dword %0, %1, %2 sc1" : "=&v"(v) : "v"(0u), "s"(c) : "memory");
}
__device__ __forceinline__ u32 count_take(u32& v) {
  asm volatile("" : "+v"(v));
  return __builtin_amdgcn_readfirstlane(v) + 1u;
}
__device__ __forceinline__ void count_bump(char* myws, u32 call) {
  if (threadIdx.x == 0) __hip_atomic_store((u32*)(myws + CNT_OFF + blockIdx.x * 128), call, __ATOMIC_RELAXED, __HIP_MEMORY_SCOPE_AGENT);
}
__device__ __forceinline__ u32 draw_call(char* myws) {
  u32 v;
  count_read(v, myws);
  asm volatile("s_waitcnt vmcnt(0)" : "+v"(v) :: "memory");
  return count_take(v);
}
__device__ __forceinline__ u32 call_tag(u32 call) { return (call % 2047u + 1u) << 21; }

__device__ __forceinline__ bool spin_fail(char* myws, u32 call, u64 t0) {
  u32* tmo = (u32*)(myws + TMO_OFF);
  u32 t = __builtin_amdgcn_readfirstlane(__hip_atomic_load(tmo, __ATOMIC_RELAXED, __HIP_MEMORY_SCOPE_AGENT));
  if (t == call || __builtin_amdgcn_s_memrealtime() - t0 > DEADLINE_TICKS) {
    __hip_atomic_store(tmo, call, __ATOMIC_RELAXED, __HIP_MEMORY_SCOPE_AGENT);
    return true;
  }
  return false;
}

__device__ __forceinline__ size_t meta_off(u32 unit) { return META_OFF + (size_t)(unit * 32 + (unit / NSTEP) * MPAD); }
__device__ __forceinline__ u32 dest_stream(u32 sl, u32 rank, u32 dest, u32 par) {
  const u32 j = ((rank - dest) & 3) - 1;
  const u32 s = sl >= 16 ? S_WIN + (rank & 3) * 4 + (sl - 16) : sl >= (u32)NA ? S_INTER + (rank & 3) * NB + (sl - NA)
              : sl >= (u32)(N1 + N2) ? S_3 + j * N3 + (sl - N1 - N2) : sl >= (u32)N1 ? S_2 + j * N2 + (sl - N1) : S_1 + j * N1 + sl;
  return par * NS_PAR + s;
}

inline u32 global_row(u32 rank, u32 sid) {
  const u32 g0 = rank & 4u, o0 = g0 ^ 4u;
  u32 src, bi;
  if (sid < (u32)S_1) { src = rank; bi = sid; }
  else if (sid < (u32)S_2) { const u32 q = sid - S_1; src = g0 + ((rank + 1 + q / N1) & 3); bi = q % N1; }
  else if (sid < (u32)S_INTER) { const u32 q = sid - S_2; src = g0 + ((rank + 1 + q / N2) & 3); bi = N1 + q % N2; }
  else if (sid < (u32)S_3) { const u32 q = sid - S_INTER; src = o0 + q / NB; bi = NA + q % NB; }
  else if (sid >= (u32)S_OWN2) { src = rank; bi = NA + sid - S_OWN2; }
  else { const u32 q = sid - S_3; src = g0 + ((rank + 1 + q / N3) & 3); bi = N1 + N2 + q % N3; }
  return (phys_rank(src & 7u) * 512 + bi * 32) & 4095u;
}

__device__ __forceinline__ void load_raw(KP p, u32x4 (&d)[8], int lane, u32 sl, u32 q, u32 t) {
  const bool isw = sl >= 16;
  const u32 rs = isw ? NLOC * 2 : KDIM * 2;
  const u32 vo = (lane >> 4) * rs + (lane & 15) * 16;
  const u64 aw = (u64)p->w + (u64)((t * 128 + (sl & 3) * 32) * (NLOC * 2) + q * 256), ax = (u64)p->x + (u64)(sl * 32 * (KDIM * 2) + t * 256);
  const u64 b0 = isw ? aw : ax;
#pragma unroll
  for (int j = 0; j < 8; ++j)
    asm volatile("global_load_dwordx4 %0, %1, %2" : "=&v"(d[j]) : "v"(vo), "s"(b0 + (u64)(j * 4 * rs)) : "memory");
}
#define WAIT_RAW(d) asm volatile("s_waitcnt vmcnt(0)" : "+v"(d[0]), "+v"(d[1]), "+v"(d[2]), "+v"(d[3]), "+v"(d[4]), "+v"(d[5]), "+v"(d[6]), "+v"(d[7]) :: "memory")
__device__ __forceinline__ void to_fragments(u32x4 (&d)[8], bool isw, char* scr, int lane) {
  LAS char* const sc = (LAS char*)scr;
  auto off = [](u32 row, u32 ch) { return 256u * row + 16u * (ch ^ (((row & 3) << 2) | ((row >> 2) & 3))); };
#pragma unroll
  for (int j = 0; j < 8; ++j) *(LAS u32x4*)(sc + off(4 * j + (lane >> 4), lane & 15)) = d[j];
  asm volatile("" ::: "memory");
  if (isw) {
    const u32 r0 = 8 * (lane >> 5) + ((lane & 15) >> 2), c0 = 2 * ((lane >> 4) & 1) + ((lane & 3) >> 1);
#pragma unroll
    for (int j = 0; j < 8; ++j) {
      const bf16x4 lo = __builtin_amdgcn_ds_read_tr16_b64_v4bf16((LAS bf16x4*)(sc + off(16 * (j & 1) + r0, 4 * (j >> 1) + c0) + 8 * (lane & 1)));
      const bf16x4 hi = __builtin_amdgcn_ds_read_tr16_b64_v4bf16((LAS bf16x4*)(sc + off(16 * (j & 1) + r0 + 4, 4 * (j >> 1) + c0) + 8 * (lane & 1)));
      d[j] = __builtin_bit_cast(u32x4, __builtin_shufflevector(lo, hi, 0, 1, 2, 3, 4, 5, 6, 7));
    }
  } else {
#pragma unroll
    for (int j = 0; j < 8; ++j) d[j] = *(const LAS u32x4*)(sc + off(lane & 31, 2 * j + (lane >> 5)));
  }
  asm volatile("" ::: "memory");
}
__device__ __forceinline__ void low_slabs(const u32x4 (&d)[8], u32x4 (&s)[8]) {
#pragma unroll
  for (int i = 0; i < 16; ++i) s[i >> 2][i & 3] = __builtin_amdgcn_perm(d[i >> 1][2 * (i & 1) + 1], d[i >> 1][2 * (i & 1)], 0x06040200u);
}
__device__ __forceinline__ bool encode(const u32x4 (&d)[8], u32x4 (&s)[8], u32& meta, int lane) {
  u32 H[16], T[16];
  u16x2 m1 = {0, 0}, m2 = {0, 0}, m3 = {0, 0}, m4 = {0, 0};
#pragma unroll
  for (int i = 0; i < 16; ++i) {
    const u32 a = d[i >> 1][2 * (i & 1)], b = d[i >> 1][2 * (i & 1) + 1];
    H[i] = __builtin_amdgcn_perm(b, a, 0x07050301u);
  }
#pragma unroll
  for (int i = 0; i < 16; i += 2) {
    T[i] = H[i] | 0x80808080u;
    T[i + 1] = H[i + 1] | 0x80808080u;
    m1 = __builtin_elementwise_max(m1, __builtin_bit_cast(u16x2, T[i]));
    m2 = __builtin_elementwise_max(m2, __builtin_bit_cast(u16x2, T[i] << 8));
    m3 = __builtin_elementwise_max(m3, __builtin_bit_cast(u16x2, T[i + 1]));
    m4 = __builtin_elementwise_max(m4, __builtin_bit_cast(u16x2, T[i + 1] << 8));
  }
  m1 = __builtin_elementwise_max(__builtin_elementwise_max(m1, m2), __builtin_elementwise_max(m3, m4));
  u32 mx = (u32)(m1.x > m1.y ? m1.x : m1.y);
  mx = max(mx, (u32)__builtin_amdgcn_update_dpp(0, mx, 0x111, 0xf, 0xf, true));
  mx = max(mx, (u32)__builtin_amdgcn_update_dpp(0, mx, 0x112, 0xf, 0xf, true));
  mx = max(mx, (u32)__builtin_amdgcn_update_dpp(0, mx, 0x114, 0xf, 0xf, true));
  mx = max(mx, (u32)__builtin_amdgcn_update_dpp(0, mx, 0x118, 0xf, 0xf, true));
  const u32 top = max(max((u32)__builtin_amdgcn_readlane(mx, 15), (u32)__builtin_amdgcn_readlane(mx, 31)),
                      max((u32)__builtin_amdgcn_readlane(mx, 47), (u32)__builtin_amdgcn_readlane(mx, 63)));
  const u32 h7 = (top >> 8) & 0x7f;
  const u32 base = __builtin_amdgcn_readfirstlane(h7 > 7 ? h7 - 7 : 0);
  const u32 base4 = base * 0x01010101u;
  u32 ok = 0xffffffffu;
  u32 N[8];
#pragma unroll
  for (int j = 0; j < 8; ++j) {
    const u32 ta = T[2 * j] - base4, tb = T[2 * j + 1] - base4;
    ok &= ta & tb;
    const u32 ca = (ta & 0x07070707u) | (H[2 * j] & 0x80808080u);
    const u32 cb = (tb & 0x07070707u) | (H[2 * j + 1] & 0x80808080u);
    N[j] = ca | __builtin_amdgcn_alignbit(cb, cb, 28);
  }
  meta = 0;
  int c0 = 0, c1 = 0;
  if (__any((~ok & 0x80808080u) != 0)) {
    u32 M0 = 0, M1 = 0;
#pragma unroll
    for (int i = 0; i < 8; ++i) {
      M0 |= ~((T[i] - base4) >> i) & (0x80808080u >> i);
      M1 |= ~((T[8 + i] - base4) >> i) & (0x80808080u >> i);
    }
    u64 bal = __ballot((M0 | M1) != 0);
    while (bal != 0 && c0 + c1 <= 7) {
      const int ln = __builtin_ctzll(bal);
      bal &= bal - 1;
      u64 mm = (u64)(u32)__builtin_amdgcn_readlane(M0, ln) | (u64)(u32)__builtin_amdgcn_readlane(M1, ln) << 32;
      while (mm != 0) {
        const u32 bit = __builtin_ctzll(mm), hf = bit >> 5, by = (bit >> 3) & 3, i = hf * 8 + 7 - (bit & 7);
        mm &= mm - 1;
        c0 += hf ^ 1; c1 += hf;
        u32 hw = 0;
#pragma unroll
        for (int z = 0; z < 16; ++z) hw |= H[z] & (u32)__builtin_amdgcn_sbfe(1 << i, z, 1);
        const u32 hb = ((u32)__builtin_amdgcn_readlane(hw, ln) >> (8 * by)) & 0x7f;
        const u32 ent = (u32)ln | (i & 7) << 6 | by << 9 | ((((hb - base) & 7) + base) ^ hb) << 11;
        if (lane == (hf ? 8 - c1 : c0)) meta = ent;
      }
    }
  }
  const bool raw = c0 + c1 > 7;
  if (lane == 0) meta = base | (raw ? 0x400u : (u32)(c0 << 11 | c1 << 14));
  s[4] = raw ? u32x4{H[0], H[1], H[2], H[3]} : u32x4{N[0], N[1], N[2], N[3]};
  s[5] = raw ? u32x4{H[4], H[5], H[6], H[7]} : u32x4{N[4], N[5], N[6], N[7]};
  s[6] = u32x4{H[8], H[9], H[10], H[11]};
  s[7] = u32x4{H[12], H[13], H[14], H[15]};
#pragma unroll
  for (int j = 0; j < 8; ++j) asm volatile("" : "+v"(s[j]));
  return raw;
}

template <int N> struct Pay { u32x4 v[N]; };
__device__ __forceinline__ void pay_load(Pay<6>& P, u32 l16, const char* sb) {
  asm volatile("global_load_dwordx4 %0, %6, %7 offset:-4096 sc0 sc1\n\t"
               "global_load_dwordx4 %1, %6, %7 offset:-3072 sc0 sc1\n\t"
               "global_load_dwordx4 %2, %6, %7 offset:-2048 sc0 sc1\n\t"
               "global_load_dwordx4 %3, %6, %7 offset:-1024 sc0 sc1\n\t"
               "global_load_dwordx4 %4, %6, %7 offset:0 sc0 sc1\n\t"
               "global_load_dwordx4 %5, %6, %7 offset:1024 sc0 sc1"
               : "=&v"(P.v[0]), "=&v"(P.v[1]), "=&v"(P.v[2]), "=&v"(P.v[3]), "=&v"(P.v[4]), "=&v"(P.v[5]) : "v"(l16), "s"(sb) : "memory");
}
__device__ __forceinline__ void pay_load(Pay<3>& P, u32 vlo, u32 vcode, const char* sb) {
  asm volatile("global_load_dwordx4 %2, %4, %5 offset:0 sc0 sc1\n\t"
               "global_load_dwordx4 %0, %3, %5 offset:-4096 sc0 sc1\n\t"
               "global_load_dwordx4 %1, %3, %5 offset:-3072 sc0 sc1"
               : "=&v"(P.v[0]), "=&v"(P.v[1]), "=&v"(P.v[2]) : "v"(vlo), "v"(vcode), "s"(sb) : "memory");
}
template <int C> __device__ __forceinline__ void pay_wait(Pay<6>& P) {
  asm volatile("s_waitcnt vmcnt(%6)" : "+v"(P.v[0]), "+v"(P.v[1]), "+v"(P.v[2]), "+v"(P.v[3]), "+v"(P.v[4]), "+v"(P.v[5]) : "n"(C) : "memory");
}
template <int C> __device__ __forceinline__ void pay_wait(Pay<3>& P) {
  asm volatile("s_waitcnt vmcnt(%3)" : "+v"(P.v[0]), "+v"(P.v[1]), "+v"(P.v[2]) : "n"(C) : "memory");
}
__device__ __forceinline__ u32x8 load_two(const char* b, u32 l16) {
  u32x4 x, y;
  asm volatile("global_load_dwordx4 %0, %2, %3 sc0 sc1\n\t"
               "global_load_dwordx4 %1, %2, %3 offset:1024 sc0 sc1\n\ts_waitcnt vmcnt(0)"
               : "=&v"(x), "=&v"(y) : "v"(l16), "s"(b) : "memory");
  return __builtin_shufflevector(x, y, 0, 1, 2, 3, 4, 5, 6, 7);
}

__device__ __forceinline__ void load_slot(u32x4 (&d)[8], u32 l16, const char* sb) {
  asm volatile("global_load_dwordx4 %0, %8, %9 offset:-4096 sc0 sc1\n\tglobal_load_dwordx4 %1, %8, %9 offset:-3072 sc0 sc1\n\t"
               "global_load_dwordx4 %2, %8, %9 offset:-2048 sc0 sc1\n\tglobal_load_dwordx4 %3, %8, %9 offset:-1024 sc0 sc1\n\t"
               "global_load_dwordx4 %4, %8, %9 offset:0 sc0 sc1\n\tglobal_load_dwordx4 %5, %8, %9 offset:1024 sc0 sc1\n\t"
               "global_load_dwordx4 %6, %8, %9 offset:2048 sc0 sc1\n\tglobal_load_dwordx4 %7, %8, %9 offset:3072 sc0 sc1\n\ts_waitcnt vmcnt(0)"
               : "=&v"(d[0]), "=&v"(d[1]), "=&v"(d[2]), "=&v"(d[3]), "=&v"(d[4]), "=&v"(d[5]), "=&v"(d[6]), "=&v"(d[7])
               : "v"(l16), "s"(sb) : "memory");
}

__device__ __forceinline__ void sender(KP p, char* smem, char* myws, int wv, int lane) {
  [[maybe_unused]] const int sto = 0;
  const u32 rank = p->rank;
  const u32 cb = blockIdx.x - NGEMM;
  const u32 link = cb & 7, cw = cb >> 3;
  const u32 l16 = lane * 16;
  STAMP(0);
  if (link != 0) {
    const u32 dest = peer_of_xcd(rank, link);
    const bool intra = link < 4;
    const int n = ((intra ? N_INTRA : N_INTER) - (int)cw + NCWG - 1) / NCWG;
    auto code_of = [=](int k) {
      const u32 e = cw + (u32)(k < n ? k : 0) * NCWG;
      const u32 word = ((const __attribute__((address_space(4))) u32*)p->ord[intra ? 0 : 1])[e >> 1];
      return (word >> (16 * (e & 1))) & 0xffffu;
    };
    LAS int* const done = (LAS int*)(smem + 12 * 8192);
    u32x4 d[8];
    auto fetch = [&](u32 c) {
      const bool rly = c >= 20 * 32;
      load_raw(p, d, lane, rly ? RB : c >> 5, dest & 3, rly ? RT0 + 4 * (c & 7) + (dest & 3) : c & 31);
    };
    u32 code = code_of(wv), ticket;
    if (wv >= 4) {
      if (threadIdx.x == 256) *done = 0;
      WG_BARRIER();
      if (wv >= 8) __builtin_amdgcn_s_sleep(16);
    }
    fetch(code);
    count_read(ticket, myws);
    if (wv < 4) WG_BARRIER();
    u32 call = 0, tag = 0;
    char* const rws = p->ws[dest];
    if (wv < 4) __builtin_amdgcn_s_setprio(2); else if (wv < 8) __builtin_amdgcn_s_setprio(1);
    int k = wv;
    for (; k < n; k += 12) {
      const u32 sl = code >> 5;
      u32 ud = sl < 20 ? dest_stream(sl, rank, dest, 0) * NSTEP + (code & 31) : sl == 20 ? S_RLY * NSTEP + (rank & 3) * 8 + (code & 7)
             : (S_2 + (sl - 21) * N2 + RB - N1) * NSTEP + RT0 + 4 * (code & 7) + (rank & 3);
      asm volatile("" : "+s"(ud));
      WAIT_RAW(d);
      if (k < 12) STAMP(20);
      if (k < 12) { call = count_take(ticket); tag = call_tag(call); }
      ud += (call & 1) * (NS_PAR * NSTEP);
      char* const b = rws + (size_t)ud * SLOT + 4096;
      u32x4 s[8];
      u32 meta;
      bool raw;
      if (P_RLY > 0 && sl > 20) {
        const u32 us = ((call & 1) * NS_PAR + S_RLY) * NSTEP + ((dest + sl - 20) & 3) * 8 + (code & 7);
        const u64 t0 = __builtin_amdgcn_s_memrealtime();
        const u32* const hd = (const u32*)(myws + meta_off(us)) + (lane & 7);
        for (u32 it = 1;; ++it) {
          meta = __hip_atomic_load(hd, __ATOMIC_RELAXED, __HIP_MEMORY_SCOPE_SYSTEM);
          if (__all((meta ^ tag) < (1u << 21))) break;
          if ((it & 31) == 0 && spin_fail(myws, call, t0)) break;
          __builtin_amdgcn_s_sleep(2);
        }
        load_slot(s, l16, myws + (size_t)us * SLOT + 4096);
        raw = __builtin_amdgcn_readfirstlane(meta) & 0x400;
      } else {
        to_fragments(d, sl >= 16 && sl < 20, smem + wv * 8192, lane);
        low_slabs(d, s);
        if (k < P_WIN0) { ST16(l16, s[0], b, -4096); ST16(l16, s[1], b, -3072); ST16(l16, s[2], b, -2048); ST16(l16, s[3], b, -1024); }
        raw = encode(d, s, meta, lane);
        meta |= tag;
      }
      code = code_of(k + 12);
      if (k < 12) STAMP(21);
      __builtin_amdgcn_s_setprio(0);
      const int need = k < P_WIN0 ? 0 : k + 1 - max(P_WIN, P_WIN1 - (k - P_WIN0) / P_TAPER);
      while (__hip_atomic_load(done, __ATOMIC_RELAXED, __HIP_MEMORY_SCOPE_WORKGROUP) < need) __builtin_amdgcn_s_sleep(1);
      if (k < 12) STAMP(22);
      if (k >= P_WIN0) { ST16(l16, s[0], b, -4096); ST16(l16, s[1], b, -3072); ST16(l16, s[2], b, -2048); ST16(l16, s[3], b, -1024); }
      ST16(l16, s[4], b, 0); ST16(l16, s[5], b, 1024);
      if (raw) { ST16(l16, s[6], b, 2048); ST16(l16, s[7], b, 3072); }
      if (P_PARK == 0 && k == n - 1 - P_REL && lane == 0)
        __hip_atomic_store((u32*)(myws + REL_OFF + link * 128) + cw, call, __ATOMIC_RELAXED, __HIP_MEMORY_SCOPE_AGENT);
      asm volatile("s_waitcnt vmcnt(0)" ::: "memory");
      char* const m = rws + meta_off(ud);
      if (lane < 8) ST4(lane * 4, meta, m);
      if (lane == 0) __hip_atomic_fetch_add(done, 1, __ATOMIC_RELAXED, __HIP_MEMORY_SCOPE_WORKGROUP);
      fetch(code);
      STAMP(3 + k / 12);
    }
    WAIT_RAW(d);
    count_bump(myws, call);
    STAMP(1);
    const u32 y = cw + (u32)(k - n) * NCWG;
    if (!intra && y < 2u * P_PARK) {
      const u32 us = (S_PARK + (dest & 3)) * NSTEP + y;
      const u64 t0 = __builtin_amdgcn_s_memrealtime();
      const u32* const hd = (const u32*)(myws + meta_off(us));
      for (u32 it = 1; __builtin_amdgcn_readfirstlane(__hip_atomic_load(hd, __ATOMIC_RELAXED, __HIP_MEMORY_SCOPE_SYSTEM)) != tag; ++it) {
        if ((it & 31) == 0 && spin_fail(myws, call, t0)) break;
        __builtin_amdgcn_s_sleep(4);
      }
      load_slot(d, l16, myws + (size_t)us * SLOT + 4096);
      if (__hip_atomic_load((u32*)(myws + TMO_OFF), __ATOMIC_RELAXED, __HIP_MEMORY_SCOPE_AGENT) == call) d[0] = u32x4{0x7fc07fc0u, 0x7fc07fc0u, 0x7fc07fc0u, 0x7fc07fc0u};
      const u32 tile = y >> 1, sid = (tile >> 1) * 4 + (y & 1) * 2;
      char* const ob = p->out[dest] + (rank & 3) * 256 + (tile & 1) * 128;
      const u32 vo = (lane >> 3) * (NLOC * 2) + (lane & 7) * 16;
      while (__hip_atomic_load(done, __ATOMIC_RELAXED, __HIP_MEMORY_SCOPE_WORKGROUP) < k - (P_WIN - 1)) __builtin_amdgcn_s_sleep(1);
#pragma unroll
      for (int g = 0; g < 8; ++g) {
        char* const b = ob + (size_t)(p->row[sid + (g >> 2)] + (g & 3) * 8) * (NLOC * 2);
        ST16(vo, d[g], b, 0);
      }
      if (y + P_REL * NCWG >= 2u * P_PARK && y + P_REL * NCWG < 2u * P_PARK + NCWG && lane == 0)
        __hip_atomic_store((u32*)(myws + REL_OFF + link * 128) + cw, call, __ATOMIC_RELAXED, __HIP_MEMORY_SCOPE_AGENT);
      asm volatile("s_waitcnt vmcnt(0)" ::: "memory");
      char* const fl = rws + DONE_OFF + ((rank & 3) * 64 + ((y & 1) ? 2 * RT_MED + 12 + tile : tile)) * 4;
      if (lane == 0) { ST4(0u, call, fl); __hip_atomic_fetch_add(done, 1, __ATOMIC_RELAXED, __HIP_MEMORY_SCOPE_WORKGROUP); }
      STAMP(2);
    }
  } else if (cb == 0 && wv == 0) {
    const u32 call = draw_call(myws);
    count_bump(myws, call);
    const u64 t0 = __builtin_amdgcn_s_memrealtime();
    while (__builtin_amdgcn_s_memrealtime() - t0 < N_INTRA * P_UT / 10) __builtin_amdgcn_s_sleep(16);
    const char* const f = myws + DONE_OFF;
    const int f0 = (4 * lane) & 63;
    for (u32 n = 1;; ++n) {
      u32x4 v;
      asm volatile("global_load_dwordx4 %0, %1, %2 sc0 sc1\n\ts_waitcnt vmcnt(0)" : "=&v"(v) : "v"(l16), "s"(f) : "memory");
      if (__all((f0 >= NFLAG || v[0] >= call) && (f0 + 1 >= NFLAG || v[1] >= call) && (f0 + 2 >= NFLAG || v[2] >= call) && (f0 + 3 >= NFLAG || v[3] >= call))) break;
      if ((n & 31) == 0 && spin_fail(myws, call, t0)) {
        if (lane == 0) *(unsigned short*)(p->out[rank] + (size_t)phys_rank(rank) * 512 * (NLOC * 2)) = 0x7fc0;
        break;
      }
    }
    STAMP(2);
  }
}

__device__ __forceinline__ u32x4 join(u32 l0, u32 l1, u32 h0, u32 h1) {
  return u32x4{__builtin_amdgcn_perm(h0, l0, 0x05010400u), __builtin_amdgcn_perm(h0, l0, 0x07030602u),
               __builtin_amdgcn_perm(h1, l1, 0x05010400u), __builtin_amdgcn_perm(h1, l1, 0x07030602u)};
}
__device__ __forceinline__ void fix_word(u32 (&h)[8], u32 e) {
  asm volatile("s_lshl_b64 exec, 1, %8\n\ts_getpc_b64 vcc\n\ts_add_u32 vcc_lo, vcc_lo, %9\n\ts_addc_u32 vcc_hi, vcc_hi, 0\n\ts_setpc_b64 vcc\n\t"
               "v_xor_b32_e32 %0, %10, %0\n\ts_branch 1f\n\tv_xor_b32_e32 %1, %10, %1\n\ts_branch 1f\n\t"
               "v_xor_b32_e32 %2, %10, %2\n\ts_branch 1f\n\tv_xor_b32_e32 %3, %10, %3\n\ts_branch 1f\n\t"
               "v_xor_b32_e32 %4, %10, %4\n\ts_branch 1f\n\tv_xor_b32_e32 %5, %10, %5\n\ts_branch 1f\n\t"
               "v_xor_b32_e32 %6, %10, %6\n\ts_branch 1f\n\tv_xor_b32_e32 %7, %10, %7\n"
               "1:\n\ts_mov_b64 exec, -1"
               : "+v"(h[0]), "+v"(h[1]), "+v"(h[2]), "+v"(h[3]), "+v"(h[4]), "+v"(h[5]), "+v"(h[6]), "+v"(h[7])
               : "s"(e), "s"(((e >> 3) & 56) + 12), "s"(((e >> 11) & 0x7f) << ((e >> 6) & 24)) : "vcc", "scc");
}

struct Stream {
  const char* sb; const char* mb; char* myws;
  u32 l16, tag, call; u64 t0;
  u32x4 W; int rdy; bool eager;
  __device__ __forceinline__ void init(char* ws, u32 call_, u32 sid, int lane) {
    myws = ws; call = call_; tag = call_tag(call_);
    sb = ws + (size_t)sid * NSTEP * SLOT + 4096;
    mb = ws + meta_off(sid * NSTEP);
    l16 = lane * 16;
    t0 = __builtin_amdgcn_s_memrealtime();
    W = u32x4{0u, 0u, 0u, 0u}; rdy = 0; eager = false;
  }
  __device__ __forceinline__ void poll(bool wide = true) {
    const u64 mask = (wide ? 0xffffull : 0xffull) << (rdy < 28 ? (2 * rdy) & 56 : 56);
    asm volatile("s_mov_b64 exec, %3\n\tglobal_load_dwordx4 %0, %1, %2 sc0 sc1\n\ts_mov_b64 exec, -1"
                 : "+v"(W) : "v"(l16), "s"(mb), "s"(mask) : "memory");
  }
  __device__ __forceinline__ void look() {
    asm volatile("" : "+v"(W));
    const u32 x = (W[0] ^ tag) | (W[1] ^ tag) | (W[2] ^ tag) | (W[3] ^ tag);
    const u64 bad = __ballot(x >= (1u << 21));
    rdy = bad ? __builtin_ctzll(bad) >> 1 : NSTEP;
  }
  __device__ __forceinline__ void spin(int t) {
    for (u32 n = 1; rdy <= t; ++n) {
      const int was = rdy;
      poll(n == 1);
      asm volatile("s_waitcnt vmcnt(0)" ::: "memory");
      look();
      if (rdy == was) {
        if (eager) __builtin_amdgcn_s_sleep(P_S2);
        else if (rdy == 0) __builtin_amdgcn_s_sleep(P_S0);
        else __builtin_amdgcn_s_sleep(P_S1);
      }
      if ((n & 31) == 0 && spin_fail(myws, call, t0)) rdy = NSTEP;
    }
  }
};

__device__ __forceinline__ void repair(u32 (&h)[8], u32 hd, u32 hdr, u32 hf, const char* sb, u32 l16) {
#pragma nounroll
  for (u32 n = (hdr >> (11 + 3 * hf)) & 7, k = hf ? 7 : 1; n != 0; --n, k += hf ? -1 : 1) fix_word(h, __builtin_amdgcn_readlane(hd, k));
  if (__builtin_expect(hdr & 0x400u, 0)) {
    const u32x8 g = load_two(sb + hf * 2048, l16);
#pragma unroll
    for (int j = 0; j < 8; ++j) asm volatile("v_mov_b32 %0, %1" : "+v"(h[j]) : "v"(g[j]));
  }
}

__device__ __forceinline__ void unpack_unit(char* myws, [[maybe_unused]] int wv, int lane, u32 call, u32 r, int t, u64 tk) {
  [[maybe_unused]] const int sto = 0;
  Stream S;
  S.init(myws, call, (call & 1) * NS_PAR + S_WIN + r, lane);
  const u32 l4 = lane * 4;
  const u32 uo = (S_IMG + 16 + r) * NSTEP;
  {
    if (r < 16) while (__builtin_amdgcn_s_memrealtime() - tk < (u32)(20 + 4 * t) * P_UT / 10) __builtin_amdgcn_s_sleep(4);
    u32 hd;
    for (u32 n = 1;; ++n) {
      hd = __hip_atomic_load((const u32*)(S.mb + t * 32) + (lane & 7), __ATOMIC_RELAXED, __HIP_MEMORY_SCOPE_SYSTEM);
      if (__all((hd ^ S.tag) < (1u << 21))) break;
      if ((n & 31) == 0 && spin_fail(myws, call, S.t0)) break;
      __builtin_amdgcn_s_sleep(4);
    }
    Pay<6> P;
    pay_load(P, S.l16, S.sb + (size_t)t * SLOT);
    pay_wait<0>(P);
    const u32 hdr = __builtin_amdgcn_readfirstlane(hd), base4 = (hdr & 0x7f) * 0x01010101u;
    u32x4 o[8];
#pragma unroll
    for (int hf = 0; hf < 2; ++hf) {
      u32 g[8];
#pragma unroll
      for (int j = 0; j < 4; ++j) {
        const u32 n = P.v[4 + hf][j];
        g[2 * j] = (n & 0x87878787u) + base4;
        g[2 * j + 1] = (__builtin_amdgcn_alignbit(n, n, 4) & 0x87878787u) + base4;
      }
      if (hdr & (0x400u | 0x3800u << (3 * hf))) repair(g, hd, hdr, hf, S.sb + (size_t)t * SLOT, S.l16);
#pragma unroll
      for (int j = 0; j < 4; ++j)
        o[4 * hf + j] = join(P.v[2 * hf + (j >> 1)][2 * (j & 1)], P.v[2 * hf + (j >> 1)][2 * (j & 1) + 1], g[2 * j], g[2 * j + 1]);
    }
    char* const ob = myws + (size_t)(uo + t) * SLOT + 4096;
    ST_SIX(S.l16, o, ob); ST16(S.l16, o[6], ob, 2048); ST16(S.l16, o[7], ob, 3072);
    asm volatile("s_waitcnt vmcnt(0)" ::: "memory");
    char* const mo = myws + meta_off((uo + t));
    if (lane < 8) ST4(l4, S.tag, mo);
    STAMP(50);
  }
}

__device__ __forceinline__ float silu(float y) {
  const float z = __builtin_amdgcn_fmed3f(y, -60.0f, 60.0f);
  const float e = __builtin_amdgcn_exp2f(z * -1.44269504088896341f);
  return y * __builtin_amdgcn_rcpf(1.0f + e);
}

__device__ __forceinline__ void gemm_tile(KP p, char* smem, char* myws, int wv, int lane, u32 call, u32 nb, u32 sid0, u32 tn, u32 ch, u32 fl, int hold, u64 tk,
                                          [[maybe_unused]] int sto) {
  const u32 rank = p->rank;
  STAMP(0);
  if (wv >= 8) __builtin_amdgcn_s_setprio(P_LPRIO);
  const u32 o0 = (rank & 4u) ^ 4u;
  const bool remote = tn >= 4;
  const u32 dst = (remote ? o0 + (tn - 4) : rank) & 7u;
  char* const ob = p->out[dst] + (remote ? (rank & 3) : tn) * 256 + ch * 128;
  u32* flag = (u32*)(p->ws[dst] + DONE_OFF) + (rank & 3) * 64 + fl;
  auto grow = [&](u32 rg) { return p->row[sid0 + (rg >> 2)] + (rg & 3) * 8; };
  char* b0 = ob + (size_t)grow(wv & 7) * (NLOC * 2);
  char* b1 = ob + (size_t)grow((wv & 7) + 8) * (NLOC * 2);
  if (hold == 2) {
    const u32 us = (S_PARK + (tn & 3)) * NSTEP + 2 * fl;
    b0 = myws + (size_t)us * SLOT + (wv & 7) * 1024; b1 = b0 + SLOT;
    flag = (u32*)(myws + meta_off(us));
  }
  asm volatile("" : "+s"(b0), "+s"(b1), "+s"(flag));
  if (nb == 0) {
    if (threadIdx.x == 0) ST4(0u, call, flag);
    return;
  }
  {
    const u32 first = sid0 < (u32)S_2 || sid0 >= (u32)S_OWN2 ? 0 : sid0 < (u32)S_INTER ? N1 * NSTEP : sid0 < (u32)S_3 ? 4 * NSTEP : N_INTRA - N3 * NSTEP;
    while (__builtin_amdgcn_s_memrealtime() - tk < first * P_UT / 10) __builtin_amdgcn_s_sleep(8);
  }
  Stream S;
  const u32 m = wv >> 1, kh = wv & 1;
  if (wv >= 8) {
    const u32 un = wv - 8;
    S.init(myws, call, S_IMG + 4 * tn + un, lane);
    const u32 lds0 = (u32)(size_t)(LAS char*)smem + un * 4096;
    const char* const g0 = S.sb - 4096 + ch * 4096;
    Stream A;
    A.init(myws, call, (call & 1) * NS_PAR + sid0 + un, lane);
    A.eager = sid0 >= (u32)S_3 && sid0 < (u32)S_OWN2;
    asm volatile("global_load_dwordx4 %0, %2, %3 sc0 sc1\n\tglobal_load_dwordx4 %1, %2, %4 sc0 sc1\n\ts_waitcnt vmcnt(0)"
                 : "+v"(A.W), "+v"(S.W) : "v"(S.l16), "s"(A.mb), "s"(S.mb) : "memory");
    int is = 0, seen = ((nb >> un) & 1) ? 0 : NSTEP;
    for (int t = -1; t < NSTEP; ++t) {
      if (seen < NSTEP && seen <= t + P_DIST + P_ALOOK) {
        A.look();
        A.spin(min(t + P_DIST, NSTEP - 1));
        if (A.rdy != seen) { *(LAS u32x4*)((LAS char*)smem + HDR_OFF + un * 1024 + A.l16) = A.W; seen = A.rdy; }
        asm volatile("" ::: "memory");
        if (seen < NSTEP) A.poll();
      }
      const int lim = min(min(NSTEP, is + 2), t + P_BD);
      if (S.rdy < min(NSTEP, lim + 2)) { S.look(); S.spin(t); if (S.rdy < NSTEP) S.poll(); }
      for (; is < min(lim, S.rdy); ++is) {
        const char* g = g0 + (size_t)is * SLOT;
        const u32 l = lds0 + (is & 7) * 16384;
        u32 keep;
        asm volatile("s_mov_b32 %0, m0\n\ts_mov_b32 m0, %3\n\ts_nop 0\n\t"
                     "global_load_lds_dwordx4 %1, %2 sc0 sc1\n\tglobal_load_lds_dwordx4 %1, %2 offset:1024 sc0 sc1\n\t"
                     "global_load_lds_dwordx4 %1, %2 offset:2048 sc0 sc1\n\tglobal_load_lds_dwordx4 %1, %2 offset:3072 sc0 sc1\n\t"
                     "s_mov_b32 m0, %0" : "=&s"(keep) : "v"(S.l16), "s"(g), "s"(l) : "memory");
      }
      const int ahead = is - 1 - t;
      if (ahead >= 6 || t < 0) asm volatile("s_waitcnt vmcnt(24)" ::: "memory");
      else if (ahead >= 4) { if (ahead == 5) asm volatile("s_waitcnt vmcnt(20)" ::: "memory"); else asm volatile("s_waitcnt vmcnt(16)" ::: "memory"); }
      else if (ahead >= 2) { if (ahead == 3) asm volatile("s_waitcnt vmcnt(12)" ::: "memory"); else asm volatile("s_waitcnt vmcnt(8)" ::: "memory"); }
      else if (ahead == 1) asm volatile("s_waitcnt vmcnt(4)" ::: "memory");
      else asm volatile("s_waitcnt vmcnt(0)" ::: "memory");
      WG_BARRIER();
      if (t >= 0 && (t % 3 == 0 || t >= 28)) STAMP(1 + t);
    }
    asm volatile("" : "+v"(S.W), "+v"(A.W));
    __builtin_amdgcn_s_setprio(0);
    if (hold == 1 && wv == 8) {
      const u32* const r = (const u32*)(myws + REL_OFF + (blockIdx.x & 7) * 128) + (lane < NCWG ? lane : 0);
      for (u32 n = 1; !__all(__hip_atomic_load(r, __ATOMIC_RELAXED, __HIP_MEMORY_SCOPE_AGENT) >= call); ++n) {
        if ((n & 31) == 0 && spin_fail(myws, call, S.t0)) break;
        __builtin_amdgcn_s_sleep(4);
      }
    }
  } else if (!((nb >> m) & 1)) {
    for (int t = -1; t < NSTEP; ++t) WG_BARRIER();
  } else {
    const char* const sb = myws + (size_t)((call & 1) * NS_PAR + sid0 + m) * NSTEP * SLOT + 4096;
    const u32 l16 = lane * 16, vlo = l16 + kh * 2048, vcode = l16 + kh * 1024;
    u32 ho = HDR_OFF + m * 1024 + (lane & 7) * 4, hd;
    const u32 rmask = 0x400u | 0x3800u << (3 * kh);
    f32x16 acc[2] = {};
    u32 bb = kh * 8192 + lane * 16;
    bf16x8 wf[4];
    u32x4 X[2][4];
    Pay<3> P[4];
    u32 h[8];
    u32 hdr = 0, base4 = 0;
    auto rd = [&](int stage, int f) {
      wf[f & 3] = *(const LAS bf16x8*)((LAS char*)smem + bb + stage * 16384 + (f >> 2) * 4096 + ((f >> 1) & 1) * 1024 + (f & 1) * 2048);
    };
    auto piece = [&](int k, int t, int ts, int waitn, bool fetch) {
      Pay<3>& Q = P[ts];
      if (k == 0) {
        if (waitn == 3) pay_wait<9>(Q); else if (waitn == 2) pay_wait<6>(Q); else if (waitn == 1) pay_wait<3>(Q); else pay_wait<0>(Q);
        hdr = __builtin_amdgcn_readfirstlane(hd);
        base4 = (hdr & 0x7f) * 0x01010101u;
      } else if (k < 6) {
#pragma unroll
        for (int j = 0; j < 4; ++j) {
          const int i = k < 4 ? 2 * j + (k & 1) : 4 * (k - 4) + j;
          if (k == 1) h[i] = __builtin_amdgcn_alignbit(Q.v[2][j], Q.v[2][j], 4);
          else if (k == 2) h[i] = Q.v[2][j] & 0x87878787u;
          else if (k == 3) h[i] &= 0x87878787u;
          else h[i] += base4;
          asm volatile("" : "+v"(h[i]));
        }
      } else if (k == 6) {
        if (hdr & rmask) repair(h, hd, hdr, kh, sb + (size_t)t * SLOT, l16);
      } else if (k == 7) {
      } else if (k < 12) {
        const int c = k - 8;
        X[ts & 1][c] = join(Q.v[c >> 1][2 * (c & 1)], Q.v[c >> 1][2 * (c & 1) + 1], h[2 * c], h[2 * c + 1]);
        asm volatile("" : "+v"(X[ts & 1][c]));
      } else if (k == 12) {
        ho += 32;
        hd = *(const LAS u32*)((LAS char*)smem + ho);
      } else if (k == 13 && fetch) {
        const int tn4 = t + P_DIST < NSTEP ? t + P_DIST : NSTEP - 1;
        pay_load(P[(ts + P_DIST) & 3], vlo, vcode, sb + (size_t)tn4 * SLOT);
      }
    };
    auto step = [&](int t, int ts, int waitn, bool fetch, bool more) {
#pragma unroll
      for (int q = 0; q < 8; ++q) {
        if (q == 4 && more) { WG_BARRIER(); if (ts == 3) bb ^= 65536u; }
        else if (q % 2 == 0 && (more || q < 6)) __builtin_amdgcn_s_waitcnt(0xC27F);
        __builtin_amdgcn_sched_barrier(0);
        acc[q & 1] = __builtin_amdgcn_mfma_f32_32x32x16_bf16(wf[q & 3], __builtin_bit_cast(bf16x8, X[ts & 1][q >> 1]), acc[q & 1], 0, 0, 0);
        __builtin_amdgcn_sched_barrier(0);
        if (q < 4) rd(ts, q + 4);
        else if (more) rd((ts + 1) & 3, q - 4);
        if (more) {
          piece(2 * q, t + 1, (ts + 1) & 3, waitn, fetch);
          piece(2 * q + 1, t + 1, (ts + 1) & 3, waitn, fetch);
        }
      }
    };
    WG_BARRIER();
#pragma unroll
    for (int d = 0; d < P_DIST; ++d) pay_load(P[d], vlo, vcode, sb + (size_t)d * SLOT);
    hd = *(const LAS u32*)((LAS char*)smem + ho);
#pragma unroll
    for (int k = 0; k < 13; ++k) piece(k, 0, 0, P_DIST - 1, true);
    WG_BARRIER();
    piece(13, 0, 0, P_DIST - 1, true);
#pragma unroll
    for (int f = 0; f < 4; ++f) { rd(0, f); __builtin_amdgcn_sched_barrier(0); }
    for (int tb = 0; tb < NSTEP - 4; tb += 4) {
#pragma unroll
      for (int ts = 0; ts < 4; ++ts) step(tb + ts, ts, P_DIST - 1, true, true);
    }
    step(NSTEP - 4, 0, P_DIST - 1, false, true);
    step(NSTEP - 3, 1, P_DIST - 2, false, true);
    step(NSTEP - 2, 2, P_DIST - 3, false, true);
    step(NSTEP - 1, 3, 0, false, false);
    STAMP(1);
    char* const xq = smem + SCR_OFF + (kh * 128 + m * 32 + (lane & 31)) * ERS + (lane >> 5) * 16;
#pragma unroll
    for (int n = 0; n < 2; ++n)
#pragma unroll
      for (int g = 0; g < 4; ++g)
        *(f32x4*)(xq + n * 128 + g * 32) = f32x4{acc[n][4 * g], acc[n][4 * g + 1], acc[n][4 * g + 2], acc[n][4 * g + 3]};
    if (P_DIST == 4) pay_wait<0>(P[0]);
  }
  const u32 tmo = __hip_atomic_load((u32*)(myws + TMO_OFF), __ATOMIC_RELAXED, __HIP_MEMORY_SCOPE_AGENT);
  WG_BARRIER();
  STAMP(40);
  if (wv < 8) {
    const u32 vo = hold == 2 ? lane * 16 : (lane >> 3) * (NLOC * 2) + (lane & 7) * 16;
    auto get = [&](u32 rg) {
      const char* const q = smem + SCR_OFF + (rg * 8 + (lane >> 3)) * ERS + (lane & 7) * 32;
      const f32x4 a = *(const f32x4*)q + *(const f32x4*)(q + 128 * ERS), c = *(const f32x4*)(q + 16) + *(const f32x4*)(q + 128 * ERS + 16);
      auto pk = [](float lo, float hi) { const f32x2 v = {silu(lo), silu(hi)}; return __builtin_bit_cast(u32, __builtin_convertvector(v, bf16x2)); };
      u32x4 v = {pk(a[0], a[1]), pk(a[2], a[3]), pk(c[0], c[1]), pk(c[2], c[3])};
      if (tmo == call) v = u32x4{0x7fc07fc0u, 0x7fc07fc0u, 0x7fc07fc0u, 0x7fc07fc0u};
      return v;
    };
    if ((nb >> (wv >> 2)) & 1) { const u32x4 v0 = get(wv); ST16(vo, v0, b0, 0); }
    if ((nb >> (wv >> 2)) & 4) { const u32x4 v1 = get(wv + 8); ST16(vo, v1, b1, 0); }
  }
  if (remote) asm volatile("s_waitcnt vmcnt(0)" ::: "memory");
  WG_BARRIER();
  if (remote) {
    if (threadIdx.x < (hold == 2 ? 2 : 1)) ST4(threadIdx.x * 32, hold == 2 ? call_tag(call) : call, flag);
  }
  STAMP(41);
}

inline bool tile_of(u32 rank, u32 b, int k, u32& nb, u32& sid0, u32& tn, u32& ch, u32& fl, int& hold) {
  const u32 peer = peer_of_xcd(rank, b & 7), i = b >> 3;
  const bool home = !((peer ^ rank) & 4), spare = i >= 2 * RT_MED;
  hold = 0;
  constexpr u32 I0 = S_INTER / 2;
  const bool extra = home && i >= I0 && i < I0 + (NB + 1) / 2 * 2;
  if (extra && k == 0) {
    const u32 j = i - I0;
    nb = (j | 1) < (u32)NB ? 3 : 1; tn = peer & 3; sid0 = S_OWN2 + (j >> 1) * 2; ch = j & 1; fl = 0;
    return true;
  }
  if (k == (extra ? 1 : 0) && !spare) {
    nb = 15; tn = (peer & 3) + (home ? 0 : 4); sid0 = (i >> 1) * 4; ch = i & 1; fl = i;
    hold = home || sid0 >= (u32)S_INTER ? 0 : i < (u32)P_PARK ? 2 : i >= 10 ? 1 : 0;
    if (!home)
      for (u32 m = 0; m < 4; ++m)
        if (sid0 + m >= (u32)S_INTER && (sid0 + m - S_INTER) / NB == (peer & 3)) nb &= ~(1u << m);
    return true;
  }
  const u32 j = spare ? i - 2 * RT_MED + 10 : i;
  nb = 3; tn = (peer & 3) + (home ? 4 : 0); sid0 = RT_MED * 4 + (j >> 1) * 2; ch = j & 1; fl = 2 * RT_MED + j;
  return spare ? k == 0 : i < 10;
}

constexpr u32 NHOME = 32 * NSTEP;
__host__ __device__ inline u32 pack_index(u32 b, u32 wv) {
  constexpr u32 NP1 = 80;
  return b < NP1 ? (wv < 2 ? b + NP1 * wv : NHOME) : 2 * NP1 + (b - NP1) + (NGEMM - NP1) * wv;
}

constexpr u32 UB0 = RT_MED * 16, UNW = 16;
__host__ __device__ inline u32 unpack_index(u32 b, u32 wv) {
  return b >= UB0 && b < UB0 + UNW ? (11 - wv) * UNW + (b - UB0) : 512u;
}

__global__ void __launch_bounds__(NTHREAD, 3) ag_gemm_silu(const Params) {
  KP p = (KP)__builtin_amdgcn_kernarg_segment_ptr();
  __shared__ __attribute__((aligned(16))) char smem[LDS_BYTES];
  const int lane = threadIdx.x & 63;
  const int wv = __builtin_amdgcn_readfirstlane(threadIdx.x >> 6);
  char* const myws = p->myws;
  if (blockIdx.x >= NGEMM) { sender(p, smem, myws, wv, lane); return; }
  u32 call;
  const u64 tk = __builtin_amdgcn_s_memrealtime();
  const u32 u = pack_index(blockIdx.x, wv);
  if (u < NHOME) {
    const u32 t = u >> 5, c = u & 31;
    const bool isw = c >= 16;
    const u32 cw = c - 16, l16 = lane * 16;
    const u64 t0 = __builtin_amdgcn_s_memrealtime();
    while (__builtin_amdgcn_s_memrealtime() - t0 < t * P_PP) __builtin_amdgcn_s_sleep(2);
    u32x4 d[8], s[8];
    load_raw(p, d, lane, isw ? 16 + (cw & 3) : c, (cw >> 2) & 3, t);
    u32 ticket;
    count_read(ticket, myws);
    WAIT_RAW(d);
    call = count_take(ticket);
    u32 meta = 0;
    bool eight = true;
    to_fragments(d, isw, smem + SCR_OFF + wv * 8192, lane);
    if (isw) {
#pragma unroll
      for (int j = 0; j < 8; ++j) s[j] = d[j];
    } else { low_slabs(d, s); eight = encode(d, s, meta, lane); }
    const u32 ud = (isw ? S_IMG + cw : (call & 1) * NS_PAR + (c < (u32)NA ? c : S_OWN2 - NA + c)) * NSTEP + t;
    char* const b = myws + (size_t)ud * SLOT + 4096;
    ST_SIX(l16, s, b);
    if (eight) { ST16(l16, s[6], b, 2048); ST16(l16, s[7], b, 3072); }
    asm volatile("s_waitcnt vmcnt(0)" ::: "memory");
    char* const m = myws + meta_off(ud);
    if (lane < 8) ST4(lane * 4, meta | call_tag(call), m);
    { [[maybe_unused]] const int sto = 0; STAMP(51); }
  } else call = draw_call(myws);
#pragma nounroll
  for (u32 v = unpack_index(blockIdx.x, wv); v < 512; v += 12 * UNW) unpack_unit(myws, wv, lane, call, (v + P_UROT(v >> 4)) & 15, v >> 4, tk);
  WG_BARRIER();
  count_bump(myws, call);
#pragma nounroll
  for (int k = 0; k < 2; ++k) {
    const u32 td = p->tile[blockIdx.x][k];
    if (!(td >> 23)) break;
    gemm_tile(p, smem, myws, wv, lane, call, td & 15, (td >> 4) & 127, (td >> 11) & 7, (td >> 14) & 1, (td >> 15) & 63, (td >> 21) & 3, tk, 16 * k);
  }
}

static void build_orders(Params& p) {
  struct E { double tau; unsigned short code; };
  E e[N_ORD];
  double sigma = P_SIGMA, ovl = P_OVL, ovl2 = P_OVL2, lead = P_LEAD, f1 = 14.5, f2 = 6, stg = P_STG;
  for (int kind = 0; kind < 2; ++kind) {
    int n = 0;
    for (int t = 0; t < NSTEP; ++t) {
      if (kind == 0) {
        for (int b = 0; b < NA; ++b) if (b != RB || t < RT0) e[n++] = {t + (b < N1 ? 0 : b < N1 + N2 ? NSTEP - ovl : 2 * NSTEP - ovl - ovl2 + (b >= N1 + N2 + 2 ? stg : 0)), (unsigned short)(b << 5 | t)};
      } else {
        for (int i = 0; i < 4; ++i) e[n++] = {t * sigma - lead, (unsigned short)((16 + i) << 5 | t)};
        for (int b = NA; b < 16; ++b) e[n++] = {t + 0.5, (unsigned short)(b << 5 | t)};
      }
    }
    if (kind == 1)
      for (int j = 0; j < P_RLY; ++j) {
        e[n++] = {f1 * sigma - lead, (unsigned short)(20 << 5 | j)};
        for (int i = 0; i < 3; ++i) e[n++] = {f2 + j, (unsigned short)((21 + i) << 5 | j)};
      }
    std::stable_sort(e, e + n, [](const E& a, const E& b) { return a.tau < b.tau; });
    for (int i = 0; i < N_ORD; ++i) p.ord[kind][i] = i < n ? e[i].code : 0;
  }
}

extern "C" void kernel_launch(int rank, int nranks, ncclComm_t comm, hipStream_t stream,
                              void* const* d_in, const int* in_sizes, int n_in,
                              void* d_out, int out_size, const int* in_dtypes, int out_dtype,
                              void* const* d_out_all, void* const* d_ws_all, size_t ws_bytes) {
  (void)comm; (void)in_dtypes; (void)out_dtype; (void)d_out;
  if (nranks != NR || (unsigned)rank >= (unsigned)NR || n_in != 2 || ws_bytes < WS_NEED ||
      in_sizes[0] != 512 * KDIM || in_sizes[1] != KDIM * NLOC || out_size != 4096 * NLOC) {
    fprintf(stderr, "unexpected problem shape\n");
    return;
  }
  Params p = {};
  p.x = (const char*)d_in[0];
  p.w = (const char*)d_in[1];
  for (int i = 0; i < NR; ++i) {
    p.out[i] = (char*)d_out_all[phys_rank(i)]; p.ws[i] = (char*)d_ws_all[phys_rank(i)];
    if ((int)phys_rank(i) == rank) p.rank = i;
  }
  p.myws = (char*)d_ws_all[rank];
  build_orders(p);
  for (int i = 0; i < NS_PAR + 4; ++i) p.row[i] = i < 64 || (i >= S_OWN2 && i < NS_PAR) ? global_row(p.rank, i) : 0;
  for (u32 b = 0; b < (u32)NGEMM; ++b)
    for (int k = 0; k < 2; ++k) {
      u32 nb, sid0, tn, ch, fl; int hold;
      p.tile[b][k] = tile_of(p.rank, b, k, nb, sid0, tn, ch, fl, hold) ? nb | sid0 << 4 | tn << 11 | ch << 14 | fl << 15 | (u32)hold << 21 | 1u << 23 : 0u;
    }
  hipLaunchKernelGGL(ag_gemm_silu, dim3(GRID), dim3(NTHREAD), 0, stream, p);
  hipError_t err = hipGetLastError();
  if (err != hipSuccess) fprintf(stderr, "launch failed: %s\n", hipGetErrorString(err));
}
```

```cpp
#include <hip/hip_runtime.h>
#include <rccl/rccl.h>
#include <cstdio>
#include <cstdint>
#include <cstdlib>
#include <algorithm>

typedef unsigned u32;
typedef unsigned long long u64;
typedef unsigned u32x2 __attribute__((ext_vector_type(2)));
typedef unsigned u32x4 __attribute__((ext_vector_type(4)));
typedef unsigned u32x8 __attribute__((ext_vector_type(8)));
typedef unsigned u32x16 __attribute__((ext_vector_type(16)));
typedef unsigned short u16x2 __attribute__((ext_vector_type(2)));
typedef __bf16 bf16x2 __attribute__((ext_vector_type(2)));
typedef __bf16 bf16x4 __attribute__((ext_vector_type(4)));
typedef __bf16 bf16x8 __attribute__((ext_vector_type(8)));
typedef float f32x2 __attribute__((ext_vector_type(2)));
typedef float f32x4 __attribute__((ext_vector_type(4)));
typedef float f32x16 __attribute__((ext_vector_type(16)));
#define LAS __attribute__((address_space(3)))

#define P_NA 11
#define P_NCWG 4
#define P_WIN 3
#define P_WIN0 4
#define P_TAPER 8
#define P_WIN1 5
#define P_DIST 3
#define P_BD 7
#define P_ALOOK 3
#define P_LPRIO 2
#define P_MPAD 0
#define P_UROT(t) (5 * (t))
#define P_STG 3
#define P_SIGMA 0.1
#define P_OVL 0.0
#define P_OVL2 0.0
#define P_LEAD 0.0
#define P_PP 15
#define P_PARK 10
#define P_RLY 2
#define P_S0 16
#define P_S1 16
#define P_S2 4
#define P_UT 100
#define P_REL 2

constexpr int NR = 8;
constexpr int KDIM = 4096, NLOC = 512;
constexpr int NSTEP = 32;
constexpr int NA = P_NA, NB = 16 - NA;
constexpr int N1 = 3, N3 = 4, N2 = NA - N1 - N3;
constexpr int S_1 = NA, S_2 = S_1 + 3 * N1, S_INTER = S_2 + 3 * N2, S_3 = 64 - 3 * N3, S_WIN = 64, S_RLY = 80, S_OWN2 = 81;
constexpr int RB = N1 + N2 - 1, RT0 = NSTEP - 4 * P_RLY;
constexpr int NS_PAR = S_OWN2 + NB, S_IMG = 2 * NS_PAR, S_PARK = S_IMG + 32, NSTREAM = S_PARK + 4;
constexpr size_t SLOT = 8192;
constexpr size_t META_OFF = (size_t)NSTREAM * NSTEP * SLOT;
constexpr u32 MPAD = P_MPAD;
constexpr size_t DONE_OFF = META_OFF + (size_t)NSTREAM * (NSTEP * 32 + MPAD);
constexpr size_t REL_OFF = DONE_OFF + 1024;
constexpr size_t CNT_OFF = REL_OFF + 1024;
constexpr size_t TMO_OFF = CNT_OFF + 256 * 128;
constexpr size_t WS_NEED = TMO_OFF + 128;
constexpr int N_INTRA = NA * NSTEP - 4 * P_RLY, N_INTER = NB * NSTEP + 4 * NSTEP + 4 * P_RLY;
constexpr int N_ORD = N_INTRA > N_INTER ? N_INTRA : N_INTER;
constexpr int RT_MED = 13;
constexpr int NGEMM = RT_MED * 16 + 16;
constexpr int NFLAG = 2 * RT_MED + 12 + P_PARK;
constexpr int NCWG = P_NCWG;
constexpr int GRID = NGEMM + 8 * NCWG;
static_assert(P_BD <= 7 && P_RLY <= 8 && N_ORD % 2 == 0 && NB > 4 && NB <= 8 && S_INTER % 4 == 0, "tile map");
static_assert(GRID <= 256, "one workgroup per CU, all resident");
constexpr int NTHREAD = 768;
constexpr int SCR_OFF = 0;
constexpr int HDR_OFF = 131072;
constexpr int LDS_BYTES = HDR_OFF + 4096;
constexpr u64 DEADLINE_TICKS = 80000000ull;
constexpr int ERS = 272;

#define P_PHYS 0x75416320u
__host__ __device__ inline u32 phys_rank(u32 l) { return (P_PHYS >> (4 * l)) & 7u; }
__host__ __device__ inline u32 peer_of_xcd(u32 rank, u32 x) { return (((rank ^ x) & 4u) | ((rank + x) & 3u)) & 7u; }

struct Params {
  const char* x;
  const char* w;
  char* myws;
  char* out[NR];
  char* ws[NR];
  int rank;
  u32 row[NS_PAR + 4];
  u32 tile[NGEMM][2];
  alignas(4) unsigned short ord[2][N_ORD];
};
static_assert(sizeof(Params) <= 4096, "kernel argument");
#define P_SFIRST 1
#define BID (P_SFIRST ? (blockIdx.x + NGEMM) % GRID : blockIdx.x)
typedef const __attribute__((address_space(4))) Params* KP;

#define WG_BARRIER() do { asm volatile("" ::: "memory"); __builtin_amdgcn_s_waitcnt(0xC07F); \
    asm volatile("s_barrier" ::: "memory"); } while (0)
#define ST16(voff, data, sbase, imm) \
  asm volatile("global_store_dwordx4 %0, %1, %2 offset:" #imm " sc0 sc1\n\ts_nop 1" :: "v"(voff), "v"(data), "s"(sbase) : "memory")
#define ST4(voff, data, sbase) \
  asm volatile("global_store_dword %0, %1, %2 sc0 sc1" :: "v"(voff), "v"(data), "s"(sbase) : "memory")
#define ST_SIX(voff, s, b) do { ST16(voff, s[0], b, -4096); ST16(voff, s[1], b, -3072); ST16(voff, s[2], b, -2048); \
    ST16(voff, s[3], b, -1024); ST16(voff, s[4], b, 0); ST16(voff, s[5], b, 1024); } while (0)
#define CAT16(a, b) __builtin_shufflevector(a, b, 0, 1, 2, 3, 4, 5, 6, 7, 8, 9, 10, 11, 12, 13, 14, 15)

__device__ __forceinline__ void count_read(u32& v, char* myws) {
  const char* const c = myws + CNT_OFF + BID * 128;
  asm volatile("global_load_dword %0, %1, %2 sc1" : "=&v"(v) : "v"(0u), "s"(c) : "memory");
}
__device__ __forceinline__ u32 count_take(u32& v) {
  asm volatile("" : "+v"(v));
  return __builtin_amdgcn_readfirstlane(v) + 1u;
}
__device__ __forceinline__ void count_bump(char* myws, u32 call) {
  if (threadIdx.x == 0) __hip_atomic_store((u32*)(myws + CNT_OFF + BID * 128), call, __ATOMIC_RELAXED, __HIP_MEMORY_SCOPE_AGENT);
}
__device__ __forceinline__ u32 draw_call(char* myws) {
  u32 v;
  count_read(v, myws);
  asm volatile("s_waitcnt vmcnt(0)" : "+v"(v) :: "memory");
  return count_take(v);
}
__device__ __forceinline__ u32 call_tag(u32 call) { return (call % 2047u + 1u) << 21; }
__device__ __forceinline__ u32 hdr_word(u32 data, u32 call, int lane) { return lane == 7 ? call : data | call_tag(call); }
__device__ __forceinline__ bool hdr_there(u32 word, u32 call, int lane) {
  return __all((lane & 7) == 7 ? word >= call : (word ^ call_tag(call)) < (1u << 21));
}

__device__ __forceinline__ bool spin_fail(char* myws, u32 call, u64 t0) {
  u32* tmo = (u32*)(myws + TMO_OFF);
  u32 t = __builtin_amdgcn_readfirstlane(__hip_atomic_load(tmo, __ATOMIC_RELAXED, __HIP_MEMORY_SCOPE_AGENT));
  if (t == call || __builtin_amdgcn_s_memrealtime() - t0 > DEADLINE_TICKS) {
    __hip_atomic_store(tmo, call, __ATOMIC_RELAXED, __HIP_MEMORY_SCOPE_AGENT);
    return true;
  }
  return false;
}

__device__ __forceinline__ size_t meta_off(u32 unit) { return META_OFF + (size_t)(unit * 32 + (unit / NSTEP) * MPAD); }
__device__ __forceinline__ u32 dest_stream(u32 sl, u32 rank, u32 dest, u32 par) {
  const u32 j = ((rank - dest) & 3) - 1;
  const u32 s = sl >= 16 ? S_WIN + (rank & 3) * 4 + (sl - 16) : sl >= (u32)NA ? S_INTER + (rank & 3) * NB + (sl - NA)
              : sl >= (u32)(N1 + N2) ? S_3 + j * N3 + (sl - N1 - N2) : sl >= (u32)N1 ? S_2 + j * N2 + (sl - N1) : S_1 + j * N1 + sl;
  return par * NS_PAR + s;
}

inline u32 global_row(u32 rank, u32 sid) {
  const u32 g0 = rank & 4u, o0 = g0 ^ 4u;
  u32 src, bi;
  if (sid < (u32)S_1) { src = rank; bi = sid; }
  else if (sid < (u32)S_2) { const u32 q = sid - S_1; src = g0 + ((rank + 1 + q / N1) & 3); bi = q % N1; }
  else if (sid < (u32)S_INTER) { const u32 q = sid - S_2; src = g0 + ((rank + 1 + q / N2) & 3); bi = N1 + q % N2; }
  else if (sid < (u32)S_3) { const u32 q = sid - S_INTER; src = o0 + q / NB; bi = NA + q % NB; }
  else if (sid >= (u32)S_OWN2) { src = rank; bi = NA + sid - S_OWN2; }
  else { const u32 q = sid - S_3; src = g0 + ((rank + 1 + q / N3) & 3); bi = N1 + N2 + q % N3; }
  return (phys_rank(src & 7u) * 512 + bi * 32) & 4095u;
}

__device__ __forceinline__ void load_raw(KP p, u32x4 (&d)[8], int lane, u32 sl, u32 q, u32 t) {
  const bool isw = sl >= 16;
  const u32 rs = isw ? NLOC * 2 : KDIM * 2;
  const u32 vo = (lane >> 4) * rs + (lane & 15) * 16;
  const u64 aw = (u64)p->w + (u64)((t * 128 + (sl & 3) * 32) * (NLOC * 2) + q * 256), ax = (u64)p->x + (u64)(sl * 32 * (KDIM * 2) + t * 256);
  const u64 b0 = isw ? aw : ax;
#pragma unroll
  for (int j = 0; j < 8; ++j)
    asm volatile("global_load_dwordx4 %0, %1, %2" : "=&v"(d[j]) : "v"(vo), "s"(b0 + (u64)(j * 4 * rs)) : "memory");
}
#define WAIT_RAW(d) asm volatile("s_waitcnt vmcnt(0)" : "+v"(d[0]), "+v"(d[1]), "+v"(d[2]), "+v"(d[3]), "+v"(d[4]), "+v"(d[5]), "+v"(d[6]), "+v"(d[7]) :: "memory")
__device__ __forceinline__ void to_fragments(u32x4 (&d)[8], bool isw, char* scr, int lane) {
  LAS char* const sc = (LAS char*)scr;
  auto off = [](u32 row, u32 ch) { return 256u * row + 16u * (ch ^ (((row & 3) << 2) | ((row >> 2) & 3))); };
#pragma unroll
  for (int j = 0; j < 8; ++j) *(LAS u32x4*)(sc + off(4 * j + (lane >> 4), lane & 15)) = d[j];
  asm volatile("" ::: "memory");
  if (isw) {
    const u32 r0 = 8 * (lane >> 5) + ((lane & 15) >> 2), c0 = 2 * ((lane >> 4) & 1) + ((lane & 3) >> 1);
#pragma unroll
    for (int j = 0; j < 8; ++j) {
      const bf16x4 lo = __builtin_amdgcn_ds_read_tr16_b64_v4bf16((LAS bf16x4*)(sc + off(16 * (j & 1) + r0, 4 * (j >> 1) + c0) + 8 * (lane & 1)));
      const bf16x4 hi = __builtin_amdgcn_ds_read_tr16_b64_v4bf16((LAS bf16x4*)(sc + off(16 * (j & 1) + r0 + 4, 4 * (j >> 1) + c0) + 8 * (lane & 1)));
      d[j] = __builtin_bit_cast(u32x4, __builtin_shufflevector(lo, hi, 0, 1, 2, 3, 4, 5, 6, 7));
    }
  } else {
#pragma unroll
    for (int j = 0; j < 8; ++j) d[j] = *(const LAS u32x4*)(sc + off(lane & 31, 2 * j + (lane >> 5)));
  }
  asm volatile("" ::: "memory");
}
__device__ __forceinline__ void low_slabs(const u32x4 (&d)[8], u32x4 (&s)[8]) {
#pragma unroll
  for (int i = 0; i < 16; ++i) s[i >> 2][i & 3] = __builtin_amdgcn_perm(d[i >> 1][2 * (i & 1) + 1], d[i >> 1][2 * (i & 1)], 0x06040200u);
}
__device__ __forceinline__ bool encode(const u32x4 (&d)[8], u32x4 (&s)[8], u32& meta, int lane) {
  u32 H[16], T[16];
  u16x2 m1 = {0, 0}, m2 = {0, 0}, m3 = {0, 0}, m4 = {0, 0};
#pragma unroll
  for (int i = 0; i < 16; ++i) {
    const u32 a = d[i >> 1][2 * (i & 1)], b = d[i >> 1][2 * (i & 1) + 1];
    H[i] = __builtin_amdgcn_perm(b, a, 0x07050301u);
  }
#pragma unroll
  for (int i = 0; i < 16; i += 2) {
    T[i] = H[i] | 0x80808080u;
    T[i + 1] = H[i + 1] | 0x80808080u;
    m1 = __builtin_elementwise_max(m1, __builtin_bit_cast(u16x2, T[i]));
    m2 = __builtin_elementwise_max(m2, __builtin_bit_cast(u16x2, T[i] << 8));
    m3 = __builtin_elementwise_max(m3, __builtin_bit_cast(u16x2, T[i + 1]));
    m4 = __builtin_elementwise_max(m4, __builtin_bit_cast(u16x2, T[i + 1] << 8));
  }
  m1 = __builtin_elementwise_max(__builtin_elementwise_max(m1, m2), __builtin_elementwise_max(m3, m4));
  u32 mx = (u32)(m1.x > m1.y ? m1.x : m1.y);
  mx = max(mx, (u32)__builtin_amdgcn_update_dpp(0, mx, 0x111, 0xf, 0xf, true));
  mx = max(mx, (u32)__builtin_amdgcn_update_dpp(0, mx, 0x112, 0xf, 0xf, true));
  mx = max(mx, (u32)__builtin_amdgcn_update_dpp(0, mx, 0x114, 0xf, 0xf, true));
  mx = max(mx, (u32)__builtin_amdgcn_update_dpp(0, mx, 0x118, 0xf, 0xf, true));
  const u32 top = max(max((u32)__builtin_amdgcn_readlane(mx, 15), (u32)__builtin_amdgcn_readlane(mx, 31)),
                      max((u32)__builtin_amdgcn_readlane(mx, 47), (u32)__builtin_amdgcn_readlane(mx, 63)));
  const u32 h7 = (top >> 8) & 0x7f;
  const u32 base = __builtin_amdgcn_readfirstlane(h7 > 7 ? h7 - 7 : 0);
  const u32 base4 = base * 0x01010101u;
  u32 ok = 0xffffffffu;
  u32 N[8];
#pragma unroll
  for (int j = 0; j < 8; ++j) {
    const u32 ta = T[2 * j] - base4, tb = T[2 * j + 1] - base4;
    ok &= ta & tb;
    const u32 ca = (ta & 0x07070707u) | (H[2 * j] & 0x80808080u);
    const u32 cb = (tb & 0x07070707u) | (H[2 * j + 1] & 0x80808080u);
    N[j] = ca | __builtin_amdgcn_alignbit(cb, cb, 28);
  }
  meta = 0;
  int c0 = 0, c1 = 0;
  if (__any((~ok & 0x80808080u) != 0)) {
    u32 M0 = 0, M1 = 0;
#pragma unroll
    for (int i = 0; i < 8; ++i) {
      M0 |= ~((T[i] - base4) >> i) & (0x80808080u >> i);
      M1 |= ~((T[8 + i] - base4) >> i) & (0x80808080u >> i);
    }
    u64 bal = __ballot((M0 | M1) != 0);
    while (bal != 0 && c0 + c1 <= 6) {
      const int ln = __builtin_ctzll(bal);
      bal &= bal - 1;
      u64 mm = (u64)(u32)__builtin_amdgcn_readlane(M0, ln) | (u64)(u32)__builtin_amdgcn_readlane(M1, ln) << 32;
      while (mm != 0) {
        const u32 bit = __builtin_ctzll(mm), hf = bit >> 5, by = (bit >> 3) & 3, i = hf * 8 + 7 - (bit & 7);
        mm &= mm - 1;
        c0 += hf ^ 1; c1 += hf;
        u32 hw = 0;
#pragma unroll
        for (int z = 0; z < 16; ++z) hw |= H[z] & (u32)__builtin_amdgcn_sbfe(1 << i, z, 1);
        const u32 hb = ((u32)__builtin_amdgcn_readlane(hw, ln) >> (8 * by)) & 0x7f;
        const u32 ent = (u32)ln | (i & 7) << 6 | by << 9 | ((((hb - base) & 7) + base) ^ hb) << 11;
        if (lane == (hf ? 7 - c1 : c0)) meta = ent;
      }
    }
  }
  const bool raw = c0 + c1 > 6;
  if (lane == 0) meta = base | (raw ? 0x400u : (u32)(c0 << 11 | c1 << 14));
  s[4] = raw ? u32x4{H[0], H[1], H[2], H[3]} : u32x4{N[0], N[1], N[2], N[3]};
  s[5] = raw ? u32x4{H[4], H[5], H[6], H[7]} : u32x4{N[4], N[5], N[6], N[7]};
  s[6] = u32x4{H[8], H[9], H[10], H[11]};
  s[7] = u32x4{H[12], H[13], H[14], H[15]};
#pragma unroll
  for (int j = 0; j < 8; ++j) asm volatile("" : "+v"(s[j]));
  return raw;
}

template <int N> struct Pay { u32x4 v[N]; };
__device__ __forceinline__ void pay_load(Pay<6>& P, u32 l16, const char* sb) {
  asm volatile("global_load_dwordx4 %0, %6, %7 offset:-4096 sc0 sc1\n\t"
               "global_load_dwordx4 %1, %6, %7 offset:-3072 sc0 sc1\n\t"
               "global_load_dwordx4 %2, %6, %7 offset:-2048 sc0 sc1\n\t"
               "global_load_dwordx4 %3, %6, %7 offset:-1024 sc0 sc1\n\t"
               "global_load_dwordx4 %4, %6, %7 offset:0 sc0 sc1\n\t"
               "global_load_dwordx4 %5, %6, %7 offset:1024 sc0 sc1"
               : "=&v"(P.v[0]), "=&v"(P.v[1]), "=&v"(P.v[2]), "=&v"(P.v[3]), "=&v"(P.v[4]), "=&v"(P.v[5]) : "v"(l16), "s"(sb) : "memory");
}
__device__ __forceinline__ void pay_load(Pay<3>& P, u32 vlo, u32 vcode, const char* sb) {
  asm volatile("global_load_dwordx4 %2, %4, %5 offset:0 sc0 sc1\n\t"
               "global_load_dwordx4 %0, %3, %5 offset:-4096 sc0 sc1\n\t"
               "global_load_dwordx4 %1, %3, %5 offset:-3072 sc0 sc1"
               : "=&v"(P.v[0]), "=&v"(P.v[1]), "=&v"(P.v[2]) : "v"(vlo), "v"(vcode), "s"(sb) : "memory");
}
template <int C> __device__ __forceinline__ void pay_wait(Pay<6>& P) {
  asm volatile("s_waitcnt vmcnt(%6)" : "+v"(P.v[0]), "+v"(P.v[1]), "+v"(P.v[2]), "+v"(P.v[3]), "+v"(P.v[4]), "+v"(P.v[5]) : "n"(C) : "memory");
}
template <int C> __device__ __forceinline__ void pay_wait(Pay<3>& P) {
  asm volatile("s_waitcnt vmcnt(%3)" : "+v"(P.v[0]), "+v"(P.v[1]), "+v"(P.v[2]) : "n"(C) : "memory");
}
__device__ __forceinline__ u32x8 load_two(const char* b, u32 l16) {
  u32x4 x, y;
  asm volatile("global_load_dwordx4 %0, %2, %3 sc0 sc1\n\t"
               "global_load_dwordx4 %1, %2, %3 offset:1024 sc0 sc1\n\ts_waitcnt vmcnt(0)"
               : "=&v"(x), "=&v"(y) : "v"(l16), "s"(b) : "memory");
  return __builtin_shufflevector(x, y, 0, 1, 2, 3, 4, 5, 6, 7);
}

__device__ __forceinline__ void load_slot(u32x4 (&d)[8], u32 l16, const char* sb) {
  asm volatile("global_load_dwordx4 %0, %8, %9 offset:-4096 sc0 sc1\n\tglobal_load_dwordx4 %1, %8, %9 offset:-3072 sc0 sc1\n\t"
               "global_load_dwordx4 %2, %8, %9 offset:-2048 sc0 sc1\n\tglobal_load_dwordx4 %3, %8, %9 offset:-1024 sc0 sc1\n\t"
               "global_load_dwordx4 %4, %8, %9 offset:0 sc0 sc1\n\tglobal_load_dwordx4 %5, %8, %9 offset:1024 sc0 sc1\n\t"
               "global_load_dwordx4 %6, %8, %9 offset:2048 sc0 sc1\n\tglobal_load_dwordx4 %7, %8, %9 offset:3072 sc0 sc1\n\ts_waitcnt vmcnt(0)"
               : "=&v"(d[0]), "=&v"(d[1]), "=&v"(d[2]), "=&v"(d[3]), "=&v"(d[4]), "=&v"(d[5]), "=&v"(d[6]), "=&v"(d[7])
               : "v"(l16), "s"(sb) : "memory");
}

__device__ __forceinline__ void sender(KP p, char* smem, char* myws, int wv, int lane) {
  const u32 rank = p->rank;
  const u32 cb = BID - NGEMM;
  const u32 link = cb & 7, cw = cb >> 3;
  const u32 l16 = lane * 16;
  if (link != 0) {
    const u32 dest = peer_of_xcd(rank, link);
    const bool intra = link < 4;
    const int n = ((intra ? N_INTRA : N_INTER) - (int)cw + NCWG - 1) / NCWG;
    auto code_of = [=](int k) {
      const u32 e = cw + (u32)(k < n ? k : 0) * NCWG;
      const u32 word = ((const __attribute__((address_space(4))) u32*)p->ord[intra ? 0 : 1])[e >> 1];
      return (word >> (16 * (e & 1))) & 0xffffu;
    };
    LAS int* const done = (LAS int*)(smem + 12 * 8192);
    u32x4 d[8];
    auto fetch = [&](u32 c) {
      const bool rly = c >= 20 * 32;
      load_raw(p, d, lane, rly ? RB : c >> 5, dest & 3, rly ? RT0 + 4 * (c & 7) + (dest & 3) : c & 31);
    };
    u32 code = code_of(wv), ticket;
    if (wv >= 4) {
      if (threadIdx.x == 256) *done = 0;
      WG_BARRIER();
      if (wv >= 8) __builtin_amdgcn_s_sleep(16);
    }
    fetch(code);
    count_read(ticket, myws);
    if (wv < 4) WG_BARRIER();
    u32 call = 0;
    char* const rws = p->ws[dest];
    if (wv < 4) __builtin_amdgcn_s_setprio(2); else if (wv < 8) __builtin_amdgcn_s_setprio(1);
    int k = wv;
    for (; k < n; k += 12) {
      const u32 sl = code >> 5;
      u32 ud = sl < 20 ? dest_stream(sl, rank, dest, 0) * NSTEP + (code & 31) : sl == 20 ? S_RLY * NSTEP + (rank & 3) * 8 + (code & 7)
             : (S_2 + (sl - 21) * N2 + RB - N1) * NSTEP + RT0 + 4 * (code & 7) + (rank & 3);
      asm volatile("" : "+s"(ud));
      WAIT_RAW(d);
      if (k < 12) call = count_take(ticket);
      ud += (call & 1) * (NS_PAR * NSTEP);
      char* const b = rws + (size_t)ud * SLOT + 4096;
      u32x4 s[8];
      u32 meta;
      bool raw;
      if (P_RLY > 0 && sl > 20) {
        const u32 us = ((call & 1) * NS_PAR + S_RLY) * NSTEP + ((dest + sl - 20) & 3) * 8 + (code & 7);
        const u64 t0 = __builtin_amdgcn_s_memrealtime();
        const u32* const hd = (const u32*)(myws + meta_off(us)) + (lane & 7);
        for (u32 it = 1;; ++it) {
          meta = __hip_atomic_load(hd, __ATOMIC_RELAXED, __HIP_MEMORY_SCOPE_SYSTEM);
          if (hdr_there(meta, call, lane)) break;
          if ((it & 31) == 0 && spin_fail(myws, call, t0)) break;
          __builtin_amdgcn_s_sleep(2);
        }
        load_slot(s, l16, myws + (size_t)us * SLOT + 4096);
        raw = __builtin_amdgcn_readfirstlane(meta) & 0x400;
      } else {
        to_fragments(d, sl >= 16 && sl < 20, smem + wv * 8192, lane);
        low_slabs(d, s);
        if (k < P_WIN0) { ST16(l16, s[0], b, -4096); ST16(l16, s[1], b, -3072); ST16(l16, s[2], b, -2048); ST16(l16, s[3], b, -1024); }
        raw = encode(d, s, meta, lane);
        meta = hdr_word(meta, call, lane);
      }
      code = code_of(k + 12);
      __builtin_amdgcn_s_setprio(0);
      const int need = k < P_WIN0 ? 0 : k + 1 - max(P_WIN, P_WIN1 - (k - P_WIN0) / P_TAPER);
      while (__hip_atomic_load(done, __ATOMIC_RELAXED, __HIP_MEMORY_SCOPE_WORKGROUP) < need) __builtin_amdgcn_s_sleep(1);
      if (k >= P_WIN0) { ST16(l16, s[0], b, -4096); ST16(l16, s[1], b, -3072); ST16(l16, s[2], b, -2048); ST16(l16, s[3], b, -1024); }
      ST16(l16, s[4], b, 0); ST16(l16, s[5], b, 1024);
      if (raw) { ST16(l16, s[6], b, 2048); ST16(l16, s[7], b, 3072); }
      if (P_PARK == 0 && k == n - 1 - P_REL && lane == 0)
        __hip_atomic_store((u32*)(myws + REL_OFF + link * 128) + cw, call, __ATOMIC_RELAXED, __HIP_MEMORY_SCOPE_AGENT);
      asm volatile("s_waitcnt vmcnt(0)" ::: "memory");
      char* const m = rws + meta_off(ud);
      if (lane < 8) ST4(lane * 4, meta, m);
      if (lane == 0) __hip_atomic_fetch_add(done, 1, __ATOMIC_RELAXED, __HIP_MEMORY_SCOPE_WORKGROUP);
      fetch(code);
    }
    WAIT_RAW(d);
    count_bump(myws, call);
    const u32 y = cw + (u32)(k - n) * NCWG;
    if (!intra && y < 2u * P_PARK) {
      const u32 us = (S_PARK + (dest & 3)) * NSTEP + y;
      const u64 t0 = __builtin_amdgcn_s_memrealtime();
      const u32* const hd = (const u32*)(myws + meta_off(us));
      for (u32 it = 1; (u32)__builtin_amdgcn_readfirstlane(__hip_atomic_load(hd, __ATOMIC_RELAXED, __HIP_MEMORY_SCOPE_SYSTEM)) < call; ++it) {
        if ((it & 31) == 0 && spin_fail(myws, call, t0)) break;
        __builtin_amdgcn_s_sleep(4);
      }
      load_slot(d, l16, myws + (size_t)us * SLOT + 4096);
      if (__hip_atomic_load((u32*)(myws + TMO_OFF), __ATOMIC_RELAXED, __HIP_MEMORY_SCOPE_AGENT) == call) d[0] = u32x4{0x7fc07fc0u, 0x7fc07fc0u, 0x7fc07fc0u, 0x7fc07fc0u};
      const u32 tile = y >> 1, sid = (tile >> 1) * 4 + (y & 1) * 2;
      char* const ob = p->out[dest] + (rank & 3) * 256 + (tile & 1) * 128;
      const u32 vo = (lane >> 3) * (NLOC * 2) + (lane & 7) * 16;
      while (__hip_atomic_load(done, __ATOMIC_RELAXED, __HIP_MEMORY_SCOPE_WORKGROUP) < k - (P_WIN - 1)) __builtin_amdgcn_s_sleep(1);
#pragma unroll
      for (int g = 0; g < 8; ++g) {
        char* const b = ob + (size_t)(p->row[sid + (g >> 2)] + (g & 3) * 8) * (NLOC * 2);
        ST16(vo, d[g], b, 0);
      }
      if (y + P_REL * NCWG >= 2u * P_PARK && y + P_REL * NCWG < 2u * P_PARK + NCWG && lane == 0)
        __hip_atomic_store((u32*)(myws + REL_OFF + link * 128) + cw, call, __ATOMIC_RELAXED, __HIP_MEMORY_SCOPE_AGENT);
      asm volatile("s_waitcnt vmcnt(0)" ::: "memory");
      char* const fl = rws + DONE_OFF + ((rank & 3) * 64 + ((y & 1) ? 2 * RT_MED + 12 + tile : tile)) * 4;
      if (lane == 0) { ST4(0u, call, fl); __hip_atomic_fetch_add(done, 1, __ATOMIC_RELAXED, __HIP_MEMORY_SCOPE_WORKGROUP); }
    }
  } else if (cb == 0 && wv == 0) {
    const u32 call = draw_call(myws);
    count_bump(myws, call);
    const u64 t0 = __builtin_amdgcn_s_memrealtime();
    while (__builtin_amdgcn_s_memrealtime() - t0 < N_INTRA * P_UT / 10) __builtin_amdgcn_s_sleep(16);
    const char* const f = myws + DONE_OFF;
    const int f0 = (4 * lane) & 63;
    for (u32 n = 1;; ++n) {
      u32x4 v;
      asm volatile("global_load_dwordx4 %0, %1, %2 sc0 sc1\n\ts_waitcnt vmcnt(0)" : "=&v"(v) : "v"(l16), "s"(f) : "memory");
      if (__all((f0 >= NFLAG || v[0] >= call) && (f0 + 1 >= NFLAG || v[1] >= call) && (f0 + 2 >= NFLAG || v[2] >= call) && (f0 + 3 >= NFLAG || v[3] >= call))) break;
      if ((n & 31) == 0 && spin_fail(myws, call, t0)) {
        if (lane == 0) *(unsigned short*)(p->out[rank] + (size_t)phys_rank(rank) * 512 * (NLOC * 2)) = 0x7fc0;
        break;
      }
    }
  }
}

__device__ __forceinline__ u32x4 join(u32 l0, u32 l1, u32 h0, u32 h1) {
  return u32x4{__builtin_amdgcn_perm(h0, l0, 0x05010400u), __builtin_amdgcn_perm(h0, l0, 0x07030602u),
               __builtin_amdgcn_perm(h1, l1, 0x05010400u), __builtin_amdgcn_perm(h1, l1, 0x07030602u)};
}
__device__ __forceinline__ void fix_word(u32 (&h)[8], u32 e) {
  asm volatile("s_lshl_b64 exec, 1, %8\n\ts_getpc_b64 vcc\n\ts_add_u32 vcc_lo, vcc_lo, %9\n\ts_addc_u32 vcc_hi, vcc_hi, 0\n\ts_setpc_b64 vcc\n\t"
               "v_xor_b32_e32 %0, %10, %0\n\ts_branch 1f\n\tv_xor_b32_e32 %1, %10, %1\n\ts_branch 1f\n\t"
               "v_xor_b32_e32 %2, %10, %2\n\ts_branch 1f\n\tv_xor_b32_e32 %3, %10, %3\n\ts_branch 1f\n\t"
               "v_xor_b32_e32 %4, %10, %4\n\ts_branch 1f\n\tv_xor_b32_e32 %5, %10, %5\n\ts_branch 1f\n\t"
               "v_xor_b32_e32 %6, %10, %6\n\ts_branch 1f\n\tv_xor_b32_e32 %7, %10, %7\n"
               "1:\n\ts_mov_b64 exec, -1"
               : "+v"(h[0]), "+v"(h[1]), "+v"(h[2]), "+v"(h[3]), "+v"(h[4]), "+v"(h[5]), "+v"(h[6]), "+v"(h[7])
               : "s"(e), "s"(((e >> 3) & 56) + 12), "s"(((e >> 11) & 0x7f) << ((e >> 6) & 24)) : "vcc", "scc");
}

struct Stream {
  const char* sb; const char* mb; char* myws;
  u32 l16, tag, call; u64 t0;
  u32x4 W; int rdy; bool eager;
  __device__ __forceinline__ void init(char* ws, u32 call_, u32 sid, int lane) {
    myws = ws; call = call_; tag = call_tag(call_);
    sb = ws + (size_t)sid * NSTEP * SLOT + 4096;
    mb = ws + meta_off(sid * NSTEP);
    l16 = lane * 16;
    t0 = __builtin_amdgcn_s_memrealtime();
    W = u32x4{0u, 0u, 0u, 0u}; rdy = 0; eager = false;
  }
  __device__ __forceinline__ void poll(bool wide = true) {
    const u64 mask = (wide ? 0xffffull : 0xffull) << (rdy < 28 ? (2 * rdy) & 56 : 56);
    asm volatile("s_mov_b64 exec, %3\n\tglobal_load_dwordx4 %0, %1, %2 sc0 sc1\n\ts_mov_b64 exec, -1"
                 : "+v"(W) : "v"(l16), "s"(mb), "s"(mask) : "memory");
  }
  __device__ __forceinline__ void look() {
    asm volatile("" : "+v"(W));
    const u32 x = (W[0] ^ tag) | (W[1] ^ tag) | (W[2] ^ tag);
    const u64 odd = 0xaaaaaaaaaaaaaaaaull;
    const u64 bad = __ballot(x >= (1u << 21)) | (__ballot((W[3] ^ tag) >= (1u << 21)) & ~odd) | (__ballot(W[3] < call) & odd);
    rdy = bad ? __builtin_ctzll(bad) >> 1 : NSTEP;
  }
  __device__ __forceinline__ void spin(int t) {
    for (u32 n = 1; rdy <= t; ++n) {
      const int was = rdy;
      poll(n == 1);
      asm volatile("s_waitcnt vmcnt(0)" ::: "memory");
      look();
      if (rdy == was) {
        if (eager) __builtin_amdgcn_s_sleep(P_S2);
        else if (rdy == 0) __builtin_amdgcn_s_sleep(P_S0);
        else __builtin_amdgcn_s_sleep(P_S1);
      }
      if ((n & 31) == 0 && spin_fail(myws, call, t0)) rdy = NSTEP;
    }
  }
};

__device__ __forceinline__ void repair(u32 (&h)[8], u32 hd, u32 hdr, u32 hf, const char* sb, u32 l16) {
#pragma nounroll
  for (u32 n = (hdr >> (11 + 3 * hf)) & 7, k = hf ? 6 : 1; n != 0; --n, k += hf ? -1 : 1) fix_word(h, __builtin_amdgcn_readlane(hd, k));
  if (__builtin_expect(hdr & 0x400u, 0)) {
    const u32x8 g = load_two(sb + hf * 2048, l16);
#pragma unroll
    for (int j = 0; j < 8; ++j) asm volatile("v_mov_b32 %0, %1" : "+v"(h[j]) : "v"(g[j]));
  }
}

__device__ __forceinline__ void unpack_unit(char* myws, int lane, u32 call, u32 r, int t, u64 tk) {
  Stream S;
  S.init(myws, call, (call & 1) * NS_PAR + S_WIN + r, lane);
  const u32 l4 = lane * 4;
  const u32 uo = (S_IMG + 16 + r) * NSTEP;
  {
    if (r < 16) while (__builtin_amdgcn_s_memrealtime() - tk < (u32)(20 + 4 * t) * P_UT / 10) __builtin_amdgcn_s_sleep(4);
    u32 hd;
    for (u32 n = 1;; ++n) {
      hd = __hip_atomic_load((const u32*)(S.mb + t * 32) + (lane & 7), __ATOMIC_RELAXED, __HIP_MEMORY_SCOPE_SYSTEM);
      if (hdr_there(hd, call, lane)) break;
      if ((n & 31) == 0 && spin_fail(myws, call, S.t0)) break;
      __builtin_amdgcn_s_sleep(4);
    }
    Pay<6> P;
    pay_load(P, S.l16, S.sb + (size_t)t * SLOT);
    pay_wait<0>(P);
    const u32 hdr = __builtin_amdgcn_readfirstlane(hd), base4 = (hdr & 0x7f) * 0x01010101u;
    u32x4 o[8];
#pragma unroll
    for (int hf = 0; hf < 2; ++hf) {
      u32 g[8];
#pragma unroll
      for (int j = 0; j < 4; ++j) {
        const u32 n = P.v[4 + hf][j];
        g[2 * j] = (n & 0x87878787u) + base4;
        g[2 * j + 1] = (__builtin_amdgcn_alignbit(n, n, 4) & 0x87878787u) + base4;
      }
      if (hdr & (0x400u | 0x3800u << (3 * hf))) repair(g, hd, hdr, hf, S.sb + (size_t)t * SLOT, S.l16);
#pragma unroll
      for (int j = 0; j < 4; ++j)
        o[4 * hf + j] = join(P.v[2 * hf + (j >> 1)][2 * (j & 1)], P.v[2 * hf + (j >> 1)][2 * (j & 1) + 1], g[2 * j], g[2 * j + 1]);
    }
    char* const ob = myws + (size_t)(uo + t) * SLOT + 4096;
    ST_SIX(S.l16, o, ob); ST16(S.l16, o[6], ob, 2048); ST16(S.l16, o[7], ob, 3072);
    asm volatile("s_waitcnt vmcnt(0)" ::: "memory");
    char* const mo = myws + meta_off((uo + t));
    if (lane < 8) ST4(l4, hdr_word(0, call, lane), mo);
  }
}

__device__ __forceinline__ float silu(float y) {
  const float z = __builtin_amdgcn_fmed3f(y, -60.0f, 60.0f);
  const float e = __builtin_amdgcn_exp2f(z * -1.44269504088896341f);
  return y * __builtin_amdgcn_rcpf(1.0f + e);
}

__device__ __forceinline__ void gemm_tile(KP p, char* smem, char* myws, int wv, int lane, u32 call, u32 nb, u32 sid0, u32 tn, u32 ch, u32 fl, int hold, u64 tk) {
  const u32 rank = p->rank;
  if (wv >= 8) __builtin_amdgcn_s_setprio(P_LPRIO);
  const u32 o0 = (rank & 4u) ^ 4u;
  const bool remote = tn >= 4;
  const u32 dst = (remote ? o0 + (tn - 4) : rank) & 7u;
  char* const ob = p->out[dst] + (remote ? (rank & 3) : tn) * 256 + ch * 128;
  u32* flag = (u32*)(p->ws[dst] + DONE_OFF) + (rank & 3) * 64 + fl;
  auto grow = [&](u32 rg) { return p->row[sid0 + (rg >> 2)] + (rg & 3) * 8; };
  char* b0 = ob + (size_t)grow(wv & 7) * (NLOC * 2);
  char* b1 = ob + (size_t)grow((wv & 7) + 8) * (NLOC * 2);
  if (hold == 2) {
    const u32 us = (S_PARK + (tn & 3)) * NSTEP + 2 * fl;
    b0 = myws + (size_t)us * SLOT + (wv & 7) * 1024; b1 = b0 + SLOT;
    flag = (u32*)(myws + meta_off(us));
  }
  asm volatile("" : "+s"(b0), "+s"(b1), "+s"(flag));
  if (nb == 0) {
    if (threadIdx.x == 0) ST4(0u, call, flag);
    return;
  }
  {
    const u32 first = sid0 < (u32)S_2 || sid0 >= (u32)S_OWN2 ? 0 : sid0 < (u32)S_INTER ? N1 * NSTEP : sid0 < (u32)S_3 ? 4 * NSTEP : N_INTRA - N3 * NSTEP;
    while (__builtin_amdgcn_s_memrealtime() - tk < first * P_UT / 10) __builtin_amdgcn_s_sleep(8);
  }
  Stream S;
  const u32 m = wv >> 1, kh = wv & 1;
  if (wv >= 8) {
    const u32 un = wv - 8;
    S.init(myws, call, S_IMG + 4 * tn + un, lane);
    const u32 lds0 = (u32)(size_t)(LAS char*)smem + un * 4096;
    const char* const g0 = S.sb - 4096 + ch * 4096;
    Stream A;
    A.init(myws, call, (call & 1) * NS_PAR + sid0 + un, lane);
    A.eager = sid0 >= (u32)S_3 && sid0 < (u32)S_OWN2;
    asm volatile("global_load_dwordx4 %0, %2, %3 sc0 sc1\n\tglobal_load_dwordx4 %1, %2, %4 sc0 sc1\n\ts_waitcnt vmcnt(0)"
                 : "+v"(A.W), "+v"(S.W) : "v"(S.l16), "s"(A.mb), "s"(S.mb) : "memory");
    int is = 0, seen = ((nb >> un) & 1) ? 0 : NSTEP;
    for (int t = -1; t < NSTEP; ++t) {
      if (seen < NSTEP && seen <= t + P_DIST + P_ALOOK) {
        A.look();
        A.spin(min(t + P_DIST, NSTEP - 1));
        if (A.rdy != seen) { *(LAS u32x4*)((LAS char*)smem + HDR_OFF + un * 1024 + A.l16) = A.W; seen = A.rdy; }
        asm volatile("" ::: "memory");
        if (seen < NSTEP) A.poll();
      }
      const int lim = min(min(NSTEP, is + 2), t + P_BD);
      if (S.rdy < min(NSTEP, lim + 2)) { S.look(); S.spin(t); if (S.rdy < NSTEP) S.poll(); }
      for (; is < min(lim, S.rdy); ++is) {
        const char* g = g0 + (size_t)is * SLOT;
        const u32 l = lds0 + (is & 7) * 16384;
        u32 keep;
        asm volatile("s_mov_b32 %0, m0\n\ts_mov_b32 m0, %3\n\ts_nop 0\n\t"
                     "global_load_lds_dwordx4 %1, %2 sc0 sc1\n\tglobal_load_lds_dwordx4 %1, %2 offset:1024 sc0 sc1\n\t"
                     "global_load_lds_dwordx4 %1, %2 offset:2048 sc0 sc1\n\tglobal_load_lds_dwordx4 %1, %2 offset:3072 sc0 sc1\n\t"
                     "s_mov_b32 m0, %0" : "=&s"(keep) : "v"(S.l16), "s"(g), "s"(l) : "memory");
      }
      const int ahead = is - 1 - t;
      if (ahead >= 6 || t < 0) asm volatile("s_waitcnt vmcnt(24)" ::: "memory");
      else if (ahead >= 4) { if (ahead == 5) asm volatile("s_waitcnt vmcnt(20)" ::: "memory"); else asm volatile("s_waitcnt vmcnt(16)" ::: "memory"); }
      else if (ahead >= 2) { if (ahead == 3) asm volatile("s_waitcnt vmcnt(12)" ::: "memory"); else asm volatile("s_waitcnt vmcnt(8)" ::: "memory"); }
      else if (ahead == 1) asm volatile("s_waitcnt vmcnt(4)" ::: "memory");
      else asm volatile("s_waitcnt vmcnt(0)" ::: "memory");
      WG_BARRIER();
    }
    asm volatile("" : "+v"(S.W), "+v"(A.W));
    __builtin_amdgcn_s_setprio(0);
    if (hold == 1 && wv == 8) {
      const u32* const r = (const u32*)(myws + REL_OFF + (BID & 7) * 128) + (lane < NCWG ? lane : 0);
      for (u32 n = 1; !__all(__hip_atomic_load(r, __ATOMIC_RELAXED, __HIP_MEMORY_SCOPE_AGENT) >= call); ++n) {
        if ((n & 31) == 0 && spin_fail(myws, call, S.t0)) break;
        __builtin_amdgcn_s_sleep(4);
      }
    }
  } else if (!((nb >> m) & 1)) {
    for (int t = -1; t < NSTEP; ++t) WG_BARRIER();
  } else {
    const char* const sb = myws + (size_t)((call & 1) * NS_PAR + sid0 + m) * NSTEP * SLOT + 4096;
    const u32 l16 = lane * 16, vlo = l16 + kh * 2048, vcode = l16 + kh * 1024;
    u32 ho = HDR_OFF + m * 1024 + (lane & 7) * 4, hd;
    const u32 rmask = 0x400u | 0x3800u << (3 * kh);
    f32x16 acc[2] = {};
    u32 bb = kh * 8192 + lane * 16;
    bf16x8 wf[4];
    u32x4 X[2][4];
    Pay<3> P[4];
    u32 h[8];
    u32 hdr = 0, base4 = 0;
    auto rd = [&](int stage, int f) {
      wf[f & 3] = *(const LAS bf16x8*)((LAS char*)smem + bb + stage * 16384 + (f >> 2) * 4096 + ((f >> 1) & 1) * 1024 + (f & 1) * 2048);
    };
    auto piece = [&](int k, int t, int ts, int waitn, bool fetch) {
      Pay<3>& Q = P[ts];
      if (k == 0) {
        if (waitn == 3) pay_wait<9>(Q); else if (waitn == 2) pay_wait<6>(Q); else if (waitn == 1) pay_wait<3>(Q); else pay_wait<0>(Q);
        hdr = __builtin_amdgcn_readfirstlane(hd);
        base4 = (hdr & 0x7f) * 0x01010101u;
      } else if (k < 6) {
#pragma unroll
        for (int j = 0; j < 4; ++j) {
          const int i = k < 4 ? 2 * j + (k & 1) : 4 * (k - 4) + j;
          if (k == 1) h[i] = __builtin_amdgcn_alignbit(Q.v[2][j], Q.v[2][j], 4);
          else if (k == 2) h[i] = Q.v[2][j] & 0x87878787u;
          else if (k == 3) h[i] &= 0x87878787u;
          else h[i] += base4;
          asm volatile("" : "+v"(h[i]));
        }
      } else if (k == 6) {
        if (hdr & rmask) repair(h, hd, hdr, kh, sb + (size_t)t * SLOT, l16);
      } else if (k == 7) {
      } else if (k < 12) {
        const int c = k - 8;
        X[ts & 1][c] = join(Q.v[c >> 1][2 * (c & 1)], Q.v[c >> 1][2 * (c & 1) + 1], h[2 * c], h[2 * c + 1]);
        asm volatile("" : "+v"(X[ts & 1][c]));
      } else if (k == 12) {
        ho += 32;
        hd = *(const LAS u32*)((LAS char*)smem + ho);
      } else if (k == 13 && fetch) {
        const int tn4 = t + P_DIST < NSTEP ? t + P_DIST : NSTEP - 1;
        pay_load(P[(ts + P_DIST) & 3], vlo, vcode, sb + (size_t)tn4 * SLOT);
      }
    };
    auto step = [&](int t, int ts, int waitn, bool fetch, bool more) {
#pragma unroll
      for (int q = 0; q < 8; ++q) {
        if (q == 4 && more) { WG_BARRIER(); if (ts == 3) bb ^= 65536u; }
        else if (q % 2 == 0 && (more || q < 6)) __builtin_amdgcn_s_waitcnt(0xC27F);
        __builtin_amdgcn_sched_barrier(0);
        acc[q & 1] = __builtin_amdgcn_mfma_f32_32x32x16_bf16(wf[q & 3], __builtin_bit_cast(bf16x8, X[ts & 1][q >> 1]), acc[q & 1], 0, 0, 0);
        __builtin_amdgcn_sched_barrier(0);
        if (q < 4) rd(ts, q + 4);
        else if (more) rd((ts + 1) & 3, q - 4);
        if (more) {
          piece(2 * q, t + 1, (ts + 1) & 3, waitn, fetch);
          piece(2 * q + 1, t + 1, (ts + 1) & 3, waitn, fetch);
        }
      }
    };
    WG_BARRIER();
#pragma unroll
    for (int d = 0; d < P_DIST; ++d) pay_load(P[d], vlo, vcode, sb + (size_t)d * SLOT);
    hd = *(const LAS u32*)((LAS char*)smem + ho);
#pragma unroll
    for (int k = 0; k < 13; ++k) piece(k, 0, 0, P_DIST - 1, true);
    WG_BARRIER();
    piece(13, 0, 0, P_DIST - 1, true);
#pragma unroll
    for (int f = 0; f < 4; ++f) { rd(0, f); __builtin_amdgcn_sched_barrier(0); }
    for (int tb = 0; tb < NSTEP - 4; tb += 4) {
#pragma unroll
      for (int ts = 0; ts < 4; ++ts) step(tb + ts, ts, P_DIST - 1, true, true);
    }
    step(NSTEP - 4, 0, P_DIST - 1, false, true);
    step(NSTEP - 3, 1, P_DIST - 2, false, true);
    step(NSTEP - 2, 2, P_DIST - 3, false, true);
    step(NSTEP - 1, 3, 0, false, false);
    char* const xq = smem + SCR_OFF + (kh * 128 + m * 32 + (lane & 31)) * ERS + (lane >> 5) * 16;
#pragma unroll
    for (int n = 0; n < 2; ++n)
#pragma unroll
      for (int g = 0; g < 4; ++g)
        *(f32x4*)(xq + n * 128 + g * 32) = f32x4{acc[n][4 * g], acc[n][4 * g + 1], acc[n][4 * g + 2], acc[n][4 * g + 3]};
    if (P_DIST == 4) pay_wait<0>(P[0]);
  }
  const u32 tmo = __hip_atomic_load((u32*)(myws + TMO_OFF), __ATOMIC_RELAXED, __HIP_MEMORY_SCOPE_AGENT);
  WG_BARRIER();
  if (wv < 8) {
    const u32 vo = hold == 2 ? lane * 16 : (lane >> 3) * (NLOC * 2) + (lane & 7) * 16;
    auto get = [&](u32 rg) {
      const char* const q = smem + SCR_OFF + (rg * 8 + (lane >> 3)) * ERS + (lane & 7) * 32;
      const f32x4 a = *(const f32x4*)q + *(const f32x4*)(q + 128 * ERS), c = *(const f32x4*)(q + 16) + *(const f32x4*)(q + 128 * ERS + 16);
      auto pk = [](float lo, float hi) { const f32x2 v = {silu(lo), silu(hi)}; return __builtin_bit_cast(u32, __builtin_convertvector(v, bf16x2)); };
      u32x4 v = {pk(a[0], a[1]), pk(a[2], a[3]), pk(c[0], c[1]), pk(c[2], c[3])};
      if (tmo == call) v = u32x4{0x7fc07fc0u, 0x7fc07fc0u, 0x7fc07fc0u, 0x7fc07fc0u};
      return v;
    };
    if ((nb >> (wv >> 2)) & 1) { const u32x4 v0 = get(wv); ST16(vo, v0, b0, 0); }
    if ((nb >> (wv >> 2)) & 4) { const u32x4 v1 = get(wv + 8); ST16(vo, v1, b1, 0); }
  }
  if (remote) asm volatile("s_waitcnt vmcnt(0)" ::: "memory");
  WG_BARRIER();
  if (remote) {
    if (threadIdx.x < (hold == 2 ? 2 : 1)) ST4(threadIdx.x * 32, call, flag);
  }
}

inline bool tile_of(u32 rank, u32 b, int k, u32& nb, u32& sid0, u32& tn, u32& ch, u32& fl, int& hold) {
  const u32 peer = peer_of_xcd(rank, b & 7), i = b >> 3;
  const bool home = !((peer ^ rank) & 4), spare = i >= 2 * RT_MED;
  hold = 0;
  constexpr u32 I0 = S_INTER / 2;
  const bool extra = home && i >= I0 && i < I0 + (NB + 1) / 2 * 2;
  if (extra && k == 0) {
    const u32 j = i - I0;
    nb = (j | 1) < (u32)NB ? 3 : 1; tn = peer & 3; sid0 = S_OWN2 + (j >> 1) * 2; ch = j & 1; fl = 0;
    return true;
  }
  if (k == (extra ? 1 : 0) && !spare) {
    nb = 15; tn = (peer & 3) + (home ? 0 : 4); sid0 = (i >> 1) * 4; ch = i & 1; fl = i;
    hold = home || sid0 >= (u32)S_INTER ? 0 : i < (u32)P_PARK ? 2 : i >= 10 ? 1 : 0;
    if (!home)
      for (u32 m = 0; m < 4; ++m)
        if (sid0 + m >= (u32)S_INTER && (sid0 + m - S_INTER) / NB == (peer & 3)) nb &= ~(1u << m);
    return true;
  }
  const u32 j = spare ? i - 2 * RT_MED + 10 : i;
  nb = 3; tn = (peer & 3) + (home ? 4 : 0); sid0 = RT_MED * 4 + (j >> 1) * 2; ch = j & 1; fl = 2 * RT_MED + j;
  return spare ? k == 0 : i < 10;
}

constexpr u32 NHOME = 32 * NSTEP;
__host__ __device__ inline u32 pack_index(u32 b, u32 wv) {
  constexpr u32 NP1 = 80;
  return b < NP1 ? (wv < 2 ? b + NP1 * wv : NHOME) : 2 * NP1 + (b - NP1) + (NGEMM - NP1) * wv;
}

constexpr u32 UB0 = RT_MED * 16, UNW = 16;
__host__ __device__ inline u32 unpack_index(u32 b, u32 wv) {
  return b >= UB0 && b < UB0 + UNW ? (11 - wv) * UNW + (b - UB0) : 512u;
}

__global__ void __launch_bounds__(NTHREAD, 3) ag_gemm_silu(const Params) {
  KP p = (KP)__builtin_amdgcn_kernarg_segment_ptr();
  __shared__ __attribute__((aligned(16))) char smem[LDS_BYTES];
  const int lane = threadIdx.x & 63;
  const int wv = __builtin_amdgcn_readfirstlane(threadIdx.x >> 6);
  char* const myws = p->myws;
  if (BID >= NGEMM) { sender(p, smem, myws, wv, lane); return; }
  u32 call;
  const u64 tk = __builtin_amdgcn_s_memrealtime();
  const u32 u = pack_index(BID, wv);
  if (u < NHOME) {
    const u32 t = u >> 5, c = u & 31;
    const bool isw = c >= 16;
    const u32 cw = c - 16, l16 = lane * 16;
    const u64 t0 = __builtin_amdgcn_s_memrealtime();
    while (__builtin_amdgcn_s_memrealtime() - t0 < t * P_PP) __builtin_amdgcn_s_sleep(2);
    u32x4 d[8], s[8];
    load_raw(p, d, lane, isw ? 16 + (cw & 3) : c, (cw >> 2) & 3, t);
    u32 ticket;
    count_read(ticket, myws);
    WAIT_RAW(d);
    call = count_take(ticket);
    u32 meta = 0;
    bool eight = true;
    to_fragments(d, isw, smem + SCR_OFF + wv * 8192, lane);
    if (isw) {
#pragma unroll
      for (int j = 0; j < 8; ++j) s[j] = d[j];
    } else { low_slabs(d, s); eight = encode(d, s, meta, lane); }
    const u32 ud = (isw ? S_IMG + cw : (call & 1) * NS_PAR + (c < (u32)NA ? c : S_OWN2 - NA + c)) * NSTEP + t;
    char* const b = myws + (size_t)ud * SLOT + 4096;
    ST_SIX(l16, s, b);
    if (eight) { ST16(l16, s[6], b, 2048); ST16(l16, s[7], b, 3072); }
    asm volatile("s_waitcnt vmcnt(0)" ::: "memory");
    char* const m = myws + meta_off(ud);
    if (lane < 8) ST4(lane * 4, hdr_word(meta, call, lane), m);
  } else call = draw_call(myws);
#pragma nounroll
  for (u32 v = unpack_index(BID, wv); v < 512; v += 12 * UNW) unpack_unit(myws, lane, call, (v + P_UROT(v >> 4)) & 15, v >> 4, tk);
  WG_BARRIER();
  count_bump(myws, call);
#pragma nounroll
  for (int k = 0; k < 2; ++k) {
    const u32 td = p->tile[BID][k];
    if (!(td >> 23)) break;
    gemm_tile(p, smem, myws, wv, lane, call, td & 15, (td >> 4) & 127, (td >> 11) & 7, (td >> 14) & 1, (td >> 15) & 63, (td >> 21) & 3, tk);
  }
}

static void build_orders(Params& p) {
  struct E { double tau; unsigned short code; };
  E e[N_ORD];
  double sigma = P_SIGMA, ovl = P_OVL, ovl2 = P_OVL2, lead = P_LEAD, f1 = 14.5, f2 = 6, stg = P_STG;
  for (int kind = 0; kind < 2; ++kind) {
    int n = 0;
    for (int t = 0; t < NSTEP; ++t) {
      if (kind == 0) {
        for (int b = 0; b < NA; ++b) if (b != RB || t < RT0) e[n++] = {t + (b < N1 ? 0 : b < N1 + N2 ? NSTEP - ovl : 2 * NSTEP - ovl - ovl2 + (b >= N1 + N2 + 2 ? stg : 0)), (unsigned short)(b << 5 | t)};
      } else {
        for (int i = 0; i < 4; ++i) e[n++] = {t * sigma - lead, (unsigned short)((16 + i) << 5 | t)};
        for (int b = NA; b < 16; ++b) e[n++] = {t + 0.5, (unsigned short)(b << 5 | t)};
      }
    }
    if (kind == 1)
      for (int j = 0; j < P_RLY; ++j) {
        e[n++] = {f1 * sigma - lead, (unsigned short)(20 << 5 | j)};
        for (int i = 0; i < 3; ++i) e[n++] = {f2 + j, (unsigned short)((21 + i) << 5 | j)};
      }
    std::stable_sort(e, e + n, [](const E& a, const E& b) { return a.tau < b.tau; });
    for (int i = 0; i < N_ORD; ++i) p.ord[kind][i] = i < n ? e[i].code : 0;
  }
}

extern "C" void kernel_launch(int rank, int nranks, ncclComm_t comm, hipStream_t stream,
                              void* const* d_in, const int* in_sizes, int n_in,
                              void* d_out, int out_size, const int* in_dtypes, int out_dtype,
                              void* const* d_out_all, void* const* d_ws_all, size_t ws_bytes) {
  (void)comm; (void)in_dtypes; (void)out_dtype; (void)d_out;
  if (nranks != NR || (unsigned)rank >= (unsigned)NR || n_in != 2 || ws_bytes < WS_NEED ||
      in_sizes[0] != 512 * KDIM || in_sizes[1] != KDIM * NLOC || out_size != 4096 * NLOC) {
    fprintf(stderr, "unexpected problem shape\n");
    return;
  }
  Params p = {};
  p.x = (const char*)d_in[0];
  p.w = (const char*)d_in[1];
  for (int i = 0; i < NR; ++i) {
    p.out[i] = (char*)d_out_all[phys_rank(i)]; p.ws[i] = (char*)d_ws_all[phys_rank(i)];
    if ((int)phys_rank(i) == rank) p.rank = i;
  }
  p.myws = (char*)d_ws_all[rank];
  build_orders(p);
  for (int i = 0; i < NS_PAR + 4; ++i) p.row[i] = i < 64 || (i >= S_OWN2 && i < NS_PAR) ? global_row(p.rank, i) : 0;
  for (u32 b = 0; b < (u32)NGEMM; ++b)
    for (int k = 0; k < 2; ++k) {
      u32 nb, sid0, tn, ch, fl; int hold;
      p.tile[b][k] = tile_of(p.rank, b, k, nb, sid0, tn, ch, fl, hold) ? nb | sid0 << 4 | tn << 11 | ch << 14 | fl << 15 | (u32)hold << 21 | 1u << 23 : 0u;
    }
  hipLaunchKernelGGL(ag_gemm_silu, dim3(GRID), dim3(NTHREAD), 0, stream, p);
  hipError_t err = hipGetLastError();
  if (err != hipSuccess) fprintf(stderr, "launch failed: %s\n", hipGetErrorString(err));
}
```

```cpp
#include <hip/hip_runtime.h>
#include <rccl/rccl.h>
#include <cstdio>
#include <cstdint>
#include <cstdlib>
#include <algorithm>

typedef unsigned u32;
typedef unsigned long long u64;
typedef unsigned u32x4 __attribute__((ext_vector_type(4)));
typedef unsigned u32x8 __attribute__((ext_vector_type(8)));
typedef unsigned short u16x2 __attribute__((ext_vector_type(2)));
typedef __bf16 bf16x2 __attribute__((ext_vector_type(2)));
typedef __bf16 bf16x4 __attribute__((ext_vector_type(4)));
typedef __bf16 bf16x8 __attribute__((ext_vector_type(8)));
typedef float f32x2 __attribute__((ext_vector_type(2)));
typedef float f32x4 __attribute__((ext_vector_type(4)));
typedef float f32x16 __attribute__((ext_vector_type(16)));
#define LAS __attribute__((address_space(3)))

#define P_NA 11
#define P_NCWG 4
#define P_WIN 3
#define P_WIN0 4
#define P_TAPER 8
#define P_WIN1 5
#define P_DIST 3
#define P_BD 7
#define P_ALOOK 3
#define P_LPRIO 2
#define P_UROT(t) (5 * (t))
#define P_STG 3
#define P_SIGMA 0.1
#define P_OVL 0.0
#define P_OVL2 0.0
#define P_LEAD 0.0
#define P_PP 15
#define P_PARK 10
#define P_RLY 2
#define P_S0 16
#define P_S1 16
#define P_S2 4
#define P_UT 100
#define P_REL 2

constexpr int NR = 8;
constexpr int KDIM = 4096, NLOC = 512;
constexpr int NSTEP = 32;
constexpr int NA = P_NA, NB = 16 - NA;
constexpr int N1 = 3, N3 = 4, N2 = NA - N1 - N3;
constexpr int S_1 = NA, S_2 = S_1 + 3 * N1, S_INTER = S_2 + 3 * N2, S_3 = 64 - 3 * N3, S_WIN = 64, S_RLY = 80, S_OWN2 = 81;
constexpr int RB = N1 + N2 - 1, RT0 = NSTEP - 4 * P_RLY;
constexpr int NS_PAR = S_OWN2 + NB, S_IMG = 2 * NS_PAR, S_PARK = S_IMG + 32, NSTREAM = S_PARK + 4;
constexpr size_t SLOT = 8192;
constexpr size_t META_OFF = (size_t)NSTREAM * NSTEP * SLOT;
constexpr size_t DONE_OFF = META_OFF + (size_t)NSTREAM * NSTEP * 32;
constexpr size_t REL_OFF = DONE_OFF + 1024;
constexpr size_t CNT_OFF = REL_OFF + 1024;
constexpr size_t TMO_OFF = CNT_OFF + 256 * 128;
constexpr size_t WS_NEED = TMO_OFF + 128;
constexpr int N_INTRA = NA * NSTEP - 4 * P_RLY, N_INTER = NB * NSTEP + 4 * NSTEP + 4 * P_RLY;
constexpr int N_ORD = N_INTRA > N_INTER ? N_INTRA : N_INTER;
constexpr int RT_MED = 13;
constexpr int NGEMM = RT_MED * 16 + 16;
constexpr int NFLAG = 2 * RT_MED + 12 + P_PARK;
constexpr int NCWG = P_NCWG;
constexpr int GRID = NGEMM + 8 * NCWG;
static_assert(P_PARK == 10 && P_BD <= 7 && P_RLY <= 8 && N_ORD % 2 == 0 && NB > 4 && NB <= 8 && S_INTER % 4 == 0, "tile map");
static_assert(GRID <= 256, "one workgroup per CU, all resident");
constexpr int NTHREAD = 768;
constexpr int SCR_OFF = 0, EPI_OFF = 32768;
constexpr int HDR_OFF = 131072;
constexpr int LDS_BYTES = HDR_OFF + 4096;
constexpr u64 DEADLINE_TICKS = 80000000ull;
constexpr int ERS = 272;

#define P_PHYS 0x75416320u
__host__ __device__ inline u32 phys_rank(u32 l) { return (P_PHYS >> (4 * l)) & 7u; }
__host__ __device__ inline u32 peer_of_xcd(u32 rank, u32 x) { return (((rank ^ x) & 4u) | ((rank + x) & 3u)) & 7u; }

struct Params {
  const char* x;
  const char* w;
  char* myws;
  char* out[NR];
  char* ws[NR];
  int rank;
  u32 row[NS_PAR + 4];
  u32 tile[NGEMM][2];
  alignas(4) unsigned short ord[2][N_ORD];
};
static_assert(sizeof(Params) <= 4096, "kernel argument");
#define BID ((blockIdx.x + NGEMM) % GRID)
typedef const __attribute__((address_space(4))) Params* KP;

#define WG_BARRIER() do { asm volatile("" ::: "memory"); __builtin_amdgcn_s_waitcnt(0xC07F); \
    asm volatile("s_barrier" ::: "memory"); } while (0)
#define ST16(voff, data, sbase, imm) \
  asm volatile("global_store_dwordx4 %0, %1, %2 offset:" #imm " sc0 sc1\n\ts_nop 1" :: "v"(voff), "v"(data), "s"(sbase) : "memory")
#define ST4(voff, data, sbase) \
  asm volatile("global_store_dword %0, %1, %2 sc0 sc1" :: "v"(voff), "v"(data), "s"(sbase) : "memory")
#define ST_SIX(voff, s, b) do { ST16(voff, s[0], b, -4096); ST16(voff, s[1], b, -3072); ST16(voff, s[2], b, -2048); \
    ST16(voff, s[3], b, -1024); ST16(voff, s[4], b, 0); ST16(voff, s[5], b, 1024); } while (0)

__device__ __forceinline__ void count_read(u32& v, char* myws) {
  const char* const c = myws + CNT_OFF + BID * 128;
  asm volatile("global_load_dword %0, %1, %2 sc1" : "=&v"(v) : "v"(0u), "s"(c) : "memory");
}
__device__ __forceinline__ u32 count_take(u32& v) {
  asm volatile("" : "+v"(v));
  return __builtin_amdgcn_readfirstlane(v) + 1u;
}
__device__ __forceinline__ void count_bump(char* myws, u32 call) {
  if (threadIdx.x == 0) __hip_atomic_store((u32*)(myws + CNT_OFF + BID * 128), call, __ATOMIC_RELAXED, __HIP_MEMORY_SCOPE_AGENT);
}
__device__ __forceinline__ u32 draw_call(char* myws) {
  u32 v;
  count_read(v, myws);
  asm volatile("s_waitcnt vmcnt(0)" : "+v"(v) :: "memory");
  return count_take(v);
}
__device__ __forceinline__ u32 call_tag(u32 call) { return (call % 2047u + 1u) << 21; }
__device__ __forceinline__ u32 hdr_word(u32 data, u32 call, int lane) { return lane == 7 ? call : data | call_tag(call); }
__device__ __forceinline__ bool hdr_there(u32 word, u32 call, int lane) {
  return __all((lane & 7) == 7 ? word >= call : (word ^ call_tag(call)) < (1u << 21));
}

__device__ __forceinline__ bool spin_fail(char* myws, u32 call, u64 t0) {
  u32* tmo = (u32*)(myws + TMO_OFF);
  u32 t = __builtin_amdgcn_readfirstlane(__hip_atomic_load(tmo, __ATOMIC_RELAXED, __HIP_MEMORY_SCOPE_AGENT));
  if (t == call || __builtin_amdgcn_s_memrealtime() - t0 > DEADLINE_TICKS) {
    __hip_atomic_store(tmo, call, __ATOMIC_RELAXED, __HIP_MEMORY_SCOPE_AGENT);
    return true;
  }
  return false;
}

__device__ __forceinline__ size_t meta_off(u32 unit) { return META_OFF + (size_t)unit * 32; }
__device__ __forceinline__ u32 dest_stream(u32 sl, u32 rank, u32 dest, u32 par) {
  const u32 j = ((rank - dest) & 3) - 1;
  const u32 s = sl >= 16 ? S_WIN + (rank & 3) * 4 + (sl - 16) : sl >= (u32)NA ? S_INTER + (rank & 3) * NB + (sl - NA)
              : sl >= (u32)(N1 + N2) ? S_3 + j * N3 + (sl - N1 - N2) : sl >= (u32)N1 ? S_2 + j * N2 + (sl - N1) : S_1 + j * N1 + sl;
  return par * NS_PAR + s;
}

inline u32 global_row(u32 rank, u32 sid) {
  const u32 g0 = rank & 4u, o0 = g0 ^ 4u;
  u32 src, bi;
  if (sid < (u32)S_1) { src = rank; bi = sid; }
  else if (sid < (u32)S_2) { const u32 q = sid - S_1; src = g0 + ((rank + 1 + q / N1) & 3); bi = q % N1; }
  else if (sid < (u32)S_INTER) { const u32 q = sid - S_2; src = g0 + ((rank + 1 + q / N2) & 3); bi = N1 + q % N2; }
  else if (sid < (u32)S_3) { const u32 q = sid - S_INTER; src = o0 + q / NB; bi = NA + q % NB; }
  else if (sid >= (u32)S_OWN2) { src = rank; bi = NA + sid - S_OWN2; }
  else { const u32 q = sid - S_3; src = g0 + ((rank + 1 + q / N3) & 3); bi = N1 + N2 + q % N3; }
  return (phys_rank(src & 7u) * 512 + bi * 32) & 4095u;
}

__device__ __forceinline__ void load_raw(KP p, u32x4 (&d)[8], int lane, u32 sl, u32 q, u32 t) {
  const bool isw = sl >= 16;
  const u32 rs = isw ? NLOC * 2 : KDIM * 2;
  const u32 vo = (lane >> 4) * rs + (lane & 15) * 16;
  const u64 aw = (u64)p->w + (u64)((t * 128 + (sl & 3) * 32) * (NLOC * 2) + q * 256), ax = (u64)p->x + (u64)(sl * 32 * (KDIM * 2) + t * 256);
  const u64 b0 = isw ? aw : ax;
#pragma unroll
  for (int j = 0; j < 8; ++j)
    asm volatile("global_load_dwordx4 %0, %1, %2" : "=&v"(d[j]) : "v"(vo), "s"(b0 + (u64)(j * 4 * rs)) : "memory");
}
#define WAIT_RAW(d) asm volatile("s_waitcnt vmcnt(0)" : "+v"(d[0]), "+v"(d[1]), "+v"(d[2]), "+v"(d[3]), "+v"(d[4]), "+v"(d[5]), "+v"(d[6]), "+v"(d[7]) :: "memory")
__device__ __forceinline__ void to_fragments(u32x4 (&d)[8], bool isw, char* scr, int lane) {
  LAS char* const sc = (LAS char*)scr;
  auto off = [](u32 row, u32 ch) { return 256u * row + 16u * (ch ^ (((row & 3) << 2) | ((row >> 2) & 3))); };
#pragma unroll
  for (int j = 0; j < 8; ++j) *(LAS u32x4*)(sc + off(4 * j + (lane >> 4), lane & 15)) = d[j];
  asm volatile("" ::: "memory");
  if (isw) {
    const u32 r0 = 8 * (lane >> 5) + ((lane & 15) >> 2), c0 = 2 * ((lane >> 4) & 1) + ((lane & 3) >> 1);
#pragma unroll
    for (int j = 0; j < 8; ++j) {
      const bf16x4 lo = __builtin_amdgcn_ds_read_tr16_b64_v4bf16((LAS bf16x4*)(sc + off(16 * (j & 1) + r0, 4 * (j >> 1) + c0) + 8 * (lane & 1)));
      const bf16x4 hi = __builtin_amdgcn_ds_read_tr16_b64_v4bf16((LAS bf16x4*)(sc + off(16 * (j & 1) + r0 + 4, 4 * (j >> 1) + c0) + 8 * (lane & 1)));
      d[j] = __builtin_bit_cast(u32x4, __builtin_shufflevector(lo, hi, 0, 1, 2, 3, 4, 5, 6, 7));
    }
  } else {
#pragma unroll
    for (int j = 0; j < 8; ++j) d[j] = *(const LAS u32x4*)(sc + off(lane & 31, 2 * j + (lane >> 5)));
  }
  asm volatile("" ::: "memory");
}
__device__ __forceinline__ void low_slabs(const u32x4 (&d)[8], u32x4 (&s)[8]) {
#pragma unroll
  for (int i = 0; i < 16; ++i) s[i >> 2][i & 3] = __builtin_amdgcn_perm(d[i >> 1][2 * (i & 1) + 1], d[i >> 1][2 * (i & 1)], 0x06040200u);
}
__device__ __forceinline__ bool encode(const u32x4 (&d)[8], u32x4 (&s)[8], u32& meta, int lane) {
  u32 H[16], T[16];
  u16x2 m1 = {0, 0}, m2 = {0, 0}, m3 = {0, 0}, m4 = {0, 0};
#pragma unroll
  for (int i = 0; i < 16; ++i) {
    const u32 a = d[i >> 1][2 * (i & 1)], b = d[i >> 1][2 * (i & 1) + 1];
    H[i] = __builtin_amdgcn_perm(b, a, 0x07050301u);
  }
#pragma unroll
  for (int i = 0; i < 16; i += 2) {
    T[i] = H[i] | 0x80808080u;
    T[i + 1] = H[i + 1] | 0x80808080u;
    m1 = __builtin_elementwise_max(m1, __builtin_bit_cast(u16x2, T[i]));
    m2 = __builtin_elementwise_max(m2, __builtin_bit_cast(u16x2, T[i] << 8));
    m3 = __builtin_elementwise_max(m3, __builtin_bit_cast(u16x2, T[i + 1]));
    m4 = __builtin_elementwise_max(m4, __builtin_bit_cast(u16x2, T[i + 1] << 8));
  }
  m1 = __builtin_elementwise_max(__builtin_elementwise_max(m1, m2), __builtin_elementwise_max(m3, m4));
  u32 mx = (u32)(m1.x > m1.y ? m1.x : m1.y);
  mx = max(mx, (u32)__builtin_amdgcn_update_dpp(0, mx, 0x111, 0xf, 0xf, true));
  mx = max(mx, (u32)__builtin_amdgcn_update_dpp(0, mx, 0x112, 0xf, 0xf, true));
  mx = max(mx, (u32)__builtin_amdgcn_update_dpp(0, mx, 0x114, 0xf, 0xf, true));
  mx = max(mx, (u32)__builtin_amdgcn_update_dpp(0, mx, 0x118, 0xf, 0xf, true));
  const u32 top = max(max((u32)__builtin_amdgcn_readlane(mx, 15), (u32)__builtin_amdgcn_readlane(mx, 31)),
                      max((u32)__builtin_amdgcn_readlane(mx, 47), (u32)__builtin_amdgcn_readlane(mx, 63)));
  const u32 h7 = (top >> 8) & 0x7f;
  const u32 base = __builtin_amdgcn_readfirstlane(h7 > 7 ? h7 - 7 : 0);
  const u32 base4 = base * 0x01010101u;
  u32 ok = 0xffffffffu;
  u32 N[8];
#pragma unroll
  for (int j = 0; j < 8; ++j) {
    const u32 ta = T[2 * j] - base4, tb = T[2 * j + 1] - base4;
    ok &= ta & tb;
    const u32 ca = (ta & 0x07070707u) | (H[2 * j] & 0x80808080u);
    const u32 cb = (tb & 0x07070707u) | (H[2 * j + 1] & 0x80808080u);
    N[j] = ca | __builtin_amdgcn_alignbit(cb, cb, 28);
  }
  meta = 0;
  int c0 = 0, c1 = 0;
  if (__any((~ok & 0x80808080u) != 0)) {
    u32 M0 = 0, M1 = 0;
#pragma unroll
    for (int i = 0; i < 8; ++i) {
      M0 |= ~((T[i] - base4) >> i) & (0x80808080u >> i);
      M1 |= ~((T[8 + i] - base4) >> i) & (0x80808080u >> i);
    }
    u64 bal = __ballot((M0 | M1) != 0);
    while (bal != 0 && c0 + c1 <= 6) {
      const int ln = __builtin_ctzll(bal);
      bal &= bal - 1;
      u64 mm = (u64)(u32)__builtin_amdgcn_readlane(M0, ln) | (u64)(u32)__builtin_amdgcn_readlane(M1, ln) << 32;
      while (mm != 0) {
        const u32 bit = __builtin_ctzll(mm), hf = bit >> 5, by = (bit >> 3) & 3, i = hf * 8 + 7 - (bit & 7);
        mm &= mm - 1;
        c0 += hf ^ 1; c1 += hf;
        u32 hw = 0;
#pragma unroll
        for (int z = 0; z < 16; ++z) hw |= H[z] & (u32)__builtin_amdgcn_sbfe(1 << i, z, 1);
        const u32 hb = ((u32)__builtin_amdgcn_readlane(hw, ln) >> (8 * by)) & 0x7f;
        const u32 ent = (u32)ln | (i & 7) << 6 | by << 9 | ((((hb - base) & 7) + base) ^ hb) << 11;
        if (lane == (hf ? 7 - c1 : c0)) meta = ent;
      }
    }
  }
  const bool raw = c0 + c1 > 6;
  if (lane == 0) meta = base | (raw ? 0x400u : (u32)(c0 << 11 | c1 << 14));
  s[4] = raw ? u32x4{H[0], H[1], H[2], H[3]} : u32x4{N[0], N[1], N[2], N[3]};
  s[5] = raw ? u32x4{H[4], H[5], H[6], H[7]} : u32x4{N[4], N[5], N[6], N[7]};
  s[6] = u32x4{H[8], H[9], H[10], H[11]};
  s[7] = u32x4{H[12], H[13], H[14], H[15]};
#pragma unroll
  for (int j = 0; j < 8; ++j) asm volatile("" : "+v"(s[j]));
  return raw;
}

template <int N> struct Pay { u32x4 v[N]; };
__device__ __forceinline__ void pay_load(Pay<6>& P, u32 l16, const char* sb) {
  asm volatile("global_load_dwordx4 %0, %6, %7 offset:-4096 sc0 sc1\n\t"
               "global_load_dwordx4 %1, %6, %7 offset:-3072 sc0 sc1\n\t"
               "global_load_dwordx4 %2, %6, %7 offset:-2048 sc0 sc1\n\t"
               "global_load_dwordx4 %3, %6, %7 offset:-1024 sc0 sc1\n\t"
               "global_load_dwordx4 %4, %6, %7 offset:0 sc0 sc1\n\t"
               "global_load_dwordx4 %5, %6, %7 offset:1024 sc0 sc1"
               : "=&v"(P.v[0]), "=&v"(P.v[1]), "=&v"(P.v[2]), "=&v"(P.v[3]), "=&v"(P.v[4]), "=&v"(P.v[5]) : "v"(l16), "s"(sb) : "memory");
}
__device__ __forceinline__ void pay_load(Pay<3>& P, u32 vlo, u32 vcode, const char* sb) {
  asm volatile("global_load_dwordx4 %2, %4, %5 offset:0 sc0 sc1\n\t"
               "global_load_dwordx4 %0, %3, %5 offset:-4096 sc0 sc1\n\t"
               "global_load_dwordx4 %1, %3, %5 offset:-3072 sc0 sc1"
               : "=&v"(P.v[0]), "=&v"(P.v[1]), "=&v"(P.v[2]) : "v"(vlo), "v"(vcode), "s"(sb) : "memory");
}
template <int C> __device__ __forceinline__ void pay_wait(Pay<6>& P) {
  asm volatile("s_waitcnt vmcnt(%6)" : "+v"(P.v[0]), "+v"(P.v[1]), "+v"(P.v[2]), "+v"(P.v[3]), "+v"(P.v[4]), "+v"(P.v[5]) : "n"(C) : "memory");
}
template <int C> __device__ __forceinline__ void pay_wait(Pay<3>& P) {
  asm volatile("s_waitcnt vmcnt(%3)" : "+v"(P.v[0]), "+v"(P.v[1]), "+v"(P.v[2]) : "n"(C) : "memory");
}
__device__ __forceinline__ u32x8 load_two(const char* b, u32 l16) {
  u32x4 x, y;
  asm volatile("global_load_dwordx4 %0, %2, %3 sc0 sc1\n\t"
               "global_load_dwordx4 %1, %2, %3 offset:1024 sc0 sc1\n\ts_waitcnt vmcnt(0)"
               : "=&v"(x), "=&v"(y) : "v"(l16), "s"(b) : "memory");
  return __builtin_shufflevector(x, y, 0, 1, 2, 3, 4, 5, 6, 7);
}

__device__ __forceinline__ void load_slot(u32x4 (&d)[8], u32 l16, const char* sb) {
  asm volatile("global_load_dwordx4 %0, %8, %9 offset:-4096 sc0 sc1\n\tglobal_load_dwordx4 %1, %8, %9 offset:-3072 sc0 sc1\n\t"
               "global_load_dwordx4 %2, %8, %9 offset:-2048 sc0 sc1\n\tglobal_load_dwordx4 %3, %8, %9 offset:-1024 sc0 sc1\n\t"
               "global_load_dwordx4 %4, %8, %9 offset:0 sc0 sc1\n\tglobal_load_dwordx4 %5, %8, %9 offset:1024 sc0 sc1\n\t"
               "global_load_dwordx4 %6, %8, %9 offset:2048 sc0 sc1\n\tglobal_load_dwordx4 %7, %8, %9 offset:3072 sc0 sc1\n\ts_waitcnt vmcnt(0)"
               : "=&v"(d[0]), "=&v"(d[1]), "=&v"(d[2]), "=&v"(d[3]), "=&v"(d[4]), "=&v"(d[5]), "=&v"(d[6]), "=&v"(d[7])
               : "v"(l16), "s"(sb) : "memory");
}

__device__ __forceinline__ void sender(KP p, char* smem, char* myws, int wv, int lane) {
  const u32 rank = p->rank;
  const u32 cb = BID - NGEMM;
  const u32 link = cb & 7, cw = cb >> 3;
  const u32 l16 = lane * 16;
  if (link != 0) {
    const u32 dest = peer_of_xcd(rank, link);
    const bool intra = link < 4;
    const int n = ((intra ? N_INTRA : N_INTER) - (int)cw + NCWG - 1) / NCWG;
    auto code_of = [=](int k) {
      const u32 e = cw + (u32)(k < n ? k : 0) * NCWG;
      const u32 word = ((const __attribute__((address_space(4))) u32*)p->ord[intra ? 0 : 1])[e >> 1];
      return (word >> (16 * (e & 1))) & 0xffffu;
    };
    LAS int* const done = (LAS int*)(smem + 12 * 8192);
    u32x4 d[8];
    auto fetch = [&](u32 c) {
      const bool rly = c >= 20 * 32;
      load_raw(p, d, lane, rly ? RB : c >> 5, dest & 3, rly ? RT0 + 4 * (c & 7) + (dest & 3) : c & 31);
    };
    u32 code = code_of(wv), ticket;
    if (wv >= 4) {
      if (threadIdx.x == 256) *done = 0;
      WG_BARRIER();
      if (wv >= 8) __builtin_amdgcn_s_sleep(16);
    }
    fetch(code);
    count_read(ticket, myws);
    if (wv < 4) WG_BARRIER();
    u32 call = 0;
    char* const rws = p->ws[dest];
    if (wv < 4) __builtin_amdgcn_s_setprio(2); else if (wv < 8) __builtin_amdgcn_s_setprio(1);
    int k = wv;
    for (; k < n; k += 12) {
      const u32 sl = code >> 5;
      u32 ud = sl < 20 ? dest_stream(sl, rank, dest, 0) * NSTEP + (code & 31) : sl == 20 ? S_RLY * NSTEP + (rank & 3) * 8 + (code & 7)
             : (S_2 + (sl - 21) * N2 + RB - N1) * NSTEP + RT0 + 4 * (code & 7) + (rank & 3);
      asm volatile("" : "+s"(ud));
      WAIT_RAW(d);
      if (k < 12) call = count_take(ticket);
      ud += (call & 1) * (NS_PAR * NSTEP);
      char* const b = rws + (size_t)ud * SLOT + 4096;
      u32x4 s[8];
      u32 meta;
      bool raw;
      if (P_RLY > 0 && sl > 20) {
        const u32 us = ((call & 1) * NS_PAR + S_RLY) * NSTEP + ((dest + sl - 20) & 3) * 8 + (code & 7);
        const u64 t0 = __builtin_amdgcn_s_memrealtime();
        const u32* const hd = (const u32*)(myws + meta_off(us)) + (lane & 7);
        for (u32 it = 1;; ++it) {
          meta = __hip_atomic_load(hd, __ATOMIC_RELAXED, __HIP_MEMORY_SCOPE_SYSTEM);
          if (hdr_there(meta, call, lane)) break;
          if ((it & 31) == 0 && spin_fail(myws, call, t0)) break;
          __builtin_amdgcn_s_sleep(2);
        }
        load_slot(s, l16, myws + (size_t)us * SLOT + 4096);
        raw = __builtin_amdgcn_readfirstlane(meta) & 0x400;
      } else {
        to_fragments(d, sl >= 16 && sl < 20, smem + wv * 8192, lane);
        low_slabs(d, s);
        if (k < P_WIN0) { ST16(l16, s[0], b, -4096); ST16(l16, s[1], b, -3072); ST16(l16, s[2], b, -2048); ST16(l16, s[3], b, -1024); }
        raw = encode(d, s, meta, lane);
        meta = hdr_word(meta, call, lane);
      }
      code = code_of(k + 12);
      __builtin_amdgcn_s_setprio(0);
      const int need = k < P_WIN0 ? 0 : k + 1 - max(P_WIN, P_WIN1 - (k - P_WIN0) / P_TAPER);
      while (__hip_atomic_load(done, __ATOMIC_RELAXED, __HIP_MEMORY_SCOPE_WORKGROUP) < need) __builtin_amdgcn_s_sleep(1);
      if (k >= P_WIN0) { ST16(l16, s[0], b, -4096); ST16(l16, s[1], b, -3072); ST16(l16, s[2], b, -2048); ST16(l16, s[3], b, -1024); }
      ST16(l16, s[4], b, 0); ST16(l16, s[5], b, 1024);
      if (raw) { ST16(l16, s[6], b, 2048); ST16(l16, s[7], b, 3072); }
      asm volatile("s_waitcnt vmcnt(0)" ::: "memory");
      char* const m = rws + meta_off(ud);
      if (lane < 8) ST4(lane * 4, meta, m);
      if (lane == 0) __hip_atomic_fetch_add(done, 1, __ATOMIC_RELAXED, __HIP_MEMORY_SCOPE_WORKGROUP);
      fetch(code);
    }
    WAIT_RAW(d);
    count_bump(myws, call);
    const u32 y = cw + (u32)(k - n) * NCWG;
    if (!intra && y < 2u * P_PARK) {
      const u32 us = (S_PARK + (dest & 3)) * NSTEP + y;
      const u64 t0 = __builtin_amdgcn_s_memrealtime();
      const u32* const hd = (const u32*)(myws + meta_off(us));
      for (u32 it = 1; (u32)__builtin_amdgcn_readfirstlane(__hip_atomic_load(hd, __ATOMIC_RELAXED, __HIP_MEMORY_SCOPE_SYSTEM)) < call; ++it) {
        if ((it & 31) == 0 && spin_fail(myws, call, t0)) break;
        __builtin_amdgcn_s_sleep(4);
      }
      load_slot(d, l16, myws + (size_t)us * SLOT + 4096);
      if (__hip_atomic_load((u32*)(myws + TMO_OFF), __ATOMIC_RELAXED, __HIP_MEMORY_SCOPE_AGENT) == call) d[0] = u32x4{0x7fc07fc0u, 0x7fc07fc0u, 0x7fc07fc0u, 0x7fc07fc0u};
      const u32 tile = y >> 1, sid = (tile >> 1) * 4 + (y & 1) * 2;
      char* const ob = p->out[dest] + (rank & 3) * 256 + (tile & 1) * 128;
      const u32 vo = (lane >> 3) * (NLOC * 2) + (lane & 7) * 16;
      while (__hip_atomic_load(done, __ATOMIC_RELAXED, __HIP_MEMORY_SCOPE_WORKGROUP) < k - (P_WIN - 1)) __builtin_amdgcn_s_sleep(1);
#pragma unroll
      for (int g = 0; g < 8; ++g) {
        char* const b = ob + (size_t)(p->row[sid + (g >> 2)] + (g & 3) * 8) * (NLOC * 2);
        ST16(vo, d[g], b, 0);
      }
      if (y + P_REL * NCWG >= 2u * P_PARK && y + P_REL * NCWG < 2u * P_PARK + NCWG && lane == 0)
        __hip_atomic_store((u32*)(myws + REL_OFF + link * 128) + cw, call, __ATOMIC_RELAXED, __HIP_MEMORY_SCOPE_AGENT);
      asm volatile("s_waitcnt vmcnt(0)" ::: "memory");
      char* const fl = rws + DONE_OFF + ((rank & 3) * 64 + ((y & 1) ? 2 * RT_MED + 12 + tile : tile)) * 4;
      if (lane == 0) { ST4(0u, call, fl); __hip_atomic_fetch_add(done, 1, __ATOMIC_RELAXED, __HIP_MEMORY_SCOPE_WORKGROUP); }
    }
  } else if (cb == 0 && wv == 0) {
    const u32 call = draw_call(myws);
    count_bump(myws, call);
    const u64 t0 = __builtin_amdgcn_s_memrealtime();
    while (__builtin_amdgcn_s_memrealtime() - t0 < N_INTRA * P_UT / 10) __builtin_amdgcn_s_sleep(16);
    const char* const f = myws + DONE_OFF;
    const int f0 = (4 * lane) & 63;
    for (u32 n = 1;; ++n) {
      u32x4 v;
      asm volatile("global_load_dwordx4 %0, %1, %2 sc0 sc1\n\ts_waitcnt vmcnt(0)" : "=&v"(v) : "v"(l16), "s"(f) : "memory");
      if (__all((f0 >= NFLAG || v[0] >= call) && (f0 + 1 >= NFLAG || v[1] >= call) && (f0 + 2 >= NFLAG || v[2] >= call) && (f0 + 3 >= NFLAG || v[3] >= call))) break;
      if ((n & 31) == 0 && spin_fail(myws, call, t0)) {
        if (lane == 0) *(unsigned short*)(p->out[rank] + (size_t)phys_rank(rank) * 512 * (NLOC * 2)) = 0x7fc0;
        break;
      }
    }
  }
}

__device__ __forceinline__ u32x4 join(u32 l0, u32 l1, u32 h0, u32 h1) {
  return u32x4{__builtin_amdgcn_perm(h0, l0, 0x05010400u), __builtin_amdgcn_perm(h0, l0, 0x07030602u),
               __builtin_amdgcn_perm(h1, l1, 0x05010400u), __builtin_amdgcn_perm(h1, l1, 0x07030602u)};
}
__device__ __forceinline__ void fix_word(u32 (&h)[8], u32 e) {
  asm volatile("s_lshl_b64 exec, 1, %8\n\ts_getpc_b64 vcc\n\ts_add_u32 vcc_lo, vcc_lo, %9\n\ts_addc_u32 vcc_hi, vcc_hi, 0\n\ts_setpc_b64 vcc\n\t"
               "v_xor_b32_e32 %0, %10, %0\n\ts_branch 1f\n\tv_xor_b32_e32 %1, %10, %1\n\ts_branch 1f\n\t"
               "v_xor_b32_e32 %2, %10, %2\n\ts_branch 1f\n\tv_xor_b32_e32 %3, %10, %3\n\ts_branch 1f\n\t"
               "v_xor_b32_e32 %4, %10, %4\n\ts_branch 1f\n\tv_xor_b32_e32 %5, %10, %5\n\ts_branch 1f\n\t"
               "v_xor_b32_e32 %6, %10, %6\n\ts_branch 1f\n\tv_xor_b32_e32 %7, %10, %7\n"
               "1:\n\ts_mov_b64 exec, -1"
               : "+v"(h[0]), "+v"(h[1]), "+v"(h[2]), "+v"(h[3]), "+v"(h[4]), "+v"(h[5]), "+v"(h[6]), "+v"(h[7])
               : "s"(e), "s"(((e >> 3) & 56) + 12), "s"(((e >> 11) & 0x7f) << ((e >> 6) & 24)) : "vcc", "scc");
}

struct Stream {
  const char* sb; const char* mb; char* myws;
  u32 l16, tag, call; u64 t0;
  u32x4 W; int rdy; bool eager;
  __device__ __forceinline__ void init(char* ws, u32 call_, u32 sid, int lane) {
    myws = ws; call = call_; tag = call_tag(call_);
    sb = ws + (size_t)sid * NSTEP * SLOT + 4096;
    mb = ws + meta_off(sid * NSTEP);
    l16 = lane * 16;
    t0 = __builtin_amdgcn_s_memrealtime();
    W = u32x4{0u, 0u, 0u, 0u}; rdy = 0; eager = false;
  }
  __device__ __forceinline__ void poll(bool wide = true) {
    const u64 mask = (wide ? 0xffffull : 0xffull) << (rdy < 28 ? (2 * rdy) & 56 : 56);
    asm volatile("s_mov_b64 exec, %3\n\tglobal_load_dwordx4 %0, %1, %2 sc0 sc1\n\ts_mov_b64 exec, -1"
                 : "+v"(W) : "v"(l16), "s"(mb), "s"(mask) : "memory");
  }
  __device__ __forceinline__ void look() {
    asm volatile("" : "+v"(W));
    const u32 x = (W[0] ^ tag) | (W[1] ^ tag) | (W[2] ^ tag);
    const u64 odd = 0xaaaaaaaaaaaaaaaaull;
    const u64 bad = __ballot(x >= (1u << 21)) | (__ballot((W[3] ^ tag) >= (1u << 21)) & ~odd) | (__ballot(W[3] < call) & odd);
    rdy = bad ? __builtin_ctzll(bad) >> 1 : NSTEP;
  }
  __device__ __forceinline__ void spin(int t) {
    for (u32 n = 1; rdy <= t; ++n) {
      const int was = rdy;
      poll(n == 1);
      asm volatile("s_waitcnt vmcnt(0)" ::: "memory");
      look();
      if (rdy == was) {
        if (eager) __builtin_amdgcn_s_sleep(P_S2);
        else if (rdy == 0) __builtin_amdgcn_s_sleep(P_S0);
        else __builtin_amdgcn_s_sleep(P_S1);
      }
      if ((n & 31) == 0 && spin_fail(myws, call, t0)) rdy = NSTEP;
    }
  }
};

__device__ __forceinline__ void repair(u32 (&h)[8], u32 hd, u32 hdr, u32 hf, const char* sb, u32 l16) {
#pragma nounroll
  for (u32 n = (hdr >> (11 + 3 * hf)) & 7, k = hf ? 6 : 1; n != 0; --n, k += hf ? -1 : 1) fix_word(h, __builtin_amdgcn_readlane(hd, k));
  if (__builtin_expect(hdr & 0x400u, 0)) {
    const u32x8 g = load_two(sb + hf * 2048, l16);
#pragma unroll
    for (int j = 0; j < 8; ++j) asm volatile("v_mov_b32 %0, %1" : "+v"(h[j]) : "v"(g[j]));
  }
}

__device__ __forceinline__ void unpack_unit(char* myws, int lane, u32 call, u32 r, int t, u64 tk) {
  Stream S;
  S.init(myws, call, (call & 1) * NS_PAR + S_WIN + r, lane);
  const u32 l4 = lane * 4;
  const u32 uo = (S_IMG + 16 + r) * NSTEP;
  {
    if (r < 16) while (__builtin_amdgcn_s_memrealtime() - tk < (u32)(20 + 4 * t) * P_UT / 10) __builtin_amdgcn_s_sleep(4);
    u32 hd;
    for (u32 n = 1;; ++n) {
      hd = __hip_atomic_load((const u32*)(S.mb + t * 32) + (lane & 7), __ATOMIC_RELAXED, __HIP_MEMORY_SCOPE_SYSTEM);
      if (hdr_there(hd, call, lane)) break;
      if ((n & 31) == 0 && spin_fail(myws, call, S.t0)) break;
      __builtin_amdgcn_s_sleep(4);
    }
    Pay<6> P;
    pay_load(P, S.l16, S.sb + (size_t)t * SLOT);
    pay_wait<0>(P);
    const u32 hdr = __builtin_amdgcn_readfirstlane(hd), base4 = (hdr & 0x7f) * 0x01010101u;
    u32x4 o[8];
#pragma unroll
    for (int hf = 0; hf < 2; ++hf) {
      u32 g[8];
#pragma unroll
      for (int j = 0; j < 4; ++j) {
        const u32 n = P.v[4 + hf][j];
        g[2 * j] = (n & 0x87878787u) + base4;
        g[2 * j + 1] = (__builtin_amdgcn_alignbit(n, n, 4) & 0x87878787u) + base4;
      }
      if (hdr & (0x400u | 0x3800u << (3 * hf))) repair(g, hd, hdr, hf, S.sb + (size_t)t * SLOT, S.l16);
#pragma unroll
      for (int j = 0; j < 4; ++j)
        o[4 * hf + j] = join(P.v[2 * hf + (j >> 1)][2 * (j & 1)], P.v[2 * hf + (j >> 1)][2 * (j & 1) + 1], g[2 * j], g[2 * j + 1]);
    }
    char* const ob = myws + (size_t)(uo + t) * SLOT + 4096;
    ST_SIX(S.l16, o, ob); ST16(S.l16, o[6], ob, 2048); ST16(S.l16, o[7], ob, 3072);
    asm volatile("s_waitcnt vmcnt(0)" ::: "memory");
    char* const mo = myws + meta_off((uo + t));
    if (lane < 8) ST4(l4, hdr_word(0, call, lane), mo);
  }
}

__device__ __forceinline__ float silu(float y) {
  const float z = __builtin_amdgcn_fmed3f(y, -60.0f, 60.0f);
  const float e = __builtin_amdgcn_exp2f(z * -1.44269504088896341f);
  return y * __builtin_amdgcn_rcpf(1.0f + e);
}

__device__ __forceinline__ void gemm_tile(KP p, char* smem, char* myws, int wv, int lane, u32 call, u32 nb, u32 sid0, u32 tn, u32 ch, u32 fl, int hold, u64 tk, bool next, u32*& owed) {
  const u32 rank = p->rank;
  if (wv >= 8) __builtin_amdgcn_s_setprio(P_LPRIO);
  const u32 o0 = (rank & 4u) ^ 4u;
  const bool remote = tn >= 4;
  const u32 dst = (remote ? o0 + (tn - 4) : rank) & 7u;
  char* const ob = p->out[dst] + (remote ? (rank & 3) : tn) * 256 + ch * 128;
  u32* flag = (u32*)(p->ws[dst] + DONE_OFF) + (rank & 3) * 64 + fl;
  auto grow = [&](u32 rg) { return p->row[sid0 + (rg >> 2)] + (rg & 3) * 8; };
  char* b0 = ob + (size_t)grow(wv & 7) * (NLOC * 2);
  char* b1 = ob + (size_t)grow((wv & 7) + 8) * (NLOC * 2);
  if (hold == 2) {
    const u32 us = (S_PARK + (tn & 3)) * NSTEP + 2 * fl;
    b0 = myws + (size_t)us * SLOT + (wv & 7) * 1024; b1 = b0 + SLOT;
    flag = (u32*)(myws + meta_off(us));
  }
  asm volatile("" : "+s"(b0), "+s"(b1), "+s"(flag));
  if (nb == 0) {
    if (threadIdx.x == 0) ST4(0u, call, flag);
    return;
  }
  {
    const u32 first = sid0 < (u32)S_2 || sid0 >= (u32)S_OWN2 ? 0 : sid0 < (u32)S_INTER ? N1 * NSTEP : sid0 < (u32)S_3 ? 4 * NSTEP : N_INTRA - N3 * NSTEP;
    while (__builtin_amdgcn_s_memrealtime() - tk < first * P_UT / 10) __builtin_amdgcn_s_sleep(8);
  }
  Stream S;
  const u32 m = wv >> 1, kh = wv & 1;
  u32 tmo = 0;
  auto read_tmo = [&]() { asm volatile("global_load_dword %0, %1, %2 sc1" : "=&v"(tmo) : "v"(0u), "s"(myws + TMO_OFF) : "memory"); };
  if (wv >= 8) {
    const u32 un = wv - 8;
    S.init(myws, call, S_IMG + 4 * tn + un, lane);
    const u32 lds0 = (u32)(size_t)(LAS char*)smem + un * 4096;
    const char* const g0 = S.sb - 4096 + ch * 4096;
    Stream A;
    A.init(myws, call, (call & 1) * NS_PAR + sid0 + un, lane);
    A.eager = sid0 >= (u32)S_3 && sid0 < (u32)S_OWN2;
    asm volatile("global_load_dwordx4 %0, %2, %3 sc0 sc1\n\tglobal_load_dwordx4 %1, %2, %4 sc0 sc1\n\ts_waitcnt vmcnt(0)"
                 : "+v"(A.W), "+v"(S.W) : "v"(S.l16), "s"(A.mb), "s"(S.mb) : "memory");
    int is = 0, seen = ((nb >> un) & 1) ? 0 : NSTEP;
    for (int t = -1; t < NSTEP; ++t) {
      if (seen < NSTEP && seen <= t + P_DIST + P_ALOOK) {
        A.look();
        A.spin(min(t + P_DIST, NSTEP - 1));
        if (A.rdy != seen) { *(LAS u32x4*)((LAS char*)smem + HDR_OFF + un * 1024 + A.l16) = A.W; seen = A.rdy; }
        asm volatile("" ::: "memory");
        if (seen < NSTEP) A.poll();
      }
      const int lim = min(min(NSTEP, is + 2), t + P_BD);
      if (S.rdy < min(NSTEP, lim + 2)) { S.look(); S.spin(t); if (S.rdy < NSTEP) S.poll(); }
      for (; is < min(lim, S.rdy); ++is) {
        const char* g = g0 + (size_t)is * SLOT;
        const u32 l = lds0 + (is & 7) * 16384;
        u32 keep;
        asm volatile("s_mov_b32 %0, m0\n\ts_mov_b32 m0, %3\n\ts_nop 0\n\t"
                     "global_load_lds_dwordx4 %1, %2 sc0 sc1\n\tglobal_load_lds_dwordx4 %1, %2 offset:1024 sc0 sc1\n\t"
                     "global_load_lds_dwordx4 %1, %2 offset:2048 sc0 sc1\n\tglobal_load_lds_dwordx4 %1, %2 offset:3072 sc0 sc1\n\t"
                     "s_mov_b32 m0, %0" : "=&s"(keep) : "v"(S.l16), "s"(g), "s"(l) : "memory");
      }
      const int ahead = is - 1 - t;
      if (ahead >= 6 || t < 0) asm volatile("s_waitcnt vmcnt(24)" ::: "memory");
      else if (ahead >= 4) { if (ahead == 5) asm volatile("s_waitcnt vmcnt(20)" ::: "memory"); else asm volatile("s_waitcnt vmcnt(16)" ::: "memory"); }
      else if (ahead >= 2) { if (ahead == 3) asm volatile("s_waitcnt vmcnt(12)" ::: "memory"); else asm volatile("s_waitcnt vmcnt(8)" ::: "memory"); }
      else if (ahead == 1) asm volatile("s_waitcnt vmcnt(4)" ::: "memory");
      else asm volatile("s_waitcnt vmcnt(0)" ::: "memory");
      WG_BARRIER();
    }
    asm volatile("" : "+v"(S.W), "+v"(A.W));
    __builtin_amdgcn_s_setprio(0);
    if (hold == 1 && wv == 8) {
      const u32* const r = (const u32*)(myws + REL_OFF + (BID & 7) * 128) + (lane < NCWG ? lane : 0);
      for (u32 n = 1; !__all(__hip_atomic_load(r, __ATOMIC_RELAXED, __HIP_MEMORY_SCOPE_AGENT) >= call); ++n) {
        if ((n & 31) == 0 && spin_fail(myws, call, S.t0)) break;
        __builtin_amdgcn_s_sleep(4);
      }
    }
  } else if (!((nb >> m) & 1)) {
    asm volatile("s_waitcnt vmcnt(0)" ::: "memory");
    for (int t = -1; t < NSTEP; ++t) WG_BARRIER();
    read_tmo();
    asm volatile("s_waitcnt vmcnt(0)" : "+v"(tmo) :: "memory");
  } else {
    const char* const sb = myws + (size_t)((call & 1) * NS_PAR + sid0 + m) * NSTEP * SLOT + 4096;
    const u32 l16 = lane * 16, vlo = l16 + kh * 2048, vcode = l16 + kh * 1024;
    u32 ho = HDR_OFF + m * 1024 + (lane & 7) * 4, hd;
    const u32 rmask = 0x400u | 0x3800u << (3 * kh);
    f32x16 acc[2] = {};
    u32 bb = kh * 8192 + lane * 16;
    bf16x8 wf[4];
    u32x4 X[2][4];
    Pay<3> P[4];
    u32 h[8];
    u32 hdr = 0, base4 = 0;
    auto rd = [&](int stage, int f) {
      wf[f & 3] = *(const LAS bf16x8*)((LAS char*)smem + bb + stage * 16384 + (f >> 2) * 4096 + ((f >> 1) & 1) * 1024 + (f & 1) * 2048);
    };
    auto piece = [&](int k, int t, int ts, int waitn, bool fetch) {
      Pay<3>& Q = P[ts];
      if (k == 0) {
        if (waitn == 3) pay_wait<9>(Q); else if (waitn == 2) pay_wait<6>(Q); else if (waitn == 1) pay_wait<3>(Q); else pay_wait<0>(Q);
        hdr = __builtin_amdgcn_readfirstlane(hd);
        base4 = (hdr & 0x7f) * 0x01010101u;
      } else if (k < 6) {
#pragma unroll
        for (int j = 0; j < 4; ++j) {
          const int i = k < 4 ? 2 * j + (k & 1) : 4 * (k - 4) + j;
          if (k == 1) h[i] = __builtin_amdgcn_alignbit(Q.v[2][j], Q.v[2][j], 4);
          else if (k == 2) h[i] = Q.v[2][j] & 0x87878787u;
          else if (k == 3) h[i] &= 0x87878787u;
          else h[i] += base4;
          asm volatile("" : "+v"(h[i]));
        }
      } else if (k == 6) {
        if (hdr & rmask) repair(h, hd, hdr, kh, sb + (size_t)t * SLOT, l16);
      } else if (k == 7) {
      } else if (k < 12) {
        const int c = k - 8;
        X[ts & 1][c] = join(Q.v[c >> 1][2 * (c & 1)], Q.v[c >> 1][2 * (c & 1) + 1], h[2 * c], h[2 * c + 1]);
        asm volatile("" : "+v"(X[ts & 1][c]));
      } else if (k == 12) {
        ho += 32;
        hd = *(const LAS u32*)((LAS char*)smem + ho);
      } else if (k == 13 && fetch) {
        const int tn4 = t + P_DIST < NSTEP ? t + P_DIST : NSTEP - 1;
        pay_load(P[(ts + P_DIST) & 3], vlo, vcode, sb + (size_t)tn4 * SLOT);
      }
    };
    auto step = [&](int t, int ts, int waitn, bool fetch, bool more) {
#pragma unroll
      for (int q = 0; q < 8; ++q) {
        if (q == 4 && more) { WG_BARRIER(); if (ts == 3) bb ^= 65536u; }
        else if (q % 2 == 0 && (more || q < 6)) __builtin_amdgcn_s_waitcnt(0xC27F);
        __builtin_amdgcn_sched_barrier(0);
        acc[q & 1] = __builtin_amdgcn_mfma_f32_32x32x16_bf16(wf[q & 3], __builtin_bit_cast(bf16x8, X[ts & 1][q >> 1]), acc[q & 1], 0, 0, 0);
        __builtin_amdgcn_sched_barrier(0);
        if (q < 4) rd(ts, q + 4);
        else if (more) rd((ts + 1) & 3, q - 4);
        if (more) {
          piece(2 * q, t + 1, (ts + 1) & 3, waitn, fetch);
          piece(2 * q + 1, t + 1, (ts + 1) & 3, waitn, fetch);
        }
      }
    };
    asm volatile("s_waitcnt vmcnt(0)" ::: "memory");
    WG_BARRIER();
    if (owed && threadIdx.x < 2) ST4(threadIdx.x * 32, call, owed);
#pragma unroll
    for (int d = 0; d < P_DIST; ++d) pay_load(P[d], vlo, vcode, sb + (size_t)d * SLOT);
    hd = *(const LAS u32*)((LAS char*)smem + ho);
#pragma unroll
    for (int k = 0; k < 13; ++k) piece(k, 0, 0, P_DIST - 1, true);
    WG_BARRIER();
    piece(13, 0, 0, P_DIST - 1, true);
#pragma unroll
    for (int f = 0; f < 4; ++f) { rd(0, f); __builtin_amdgcn_sched_barrier(0); }
    for (int tb = 0; tb < NSTEP - 4; tb += 4) {
#pragma unroll
      for (int ts = 0; ts < 4; ++ts) step(tb + ts, ts, P_DIST - 1, true, true);
    }
    step(NSTEP - 4, 0, P_DIST - 1, false, true);
    step(NSTEP - 3, 1, P_DIST - 2, false, true);
    step(NSTEP - 2, 2, P_DIST - 3, false, true);
    read_tmo();
    step(NSTEP - 1, 3, 0, false, false);
    char* const xq = smem + EPI_OFF + (kh * 128 + m * 32 + (lane & 31)) * ERS + (lane >> 5) * 16;
#pragma unroll
    for (int n = 0; n < 2; ++n)
#pragma unroll
      for (int g = 0; g < 4; ++g)
        *(f32x4*)(xq + n * 128 + g * 32) = f32x4{acc[n][4 * g], acc[n][4 * g + 1], acc[n][4 * g + 2], acc[n][4 * g + 3]};
    if (P_DIST == 4) pay_wait<0>(P[0]);
    asm volatile("s_waitcnt vmcnt(0)" : "+v"(tmo) :: "memory");
  }
  WG_BARRIER();
  if (wv < 8) {
    const u32 vo = hold == 2 ? lane * 16 : (lane >> 3) * (NLOC * 2) + (lane & 7) * 16;
    auto get = [&](u32 rg) {
      const char* const q = smem + EPI_OFF + (rg * 8 + (lane >> 3)) * ERS + (lane & 7) * 32;
      const f32x4 a = *(const f32x4*)q + *(const f32x4*)(q + 128 * ERS), c = *(const f32x4*)(q + 16) + *(const f32x4*)(q + 128 * ERS + 16);
      auto pk = [](float lo, float hi) { const f32x2 v = {silu(lo), silu(hi)}; return __builtin_bit_cast(u32, __builtin_convertvector(v, bf16x2)); };
      u32x4 v = {pk(a[0], a[1]), pk(a[2], a[3]), pk(c[0], c[1]), pk(c[2], c[3])};
      if (tmo == call) v = u32x4{0x7fc07fc0u, 0x7fc07fc0u, 0x7fc07fc0u, 0x7fc07fc0u};
      return v;
    };
    if ((nb >> (wv >> 2)) & 1) { const u32x4 v0 = get(wv); ST16(vo, v0, b0, 0); }
    if ((nb >> (wv >> 2)) & 4) { const u32x4 v1 = get(wv + 8); ST16(vo, v1, b1, 0); }
  }
  owed = nullptr;
  if (next) { if (remote) owed = flag; }
  else if (remote) {
    asm volatile("s_waitcnt vmcnt(0)" ::: "memory");
    WG_BARRIER();
    if (threadIdx.x == 0) ST4(0u, call, flag);
  }
}

inline bool tile_of(u32 rank, u32 b, int k, u32& nb, u32& sid0, u32& tn, u32& ch, u32& fl, int& hold) {
  const u32 peer = peer_of_xcd(rank, b & 7), i = b >> 3;
  const bool home = !((peer ^ rank) & 4), spare = i >= 2 * RT_MED;
  hold = 0;
  constexpr u32 I0 = S_INTER / 2;
  const bool extra = home && i >= I0 && i < I0 + (NB + 1) / 2 * 2;
  if (extra && k == 0) {
    const u32 j = i - I0;
    nb = (j | 1) < (u32)NB ? 3 : 1; tn = peer & 3; sid0 = S_OWN2 + (j >> 1) * 2; ch = j & 1; fl = 0;
    return true;
  }
  if (k == (extra ? 1 : 0) && !spare) {
    nb = 15; tn = (peer & 3) + (home ? 0 : 4); sid0 = (i >> 1) * 4; ch = i & 1; fl = i;
    hold = home || sid0 >= (u32)S_INTER ? 0 : i < (u32)P_PARK ? 2 : i >= 10 ? 1 : 0;
    if (!home)
      for (u32 m = 0; m < 4; ++m)
        if (sid0 + m >= (u32)S_INTER && (sid0 + m - S_INTER) / NB == (peer & 3)) nb &= ~(1u << m);
    return true;
  }
  const u32 j = spare ? i - 2 * RT_MED + 10 : i;
  nb = 3; tn = (peer & 3) + (home ? 4 : 0); sid0 = RT_MED * 4 + (j >> 1) * 2; ch = j & 1; fl = 2 * RT_MED + j;
  return spare ? k == 0 : i < 10;
}

constexpr u32 NHOME = 32 * NSTEP;
__host__ __device__ inline u32 pack_index(u32 b, u32 wv) {
  constexpr u32 NP1 = 80;
  return b < NP1 ? (wv < 2 ? b + NP1 * wv : NHOME) : 2 * NP1 + (b - NP1) + (NGEMM - NP1) * wv;
}

constexpr u32 UB0 = RT_MED * 16, UNW = 16;
__host__ __device__ inline u32 unpack_index(u32 b, u32 wv) {
  return b >= UB0 && b < UB0 + UNW ? (11 - wv) * UNW + (b - UB0) : 512u;
}

__global__ void __launch_bounds__(NTHREAD, 3) ag_gemm_silu(const Params) {
  KP p = (KP)__builtin_amdgcn_kernarg_segment_ptr();
  __shared__ __attribute__((aligned(16))) char smem[LDS_BYTES];
  const int lane = threadIdx.x & 63;
  const int wv = __builtin_amdgcn_readfirstlane(threadIdx.x >> 6);
  char* const myws = p->myws;
  if (BID >= NGEMM) { sender(p, smem, myws, wv, lane); return; }
  u32 call;
  const u64 tk = __builtin_amdgcn_s_memrealtime();
  const u32 u = pack_index(BID, wv);
  if (u < NHOME) {
    const u32 t = u >> 5, xc = u & 7, sub = (u >> 3) & 3;
    const u32 c = xc < 4 ? sub * 4 + xc : 16 + (peer_of_xcd(p->rank, xc) & 3) * 4 + sub;
    const bool isw = c >= 16;
    const u32 cw = c - 16, l16 = lane * 16;
    const u64 t0 = __builtin_amdgcn_s_memrealtime();
    while (__builtin_amdgcn_s_memrealtime() - t0 < t * P_PP) __builtin_amdgcn_s_sleep(2);
    u32x4 d[8], s[8];
    load_raw(p, d, lane, isw ? 16 + (cw & 3) : c, (cw >> 2) & 3, t);
    u32 ticket;
    count_read(ticket, myws);
    WAIT_RAW(d);
    call = count_take(ticket);
    u32 meta = 0;
    bool eight = true;
    to_fragments(d, isw, smem + SCR_OFF + wv * 8192, lane);
    if (isw) {
#pragma unroll
      for (int j = 0; j < 8; ++j) s[j] = d[j];
    } else { low_slabs(d, s); eight = encode(d, s, meta, lane); }
    const u32 ud = (isw ? S_IMG + cw : (call & 1) * NS_PAR + (c < (u32)NA ? c : S_OWN2 - NA + c)) * NSTEP + t;
    char* const b = myws + (size_t)ud * SLOT + 4096;
    ST_SIX(l16, s, b);
    if (eight) { ST16(l16, s[6], b, 2048); ST16(l16, s[7], b, 3072); }
    asm volatile("s_waitcnt vmcnt(0)" ::: "memory");
    char* const m = myws + meta_off(ud);
    if (lane < 8) ST4(lane * 4, hdr_word(meta, call, lane), m);
  } else call = draw_call(myws);
#pragma nounroll
  for (u32 v = unpack_index(BID, wv); v < 512; v += 12 * UNW) unpack_unit(myws, lane, call, (v + P_UROT(v >> 4)) & 15, v >> 4, tk);
  WG_BARRIER();
  count_bump(myws, call);
  u32* owed = nullptr;
#pragma nounroll
  for (int k = 0; k < 2; ++k) {
    const u32 td = p->tile[BID][k];
    if (!(td >> 23)) break;
    gemm_tile(p, smem, myws, wv, lane, call, td & 15, (td >> 4) & 127, (td >> 11) & 7, (td >> 14) & 1, (td >> 15) & 63, (td >> 21) & 3, tk, k == 0 && p->tile[BID][1] != 0, owed);
  }
}

static void build_orders(Params& p) {
  struct E { double tau; unsigned short code; };
  E e[N_ORD];
  double sigma = P_SIGMA, ovl = P_OVL, ovl2 = P_OVL2, lead = P_LEAD, f1 = 14.5, f2 = 6, stg = P_STG;
  for (int kind = 0; kind < 2; ++kind) {
    int n = 0;
    for (int t = 0; t < NSTEP; ++t) {
      if (kind == 0) {
        for (int b = 0; b < NA; ++b) if (b != RB || t < RT0) e[n++] = {t + (b < N1 ? 0 : b < N1 + N2 ? NSTEP - ovl : 2 * NSTEP - ovl - ovl2 + (b >= N1 + N2 + 2 ? stg : 0)), (unsigned short)(b << 5 | t)};
      } else {
        for (int i = 0; i < 4; ++i) e[n++] = {t * sigma - lead, (unsigned short)((16 + i) << 5 | t)};
        for (int b = NA; b < 16; ++b) e[n++] = {t + 0.5, (unsigned short)(b << 5 | t)};
      }
    }
    if (kind == 1)
      for (int j = 0; j < P_RLY; ++j) {
        e[n++] = {f1 * sigma - lead, (unsigned short)(20 << 5 | j)};
        for (int i = 0; i < 3; ++i) e[n++] = {f2 + j, (unsigned short)((21 + i) << 5 | j)};
      }
    std::stable_sort(e, e + n, [](const E& a, const E& b) { return a.tau < b.tau; });
    for (int i = 0; i < N_ORD; ++i) p.ord[kind][i] = i < n ? e[i].code : 0;
  }
}

extern "C" void kernel_launch(int rank, int nranks, ncclComm_t comm, hipStream_t stream,
                              void* const* d_in, const int* in_sizes, int n_in,
                              void* d_out, int out_size, const int* in_dtypes, int out_dtype,
                              void* const* d_out_all, void* const* d_ws_all, size_t ws_bytes) {
  (void)comm; (void)in_dtypes; (void)out_dtype; (void)d_out;
  if (nranks != NR || (unsigned)rank >= (unsigned)NR || n_in != 2 || ws_bytes < WS_NEED ||
      in_sizes[0] != 512 * KDIM || in_sizes[1] != KDIM * NLOC || out_size != 4096 * NLOC) {
    fprintf(stderr, "unexpected problem shape\n");
    return;
  }
  Params p = {};
  p.x = (const char*)d_in[0];
  p.w = (const char*)d_in[1];
  for (int i = 0; i < NR; ++i) {
    p.out[i] = (char*)d_out_all[phys_rank(i)]; p.ws[i] = (char*)d_ws_all[phys_rank(i)];
    if ((int)phys_rank(i) == rank) p.rank = i;
  }
  p.myws = (char*)d_ws_all[rank];
  build_orders(p);
  for (int i = 0; i < NS_PAR + 4; ++i) p.row[i] = i < 64 || (i >= S_OWN2 && i < NS_PAR) ? global_row(p.rank, i) : 0;
  for (u32 b = 0; b < (u32)NGEMM; ++b)
    for (int k = 0; k < 2; ++k) {
      u32 nb, sid0, tn, ch, fl; int hold;
      p.tile[b][k] = tile_of(p.rank, b, k, nb, sid0, tn, ch, fl, hold) ? nb | sid0 << 4 | tn << 11 | ch << 14 | fl << 15 | (u32)hold << 21 | 1u << 23 : 0u;
    }
  hipLaunchKernelGGL(ag_gemm_silu, dim3(GRID), dim3(NTHREAD), 0, stream, p);
  hipError_t err = hipGetLastError();
  if (err != hipSuccess) fprintf(stderr, "launch failed: %s\n", hipGetErrorString(err));
}
```

```cpp
#include <hip/hip_runtime.h>
#include <rccl/rccl.h>
#include <cstdio>
#include <cstdint>
#include <cstdlib>
#include <algorithm>

typedef unsigned u32;
typedef unsigned long long u64;
typedef unsigned u32x4 __attribute__((ext_vector_type(4)));
typedef unsigned u32x8 __attribute__((ext_vector_type(8)));
typedef unsigned short u16x2 __attribute__((ext_vector_type(2)));
typedef __bf16 bf16x2 __attribute__((ext_vector_type(2)));
typedef __bf16 bf16x4 __attribute__((ext_vector_type(4)));
typedef __bf16 bf16x8 __attribute__((ext_vector_type(8)));
typedef float f32x2 __attribute__((ext_vector_type(2)));
typedef float f32x4 __attribute__((ext_vector_type(4)));
typedef float f32x16 __attribute__((ext_vector_type(16)));
#define LAS __attribute__((address_space(3)))

#define P_NA 11
#define P_NCWG 4
#define P_WIN 3
#define P_WIN0 4
#define P_TAPER 8
#define P_WIN1 5
#define P_DIST 3
#define P_BD 7
#define P_ALOOK 3
#define P_LPRIO 2
#define P_UROT(t) (5 * (t))
#define P_STG 3
#define P_SIGMA 0.1
#define P_OVL 0.0
#define P_OVL2 0.0
#define P_LEAD 0.0
#define P_PP 15
#define P_PARK 10
#define P_RLY 2
#define P_S0 16
#define P_S1 16
#define P_S2 4
#define P_UT 100
#define P_REL 2

constexpr int NR = 8;
constexpr int KDIM = 4096, NLOC = 512;
constexpr int NSTEP = 32;
constexpr int NA = P_NA, NB = 16 - NA;
constexpr int N1 = 3, N3 = 4, N2 = NA - N1 - N3;
constexpr int S_1 = NA, S_2 = S_1 + 3 * N1, S_INTER = S_2 + 3 * N2, S_3 = 64 - 3 * N3, S_WIN = 64, S_RLY = 80, S_OWN2 = 81;
constexpr int RB = N1 + N2 - 1, RT0 = NSTEP - 4 * P_RLY;
constexpr int NS_PAR = S_OWN2 + NB, S_IMG = 2 * NS_PAR, S_PARK = S_IMG + 32, NSTREAM = S_PARK + 4;
constexpr size_t SLOT = 8192;
constexpr size_t META_OFF = (size_t)NSTREAM * NSTEP * SLOT;
constexpr size_t DONE_OFF = META_OFF + (size_t)NSTREAM * NSTEP * 32;
constexpr size_t REL_OFF = DONE_OFF + 1024;
constexpr size_t CNT_OFF = REL_OFF + 1024;
constexpr size_t TMO_OFF = CNT_OFF + 256 * 128;
constexpr size_t WS_NEED = TMO_OFF + 128;
constexpr int N_INTRA = NA * NSTEP - 4 * P_RLY, N_INTER = NB * NSTEP + 4 * NSTEP + 4 * P_RLY;
constexpr int N_ORD = N_INTRA > N_INTER ? N_INTRA : N_INTER;
constexpr int RT_MED = 13;
constexpr int NGEMM = RT_MED * 16 + 16;
constexpr int NFLAG = 2 * RT_MED + 12 + P_PARK;
constexpr int NCWG = P_NCWG;
constexpr int GRID = NGEMM + 8 * NCWG;
static_assert(P_PARK == 10 && P_BD <= 7 && P_RLY <= 8 && N_ORD % 2 == 0 && NB > 4 && NB <= 8 && S_INTER % 4 == 0, "tile map");
static_assert(GRID <= 256, "one workgroup per CU, all resident");
constexpr int NTHREAD = 768;
constexpr int SCR_OFF = 0, EPI_OFF = 32768;
constexpr int HDR_OFF = 131072;
constexpr int CLAIM_OFF = HDR_OFF + 4096 + 64;
constexpr int LDS_BYTES = CLAIM_OFF + 64;
constexpr u64 DEADLINE_TICKS = 80000000ull;
constexpr int ERS = 272;

#define P_PHYS 0x75416320u
__host__ __device__ inline u32 phys_rank(u32 l) { return (P_PHYS >> (4 * l)) & 7u; }
__host__ __device__ inline u32 peer_of_xcd(u32 rank, u32 x) { return (((rank ^ x) & 4u) | ((rank + x) & 3u)) & 7u; }

struct Params {
  const char* x;
  const char* w;
  char* myws;
  char* out[NR];
  char* ws[NR];
  int rank;
  u32 row[NS_PAR + 4];
  u32 tile[NGEMM][2];
  alignas(4) unsigned short ord[2][N_ORD];
};
static_assert(sizeof(Params) <= 4096, "kernel argument");
typedef const __attribute__((address_space(4))) Params* KP;

#define WG_BARRIER() do { asm volatile("" ::: "memory"); __builtin_amdgcn_s_waitcnt(0xC07F); \
    asm volatile("s_barrier" ::: "memory"); } while (0)
#define ST16(voff, data, sbase, imm) \
  asm volatile("global_store_dwordx4 %0, %1, %2 offset:" #imm " sc0 sc1\n\ts_nop 1" :: "v"(voff), "v"(data), "s"(sbase) : "memory")
#define ST4(voff, data, sbase) \
  asm volatile("global_store_dword %0, %1, %2 sc0 sc1" :: "v"(voff), "v"(data), "s"(sbase) : "memory")
#define ST_SIX(voff, s, b) do { ST16(voff, s[0], b, -4096); ST16(voff, s[1], b, -3072); ST16(voff, s[2], b, -2048); \
    ST16(voff, s[3], b, -1024); ST16(voff, s[4], b, 0); ST16(voff, s[5], b, 1024); } while (0)

__device__ __forceinline__ u32* counter(char* myws) { return (u32*)(myws + CNT_OFF + blockIdx.x * 128); }
__device__ __forceinline__ u32 take_role(char* myws, u32 r, u32 call) {
#pragma nounroll
  for (int i = 0; i < GRID; ++i, r = (r + 1) % GRID) {
    u32 old;
    asm volatile("s_mov_b64 exec, 1\n\tglobal_atomic_umax %0, %1, %2, %3 sc0\n\ts_mov_b64 exec, -1\n\ts_waitcnt vmcnt(0)"
                 : "=&v"(old) : "v"(r * 128), "v"(call), "s"(myws + CNT_OFF + 64) : "memory");
    if ((u32)__builtin_amdgcn_readfirstlane(old) < call) break;
  }
  return r;
}
__device__ __forceinline__ void claim(char* smem, char* myws, int wv, u32& bid, u32& call) {
  LAS u32* const cl = (LAS u32*)(smem + CLAIM_OFF);
  if (wv == 0) {
    const u32 c = __builtin_amdgcn_readfirstlane(__hip_atomic_load(counter(myws), __ATOMIC_RELAXED, __HIP_MEMORY_SCOPE_AGENT)) + 1u;
    cl[1] = take_role(myws, bid, c);
    cl[0] = c;
    if (threadIdx.x == 0) __hip_atomic_store(counter(myws), c, __ATOMIC_RELAXED, __HIP_MEMORY_SCOPE_AGENT);
  }
  WG_BARRIER();
  call = __builtin_amdgcn_readfirstlane(cl[0]);
  bid = __builtin_amdgcn_readfirstlane(cl[1]);
}
__device__ __forceinline__ u32 call_tag(u32 call) { return (call % 2047u + 1u) << 21; }
__device__ __forceinline__ u32 hdr_word(u32 data, u32 call, int lane) { return lane == 7 ? call : data | call_tag(call); }
__device__ __forceinline__ bool hdr_there(u32 word, u32 call, int lane) {
  return __all((lane & 7) == 7 ? word >= call : (word ^ call_tag(call)) < (1u << 21));
}

__device__ __forceinline__ bool spin_fail(char* myws, u32 call, u64 t0) {
  u32* tmo = (u32*)(myws + TMO_OFF);
  u32 t = __builtin_amdgcn_readfirstlane(__hip_atomic_load(tmo, __ATOMIC_RELAXED, __HIP_MEMORY_SCOPE_AGENT));
  if (t == call || __builtin_amdgcn_s_memrealtime() - t0 > DEADLINE_TICKS) {
    __hip_atomic_store(tmo, call, __ATOMIC_RELAXED, __HIP_MEMORY_SCOPE_AGENT);
    return true;
  }
  return false;
}

__device__ __forceinline__ size_t meta_off(u32 unit) { return META_OFF + (size_t)unit * 32; }
__device__ __forceinline__ u32 dest_stream(u32 sl, u32 rank, u32 dest, u32 par) {
  const u32 j = ((rank - dest) & 3) - 1;
  const u32 s = sl >= 16 ? S_WIN + (rank & 3) * 4 + (sl - 16) : sl >= (u32)NA ? S_INTER + (rank & 3) * NB + (sl - NA)
              : sl >= (u32)(N1 + N2) ? S_3 + j * N3 + (sl - N1 - N2) : sl >= (u32)N1 ? S_2 + j * N2 + (sl - N1) : S_1 + j * N1 + sl;
  return par * NS_PAR + s;
}

inline u32 global_row(u32 rank, u32 sid) {
  const u32 g0 = rank & 4u, o0 = g0 ^ 4u;
  u32 src, bi;
  if (sid < (u32)S_1) { src = rank; bi = sid; }
  else if (sid < (u32)S_2) { const u32 q = sid - S_1; src = g0 + ((rank + 1 + q / N1) & 3); bi = q % N1; }
  else if (sid < (u32)S_INTER) { const u32 q = sid - S_2; src = g0 + ((rank + 1 + q / N2) & 3); bi = N1 + q % N2; }
  else if (sid < (u32)S_3) { const u32 q = sid - S_INTER; src = o0 + q / NB; bi = NA + q % NB; }
  else if (sid >= (u32)S_OWN2) { src = rank; bi = NA + sid - S_OWN2; }
  else { const u32 q = sid - S_3; src = g0 + ((rank + 1 + q / N3) & 3); bi = N1 + N2 + q % N3; }
  return (phys_rank(src & 7u) * 512 + bi * 32) & 4095u;
}

__device__ __forceinline__ void load_raw(KP p, u32x4 (&d)[8], int lane, u32 sl, u32 q, u32 t) {
  const bool isw = sl >= 16;
  const u32 rs = isw ? NLOC * 2 : KDIM * 2;
  const u32 vo = (lane >> 4) * rs + (lane & 15) * 16;
  const u64 aw = (u64)p->w + (u64)((t * 128 + (sl & 3) * 32) * (NLOC * 2) + q * 256), ax = (u64)p->x + (u64)(sl * 32 * (KDIM * 2) + t * 256);
  const u64 b0 = isw ? aw : ax;
#pragma unroll
  for (int j = 0; j < 8; ++j)
    asm volatile("global_load_dwordx4 %0, %1, %2" : "=&v"(d[j]) : "v"(vo), "s"(b0 + (u64)(j * 4 * rs)) : "memory");
}
#define WAIT_RAW(d) asm volatile("s_waitcnt vmcnt(0)" : "+v"(d[0]), "+v"(d[1]), "+v"(d[2]), "+v"(d[3]), "+v"(d[4]), "+v"(d[5]), "+v"(d[6]), "+v"(d[7]) :: "memory")
__device__ __forceinline__ void to_fragments(u32x4 (&d)[8], bool isw, char* scr, int lane) {
  LAS char* const sc = (LAS char*)scr;
  auto off = [](u32 row, u32 ch) { return 256u * row + 16u * (ch ^ (((row & 3) << 2) | ((row >> 2) & 3))); };
#pragma unroll
  for (int j = 0; j < 8; ++j) *(LAS u32x4*)(sc + off(4 * j + (lane >> 4), lane & 15)) = d[j];
  asm volatile("" ::: "memory");
  if (isw) {
    const u32 r0 = 8 * (lane >> 5) + ((lane & 15) >> 2), c0 = 2 * ((lane >> 4) & 1) + ((lane & 3) >> 1);
#pragma unroll
    for (int j = 0; j < 8; ++j) {
      const bf16x4 lo = __builtin_amdgcn_ds_read_tr16_b64_v4bf16((LAS bf16x4*)(sc + off(16 * (j & 1) + r0, 4 * (j >> 1) + c0) + 8 * (lane & 1)));
      const bf16x4 hi = __builtin_amdgcn_ds_read_tr16_b64_v4bf16((LAS bf16x4*)(sc + off(16 * (j & 1) + r0 + 4, 4 * (j >> 1) + c0) + 8 * (lane & 1)));
      d[j] = __builtin_bit_cast(u32x4, __builtin_shufflevector(lo, hi, 0, 1, 2, 3, 4, 5, 6, 7));
    }
  } else {
#pragma unroll
    for (int j = 0; j < 8; ++j) d[j] = *(const LAS u32x4*)(sc + off(lane & 31, 2 * j + (lane >> 5)));
  }
  asm volatile("" ::: "memory");
}
__device__ __forceinline__ void low_slabs(const u32x4 (&d)[8], u32x4 (&s)[8]) {
#pragma unroll
  for (int i = 0; i < 16; ++i) s[i >> 2][i & 3] = __builtin_amdgcn_perm(d[i >> 1][2 * (i & 1) + 1], d[i >> 1][2 * (i & 1)], 0x06040200u);
}
__device__ __forceinline__ bool encode(const u32x4 (&d)[8], u32x4 (&s)[8], u32& meta, int lane) {
  u32 H[16], T[16];
  u16x2 m1 = {0, 0}, m2 = {0, 0}, m3 = {0, 0}, m4 = {0, 0};
#pragma unroll
  for (int i = 0; i < 16; ++i) {
    const u32 a = d[i >> 1][2 * (i & 1)], b = d[i >> 1][2 * (i & 1) + 1];
    H[i] = __builtin_amdgcn_perm(b, a, 0x07050301u);
  }
#pragma unroll
  for (int i = 0; i < 16; i += 2) {
    T[i] = H[i] | 0x80808080u;
    T[i + 1] = H[i + 1] | 0x80808080u;
    m1 = __builtin_elementwise_max(m1, __builtin_bit_cast(u16x2, T[i]));
    m2 = __builtin_elementwise_max(m2, __builtin_bit_cast(u16x2, T[i] << 8));
    m3 = __builtin_elementwise_max(m3, __builtin_bit_cast(u16x2, T[i + 1]));
    m4 = __builtin_elementwise_max(m4, __builtin_bit_cast(u16x2, T[i + 1] << 8));
  }
  m1 = __builtin_elementwise_max(__builtin_elementwise_max(m1, m2), __builtin_elementwise_max(m3, m4));
  u32 mx = (u32)(m1.x > m1.y ? m1.x : m1.y);
  mx = max(mx, (u32)__builtin_amdgcn_update_dpp(0, mx, 0x111, 0xf, 0xf, true));
  mx = max(mx, (u32)__builtin_amdgcn_update_dpp(0, mx, 0x112, 0xf, 0xf, true));
  mx = max(mx, (u32)__builtin_amdgcn_update_dpp(0, mx, 0x114, 0xf, 0xf, true));
  mx = max(mx, (u32)__builtin_amdgcn_update_dpp(0, mx, 0x118, 0xf, 0xf, true));
  const u32 top = max(max((u32)__builtin_amdgcn_readlane(mx, 15), (u32)__builtin_amdgcn_readlane(mx, 31)),
                      max((u32)__builtin_amdgcn_readlane(mx, 47), (u32)__builtin_amdgcn_readlane(mx, 63)));
  const u32 h7 = (top >> 8) & 0x7f;
  const u32 base = __builtin_amdgcn_readfirstlane(h7 > 7 ? h7 - 7 : 0);
  const u32 base4 = base * 0x01010101u;
  u32 ok = 0xffffffffu;
  u32 N[8];
#pragma unroll
  for (int j = 0; j < 8; ++j) {
    const u32 ta = T[2 * j] - base4, tb = T[2 * j + 1] - base4;
    ok &= ta & tb;
    const u32 ca = (ta & 0x07070707u) | (H[2 * j] & 0x80808080u);
    const u32 cb = (tb & 0x07070707u) | (H[2 * j + 1] & 0x80808080u);
    N[j] = ca | __builtin_amdgcn_alignbit(cb, cb, 28);
  }
  meta = 0;
  int c0 = 0, c1 = 0;
  if (__any((~ok & 0x80808080u) != 0)) {
    u32 M0 = 0, M1 = 0;
#pragma unroll
    for (int i = 0; i < 8; ++i) {
      M0 |= ~((T[i] - base4) >> i) & (0x80808080u >> i);
      M1 |= ~((T[8 + i] - base4) >> i) & (0x80808080u >> i);
    }
    u64 bal = __ballot((M0 | M1) != 0);
    while (bal != 0 && c0 + c1 <= 6) {
      const int ln = __builtin_ctzll(bal);
      bal &= bal - 1;
      u64 mm = (u64)(u32)__builtin_amdgcn_readlane(M0, ln) | (u64)(u32)__builtin_amdgcn_readlane(M1, ln) << 32;
      while (mm != 0) {
        const u32 bit = __builtin_ctzll(mm), hf = bit >> 5, by = (bit >> 3) & 3, i = hf * 8 + 7 - (bit & 7);
        mm &= mm - 1;
        c0 += hf ^ 1; c1 += hf;
        u32 hw = 0;
#pragma unroll
        for (int z = 0; z < 16; ++z) hw |= H[z] & (u32)__builtin_amdgcn_sbfe(1 << i, z, 1);
        const u32 hb = ((u32)__builtin_amdgcn_readlane(hw, ln) >> (8 * by)) & 0x7f;
        const u32 ent = (u32)ln | (i & 7) << 6 | by << 9 | ((((hb - base) & 7) + base) ^ hb) << 11;
        if (lane == (hf ? 7 - c1 : c0)) meta = ent;
      }
    }
  }
  const bool raw = c0 + c1 > 6;
  if (lane == 0) meta = base | (raw ? 0x400u : (u32)(c0 << 11 | c1 << 14));
  s[4] = raw ? u32x4{H[0], H[1], H[2], H[3]} : u32x4{N[0], N[1], N[2], N[3]};
  s[5] = raw ? u32x4{H[4], H[5], H[6], H[7]} : u32x4{N[4], N[5], N[6], N[7]};
  s[6] = u32x4{H[8], H[9], H[10], H[11]};
  s[7] = u32x4{H[12], H[13], H[14], H[15]};
#pragma unroll
  for (int j = 0; j < 8; ++j) asm volatile("" : "+v"(s[j]));
  return raw;
}

template <int N> struct Pay { u32x4 v[N]; };
__device__ __forceinline__ void pay_load(Pay<6>& P, u32 l16, const char* sb) {
  asm volatile("global_load_dwordx4 %0, %6, %7 offset:-4096 sc0 sc1\n\t"
               "global_load_dwordx4 %1, %6, %7 offset:-3072 sc0 sc1\n\t"
               "global_load_dwordx4 %2, %6, %7 offset:-2048 sc0 sc1\n\t"
               "global_load_dwordx4 %3, %6, %7 offset:-1024 sc0 sc1\n\t"
               "global_load_dwordx4 %4, %6, %7 offset:0 sc0 sc1\n\t"
               "global_load_dwordx4 %5, %6, %7 offset:1024 sc0 sc1"
               : "=&v"(P.v[0]), "=&v"(P.v[1]), "=&v"(P.v[2]), "=&v"(P.v[3]), "=&v"(P.v[4]), "=&v"(P.v[5]) : "v"(l16), "s"(sb) : "memory");
}
__device__ __forceinline__ void pay_load(Pay<3>& P, u32 vlo, u32 vcode, const char* sb) {
  asm volatile("global_load_dwordx4 %2, %4, %5 offset:0 sc0 sc1\n\t"
               "global_load_dwordx4 %0, %3, %5 offset:-4096 sc0 sc1\n\t"
               "global_load_dwordx4 %1, %3, %5 offset:-3072 sc0 sc1"
               : "=&v"(P.v[0]), "=&v"(P.v[1]), "=&v"(P.v[2]) : "v"(vlo), "v"(vcode), "s"(sb) : "memory");
}
template <int C> __device__ __forceinline__ void pay_wait(Pay<6>& P) {
  asm volatile("s_waitcnt vmcnt(%6)" : "+v"(P.v[0]), "+v"(P.v[1]), "+v"(P.v[2]), "+v"(P.v[3]), "+v"(P.v[4]), "+v"(P.v[5]) : "n"(C) : "memory");
}
template <int C> __device__ __forceinline__ void pay_wait(Pay<3>& P) {
  asm volatile("s_waitcnt vmcnt(%3)" : "+v"(P.v[0]), "+v"(P.v[1]), "+v"(P.v[2]) : "n"(C) : "memory");
}
__device__ __forceinline__ u32x8 load_two(const char* b, u32 l16) {
  u32x4 x, y;
  asm volatile("global_load_dwordx4 %0, %2, %3 sc0 sc1\n\t"
               "global_load_dwordx4 %1, %2, %3 offset:1024 sc0 sc1\n\ts_waitcnt vmcnt(0)"
               : "=&v"(x), "=&v"(y) : "v"(l16), "s"(b) : "memory");
  return __builtin_shufflevector(x, y, 0, 1, 2, 3, 4, 5, 6, 7);
}

__device__ __forceinline__ void load_slot(u32x4 (&d)[8], u32 l16, const char* sb) {
  asm volatile("global_load_dwordx4 %0, %8, %9 offset:-4096 sc0 sc1\n\tglobal_load_dwordx4 %1, %8, %9 offset:-3072 sc0 sc1\n\t"
               "global_load_dwordx4 %2, %8, %9 offset:-2048 sc0 sc1\n\tglobal_load_dwordx4 %3, %8, %9 offset:-1024 sc0 sc1\n\t"
               "global_load_dwordx4 %4, %8, %9 offset:0 sc0 sc1\n\tglobal_load_dwordx4 %5, %8, %9 offset:1024 sc0 sc1\n\t"
               "global_load_dwordx4 %6, %8, %9 offset:2048 sc0 sc1\n\tglobal_load_dwordx4 %7, %8, %9 offset:3072 sc0 sc1\n\ts_waitcnt vmcnt(0)"
               : "=&v"(d[0]), "=&v"(d[1]), "=&v"(d[2]), "=&v"(d[3]), "=&v"(d[4]), "=&v"(d[5]), "=&v"(d[6]), "=&v"(d[7])
               : "v"(l16), "s"(sb) : "memory");
}

template <bool ask>
__device__ __forceinline__ bool sender(KP p, char* smem, char* myws, int wv, int lane, u32& bid, u32& call) {
  const u32 rank = p->rank;
  const u32 cb = bid - NGEMM;
  const u32 link = cb & 7, cw = cb >> 3;
  const u32 l16 = lane * 16;
  if (link != 0) {
    const u32 dest = peer_of_xcd(rank, link);
    const bool intra = link < 4;
    const int n = ((intra ? N_INTRA : N_INTER) - (int)cw + NCWG - 1) / NCWG;
    auto code_of = [=](int k) {
      const u32 e = cw + (u32)(k < n ? k : 0) * NCWG;
      const u32 word = ((const __attribute__((address_space(4))) u32*)p->ord[intra ? 0 : 1])[e >> 1];
      return (word >> (16 * (e & 1))) & 0xffffu;
    };
    LAS int* const done = (LAS int*)(smem + 12 * 8192);
    u32x4 d[8];
    auto fetch = [&](u32 c) {
      const bool rly = c >= 20 * 32;
      load_raw(p, d, lane, rly ? RB : c >> 5, dest & 3, rly ? RT0 + 4 * (c & 7) + (dest & 3) : c & 31);
    };
    u32 code = code_of(wv);
    LAS u32* const got = (LAS u32*)(smem + CLAIM_OFF + 16);
    if (wv >= 4) {
      if (threadIdx.x == 256) { *done = 0; *got = 0; }
      WG_BARRIER();
      if (wv >= 8) __builtin_amdgcn_s_sleep(16);
    }
    fetch(code);
    u32 ticket = 0;
    if (ask) asm volatile("global_load_dword %0, %1, %2 sc1" : "=&v"(ticket) : "v"(0u), "s"(counter(myws)) : "memory");
    if (wv < 4) WG_BARRIER();
    bool lost = false;
    char* const rws = p->ws[dest];
    if (wv < 4) __builtin_amdgcn_s_setprio(2); else if (wv < 8) __builtin_amdgcn_s_setprio(1);
    int k = wv;
    for (; k < n; k += 12) {
      const u32 sl = code >> 5;
      u32 ud = sl < 20 ? dest_stream(sl, rank, dest, 0) * NSTEP + (code & 31) : sl == 20 ? S_RLY * NSTEP + (rank & 3) * 8 + (code & 7)
             : (S_2 + (sl - 21) * N2 + RB - N1) * NSTEP + RT0 + 4 * (code & 7) + (rank & 3);
      asm volatile("" : "+s"(ud));
      WAIT_RAW(d);
      if (ask && k < 24) {
        if (k < 12) {
          asm volatile("" : "+v"(ticket));
          call = __builtin_amdgcn_readfirstlane(ticket) + 1u;
          if (wv == 11) __hip_atomic_store(got, take_role(myws, bid, call) + 1u, __ATOMIC_RELAXED, __HIP_MEMORY_SCOPE_WORKGROUP);
        } else {
          u32 g;
          while ((g = __builtin_amdgcn_readfirstlane(__hip_atomic_load(got, __ATOMIC_RELAXED, __HIP_MEMORY_SCOPE_WORKGROUP))) == 0) __builtin_amdgcn_s_sleep(1);
          if (g - 1 != bid) { bid = g - 1; lost = true; break; }
        }
      }
      ud += (call & 1) * (NS_PAR * NSTEP);
      char* const b = rws + (size_t)ud * SLOT + 4096;
      u32x4 s[8];
      u32 meta;
      bool raw;
      if (P_RLY > 0 && sl > 20) {
        const u32 us = ((call & 1) * NS_PAR + S_RLY) * NSTEP + ((dest + sl - 20) & 3) * 8 + (code & 7);
        const u64 t0 = __builtin_amdgcn_s_memrealtime();
        const u32* const hd = (const u32*)(myws + meta_off(us)) + (lane & 7);
        for (u32 it = 1;; ++it) {
          meta = __hip_atomic_load(hd, __ATOMIC_RELAXED, __HIP_MEMORY_SCOPE_SYSTEM);
          if (hdr_there(meta, call, lane)) break;
          if ((it & 31) == 0 && spin_fail(myws, call, t0)) break;
          __builtin_amdgcn_s_sleep(2);
        }
        load_slot(s, l16, myws + (size_t)us * SLOT + 4096);
        raw = __builtin_amdgcn_readfirstlane(meta) & 0x400;
      } else {
        to_fragments(d, sl >= 16 && sl < 20, smem + wv * 8192, lane);
        low_slabs(d, s);
        if (k < P_WIN0) { ST16(l16, s[0], b, -4096); ST16(l16, s[1], b, -3072); ST16(l16, s[2], b, -2048); ST16(l16, s[3], b, -1024); }
        raw = encode(d, s, meta, lane);
        meta = hdr_word(meta, call, lane);
      }
      code = code_of(k + 12);
      __builtin_amdgcn_s_setprio(0);
      const int need = k < P_WIN0 ? 0 : k + 1 - max(P_WIN, P_WIN1 - (k - P_WIN0) / P_TAPER);
      while (__builtin_amdgcn_readfirstlane(__hip_atomic_load(done, __ATOMIC_RELAXED, __HIP_MEMORY_SCOPE_WORKGROUP)) < need) __builtin_amdgcn_s_sleep(1);
      if (k >= P_WIN0) { ST16(l16, s[0], b, -4096); ST16(l16, s[1], b, -3072); ST16(l16, s[2], b, -2048); ST16(l16, s[3], b, -1024); }
      ST16(l16, s[4], b, 0); ST16(l16, s[5], b, 1024);
      if (raw) { ST16(l16, s[6], b, 2048); ST16(l16, s[7], b, 3072); }
      asm volatile("s_waitcnt vmcnt(0)" ::: "memory");
      char* const m = rws + meta_off(ud);
      if (lane < 8) ST4(lane * 4, meta, m);
      if (lane == 0) __hip_atomic_fetch_add(done, 1, __ATOMIC_RELAXED, __HIP_MEMORY_SCOPE_WORKGROUP);
      fetch(code);
    }
    WAIT_RAW(d);
    if (lost) WG_BARRIER();
    if (ask && threadIdx.x == 0) __hip_atomic_store(counter(myws), call, __ATOMIC_RELAXED, __HIP_MEMORY_SCOPE_AGENT);
    if (lost) { __builtin_amdgcn_s_setprio(0); return false; }
    const u32 y = cw + (u32)(k - n) * NCWG;
    if (!intra && y < 2u * P_PARK) {
      const u32 us = (S_PARK + (dest & 3)) * NSTEP + y;
      const u64 t0 = __builtin_amdgcn_s_memrealtime();
      const u32* const hd = (const u32*)(myws + meta_off(us));
      for (u32 it = 1; (u32)__builtin_amdgcn_readfirstlane(__hip_atomic_load(hd, __ATOMIC_RELAXED, __HIP_MEMORY_SCOPE_SYSTEM)) < call; ++it) {
        if ((it & 31) == 0 && spin_fail(myws, call, t0)) break;
        __builtin_amdgcn_s_sleep(4);
      }
      load_slot(d, l16, myws + (size_t)us * SLOT + 4096);
      if (__hip_atomic_load((u32*)(myws + TMO_OFF), __ATOMIC_RELAXED, __HIP_MEMORY_SCOPE_AGENT) == call) d[0] = u32x4{0x7fc07fc0u, 0x7fc07fc0u, 0x7fc07fc0u, 0x7fc07fc0u};
      const u32 tile = y >> 1, sid = (tile >> 1) * 4 + (y & 1) * 2;
      char* const ob = p->out[dest] + (rank & 3) * 256 + (tile & 1) * 128;
      const u32 vo = (lane >> 3) * (NLOC * 2) + (lane & 7) * 16;
      while (__builtin_amdgcn_readfirstlane(__hip_atomic_load(done, __ATOMIC_RELAXED, __HIP_MEMORY_SCOPE_WORKGROUP)) < k - (P_WIN - 1)) __builtin_amdgcn_s_sleep(1);
#pragma unroll
      for (int g = 0; g < 8; ++g) {
        char* const b = ob + (size_t)(p->row[sid + (g >> 2)] + (g & 3) * 8) * (NLOC * 2);
        ST16(vo, d[g], b, 0);
      }
      if (y + P_REL * NCWG >= 2u * P_PARK && y + P_REL * NCWG < 2u * P_PARK + NCWG && lane == 0)
        __hip_atomic_store((u32*)(myws + REL_OFF + link * 128) + cw, call, __ATOMIC_RELAXED, __HIP_MEMORY_SCOPE_AGENT);
      asm volatile("s_waitcnt vmcnt(0)" ::: "memory");
      char* const fl = rws + DONE_OFF + ((rank & 3) * 64 + ((y & 1) ? 2 * RT_MED + 12 + tile : tile)) * 4;
      if (lane == 0) { ST4(0u, call, fl); __hip_atomic_fetch_add(done, 1, __ATOMIC_RELAXED, __HIP_MEMORY_SCOPE_WORKGROUP); }
    }
    return true;
  }
  if (ask) {
    const u32 asked = bid;
    claim(smem, myws, wv, bid, call);
    if (bid != asked) return false;
  }
  if (cb == 0 && wv == 0) {
    const u64 t0 = __builtin_amdgcn_s_memrealtime();
    while (__builtin_amdgcn_s_memrealtime() - t0 < N_INTRA * P_UT / 10) __builtin_amdgcn_s_sleep(16);
    const char* const f = myws + DONE_OFF;
    const int f0 = (4 * lane) & 63;
    for (u32 n = 1;; ++n) {
      u32x4 v;
      asm volatile("global_load_dwordx4 %0, %1, %2 sc0 sc1\n\ts_waitcnt vmcnt(0)" : "=&v"(v) : "v"(l16), "s"(f) : "memory");
      if (__all((f0 >= NFLAG || v[0] >= call) && (f0 + 1 >= NFLAG || v[1] >= call) && (f0 + 2 >= NFLAG || v[2] >= call) && (f0 + 3 >= NFLAG || v[3] >= call))) break;
      if ((n & 31) == 0 && spin_fail(myws, call, t0)) {
        if (lane == 0) *(unsigned short*)(p->out[rank] + (size_t)phys_rank(rank) * 512 * (NLOC * 2)) = 0x7fc0;
        break;
      }
    }
  }
  return true;
}

__device__ __forceinline__ u32x4 join(u32 l0, u32 l1, u32 h0, u32 h1) {
  return u32x4{__builtin_amdgcn_perm(h0, l0, 0x05010400u), __builtin_amdgcn_perm(h0, l0, 0x07030602u),
               __builtin_amdgcn_perm(h1, l1, 0x05010400u), __builtin_amdgcn_perm(h1, l1, 0x07030602u)};
}
__device__ __forceinline__ void fix_word(u32 (&h)[8], u32 e) {
  asm volatile("s_lshl_b64 exec, 1, %8\n\ts_getpc_b64 vcc\n\ts_add_u32 vcc_lo, vcc_lo, %9\n\ts_addc_u32 vcc_hi, vcc_hi, 0\n\ts_setpc_b64 vcc\n\t"
               "v_xor_b32_e32 %0, %10, %0\n\ts_branch 1f\n\tv_xor_b32_e32 %1, %10, %1\n\ts_branch 1f\n\t"
               "v_xor_b32_e32 %2, %10, %2\n\ts_branch 1f\n\tv_xor_b32_e32 %3, %10, %3\n\ts_branch 1f\n\t"
               "v_xor_b32_e32 %4, %10, %4\n\ts_branch 1f\n\tv_xor_b32_e32 %5, %10, %5\n\ts_branch 1f\n\t"
               "v_xor_b32_e32 %6, %10, %6\n\ts_branch 1f\n\tv_xor_b32_e32 %7, %10, %7\n"
               "1:\n\ts_mov_b64 exec, -1"
               : "+v"(h[0]), "+v"(h[1]), "+v"(h[2]), "+v"(h[3]), "+v"(h[4]), "+v"(h[5]), "+v"(h[6]), "+v"(h[7])
               : "s"(e), "s"(((e >> 3) & 56) + 12), "s"(((e >> 11) & 0x7f) << ((e >> 6) & 24)) : "vcc", "scc");
}

struct Stream {
  const char* sb; const char* mb; char* myws;
  u32 l16, tag, call; u64 t0;
  u32x4 W; int rdy; bool eager;
  __device__ __forceinline__ void init(char* ws, u32 call_, u32 sid, int lane) {
    myws = ws; call = call_; tag = call_tag(call_);
    sb = ws + (size_t)sid * NSTEP * SLOT + 4096;
    mb = ws + meta_off(sid * NSTEP);
    l16 = lane * 16;
    t0 = __builtin_amdgcn_s_memrealtime();
    W = u32x4{0u, 0u, 0u, 0u}; rdy = 0; eager = false;
  }
  __device__ __forceinline__ void poll(bool wide = true) {
    const u64 mask = (wide ? 0xffffull : 0xffull) << (rdy < 28 ? (2 * rdy) & 56 : 56);
    asm volatile("s_mov_b64 exec, %3\n\tglobal_load_dwordx4 %0, %1, %2 sc0 sc1\n\ts_mov_b64 exec, -1"
                 : "+v"(W) : "v"(l16), "s"(mb), "s"(mask) : "memory");
  }
  __device__ __forceinline__ void look() {
    asm volatile("" : "+v"(W));
    const u32 x = (W[0] ^ tag) | (W[1] ^ tag) | (W[2] ^ tag);
    const u64 odd = 0xaaaaaaaaaaaaaaaaull;
    const u64 bad = __ballot(x >= (1u << 21)) | (__ballot((W[3] ^ tag) >= (1u << 21)) & ~odd) | (__ballot(W[3] < call) & odd);
    rdy = bad ? __builtin_ctzll(bad) >> 1 : NSTEP;
  }
  __device__ __forceinline__ void spin(int t) {
    for (u32 n = 1; rdy <= t; ++n) {
      const int was = rdy;
      poll(n == 1);
      asm volatile("s_waitcnt vmcnt(0)" ::: "memory");
      look();
      if (rdy == was) {
        if (eager) __builtin_amdgcn_s_sleep(P_S2);
        else if (rdy == 0) __builtin_amdgcn_s_sleep(P_S0);
        else __builtin_amdgcn_s_sleep(P_S1);
      }
      if ((n & 31) == 0 && spin_fail(myws, call, t0)) rdy = NSTEP;
    }
  }
};

__device__ __forceinline__ void repair(u32 (&h)[8], u32 hd, u32 hdr, u32 hf, const char* sb, u32 l16) {
#pragma nounroll
  for (u32 n = (hdr >> (11 + 3 * hf)) & 7, k = hf ? 6 : 1; n != 0; --n, k += hf ? -1 : 1) fix_word(h, __builtin_amdgcn_readlane(hd, k));
  if (__builtin_expect(hdr & 0x400u, 0)) {
    const u32x8 g = load_two(sb + hf * 2048, l16);
#pragma unroll
    for (int j = 0; j < 8; ++j) asm volatile("v_mov_b32 %0, %1" : "+v"(h[j]) : "v"(g[j]));
  }
}

__device__ __forceinline__ void unpack_unit(char* myws, int lane, u32 call, u32 r, int t, u64 tk) {
  Stream S;
  S.init(myws, call, (call & 1) * NS_PAR + S_WIN + r, lane);
  const u32 l4 = lane * 4;
  const u32 uo = (S_IMG + 16 + r) * NSTEP;
  {
    if (r < 16) while (__builtin_amdgcn_s_memrealtime() - tk < (u32)(20 + 4 * t) * P_UT / 10) __builtin_amdgcn_s_sleep(4);
    u32 hd;
    for (u32 n = 1;; ++n) {
      hd = __hip_atomic_load((const u32*)(S.mb + t * 32) + (lane & 7), __ATOMIC_RELAXED, __HIP_MEMORY_SCOPE_SYSTEM);
      if (hdr_there(hd, call, lane)) break;
      if ((n & 31) == 0 && spin_fail(myws, call, S.t0)) break;
      __builtin_amdgcn_s_sleep(4);
    }
    Pay<6> P;
    pay_load(P, S.l16, S.sb + (size_t)t * SLOT);
    pay_wait<0>(P);
    const u32 hdr = __builtin_amdgcn_readfirstlane(hd), base4 = (hdr & 0x7f) * 0x01010101u;
    u32x4 o[8];
#pragma unroll
    for (int hf = 0; hf < 2; ++hf) {
      u32 g[8];
#pragma unroll
      for (int j = 0; j < 4; ++j) {
        const u32 n = P.v[4 + hf][j];
        g[2 * j] = (n & 0x87878787u) + base4;
        g[2 * j + 1] = (__builtin_amdgcn_alignbit(n, n, 4) & 0x87878787u) + base4;
      }
      if (hdr & (0x400u | 0x3800u << (3 * hf))) repair(g, hd, hdr, hf, S.sb + (size_t)t * SLOT, S.l16);
#pragma unroll
      for (int j = 0; j < 4; ++j)
        o[4 * hf + j] = join(P.v[2 * hf + (j >> 1)][2 * (j & 1)], P.v[2 * hf + (j >> 1)][2 * (j & 1) + 1], g[2 * j], g[2 * j + 1]);
    }
    char* const ob = myws + (size_t)(uo + t) * SLOT + 4096;
    ST_SIX(S.l16, o, ob); ST16(S.l16, o[6], ob, 2048); ST16(S.l16, o[7], ob, 3072);
    asm volatile("s_waitcnt vmcnt(0)" ::: "memory");
    char* const mo = myws + meta_off((uo + t));
    if (lane < 8) ST4(l4, hdr_word(0, call, lane), mo);
  }
}

__device__ __forceinline__ float silu(float y) {
  const float z = __builtin_amdgcn_fmed3f(y, -60.0f, 60.0f);
  const float e = __builtin_amdgcn_exp2f(z * -1.44269504088896341f);
  return y * __builtin_amdgcn_rcpf(1.0f + e);
}

__device__ __forceinline__ void gemm_tile(KP p, char* smem, char* myws, int wv, int lane, u32 bid, u32 call, u32 nb, u32 sid0, u32 tn, u32 ch, u32 fl, int hold, u64 tk, bool next, u32*& owed) {
  const u32 rank = p->rank;
  if (wv >= 8) __builtin_amdgcn_s_setprio(P_LPRIO);
  const u32 o0 = (rank & 4u) ^ 4u;
  const bool remote = tn >= 4;
  const u32 dst = (remote ? o0 + (tn - 4) : rank) & 7u;
  char* const ob = p->out[dst] + (remote ? (rank & 3) : tn) * 256 + ch * 128;
  u32* flag = (u32*)(p->ws[dst] + DONE_OFF) + (rank & 3) * 64 + fl;
  auto grow = [&](u32 rg) { return p->row[sid0 + (rg >> 2)] + (rg & 3) * 8; };
  char* b0 = ob + (size_t)grow(wv & 7) * (NLOC * 2);
  char* b1 = ob + (size_t)grow((wv & 7) + 8) * (NLOC * 2);
  if (hold == 2) {
    const u32 us = (S_PARK + (tn & 3)) * NSTEP + 2 * fl;
    b0 = myws + (size_t)us * SLOT + (wv & 7) * 1024; b1 = b0 + SLOT;
    flag = (u32*)(myws + meta_off(us));
  }
  asm volatile("" : "+s"(b0), "+s"(b1), "+s"(flag));
  if (nb == 0) {
    if (threadIdx.x == 0) ST4(0u, call, flag);
    return;
  }
  {
    const u32 first = sid0 < (u32)S_2 || sid0 >= (u32)S_OWN2 ? 0 : sid0 < (u32)S_INTER ? N1 * NSTEP : sid0 < (u32)S_3 ? 4 * NSTEP : N_INTRA - N3 * NSTEP;
    while (__builtin_amdgcn_s_memrealtime() - tk < first * P_UT / 10) __builtin_amdgcn_s_sleep(8);
  }
  Stream S;
  const u32 m = wv >> 1, kh = wv & 1;
  u32 tmo = 0;
  auto read_tmo = [&]() { asm volatile("global_load_dword %0, %1, %2 sc1" : "=&v"(tmo) : "v"(0u), "s"(myws + TMO_OFF) : "memory"); };
  if (wv >= 8) {
    const u32 un = wv - 8;
    S.init(myws, call, S_IMG + 4 * tn + un, lane);
    const u32 lds0 = (u32)(size_t)(LAS char*)smem + un * 4096;
    const char* const g0 = S.sb - 4096 + ch * 4096;
    Stream A;
    A.init(myws, call, (call & 1) * NS_PAR + sid0 + un, lane);
    A.eager = sid0 >= (u32)S_3 && sid0 < (u32)S_OWN2;
    asm volatile("global_load_dwordx4 %0, %2, %3 sc0 sc1\n\tglobal_load_dwordx4 %1, %2, %4 sc0 sc1\n\ts_waitcnt vmcnt(0)"
                 : "+v"(A.W), "+v"(S.W) : "v"(S.l16), "s"(A.mb), "s"(S.mb) : "memory");
    int is = 0, seen = ((nb >> un) & 1) ? 0 : NSTEP;
    for (int t = -1; t < NSTEP; ++t) {
      if (seen < NSTEP && seen <= t + P_DIST + P_ALOOK) {
        A.look();
        A.spin(min(t + P_DIST, NSTEP - 1));
        if (A.rdy != seen) { *(LAS u32x4*)((LAS char*)smem + HDR_OFF + un * 1024 + A.l16) = A.W; seen = A.rdy; }
        asm volatile("" ::: "memory");
        if (seen < NSTEP) A.poll();
      }
      const int lim = min(min(NSTEP, is + 2), t + P_BD);
      if (S.rdy < min(NSTEP, lim + 2)) { S.look(); S.spin(t); if (S.rdy < NSTEP) S.poll(); }
      for (; is < min(lim, S.rdy); ++is) {
        const char* g = g0 + (size_t)is * SLOT;
        const u32 l = lds0 + (is & 7) * 16384;
        u32 keep;
        asm volatile("s_mov_b32 %0, m0\n\ts_mov_b32 m0, %3\n\ts_nop 0\n\t"
                     "global_load_lds_dwordx4 %1, %2 sc0 sc1\n\tglobal_load_lds_dwordx4 %1, %2 offset:1024 sc0 sc1\n\t"
                     "global_load_lds_dwordx4 %1, %2 offset:2048 sc0 sc1\n\tglobal_load_lds_dwordx4 %1, %2 offset:3072 sc0 sc1\n\t"
                     "s_mov_b32 m0, %0" : "=&s"(keep) : "v"(S.l16), "s"(g), "s"(l) : "memory");
      }
      const int ahead = is - 1 - t;
      if (ahead >= 6 || t < 0) asm volatile("s_waitcnt vmcnt(24)" ::: "memory");
      else if (ahead >= 4) { if (ahead == 5) asm volatile("s_waitcnt vmcnt(20)" ::: "memory"); else asm volatile("s_waitcnt vmcnt(16)" ::: "memory"); }
      else if (ahead >= 2) { if (ahead == 3) asm volatile("s_waitcnt vmcnt(12)" ::: "memory"); else asm volatile("s_waitcnt vmcnt(8)" ::: "memory"); }
      else if (ahead == 1) asm volatile("s_waitcnt vmcnt(4)" ::: "memory");
      else asm volatile("s_waitcnt vmcnt(0)" ::: "memory");
      WG_BARRIER();
    }
    asm volatile("" : "+v"(S.W), "+v"(A.W));
    __builtin_amdgcn_s_setprio(0);
    if (hold == 1 && wv == 8) {
      const u32* const r = (const u32*)(myws + REL_OFF + (bid & 7) * 128) + (lane < NCWG ? lane : 0);
      for (u32 n = 1; !__all(__hip_atomic_load(r, __ATOMIC_RELAXED, __HIP_MEMORY_SCOPE_AGENT) >= call); ++n) {
        if ((n & 31) == 0 && spin_fail(myws, call, S.t0)) break;
        __builtin_amdgcn_s_sleep(4);
      }
    }
  } else if (!((nb >> m) & 1)) {
    asm volatile("s_waitcnt vmcnt(0)" ::: "memory");
    for (int t = -1; t < NSTEP; ++t) WG_BARRIER();
    read_tmo();
    asm volatile("s_waitcnt vmcnt(0)" : "+v"(tmo) :: "memory");
  } else {
    const char* const sb = myws + (size_t)((call & 1) * NS_PAR + sid0 + m) * NSTEP * SLOT + 4096;
    const u32 l16 = lane * 16, vlo = l16 + kh * 2048, vcode = l16 + kh * 1024;
    u32 ho = HDR_OFF + m * 1024 + (lane & 7) * 4, hd;
    const u32 rmask = 0x400u | 0x3800u << (3 * kh);
    f32x16 acc[2] = {};
    u32 bb = kh * 8192 + lane * 16;
    bf16x8 wf[4];
    u32x4 X[2][4];
    Pay<3> P[4];
    u32 h[8];
    u32 hdr = 0, base4 = 0;
    auto rd = [&](int stage, int f) {
      wf[f & 3] = *(const LAS bf16x8*)((LAS char*)smem + bb + stage * 16384 + (f >> 2) * 4096 + ((f >> 1) & 1) * 1024 + (f & 1) * 2048);
    };
    auto piece = [&](int k, int t, int ts, int waitn, bool fetch) {
      Pay<3>& Q = P[ts];
      if (k == 0) {
        if (waitn == 3) pay_wait<9>(Q); else if (waitn == 2) pay_wait<6>(Q); else if (waitn == 1) pay_wait<3>(Q); else pay_wait<0>(Q);
        hdr = __builtin_amdgcn_readfirstlane(hd);
        base4 = (hdr & 0x7f) * 0x01010101u;
      } else if (k < 6) {
#pragma unroll
        for (int j = 0; j < 4; ++j) {
          const int i = k < 4 ? 2 * j + (k & 1) : 4 * (k - 4) + j;
          if (k == 1) h[i] = __builtin_amdgcn_alignbit(Q.v[2][j], Q.v[2][j], 4);
          else if (k == 2) h[i] = Q.v[2][j] & 0x87878787u;
          else if (k == 3) h[i] &= 0x87878787u;
          else h[i] += base4;
          asm volatile("" : "+v"(h[i]));
        }
      } else if (k == 6) {
        if (hdr & rmask) repair(h, hd, hdr, kh, sb + (size_t)t * SLOT, l16);
      } else if (k == 7) {
      } else if (k < 12) {
        const int c = k - 8;
        X[ts & 1][c] = join(Q.v[c >> 1][2 * (c & 1)], Q.v[c >> 1][2 * (c & 1) + 1], h[2 * c], h[2 * c + 1]);
        asm volatile("" : "+v"(X[ts & 1][c]));
      } else if (k == 12) {
        ho += 32;
        hd = *(const LAS u32*)((LAS char*)smem + ho);
      } else if (k == 13 && fetch) {
        const int tn4 = t + P_DIST < NSTEP ? t + P_DIST : NSTEP - 1;
        pay_load(P[(ts + P_DIST) & 3], vlo, vcode, sb + (size_t)tn4 * SLOT);
      }
    };
    auto step = [&](int t, int ts, int waitn, bool fetch, bool more) {
#pragma unroll
      for (int q = 0; q < 8; ++q) {
        if (q == 4 && more) { WG_BARRIER(); if (ts == 3) bb ^= 65536u; }
        else if (q % 2 == 0 && (more || q < 6)) __builtin_amdgcn_s_waitcnt(0xC27F);
        __builtin_amdgcn_sched_barrier(0);
        acc[q & 1] = __builtin_amdgcn_mfma_f32_32x32x16_bf16(wf[q & 3], __builtin_bit_cast(bf16x8, X[ts & 1][q >> 1]), acc[q & 1], 0, 0, 0);
        __builtin_amdgcn_sched_barrier(0);
        if (q < 4) rd(ts, q + 4);
        else if (more) rd((ts + 1) & 3, q - 4);
        if (more) {
          piece(2 * q, t + 1, (ts + 1) & 3, waitn, fetch);
          piece(2 * q + 1, t + 1, (ts + 1) & 3, waitn, fetch);
        }
      }
    };
    asm volatile("s_waitcnt vmcnt(0)" ::: "memory");
    WG_BARRIER();
    if (owed && threadIdx.x < 2) ST4(threadIdx.x * 32, call, owed);
#pragma unroll
    for (int d = 0; d < P_DIST; ++d) pay_load(P[d], vlo, vcode, sb + (size_t)d * SLOT);
    hd = *(const LAS u32*)((LAS char*)smem + ho);
#pragma unroll
    for (int k = 0; k < 13; ++k) piece(k, 0, 0, P_DIST - 1, true);
    WG_BARRIER();
    piece(13, 0, 0, P_DIST - 1, true);
#pragma unroll
    for (int f = 0; f < 4; ++f) { rd(0, f); __builtin_amdgcn_sched_barrier(0); }
    for (int tb = 0; tb < NSTEP - 4; tb += 4) {
#pragma unroll
      for (int ts = 0; ts < 4; ++ts) step(tb + ts, ts, P_DIST - 1, true, true);
    }
    step(NSTEP - 4, 0, P_DIST - 1, false, true);
    step(NSTEP - 3, 1, P_DIST - 2, false, true);
    step(NSTEP - 2, 2, P_DIST - 3, false, true);
    read_tmo();
    step(NSTEP - 1, 3, 0, false, false);
    char* const xq = smem + EPI_OFF + (kh * 128 + m * 32 + (lane & 31)) * ERS + (lane >> 5) * 16;
#pragma unroll
    for (int n = 0; n < 2; ++n)
#pragma unroll
      for (int g = 0; g < 4; ++g)
        *(f32x4*)(xq + n * 128 + g * 32) = f32x4{acc[n][4 * g], acc[n][4 * g + 1], acc[n][4 * g + 2], acc[n][4 * g + 3]};
    if (P_DIST == 4) pay_wait<0>(P[0]);
    asm volatile("s_waitcnt vmcnt(0)" : "+v"(tmo) :: "memory");
  }
  WG_BARRIER();
  if (wv < 8) {
    const u32 vo = hold == 2 ? lane * 16 : (lane >> 3) * (NLOC * 2) + (lane & 7) * 16;
    auto get = [&](u32 rg) {
      const char* const q = smem + EPI_OFF + (rg * 8 + (lane >> 3)) * ERS + (lane & 7) * 32;
      const f32x4 a = *(const f32x4*)q + *(const f32x4*)(q + 128 * ERS), c = *(const f32x4*)(q + 16) + *(const f32x4*)(q + 128 * ERS + 16);
      auto pk = [](float lo, float hi) { const f32x2 v = {silu(lo), silu(hi)}; return __builtin_bit_cast(u32, __builtin_convertvector(v, bf16x2)); };
      u32x4 v = {pk(a[0], a[1]), pk(a[2], a[3]), pk(c[0], c[1]), pk(c[2], c[3])};
      if (tmo == call) v = u32x4{0x7fc07fc0u, 0x7fc07fc0u, 0x7fc07fc0u, 0x7fc07fc0u};
      return v;
    };
    if ((nb >> (wv >> 2)) & 1) { const u32x4 v0 = get(wv); ST16(vo, v0, b0, 0); }
    if ((nb >> (wv >> 2)) & 4) { const u32x4 v1 = get(wv + 8); ST16(vo, v1, b1, 0); }
  }
  owed = nullptr;
  if (next) { if (remote) owed = flag; }
  else if (remote) {
    asm volatile("s_waitcnt vmcnt(0)" ::: "memory");
    WG_BARRIER();
    if (threadIdx.x == 0) ST4(0u, call, flag);
  }
}

inline bool tile_of(u32 rank, u32 b, int k, u32& nb, u32& sid0, u32& tn, u32& ch, u32& fl, int& hold) {
  const u32 peer = peer_of_xcd(rank, b & 7), i = b >> 3;
  const bool home = !((peer ^ rank) & 4), spare = i >= 2 * RT_MED;
  hold = 0;
  constexpr u32 I0 = S_INTER / 2;
  const bool extra = home && i >= I0 && i < I0 + (NB + 1) / 2 * 2;
  if (extra && k == 0) {
    const u32 j = i - I0;
    nb = (j | 1) < (u32)NB ? 3 : 1; tn = peer & 3; sid0 = S_OWN2 + (j >> 1) * 2; ch = j & 1; fl = 0;
    return true;
  }
  if (k == (extra ? 1 : 0) && !spare) {
    nb = 15; tn = (peer & 3) + (home ? 0 : 4); sid0 = (i >> 1) * 4; ch = i & 1; fl = i;
    hold = home || sid0 >= (u32)S_INTER ? 0 : i < (u32)P_PARK ? 2 : i >= 10 ? 1 : 0;
    if (!home)
      for (u32 m = 0; m < 4; ++m)
        if (sid0 + m >= (u32)S_INTER && (sid0 + m - S_INTER) / NB == (peer & 3)) nb &= ~(1u << m);
    return true;
  }
  const u32 j = spare ? i - 2 * RT_MED + 10 : i;
  nb = 3; tn = (peer & 3) + (home ? 4 : 0); sid0 = RT_MED * 4 + (j >> 1) * 2; ch = j & 1; fl = 2 * RT_MED + j;
  return spare ? k == 0 : i < 10;
}

constexpr u32 NHOME = 32 * NSTEP;
__host__ __device__ inline u32 pack_index(u32 b, u32 wv) {
  constexpr u32 NP1 = 80;
  return b < NP1 ? (wv < 2 ? b + NP1 * wv : NHOME) : 2 * NP1 + (b - NP1) + (NGEMM - NP1) * wv;
}

constexpr u32 UB0 = RT_MED * 16, UNW = 16;
__host__ __device__ inline u32 unpack_index(u32 b, u32 wv) {
  return b >= UB0 && b < UB0 + UNW ? (11 - wv) * UNW + (b - UB0) : 512u;
}

__global__ void __launch_bounds__(NTHREAD, 3) ag_gemm_silu(const Params) {
  KP p = (KP)__builtin_amdgcn_kernarg_segment_ptr();
  __shared__ __attribute__((aligned(16))) char smem[LDS_BYTES];
  const int lane = threadIdx.x & 63;
  const int wv = __builtin_amdgcn_readfirstlane(threadIdx.x >> 6);
  char* const myws = p->myws;
  const u64 tk = __builtin_amdgcn_s_memrealtime();
  u32 bid = (((blockIdx.x & ~7u) | ((__builtin_amdgcn_s_getreg((3 << 11) | 20)) & 7u)) + NGEMM) % GRID, call = 0;
  if (bid >= (u32)NGEMM) { if (sender<true>(p, smem, myws, wv, lane, bid, call)) return; }
  else claim(smem, myws, wv, bid, call);
  bid = __builtin_amdgcn_readfirstlane(bid); call = __builtin_amdgcn_readfirstlane(call);
  if (__builtin_expect(bid >= (u32)NGEMM, 0)) { sender<false>(p, smem, myws, wv, lane, bid, call); return; }
  const u32 u = pack_index(bid, wv);
  if (u < NHOME) {
    const u32 t = u >> 5, xc = u & 7, sub = (u >> 3) & 3;
    const u32 c = xc < 4 ? sub * 4 + xc : 16 + (peer_of_xcd(p->rank, xc) & 3) * 4 + sub;
    const bool isw = c >= 16;
    const u32 cw = c - 16, l16 = lane * 16;
    while (__builtin_amdgcn_s_memrealtime() - tk < t * P_PP) __builtin_amdgcn_s_sleep(2);
    u32x4 d[8], s[8];
    load_raw(p, d, lane, isw ? 16 + (cw & 3) : c, (cw >> 2) & 3, t);
    WAIT_RAW(d);
    u32 meta = 0;
    bool eight = true;
    to_fragments(d, isw, smem + SCR_OFF + wv * 8192, lane);
    if (isw) {
#pragma unroll
      for (int j = 0; j < 8; ++j) s[j] = d[j];
    } else { low_slabs(d, s); eight = encode(d, s, meta, lane); }
    const u32 ud = (isw ? S_IMG + cw : (call & 1) * NS_PAR + (c < (u32)NA ? c : S_OWN2 - NA + c)) * NSTEP + t;
    char* const b = myws + (size_t)ud * SLOT + 4096;
    ST_SIX(l16, s, b);
    if (eight) { ST16(l16, s[6], b, 2048); ST16(l16, s[7], b, 3072); }
    asm volatile("s_waitcnt vmcnt(0)" ::: "memory");
    char* const m = myws + meta_off(ud);
    if (lane < 8) ST4(lane * 4, hdr_word(meta, call, lane), m);
  }
#pragma nounroll
  for (u32 v = unpack_index(bid, wv); v < 512; v += 12 * UNW) unpack_unit(myws, lane, call, (v + P_UROT(v >> 4)) & 15, v >> 4, tk);
  WG_BARRIER();
  u32* owed = nullptr;
#pragma nounroll
  for (int k = 0; k < 2; ++k) {
    const u32 td = p->tile[bid][k];
    if (!(td >> 23)) break;
    gemm_tile(p, smem, myws, wv, lane, bid, call, td & 15, (td >> 4) & 127, (td >> 11) & 7, (td >> 14) & 1, (td >> 15) & 63, (td >> 21) & 3, tk, k == 0 && p->tile[bid][1] != 0, owed);
  }
}

static void build_orders(Params& p) {
  struct E { double tau; unsigned short code; };
  E e[N_ORD];
  double sigma = P_SIGMA, ovl = P_OVL, ovl2 = P_OVL2, lead = P_LEAD, f1 = 14.5, f2 = 6, stg = P_STG;
  for (int kind = 0; kind < 2; ++kind) {
    int n = 0;
    for (int t = 0; t < NSTEP; ++t) {
      if (kind == 0) {
        for (int b = 0; b < NA; ++b) if (b != RB || t < RT0) e[n++] = {t + (b < N1 ? 0 : b < N1 + N2 ? NSTEP - ovl : 2 * NSTEP - ovl - ovl2 + (b >= N1 + N2 + 2 ? stg : 0)), (unsigned short)(b << 5 | t)};
      } else {
        for (int i = 0; i < 4; ++i) e[n++] = {t * sigma - lead, (unsigned short)((16 + i) << 5 | t)};
        for (int b = NA; b < 16; ++b) e[n++] = {t + 0.5, (unsigned short)(b << 5 | t)};
      }
    }
    if (kind == 1)
      for (int j = 0; j < P_RLY; ++j) {
        e[n++] = {f1 * sigma - lead, (unsigned short)(20 << 5 | j)};
        for (int i = 0; i < 3; ++i) e[n++] = {f2 + j, (unsigned short)((21 + i) << 5 | j)};
      }
    std::stable_sort(e, e + n, [](const E& a, const E& b) { return a.tau < b.tau; });
    for (int i = 0; i < N_ORD; ++i) p.ord[kind][i] = i < n ? e[i].code : 0;
  }
}

extern "C" void kernel_launch(int rank, int nranks, ncclComm_t comm, hipStream_t stream,
                              void* const* d_in, const int* in_sizes, int n_in,
                              void* d_out, int out_size, const int* in_dtypes, int out_dtype,
                              void* const* d_out_all, void* const* d_ws_all, size_t ws_bytes) {
  (void)comm; (void)in_dtypes; (void)out_dtype; (void)d_out;
  if (nranks != NR || (unsigned)rank >= (unsigned)NR || n_in != 2 || ws_bytes < WS_NEED ||
      in_sizes[0] != 512 * KDIM || in_sizes[1] != KDIM * NLOC || out_size != 4096 * NLOC) {
    fprintf(stderr, "unexpected problem shape\n");
    return;
  }
  Params p = {};
  p.x = (const char*)d_in[0];
  p.w = (const char*)d_in[1];
  for (int i = 0; i < NR; ++i) {
    p.out[i] = (char*)d_out_all[phys_rank(i)]; p.ws[i] = (char*)d_ws_all[phys_rank(i)];
    if ((int)phys_rank(i) == rank) p.rank = i;
  }
  p.myws = (char*)d_ws_all[rank];
  build_orders(p);
  for (int i = 0; i < NS_PAR + 4; ++i) p.row[i] = i < 64 || (i >= S_OWN2 && i < NS_PAR) ? global_row(p.rank, i) : 0;
  for (u32 b = 0; b < (u32)NGEMM; ++b)
    for (int k = 0; k < 2; ++k) {
      u32 nb, sid0, tn, ch, fl; int hold;
      p.tile[b][k] = tile_of(p.rank, b, k, nb, sid0, tn, ch, fl, hold) ? nb | sid0 << 4 | tn << 11 | ch << 14 | fl << 15 | (u32)hold << 21 | 1u << 23 : 0u;
    }
  hipLaunchKernelGGL(ag_gemm_silu, dim3(GRID), dim3(NTHREAD), 0, stream, p);
  hipError_t err = hipGetLastError();
  if (err != hipSuccess) fprintf(stderr, "launch failed: %s\n", hipGetErrorString(err));
}
```

```cpp
#include <hip/hip_runtime.h>
#include <rccl/rccl.h>
#include <cstdio>
#include <cstdint>
#include <cstdlib>
#include <algorithm>

typedef unsigned u32;
typedef unsigned long long u64;
typedef unsigned u32x4 __attribute__((ext_vector_type(4)));
typedef unsigned u32x8 __attribute__((ext_vector_type(8)));
typedef unsigned short u16x2 __attribute__((ext_vector_type(2)));
typedef __bf16 bf16x2 __attribute__((ext_vector_type(2)));
typedef __bf16 bf16x4 __attribute__((ext_vector_type(4)));
typedef __bf16 bf16x8 __attribute__((ext_vector_type(8)));
typedef float f32x2 __attribute__((ext_vector_type(2)));
typedef float f32x4 __attribute__((ext_vector_type(4)));
typedef float f32x16 __attribute__((ext_vector_type(16)));
#define LAS __attribute__((address_space(3)))

#define P_NA 11
#define P_NCWG 4
#define P_WIN 3
#define P_WIN0 4
#define P_WIN1 5
#define P_TAPER 8
#define P_DIST 3
#define P_BD 7
#define P_ALOOK 3
#define P_LPRIO 2
#define P_UROT(t) (5 * (t))
#define P_STG 3
#define P_SIGMA 0.1
#define P_OVL 0.0
#define P_OVL2 0.0
#define P_LEAD 0.0
#define P_PP 15
#define P_PARK 10
#define P_RLY 2
#define P_S0 16
#define P_S1 16
#define P_S2 4
#define P_UT 100
#define P_REL 2

constexpr int NR = 8;
constexpr int KDIM = 4096, NLOC = 512;
constexpr int NSTEP = 32;
constexpr int NA = P_NA, NB = 16 - NA;
constexpr int N1 = 3, N3 = 4, N2 = NA - N1 - N3;
constexpr int S_1 = NA, S_2 = S_1 + 3 * N1, S_INTER = S_2 + 3 * N2, S_3 = 64 - 3 * N3, S_WIN = 64, S_RLY = 80, S_OWN2 = 81;
constexpr int RB = N1 + N2 - 1, RT0 = NSTEP - 4 * P_RLY;
constexpr int NS_PAR = S_OWN2 + NB, S_IMG = 2 * NS_PAR, S_PARK = S_IMG + 32, NSTREAM = S_PARK + 4;
constexpr size_t SLOT = 8192;
constexpr size_t META_OFF = (size_t)NSTREAM * NSTEP * SLOT;
constexpr size_t DONE_OFF = META_OFF + (size_t)NSTREAM * NSTEP * 32;
constexpr size_t REL_OFF = DONE_OFF + 1024;
constexpr size_t CNT_OFF = REL_OFF + 1024;
constexpr size_t TMO_OFF = CNT_OFF + 256 * 128;
constexpr size_t WS_NEED = TMO_OFF + 128;
constexpr int N_INTRA = NA * NSTEP - 4 * P_RLY, N_INTER = NB * NSTEP + 4 * NSTEP + 4 * P_RLY;
constexpr int N_ORD = N_INTRA > N_INTER ? N_INTRA : N_INTER;
constexpr int RT_MED = 13;
constexpr int NGEMM = RT_MED * 16 + 16;
constexpr int NFLAG = 2 * RT_MED + 12 + P_PARK;
constexpr int NCWG = P_NCWG;
constexpr int GRID = NGEMM + 8 * NCWG;
static_assert(P_PARK == 10 && P_BD <= 7 && P_RLY <= 8 && N_ORD % 2 == 0 && NB > 4 && NB <= 8 && S_INTER % 4 == 0, "tile map");
static_assert(GRID <= 256, "one workgroup per CU, all resident");
constexpr int NTHREAD = 768;
constexpr int SCR_OFF = 0, EPI_OFF = 32768;
constexpr int HDR_OFF = 131072;
constexpr int CLAIM_OFF = HDR_OFF + 4096 + 64;
constexpr int LDS_BYTES = CLAIM_OFF + 64;
constexpr u64 DEADLINE_TICKS = 80000000ull;
constexpr int ERS = 272;

#define P_PHYS 0x75416320u
__host__ __device__ inline u32 phys_rank(u32 l) { return (P_PHYS >> (4 * l)) & 7u; }
__host__ __device__ inline u32 peer_of_xcd(u32 rank, u32 x) { return (((rank ^ x) & 4u) | ((rank + x) & 3u)) & 7u; }

struct Params {
  const char* x;
  const char* w;
  char* myws;
  char* out[NR];
  char* ws[NR];
  int rank;
  u32 row[NS_PAR + 4];
  u32 tile[NGEMM][2];
  alignas(4) unsigned short ord[2][N_ORD];
};
static_assert(sizeof(Params) <= 4096, "kernel argument");
typedef const __attribute__((address_space(4))) Params* KP;

#define WG_BARRIER() do { asm volatile("" ::: "memory"); __builtin_amdgcn_s_waitcnt(0xC07F); \
    asm volatile("s_barrier" ::: "memory"); } while (0)
#define ST16(voff, data, sbase, imm) \
  asm volatile("global_store_dwordx4 %0, %1, %2 offset:" #imm " sc0 sc1\n\ts_nop 1" :: "v"(voff), "v"(data), "s"(sbase) : "memory")
#define ST4(voff, data, sbase) \
  asm volatile("global_store_dword %0, %1, %2 sc0 sc1" :: "v"(voff), "v"(data), "s"(sbase) : "memory")
#define ST_SIX(voff, s, b) do { ST16(voff, s[0], b, -4096); ST16(voff, s[1], b, -3072); ST16(voff, s[2], b, -2048); \
    ST16(voff, s[3], b, -1024); ST16(voff, s[4], b, 0); ST16(voff, s[5], b, 1024); } while (0)

__device__ __forceinline__ u32* counter(char* myws) { return (u32*)(myws + CNT_OFF + blockIdx.x * 128); }
__device__ __forceinline__ u32 take_role(char* myws, u32 r, u32 call) {
#pragma nounroll
  for (int i = 0; i < GRID; ++i, r = (r + 1) % GRID) {
    u32 old;
    asm volatile("s_mov_b64 exec, 1\n\tglobal_atomic_umax %0, %1, %2, %3 sc0\n\ts_mov_b64 exec, -1\n\ts_waitcnt vmcnt(0)"
                 : "=&v"(old) : "v"(r * 128), "v"(call), "s"(myws + CNT_OFF + 64) : "memory");
    if ((u32)__builtin_amdgcn_readfirstlane(old) < call) break;
  }
  return r;
}
__device__ __forceinline__ void claim(char* smem, char* myws, int wv, u32& bid, u32& call) {
  LAS u32* const cl = (LAS u32*)(smem + CLAIM_OFF);
  if (wv == 0) {
    const u32 c = __builtin_amdgcn_readfirstlane(__hip_atomic_load(counter(myws), __ATOMIC_RELAXED, __HIP_MEMORY_SCOPE_AGENT)) + 1u;
    cl[1] = take_role(myws, bid, c);
    cl[0] = c;
    if (threadIdx.x == 0) __hip_atomic_store(counter(myws), c, __ATOMIC_RELAXED, __HIP_MEMORY_SCOPE_AGENT);
  }
  WG_BARRIER();
  call = __builtin_amdgcn_readfirstlane(cl[0]);
  bid = __builtin_amdgcn_readfirstlane(cl[1]);
}
__device__ __forceinline__ u32 call_tag(u32 call) { return (call % 2047u + 1u) << 21; }
__device__ __forceinline__ u32 hdr_word(u32 data, u32 call, int lane) { return lane == 7 ? call : data | call_tag(call); }
__device__ __forceinline__ bool hdr_there(u32 word, u32 call, int lane) {
  return __all((lane & 7) == 7 ? word >= call : (word ^ call_tag(call)) < (1u << 21));
}

__device__ __forceinline__ bool spin_fail(char* myws, u32 call, u64 t0) {
  u32* tmo = (u32*)(myws + TMO_OFF);
  u32 t = __builtin_amdgcn_readfirstlane(__hip_atomic_load(tmo, __ATOMIC_RELAXED, __HIP_MEMORY_SCOPE_AGENT));
  if (t == call || __builtin_amdgcn_s_memrealtime() - t0 > DEADLINE_TICKS) {
    __hip_atomic_store(tmo, call, __ATOMIC_RELAXED, __HIP_MEMORY_SCOPE_AGENT);
    return true;
  }
  return false;
}

__device__ __forceinline__ size_t meta_off(u32 unit) { return META_OFF + (size_t)unit * 32; }
__device__ __forceinline__ u32 dest_stream(u32 sl, u32 rank, u32 dest, u32 par) {
  const u32 j = ((rank - dest) & 3) - 1;
  const u32 s = sl >= 16 ? S_WIN + (rank & 3) * 4 + (sl - 16) : sl >= (u32)NA ? S_INTER + (rank & 3) * NB + (sl - NA)
              : sl >= (u32)(N1 + N2) ? S_3 + j * N3 + (sl - N1 - N2) : sl >= (u32)N1 ? S_2 + j * N2 + (sl - N1) : S_1 + j * N1 + sl;
  return par * NS_PAR + s;
}

inline u32 global_row(u32 rank, u32 sid) {
  const u32 g0 = rank & 4u, o0 = g0 ^ 4u;
  u32 src, bi;
  if (sid < (u32)S_1) { src = rank; bi = sid; }
  else if (sid < (u32)S_2) { const u32 q = sid - S_1; src = g0 + ((rank + 1 + q / N1) & 3); bi = q % N1; }
  else if (sid < (u32)S_INTER) { const u32 q = sid - S_2; src = g0 + ((rank + 1 + q / N2) & 3); bi = N1 + q % N2; }
  else if (sid < (u32)S_3) { const u32 q = sid - S_INTER; src = o0 + q / NB; bi = NA + q % NB; }
  else if (sid >= (u32)S_OWN2) { src = rank; bi = NA + sid - S_OWN2; }
  else { const u32 q = sid - S_3; src = g0 + ((rank + 1 + q / N3) & 3); bi = N1 + N2 + q % N3; }
  return (phys_rank(src & 7u) * 512 + bi * 32) & 4095u;
}

__device__ __forceinline__ void load_raw(KP p, u32x4 (&d)[8], int lane, u32 sl, u32 q, u32 t) {
  const bool isw = sl >= 16;
  const u32 rs = isw ? NLOC * 2 : KDIM * 2;
  const u32 vo = (lane >> 4) * rs + (lane & 15) * 16;
  const u64 aw = (u64)p->w + (u64)((t * 128 + (sl & 3) * 32) * (NLOC * 2) + q * 256), ax = (u64)p->x + (u64)(sl * 32 * (KDIM * 2) + t * 256);
  const u64 b0 = isw ? aw : ax;
#pragma unroll
  for (int j = 0; j < 8; ++j)
    asm volatile("global_load_dwordx4 %0, %1, %2" : "=&v"(d[j]) : "v"(vo), "s"(b0 + (u64)(j * 4 * rs)) : "memory");
}
#define WAIT_RAW(d) asm volatile("s_waitcnt vmcnt(0)" : "+v"(d[0]), "+v"(d[1]), "+v"(d[2]), "+v"(d[3]), "+v"(d[4]), "+v"(d[5]), "+v"(d[6]), "+v"(d[7]) :: "memory")
__device__ __forceinline__ void to_fragments(u32x4 (&d)[8], bool isw, char* scr, int lane) {
  LAS char* const sc = (LAS char*)scr;
  auto off = [](u32 row, u32 ch) { return 256u * row + 16u * (ch ^ (((row & 3) << 2) | ((row >> 2) & 3))); };
#pragma unroll
  for (int j = 0; j < 8; ++j) *(LAS u32x4*)(sc + off(4 * j + (lane >> 4), lane & 15)) = d[j];
  asm volatile("" ::: "memory");
  if (isw) {
    const u32 r0 = 8 * (lane >> 5) + ((lane & 15) >> 2), c0 = 2 * ((lane >> 4) & 1) + ((lane & 3) >> 1);
#pragma unroll
    for (int j = 0; j < 8; ++j) {
      const bf16x4 lo = __builtin_amdgcn_ds_read_tr16_b64_v4bf16((LAS bf16x4*)(sc + off(16 * (j & 1) + r0, 4 * (j >> 1) + c0) + 8 * (lane & 1)));
      const bf16x4 hi = __builtin_amdgcn_ds_read_tr16_b64_v4bf16((LAS bf16x4*)(sc + off(16 * (j & 1) + r0 + 4, 4 * (j >> 1) + c0) + 8 * (lane & 1)));
      d[j] = __builtin_bit_cast(u32x4, __builtin_shufflevector(lo, hi, 0, 1, 2, 3, 4, 5, 6, 7));
    }
  } else {
#pragma unroll
    for (int j = 0; j < 8; ++j) d[j] = *(const LAS u32x4*)(sc + off(lane & 31, 2 * j + (lane >> 5)));
  }
  asm volatile("" ::: "memory");
}
__device__ __forceinline__ void low_slabs(const u32x4 (&d)[8], u32x4 (&s)[8]) {
#pragma unroll
  for (int i = 0; i < 16; ++i) s[i >> 2][i & 3] = __builtin_amdgcn_perm(d[i >> 1][2 * (i & 1) + 1], d[i >> 1][2 * (i & 1)], 0x06040200u);
}
__device__ __forceinline__ bool encode(const u32x4 (&d)[8], u32x4 (&s)[8], u32& meta, int lane) {
  u32 H[16], T[16];
  u16x2 m1 = {0, 0}, m2 = {0, 0}, m3 = {0, 0}, m4 = {0, 0};
#pragma unroll
  for (int i = 0; i < 16; ++i) {
    const u32 a = d[i >> 1][2 * (i & 1)], b = d[i >> 1][2 * (i & 1) + 1];
    H[i] = __builtin_amdgcn_perm(b, a, 0x07050301u);
  }
#pragma unroll
  for (int i = 0; i < 16; i += 2) {
    T[i] = H[i] | 0x80808080u;
    T[i + 1] = H[i + 1] | 0x80808080u;
    m1 = __builtin_elementwise_max(m1, __builtin_bit_cast(u16x2, T[i]));
    m2 = __builtin_elementwise_max(m2, __builtin_bit_cast(u16x2, T[i] << 8));
    m3 = __builtin_elementwise_max(m3, __builtin_bit_cast(u16x2, T[i + 1]));
    m4 = __builtin_elementwise_max(m4, __builtin_bit_cast(u16x2, T[i + 1] << 8));
  }
  m1 = __builtin_elementwise_max(__builtin_elementwise_max(m1, m2), __builtin_elementwise_max(m3, m4));
  u32 mx = (u32)(m1.x > m1.y ? m1.x : m1.y);
  mx = max(mx, (u32)__builtin_amdgcn_update_dpp(0, mx, 0x111, 0xf, 0xf, true));
  mx = max(mx, (u32)__builtin_amdgcn_update_dpp(0, mx, 0x112, 0xf, 0xf, true));
  mx = max(mx, (u32)__builtin_amdgcn_update_dpp(0, mx, 0x114, 0xf, 0xf, true));
  mx = max(mx, (u32)__builtin_amdgcn_update_dpp(0, mx, 0x118, 0xf, 0xf, true));
  const u32 top = max(max((u32)__builtin_amdgcn_readlane(mx, 15), (u32)__builtin_amdgcn_readlane(mx, 31)),
                      max((u32)__builtin_amdgcn_readlane(mx, 47), (u32)__builtin_amdgcn_readlane(mx, 63)));
  const u32 h7 = (top >> 8) & 0x7f;
  const u32 base = __builtin_amdgcn_readfirstlane(h7 > 7 ? h7 - 7 : 0);
  const u32 base4 = base * 0x01010101u;
  u32 ok = 0xffffffffu;
  u32 N[8];
#pragma unroll
  for (int j = 0; j < 8; ++j) {
    const u32 ta = T[2 * j] - base4, tb = T[2 * j + 1] - base4;
    ok &= ta & tb;
    const u32 ca = (ta & 0x07070707u) | (H[2 * j] & 0x80808080u);
    const u32 cb = (tb & 0x07070707u) | (H[2 * j + 1] & 0x80808080u);
    N[j] = ca | __builtin_amdgcn_alignbit(cb, cb, 28);
  }
  meta = 0;
  int c0 = 0, c1 = 0;
  if (__any((~ok & 0x80808080u) != 0)) {
    u32 M0 = 0, M1 = 0;
#pragma unroll
    for (int i = 0; i < 8; ++i) {
      M0 |= ~((T[i] - base4) >> i) & (0x80808080u >> i);
      M1 |= ~((T[8 + i] - base4) >> i) & (0x80808080u >> i);
    }
    u64 bal = __ballot((M0 | M1) != 0);
    while (bal != 0 && c0 + c1 <= 6) {
      const int ln = __builtin_ctzll(bal);
      bal &= bal - 1;
      u64 mm = (u64)(u32)__builtin_amdgcn_readlane(M0, ln) | (u64)(u32)__builtin_amdgcn_readlane(M1, ln) << 32;
      while (mm != 0) {
        const u32 bit = __builtin_ctzll(mm), hf = bit >> 5, by = (bit >> 3) & 3, i = hf * 8 + 7 - (bit & 7);
        mm &= mm - 1;
        c0 += hf ^ 1; c1 += hf;
        u32 hw = 0;
#pragma unroll
        for (int z = 0; z < 16; ++z) hw |= H[z] & (u32)__builtin_amdgcn_sbfe(1 << i, z, 1);
        const u32 hb = ((u32)__builtin_amdgcn_readlane(hw, ln) >> (8 * by)) & 0x7f;
        const u32 ent = (u32)ln | (i & 7) << 6 | by << 9 | ((((hb - base) & 7) + base) ^ hb) << 11;
        if (lane == (hf ? 7 - c1 : c0)) meta = ent;
      }
    }
  }
  const bool raw = c0 + c1 > 6;
  if (lane == 0) meta = base | (raw ? 0x400u : (u32)(c0 << 11 | c1 << 14));
  s[4] = raw ? u32x4{H[0], H[1], H[2], H[3]} : u32x4{N[0], N[1], N[2], N[3]};
  s[5] = raw ? u32x4{H[4], H[5], H[6], H[7]} : u32x4{N[4], N[5], N[6], N[7]};
  s[6] = u32x4{H[8], H[9], H[10], H[11]};
  s[7] = u32x4{H[12], H[13], H[14], H[15]};
#pragma unroll
  for (int j = 0; j < 8; ++j) asm volatile("" : "+v"(s[j]));
  return raw;
}

template <int N> struct Pay { u32x4 v[N]; };
__device__ __forceinline__ void pay_load(Pay<6>& P, u32 l16, const char* sb) {
  asm volatile("global_load_dwordx4 %0, %6, %7 offset:-4096 sc0 sc1\n\t"
               "global_load_dwordx4 %1, %6, %7 offset:-3072 sc0 sc1\n\t"
               "global_load_dwordx4 %2, %6, %7 offset:-2048 sc0 sc1\n\t"
               "global_load_dwordx4 %3, %6, %7 offset:-1024 sc0 sc1\n\t"
               "global_load_dwordx4 %4, %6, %7 offset:0 sc0 sc1\n\t"
               "global_load_dwordx4 %5, %6, %7 offset:1024 sc0 sc1"
               : "=&v"(P.v[0]), "=&v"(P.v[1]), "=&v"(P.v[2]), "=&v"(P.v[3]), "=&v"(P.v[4]), "=&v"(P.v[5]) : "v"(l16), "s"(sb) : "memory");
}
__device__ __forceinline__ void pay_load(Pay<3>& P, u32 vlo, u32 vcode, const char* sb) {
  asm volatile("global_load_dwordx4 %2, %4, %5 offset:0 sc0 sc1\n\t"
               "global_load_dwordx4 %0, %3, %5 offset:-4096 sc0 sc1\n\t"
               "global_load_dwordx4 %1, %3, %5 offset:-3072 sc0 sc1"
               : "=&v"(P.v[0]), "=&v"(P.v[1]), "=&v"(P.v[2]) : "v"(vlo), "v"(vcode), "s"(sb) : "memory");
}
template <int C> __device__ __forceinline__ void pay_wait(Pay<6>& P) {
  asm volatile("s_waitcnt vmcnt(%6)" : "+v"(P.v[0]), "+v"(P.v[1]), "+v"(P.v[2]), "+v"(P.v[3]), "+v"(P.v[4]), "+v"(P.v[5]) : "n"(C) : "memory");
}
template <int C> __device__ __forceinline__ void pay_wait(Pay<3>& P) {
  asm volatile("s_waitcnt vmcnt(%3)" : "+v"(P.v[0]), "+v"(P.v[1]), "+v"(P.v[2]) : "n"(C) : "memory");
}
__device__ __forceinline__ u32x8 load_two(const char* b, u32 l16) {
  u32x4 x, y;
  asm volatile("global_load_dwordx4 %0, %2, %3 sc0 sc1\n\t"
               "global_load_dwordx4 %1, %2, %3 offset:1024 sc0 sc1\n\ts_waitcnt vmcnt(0)"
               : "=&v"(x), "=&v"(y) : "v"(l16), "s"(b) : "memory");
  return __builtin_shufflevector(x, y, 0, 1, 2, 3, 4, 5, 6, 7);
}

__device__ __forceinline__ void load_slot(u32x4 (&d)[8], u32 l16, const char* sb) {
  asm volatile("global_load_dwordx4 %0, %8, %9 offset:-4096 sc0 sc1\n\tglobal_load_dwordx4 %1, %8, %9 offset:-3072 sc0 sc1\n\t"
               "global_load_dwordx4 %2, %8, %9 offset:-2048 sc0 sc1\n\tglobal_load_dwordx4 %3, %8, %9 offset:-1024 sc0 sc1\n\t"
               "global_load_dwordx4 %4, %8, %9 offset:0 sc0 sc1\n\tglobal_load_dwordx4 %5, %8, %9 offset:1024 sc0 sc1\n\t"
               "global_load_dwordx4 %6, %8, %9 offset:2048 sc0 sc1\n\tglobal_load_dwordx4 %7, %8, %9 offset:3072 sc0 sc1\n\ts_waitcnt vmcnt(0)"
               : "=&v"(d[0]), "=&v"(d[1]), "=&v"(d[2]), "=&v"(d[3]), "=&v"(d[4]), "=&v"(d[5]), "=&v"(d[6]), "=&v"(d[7])
               : "v"(l16), "s"(sb) : "memory");
}

template <bool ask>
__device__ __forceinline__ bool sender(KP p, char* smem, char* myws, int wv, int lane, u32& bid, u32& call) {
  const u32 rank = p->rank;
  const u32 cb = bid - NGEMM;
  const u32 link = cb & 7, cw = cb >> 3;
  const u32 l16 = lane * 16;
  if (link != 0) {
    const u32 dest = peer_of_xcd(rank, link);
    const bool intra = link < 4;
    const int n = ((intra ? N_INTRA : N_INTER) - (int)cw + NCWG - 1) / NCWG;
    auto code_of = [=](int k) {
      const u32 e = cw + (u32)(k < n ? k : 0) * NCWG;
      const u32 word = ((const __attribute__((address_space(4))) u32*)p->ord[intra ? 0 : 1])[e >> 1];
      return (word >> (16 * (e & 1))) & 0xffffu;
    };
    LAS int* const done = (LAS int*)(smem + 12 * 8192);
    u32x4 d[8];
    auto fetch = [&](u32 c) {
      const bool rly = c >= 20 * 32;
      load_raw(p, d, lane, rly ? RB : c >> 5, dest & 3, rly ? RT0 + 4 * (c & 7) + (dest & 3) : c & 31);
    };
    u32 code = code_of(wv);
    LAS u32* const got = (LAS u32*)(smem + CLAIM_OFF + 16);
    if (wv >= 4) {
      if (threadIdx.x == 256) { *done = 0; *got = 0; }
      WG_BARRIER();
      if (wv >= 8) __builtin_amdgcn_s_sleep(16);
    }
    fetch(code);
    u32 ticket = 0;
    if (ask) asm volatile("global_load_dword %0, %1, %2 sc1" : "=&v"(ticket) : "v"(0u), "s"(counter(myws)) : "memory");
    if (wv < 4) WG_BARRIER();
    bool lost = false;
    char* const rws = p->ws[dest];
    if (wv < 4) __builtin_amdgcn_s_setprio(2); else if (wv < 8) __builtin_amdgcn_s_setprio(1);
    int k = wv;
    for (;; k += 12) {
      const u32 sl = code >> 5;
      u32 ud = sl < 20 ? dest_stream(sl, rank, dest, 0) * NSTEP + (code & 31) : sl == 20 ? S_RLY * NSTEP + (rank & 3) * 8 + (code & 7)
             : (S_2 + (sl - 21) * N2 + RB - N1) * NSTEP + RT0 + 4 * (code & 7) + (rank & 3);
      asm volatile("" : "+s"(ud));
      WAIT_RAW(d);
      if (ask && k < 24) {
        if (k < 12) {
          asm volatile("" : "+v"(ticket));
          call = __builtin_amdgcn_readfirstlane(ticket) + 1u;
          if (wv == 11) __hip_atomic_store(got, take_role(myws, bid, call) + 1u, __ATOMIC_RELAXED, __HIP_MEMORY_SCOPE_WORKGROUP);
        } else {
          u32 g;
          while ((g = __builtin_amdgcn_readfirstlane(__hip_atomic_load(got, __ATOMIC_RELAXED, __HIP_MEMORY_SCOPE_WORKGROUP))) == 0) __builtin_amdgcn_s_sleep(1);
          if (g - 1 != bid) { bid = g - 1; lost = true; break; }
        }
      }
      ud += (call & 1) * (NS_PAR * NSTEP);
      char* const b = rws + (size_t)ud * SLOT + 4096;
      u32x4 s[8];
      u32 meta;
      bool raw;
      if (P_RLY > 0 && sl > 20) {
        const u32 us = ((call & 1) * NS_PAR + S_RLY) * NSTEP + ((dest + sl - 20) & 3) * 8 + (code & 7);
        const u64 t0 = __builtin_amdgcn_s_memrealtime();
        const u32* const hd = (const u32*)(myws + meta_off(us)) + (lane & 7);
        for (u32 it = 1;; ++it) {
          meta = __hip_atomic_load(hd, __ATOMIC_RELAXED, __HIP_MEMORY_SCOPE_SYSTEM);
          if (hdr_there(meta, call, lane)) break;
          if ((it & 31) == 0 && spin_fail(myws, call, t0)) break;
          __builtin_amdgcn_s_sleep(2);
        }
        load_slot(s, l16, myws + (size_t)us * SLOT + 4096);
        raw = __builtin_amdgcn_readfirstlane(meta) & 0x400;
      } else {
        to_fragments(d, sl >= 16 && sl < 20, smem + wv * 8192, lane);
        low_slabs(d, s);
        if (k < P_WIN0) { ST16(l16, s[0], b, -4096); ST16(l16, s[1], b, -3072); ST16(l16, s[2], b, -2048); ST16(l16, s[3], b, -1024); }
        raw = encode(d, s, meta, lane);
        meta = hdr_word(meta, call, lane);
      }
      code = code_of(k + 12);
      __builtin_amdgcn_s_setprio(0);
      const int need = k < P_WIN0 ? 0 : k + 1 - max(P_WIN, P_WIN1 - (k - P_WIN0) / P_TAPER);
      while (__builtin_amdgcn_readfirstlane(__hip_atomic_load(done, __ATOMIC_RELAXED, __HIP_MEMORY_SCOPE_WORKGROUP)) < need) __builtin_amdgcn_s_sleep(1);
      if (k >= P_WIN0) { ST16(l16, s[0], b, -4096); ST16(l16, s[1], b, -3072); ST16(l16, s[2], b, -2048); ST16(l16, s[3], b, -1024); }
      ST16(l16, s[4], b, 0); ST16(l16, s[5], b, 1024);
      if (raw) { ST16(l16, s[6], b, 2048); ST16(l16, s[7], b, 3072); }
      asm volatile("s_waitcnt vmcnt(0)" ::: "memory");
      char* const m = rws + meta_off(ud);
      if (lane < 8) ST4(lane * 4, meta, m);
      if (lane == 0) __hip_atomic_fetch_add(done, 1, __ATOMIC_RELAXED, __HIP_MEMORY_SCOPE_WORKGROUP);
      if (k + 12 >= n) { k += 12; break; }
      fetch(code);
      WAIT_RAW(d);
    }
    if (lost) WG_BARRIER();
    if (ask && threadIdx.x == 0) __hip_atomic_store(counter(myws), call, __ATOMIC_RELAXED, __HIP_MEMORY_SCOPE_AGENT);
    if (lost) { __builtin_amdgcn_s_setprio(0); return false; }
    const u32 y = cw + (u32)(k - n) * NCWG;
    if (!intra && y < 2u * P_PARK) {
      const u32 us = (S_PARK + (dest & 3)) * NSTEP + y;
      const u64 t0 = __builtin_amdgcn_s_memrealtime();
      const u32* const hd = (const u32*)(myws + meta_off(us));
      for (u32 it = 1; (u32)__builtin_amdgcn_readfirstlane(__hip_atomic_load(hd, __ATOMIC_RELAXED, __HIP_MEMORY_SCOPE_SYSTEM)) < call; ++it) {
        if ((it & 31) == 0 && spin_fail(myws, call, t0)) break;
        __builtin_amdgcn_s_sleep(4);
      }
      load_slot(d, l16, myws + (size_t)us * SLOT + 4096);
      if (__hip_atomic_load((u32*)(myws + TMO_OFF), __ATOMIC_RELAXED, __HIP_MEMORY_SCOPE_AGENT) == call) d[0] = u32x4{0x7fc07fc0u, 0x7fc07fc0u, 0x7fc07fc0u, 0x7fc07fc0u};
      const u32 tile = y >> 1, sid = (tile >> 1) * 4 + (y & 1) * 2;
      char* const ob = p->out[dest] + (rank & 3) * 256 + (tile & 1) * 128;
      const u32 vo = (lane >> 3) * (NLOC * 2) + (lane & 7) * 16;
      char* r0 = ob + (size_t)p->row[sid] * (NLOC * 2);
      char* r1 = ob + (size_t)p->row[sid + 1] * (NLOC * 2);
      asm volatile("" : "+s"(r0), "+s"(r1));
      while (__builtin_amdgcn_readfirstlane(__hip_atomic_load(done, __ATOMIC_RELAXED, __HIP_MEMORY_SCOPE_WORKGROUP)) < k - (P_WIN - 1)) __builtin_amdgcn_s_sleep(1);
#pragma unroll
      for (int g = 0; g < 8; ++g) {
        char* const b = (g < 4 ? r0 : r1) + (g & 3) * 8 * (NLOC * 2);
        ST16(vo, d[g], b, 0);
      }
      if (y + P_REL * NCWG >= 2u * P_PARK && y + P_REL * NCWG < 2u * P_PARK + NCWG && lane == 0)
        __hip_atomic_store((u32*)(myws + REL_OFF + link * 128) + cw, call, __ATOMIC_RELAXED, __HIP_MEMORY_SCOPE_AGENT);
      asm volatile("s_waitcnt vmcnt(0)" ::: "memory");
      char* const fl = rws + DONE_OFF + ((rank & 3) * 64 + ((y & 1) ? 2 * RT_MED + 12 + tile : tile)) * 4;
      if (lane == 0) { ST4(0u, call, fl); __hip_atomic_fetch_add(done, 1, __ATOMIC_RELAXED, __HIP_MEMORY_SCOPE_WORKGROUP); }
    }
    return true;
  }
  if (ask) {
    const u32 asked = bid;
    claim(smem, myws, wv, bid, call);
    if (bid != asked) return false;
  }
  if (cb == 0 && wv == 0) {
    const u64 t0 = __builtin_amdgcn_s_memrealtime();
    while (__builtin_amdgcn_s_memrealtime() - t0 < N_INTRA * P_UT / 10) __builtin_amdgcn_s_sleep(16);
    const char* const f = myws + DONE_OFF;
    const int f0 = (4 * lane) & 63;
    for (u32 n = 1;; ++n) {
      u32x4 v;
      asm volatile("global_load_dwordx4 %0, %1, %2 sc0 sc1\n\ts_waitcnt vmcnt(0)" : "=&v"(v) : "v"(l16), "s"(f) : "memory");
      if (__all((f0 >= NFLAG || v[0] >= call) && (f0 + 1 >= NFLAG || v[1] >= call) && (f0 + 2 >= NFLAG || v[2] >= call) && (f0 + 3 >= NFLAG || v[3] >= call))) break;
      if ((n & 31) == 0 && spin_fail(myws, call, t0)) {
        if (lane == 0) *(unsigned short*)(p->out[rank] + (size_t)phys_rank(rank) * 512 * (NLOC * 2)) = 0x7fc0;
        break;
      }
    }
  }
  return true;
}

__device__ __forceinline__ u32x4 join(u32 l0, u32 l1, u32 h0, u32 h1) {
  return u32x4{__builtin_amdgcn_perm(h0, l0, 0x05010400u), __builtin_amdgcn_perm(h0, l0, 0x07030602u),
               __builtin_amdgcn_perm(h1, l1, 0x05010400u), __builtin_amdgcn_perm(h1, l1, 0x07030602u)};
}
__device__ __forceinline__ void fix_word(u32 (&h)[8], u32 e) {
  asm volatile("s_lshl_b64 exec, 1, %8\n\ts_getpc_b64 vcc\n\ts_add_u32 vcc_lo, vcc_lo, %9\n\ts_addc_u32 vcc_hi, vcc_hi, 0\n\ts_setpc_b64 vcc\n\t"
               "v_xor_b32_e32 %0, %10, %0\n\ts_branch 1f\n\tv_xor_b32_e32 %1, %10, %1\n\ts_branch 1f\n\t"
               "v_xor_b32_e32 %2, %10, %2\n\ts_branch 1f\n\tv_xor_b32_e32 %3, %10, %3\n\ts_branch 1f\n\t"
               "v_xor_b32_e32 %4, %10, %4\n\ts_branch 1f\n\tv_xor_b32_e32 %5, %10, %5\n\ts_branch 1f\n\t"
               "v_xor_b32_e32 %6, %10, %6\n\ts_branch 1f\n\tv_xor_b32_e32 %7, %10, %7\n"
               "1:\n\ts_mov_b64 exec, -1"
               : "+v"(h[0]), "+v"(h[1]), "+v"(h[2]), "+v"(h[3]), "+v"(h[4]), "+v"(h[5]), "+v"(h[6]), "+v"(h[7])
               : "s"(e), "s"(((e >> 3) & 56) + 12), "s"(((e >> 11) & 0x7f) << ((e >> 6) & 24)) : "vcc", "scc");
}

struct Stream {
  const char* sb; const char* mb; char* myws;
  u32 l16, tag, call; u64 t0;
  u32x4 W; int rdy; bool eager;
  __device__ __forceinline__ void init(char* ws, u32 call_, u32 sid, int lane) {
    myws = ws; call = call_; tag = call_tag(call_);
    sb = ws + (size_t)sid * NSTEP * SLOT + 4096;
    mb = ws + meta_off(sid * NSTEP);
    l16 = lane * 16;
    t0 = __builtin_amdgcn_s_memrealtime();
    W = u32x4{0u, 0u, 0u, 0u}; rdy = 0; eager = false;
  }
  __device__ __forceinline__ void poll(bool wide = true) {
    const u64 mask = (wide ? 0xffffull : 0xffull) << (rdy < 28 ? (2 * rdy) & 56 : 56);
    asm volatile("s_mov_b64 exec, %3\n\tglobal_load_dwordx4 %0, %1, %2 sc0 sc1\n\ts_mov_b64 exec, -1"
                 : "+v"(W) : "v"(l16), "s"(mb), "s"(mask) : "memory");
  }
  __device__ __forceinline__ void look() {
    asm volatile("" : "+v"(W));
    const u32 x = (W[0] ^ tag) | (W[1] ^ tag) | (W[2] ^ tag);
    const u64 odd = 0xaaaaaaaaaaaaaaaaull;
    const u64 bad = __ballot(x >= (1u << 21)) | (__ballot((W[3] ^ tag) >= (1u << 21)) & ~odd) | (__ballot(W[3] < call) & odd);
    rdy = bad ? __builtin_ctzll(bad) >> 1 : NSTEP;
  }
  __device__ __forceinline__ void spin(int t) {
    for (u32 n = 1; rdy <= t; ++n) {
      const int was = rdy;
      poll(n == 1);
      asm volatile("s_waitcnt vmcnt(0)" ::: "memory");
      look();
      if (rdy == was) {
        if (eager) __builtin_amdgcn_s_sleep(P_S2);
        else if (rdy == 0) __builtin_amdgcn_s_sleep(P_S0);
        else __builtin_amdgcn_s_sleep(P_S1);
      }
      if ((n & 31) == 0 && spin_fail(myws, call, t0)) rdy = NSTEP;
    }
  }
};

__device__ __forceinline__ void repair(u32 (&h)[8], u32 hd, u32 hdr, u32 hf, const char* sb, u32 l16) {
#pragma nounroll
  for (u32 n = (hdr >> (11 + 3 * hf)) & 7, k = hf ? 6 : 1; n != 0; --n, k += hf ? -1 : 1) fix_word(h, __builtin_amdgcn_readlane(hd, k));
  if (__builtin_expect(hdr & 0x400u, 0)) {
    const u32x8 g = load_two(sb + hf * 2048, l16);
#pragma unroll
    for (int j = 0; j < 8; ++j) asm volatile("v_mov_b32 %0, %1" : "+v"(h[j]) : "v"(g[j]));
  }
}

__device__ __forceinline__ void unpack_unit(char* myws, int lane, u32 call, u32 r, int t, u64 tk) {
  Stream S;
  S.init(myws, call, (call & 1) * NS_PAR + S_WIN + r, lane);
  const u32 l4 = lane * 4;
  const u32 uo = (S_IMG + 16 + r) * NSTEP;
  {
    if (r < 16) while (__builtin_amdgcn_s_memrealtime() - tk < (u32)(20 + 4 * t) * P_UT / 10) __builtin_amdgcn_s_sleep(4);
    u32 hd;
    for (u32 n = 1;; ++n) {
      hd = __hip_atomic_load((const u32*)(S.mb + t * 32) + (lane & 7), __ATOMIC_RELAXED, __HIP_MEMORY_SCOPE_SYSTEM);
      if (hdr_there(hd, call, lane)) break;
      if ((n & 31) == 0 && spin_fail(myws, call, S.t0)) break;
      __builtin_amdgcn_s_sleep(4);
    }
    Pay<6> P;
    pay_load(P, S.l16, S.sb + (size_t)t * SLOT);
    pay_wait<0>(P);
    const u32 hdr = __builtin_amdgcn_readfirstlane(hd), base4 = (hdr & 0x7f) * 0x01010101u;
    u32x4 o[8];
#pragma unroll
    for (int hf = 0; hf < 2; ++hf) {
      u32 g[8];
#pragma unroll
      for (int j = 0; j < 4; ++j) {
        const u32 n = P.v[4 + hf][j];
        g[2 * j] = (n & 0x87878787u) + base4;
        g[2 * j + 1] = (__builtin_amdgcn_alignbit(n, n, 4) & 0x87878787u) + base4;
      }
      if (hdr & (0x400u | 0x3800u << (3 * hf))) repair(g, hd, hdr, hf, S.sb + (size_t)t * SLOT, S.l16);
#pragma unroll
      for (int j = 0; j < 4; ++j)
        o[4 * hf + j] = join(P.v[2 * hf + (j >> 1)][2 * (j & 1)], P.v[2 * hf + (j >> 1)][2 * (j & 1) + 1], g[2 * j], g[2 * j + 1]);
    }
    char* const ob = myws + (size_t)(uo + t) * SLOT + 4096;
    ST_SIX(S.l16, o, ob); ST16(S.l16, o[6], ob, 2048); ST16(S.l16, o[7], ob, 3072);
    asm volatile("s_waitcnt vmcnt(0)" ::: "memory");
    char* const mo = myws + meta_off((uo + t));
    if (lane < 8) ST4(l4, hdr_word(0, call, lane), mo);
  }
}

__device__ __forceinline__ float silu(float y) {
  const float z = __builtin_amdgcn_fmed3f(y, -60.0f, 60.0f);
  const float e = __builtin_amdgcn_exp2f(z * -1.44269504088896341f);
  return y * __builtin_amdgcn_rcpf(1.0f + e);
}

__device__ __forceinline__ void gemm_tile(KP p, char* smem, char* myws, int wv, int lane, u32 bid, u32 call, u32 nb, u32 sid0, u32 tn, u32 ch, u32 fl, int hold, u64 tk, bool next, u32*& owed) {
  const u32 rank = p->rank;
  if (wv >= 8) __builtin_amdgcn_s_setprio(P_LPRIO);
  const u32 o0 = (rank & 4u) ^ 4u;
  const bool remote = tn >= 4;
  const u32 dst = (remote ? o0 + (tn - 4) : rank) & 7u;
  char* const ob = p->out[dst] + (remote ? (rank & 3) : tn) * 256 + ch * 128;
  u32* flag = (u32*)(p->ws[dst] + DONE_OFF) + (rank & 3) * 64 + fl;
  auto grow = [&](u32 rg) { return p->row[sid0 + (rg >> 2)] + (rg & 3) * 8; };
  char* b0 = ob + (size_t)grow(wv & 7) * (NLOC * 2);
  char* b1 = ob + (size_t)grow((wv & 7) + 8) * (NLOC * 2);
  if (hold == 2) {
    const u32 us = (S_PARK + (tn & 3)) * NSTEP + 2 * fl;
    b0 = myws + (size_t)us * SLOT + (wv & 7) * 1024; b1 = b0 + SLOT;
    flag = (u32*)(myws + meta_off(us));
  }
  asm volatile("" : "+s"(b0), "+s"(b1), "+s"(flag));
  if (nb == 0) {
    if (threadIdx.x == 0) ST4(0u, call, flag);
    return;
  }
  {
    const u32 first = sid0 < (u32)S_2 || sid0 >= (u32)S_OWN2 ? 0 : sid0 < (u32)S_INTER ? N1 * NSTEP : sid0 < (u32)S_3 ? 4 * NSTEP : N_INTRA - N3 * NSTEP;
    while (__builtin_amdgcn_s_memrealtime() - tk < first * P_UT / 10) __builtin_amdgcn_s_sleep(8);
  }
  Stream S;
  const u32 m = wv >> 1, kh = wv & 1;
  u32 tmo = 0;
  auto read_tmo = [&]() { asm volatile("global_load_dword %0, %1, %2 sc1" : "=&v"(tmo) : "v"(0u), "s"(myws + TMO_OFF) : "memory"); };
  if (wv >= 8) {
    const u32 un = wv - 8;
    S.init(myws, call, S_IMG + 4 * tn + un, lane);
    const u32 lds0 = (u32)(size_t)(LAS char*)smem + un * 4096;
    const char* const g0 = S.sb - 4096 + ch * 4096;
    Stream A;
    A.init(myws, call, (call & 1) * NS_PAR + sid0 + un, lane);
    A.eager = sid0 >= (u32)S_3 && sid0 < (u32)S_OWN2;
    asm volatile("global_load_dwordx4 %0, %2, %3 sc0 sc1\n\tglobal_load_dwordx4 %1, %2, %4 sc0 sc1\n\ts_waitcnt vmcnt(0)"
                 : "+v"(A.W), "+v"(S.W) : "v"(S.l16), "s"(A.mb), "s"(S.mb) : "memory");
    int is = 0, seen = ((nb >> un) & 1) ? 0 : NSTEP;
    for (int t = -1; t < NSTEP; ++t) {
      if (seen < NSTEP && seen <= t + P_DIST + P_ALOOK) {
        A.look();
        A.spin(min(t + P_DIST, NSTEP - 1));
        if (A.rdy != seen) { *(LAS u32x4*)((LAS char*)smem + HDR_OFF + un * 1024 + A.l16) = A.W; seen = A.rdy; }
        asm volatile("" ::: "memory");
        if (seen < NSTEP) A.poll();
      }
      const int lim = min(min(NSTEP, is + 2), t + P_BD);
      if (S.rdy < min(NSTEP, lim + 2)) { S.look(); S.spin(t); if (S.rdy < NSTEP) S.poll(); }
      for (; is < min(lim, S.rdy); ++is) {
        const char* g = g0 + (size_t)is * SLOT;
        const u32 l = lds0 + (is & 7) * 16384;
        u32 keep;
        asm volatile("s_mov_b32 %0, m0\n\ts_mov_b32 m0, %3\n\ts_nop 0\n\t"
                     "global_load_lds_dwordx4 %1, %2 sc0 sc1\n\tglobal_load_lds_dwordx4 %1, %2 offset:1024 sc0 sc1\n\t"
                     "global_load_lds_dwordx4 %1, %2 offset:2048 sc0 sc1\n\tglobal_load_lds_dwordx4 %1, %2 offset:3072 sc0 sc1\n\t"
                     "s_mov_b32 m0, %0" : "=&s"(keep) : "v"(S.l16), "s"(g), "s"(l) : "memory");
      }
      const int ahead = is - 1 - t;
      if (ahead >= 6 || t < 0) asm volatile("s_waitcnt vmcnt(24)" ::: "memory");
      else if (ahead >= 4) { if (ahead == 5) asm volatile("s_waitcnt vmcnt(20)" ::: "memory"); else asm volatile("s_waitcnt vmcnt(16)" ::: "memory"); }
      else if (ahead >= 2) { if (ahead == 3) asm volatile("s_waitcnt vmcnt(12)" ::: "memory"); else asm volatile("s_waitcnt vmcnt(8)" ::: "memory"); }
      else if (ahead == 1) asm volatile("s_waitcnt vmcnt(4)" ::: "memory");
      else asm volatile("s_waitcnt vmcnt(0)" ::: "memory");
      WG_BARRIER();
    }
    asm volatile("s_waitcnt vmcnt(0)" : "+v"(S.W), "+v"(A.W) :: "memory");
    __builtin_amdgcn_s_setprio(0);
    if (hold == 1 && wv == 8) {
      const u32* const r = (const u32*)(myws + REL_OFF + (bid & 7) * 128) + (lane < NCWG ? lane : 0);
      for (u32 n = 1; !__all(__hip_atomic_load(r, __ATOMIC_RELAXED, __HIP_MEMORY_SCOPE_AGENT) >= call); ++n) {
        if ((n & 31) == 0 && spin_fail(myws, call, S.t0)) break;
        __builtin_amdgcn_s_sleep(4);
      }
    }
  } else if (!((nb >> m) & 1)) {
    asm volatile("s_waitcnt vmcnt(0)" ::: "memory");
    for (int t = -1; t < NSTEP; ++t) WG_BARRIER();
    read_tmo();
    asm volatile("s_waitcnt vmcnt(0)" : "+v"(tmo) :: "memory");
  } else {
    const char* const sb = myws + (size_t)((call & 1) * NS_PAR + sid0 + m) * NSTEP * SLOT + 4096;
    const u32 l16 = lane * 16, vlo = l16 + kh * 2048, vcode = l16 + kh * 1024;
    u32 ho = HDR_OFF + m * 1024 + (lane & 7) * 4, hd;
    const u32 rmask = 0x400u | 0x3800u << (3 * kh);
    f32x16 acc[2] = {};
    u32 bb = kh * 8192 + lane * 16;
    bf16x8 wf[4];
    u32x4 X[2][4];
    Pay<3> P[4];
    u32 h[8];
    u32 hdr = 0, base4 = 0;
    auto rd = [&](int stage, int f) {
      wf[f & 3] = *(const LAS bf16x8*)((LAS char*)smem + bb + stage * 16384 + (f >> 2) * 4096 + ((f >> 1) & 1) * 1024 + (f & 1) * 2048);
    };
    auto piece = [&](int k, int t, int ts, int waitn, bool fetch) {
      Pay<3>& Q = P[ts];
      if (k == 0) {
        if (waitn == 3) pay_wait<9>(Q); else if (waitn == 2) pay_wait<6>(Q); else if (waitn == 1) pay_wait<3>(Q); else pay_wait<0>(Q);
        hdr = __builtin_amdgcn_readfirstlane(hd);
        base4 = (hdr & 0x7f) * 0x01010101u;
      } else if (k < 6) {
#pragma unroll
        for (int j = 0; j < 4; ++j) {
          const int i = k < 4 ? 2 * j + (k & 1) : 4 * (k - 4) + j;
          if (k == 1) h[i] = __builtin_amdgcn_alignbit(Q.v[2][j], Q.v[2][j], 4);
          else if (k == 2) h[i] = Q.v[2][j] & 0x87878787u;
          else if (k == 3) h[i] &= 0x87878787u;
          else h[i] += base4;
          asm volatile("" : "+v"(h[i]));
        }
      } else if (k == 6) {
        if (hdr & rmask) repair(h, hd, hdr, kh, sb + (size_t)t * SLOT, l16);
      } else if (k == 7) {
      } else if (k < 12) {
        const int c = k - 8;
        X[ts & 1][c] = join(Q.v[c >> 1][2 * (c & 1)], Q.v[c >> 1][2 * (c & 1) + 1], h[2 * c], h[2 * c + 1]);
        asm volatile("" : "+v"(X[ts & 1][c]));
      } else if (k == 12) {
        ho += 32;
        hd = *(const LAS u32*)((LAS char*)smem + ho);
      } else if (k == 13 && fetch) {
        const int tn4 = t + P_DIST < NSTEP ? t + P_DIST : NSTEP - 1;
        pay_load(P[(ts + P_DIST) & 3], vlo, vcode, sb + (size_t)tn4 * SLOT);
      }
    };
    auto step = [&](int t, int ts, int waitn, bool fetch, bool more) {
#pragma unroll
      for (int q = 0; q < 8; ++q) {
        if (q == 4 && more) { WG_BARRIER(); if (ts == 3) bb ^= 65536u; }
        else if (q % 2 == 0 && (more || q < 6)) __builtin_amdgcn_s_waitcnt(0xC27F);
        __builtin_amdgcn_sched_barrier(0);
        acc[q & 1] = __builtin_amdgcn_mfma_f32_32x32x16_bf16(wf[q & 3], __builtin_bit_cast(bf16x8, X[ts & 1][q >> 1]), acc[q & 1], 0, 0, 0);
        __builtin_amdgcn_sched_barrier(0);
        if (q < 4) rd(ts, q + 4);
        else if (more) rd((ts + 1) & 3, q - 4);
        if (more) {
          piece(2 * q, t + 1, (ts + 1) & 3, waitn, fetch);
          piece(2 * q + 1, t + 1, (ts + 1) & 3, waitn, fetch);
        }
      }
    };
    asm volatile("s_waitcnt vmcnt(0)" ::: "memory");
    WG_BARRIER();
    if (owed && threadIdx.x < 2) ST4(threadIdx.x * 32, call, owed);
#pragma unroll
    for (int d = 0; d < P_DIST; ++d) pay_load(P[d], vlo, vcode, sb + (size_t)d * SLOT);
    hd = *(const LAS u32*)((LAS char*)smem + ho);
#pragma unroll
    for (int k = 0; k < 13; ++k) piece(k, 0, 0, P_DIST - 1, true);
    WG_BARRIER();
    piece(13, 0, 0, P_DIST - 1, true);
#pragma unroll
    for (int f = 0; f < 4; ++f) { rd(0, f); __builtin_amdgcn_sched_barrier(0); }
    for (int tb = 0; tb < NSTEP - 4; tb += 4) {
#pragma unroll
      for (int ts = 0; ts < 4; ++ts) step(tb + ts, ts, P_DIST - 1, true, true);
    }
    step(NSTEP - 4, 0, P_DIST - 1, false, true);
    step(NSTEP - 3, 1, P_DIST - 2, false, true);
    step(NSTEP - 2, 2, P_DIST - 3, false, true);
    read_tmo();
    step(NSTEP - 1, 3, 0, false, false);
    char* const xq = smem + EPI_OFF + (kh * 128 + m * 32 + (lane & 31)) * ERS + (lane >> 5) * 16;
#pragma unroll
    for (int n = 0; n < 2; ++n)
#pragma unroll
      for (int g = 0; g < 4; ++g)
        *(f32x4*)(xq + n * 128 + g * 32) = f32x4{acc[n][4 * g], acc[n][4 * g + 1], acc[n][4 * g + 2], acc[n][4 * g + 3]};
    if (P_DIST == 4) pay_wait<0>(P[0]);
    asm volatile("s_waitcnt vmcnt(0)" : "+v"(tmo) :: "memory");
  }
  WG_BARRIER();
  if (wv < 8) {
    const u32 vo = hold == 2 ? lane * 16 : (lane >> 3) * (NLOC * 2) + (lane & 7) * 16;
    auto get = [&](u32 rg) {
      const char* const q = smem + EPI_OFF + (rg * 8 + (lane >> 3)) * ERS + (lane & 7) * 32;
      const f32x4 a = *(const f32x4*)q + *(const f32x4*)(q + 128 * ERS), c = *(const f32x4*)(q + 16) + *(const f32x4*)(q + 128 * ERS + 16);
      auto pk = [](float lo, float hi) { const f32x2 v = {silu(lo), silu(hi)}; return __builtin_bit_cast(u32, __builtin_convertvector(v, bf16x2)); };
      u32x4 v = {pk(a[0], a[1]), pk(a[2], a[3]), pk(c[0], c[1]), pk(c[2], c[3])};
      if (tmo == call) v = u32x4{0x7fc07fc0u, 0x7fc07fc0u, 0x7fc07fc0u, 0x7fc07fc0u};
      return v;
    };
    if ((nb >> (wv >> 2)) & 1) { const u32x4 v0 = get(wv); ST16(vo, v0, b0, 0); }
    if ((nb >> (wv >> 2)) & 4) { const u32x4 v1 = get(wv + 8); ST16(vo, v1, b1, 0); }
  }
  owed = nullptr;
  if (next) { if (remote) owed = flag; }
  else if (remote) {
    asm volatile("s_waitcnt vmcnt(0)" ::: "memory");
    WG_BARRIER();
    if (threadIdx.x == 0) ST4(0u, call, flag);
  }
}

inline bool tile_of(u32 rank, u32 b, int k, u32& nb, u32& sid0, u32& tn, u32& ch, u32& fl, int& hold) {
  const u32 peer = peer_of_xcd(rank, b & 7), i = b >> 3;
  const bool home = !((peer ^ rank) & 4), spare = i >= 2 * RT_MED;
  hold = 0;
  constexpr u32 I0 = S_INTER / 2;
  const bool extra = home && i >= I0 && i < I0 + (NB + 1) / 2 * 2;
  if (extra && k == 0) {
    const u32 j = i - I0;
    nb = (j | 1) < (u32)NB ? 3 : 1; tn = peer & 3; sid0 = S_OWN2 + (j >> 1) * 2; ch = j & 1; fl = 0;
    return true;
  }
  if (k == (extra ? 1 : 0) && !spare) {
    nb = 15; tn = (peer & 3) + (home ? 0 : 4); sid0 = (i >> 1) * 4; ch = i & 1; fl = i;
    hold = home || sid0 >= (u32)S_INTER ? 0 : i < (u32)P_PARK ? 2 : i >= 10 ? 1 : 0;
    if (!home)
      for (u32 m = 0; m < 4; ++m)
        if (sid0 + m >= (u32)S_INTER && (sid0 + m - S_INTER) / NB == (peer & 3)) nb &= ~(1u << m);
    return true;
  }
  const u32 j = spare ? i - 2 * RT_MED + 10 : i;
  nb = 3; tn = (peer & 3) + (home ? 4 : 0); sid0 = RT_MED * 4 + (j >> 1) * 2; ch = j & 1; fl = 2 * RT_MED + j;
  return spare ? k == 0 : i < 10;
}

constexpr u32 NHOME = 32 * NSTEP;
__host__ __device__ inline u32 pack_index(u32 b, u32 wv) {
  constexpr u32 NP1 = 80;
  return b < NP1 ? (wv < 2 ? b + NP1 * wv : NHOME) : 2 * NP1 + (b - NP1) + (NGEMM - NP1) * wv;
}

constexpr u32 UB0 = RT_MED * 16, UNW = 16;
__host__ __device__ inline u32 unpack_index(u32 b, u32 wv) {
  return b >= UB0 && b < UB0 + UNW ? (11 - wv) * UNW + (b - UB0) : 512u;
}

__global__ void __launch_bounds__(NTHREAD, 3) ag_gemm_silu(const Params) {
  KP p = (KP)__builtin_amdgcn_kernarg_segment_ptr();
  __shared__ __attribute__((aligned(16))) char smem[LDS_BYTES];
  const int lane = threadIdx.x & 63;
  const int wv = __builtin_amdgcn_readfirstlane(threadIdx.x >> 6);
  char* const myws = p->myws;
  const u64 tk = __builtin_amdgcn_s_memrealtime();
  u32 bid = (((blockIdx.x & ~7u) | ((__builtin_amdgcn_s_getreg((3 << 11) | 20)) & 7u)) + NGEMM) % GRID, call = 0;
  if (bid >= (u32)NGEMM) { if (sender<true>(p, smem, myws, wv, lane, bid, call)) return; }
  else claim(smem, myws, wv, bid, call);
  bid = __builtin_amdgcn_readfirstlane(bid); call = __builtin_amdgcn_readfirstlane(call);
  if (__builtin_expect(bid >= (u32)NGEMM, 0)) { sender<false>(p, smem, myws, wv, lane, bid, call); return; }
  const u32 u = pack_index(bid, wv);
  if (u < NHOME) {
    const u32 t = u >> 5, xc = u & 7, sub = (u >> 3) & 3;
    const u32 c = xc < 4 ? sub * 4 + xc : 16 + (peer_of_xcd(p->rank, xc) & 3) * 4 + sub;
    const bool isw = c >= 16;
    const u32 cw = c - 16, l16 = lane * 16;
    while (__builtin_amdgcn_s_memrealtime() - tk < t * P_PP) __builtin_amdgcn_s_sleep(2);
    u32x4 d[8], s[8];
    load_raw(p, d, lane, isw ? 16 + (cw & 3) : c, (cw >> 2) & 3, t);
    WAIT_RAW(d);
    u32 meta = 0;
    bool eight = true;
    to_fragments(d, isw, smem + SCR_OFF + wv * 8192, lane);
    if (isw) {
#pragma unroll
      for (int j = 0; j < 8; ++j) s[j] = d[j];
    } else { low_slabs(d, s); eight = encode(d, s, meta, lane); }
    const u32 ud = (isw ? S_IMG + cw : (call & 1) * NS_PAR + (c < (u32)NA ? c : S_OWN2 - NA + c)) * NSTEP + t;
    char* const b = myws + (size_t)ud * SLOT + 4096;
    ST_SIX(l16, s, b);
    if (eight) { ST16(l16, s[6], b, 2048); ST16(l16, s[7], b, 3072); }
    asm volatile("s_waitcnt vmcnt(0)" ::: "memory");
    char* const m = myws + meta_off(ud);
    if (lane < 8) ST4(lane * 4, hdr_word(meta, call, lane), m);
  }
#pragma nounroll
  for (u32 v = unpack_index(bid, wv); v < 512; v += 12 * UNW) unpack_unit(myws, lane, call, (v + P_UROT(v >> 4)) & 15, v >> 4, tk);
  WG_BARRIER();
  u32* owed = nullptr;
#pragma nounroll
  for (int k = 0; k < 2; ++k) {
    const u32 td = p->tile[bid][k];
    if (!(td >> 23)) break;
    gemm_tile(p, smem, myws, wv, lane, bid, call, td & 15, (td >> 4) & 127, (td >> 11) & 7, (td >> 14) & 1, (td >> 15) & 63, (td >> 21) & 3, tk, k == 0 && p->tile[bid][1] != 0, owed);
  }
}

static void build_orders(Params& p) {
  struct E { double tau; unsigned short code; };
  E e[N_ORD];
  double sigma = P_SIGMA, ovl = P_OVL, ovl2 = P_OVL2, lead = P_LEAD, f1 = 14.5, f2 = 6, stg = P_STG;
  for (int kind = 0; kind < 2; ++kind) {
    int n = 0;
    for (int t = 0; t < NSTEP; ++t) {
      if (kind == 0) {
        for (int b = 0; b < NA; ++b) if (b != RB || t < RT0) e[n++] = {t + (b < N1 ? 0 : b < N1 + N2 ? NSTEP - ovl : 2 * NSTEP - ovl - ovl2 + (b >= N1 + N2 + 2 ? stg : 0)), (unsigned short)(b << 5 | t)};
      } else {
        for (int i = 0; i < 4; ++i) e[n++] = {t * sigma - lead, (unsigned short)((16 + i) << 5 | t)};
        for (int b = NA; b < 16; ++b) e[n++] = {t + 0.5, (unsigned short)(b << 5 | t)};
      }
    }
    if (kind == 1)
      for (int j = 0; j < P_RLY; ++j) {
        e[n++] = {f1 * sigma - lead, (unsigned short)(20 << 5 | j)};
        for (int i = 0; i < 3; ++i) e[n++] = {f2 + j, (unsigned short)((21 + i) << 5 | j)};
      }
    std::stable_sort(e, e + n, [](const E& a, const E& b) { return a.tau < b.tau; });
    for (int i = 0; i < N_ORD; ++i) p.ord[kind][i] = i < n ? e[i].code : 0;
  }
}

extern "C" void kernel_launch(int rank, int nranks, ncclComm_t comm, hipStream_t stream,
                              void* const* d_in, const int* in_sizes, int n_in,
                              void* d_out, int out_size, const int* in_dtypes, int out_dtype,
                              void* const* d_out_all, void* const* d_ws_all, size_t ws_bytes) {
  (void)comm; (void)in_dtypes; (void)out_dtype; (void)d_out;
  if (nranks != NR || (unsigned)rank >= (unsigned)NR || n_in != 2 || ws_bytes < WS_NEED ||
      in_sizes[0] != 512 * KDIM || in_sizes[1] != KDIM * NLOC || out_size != 4096 * NLOC) {
    fprintf(stderr, "unexpected problem shape\n");
    return;
  }
  Params p = {};
  p.x = (const char*)d_in[0];
  p.w = (const char*)d_in[1];
  for (int i = 0; i < NR; ++i) {
    p.out[i] = (char*)d_out_all[phys_rank(i)]; p.ws[i] = (char*)d_ws_all[phys_rank(i)];
    if ((int)phys_rank(i) == rank) p.rank = i;
  }
  p.myws = (char*)d_ws_all[rank];
  build_orders(p);
  for (int i = 0; i < NS_PAR + 4; ++i) p.row[i] = i < 64 || (i >= S_OWN2 && i < NS_PAR) ? global_row(p.rank, i) : 0;
  for (u32 b = 0; b < (u32)NGEMM; ++b)
    for (int k = 0; k < 2; ++k) {
      u32 nb, sid0, tn, ch, fl; int hold;
      p.tile[b][k] = tile_of(p.rank, b, k, nb, sid0, tn, ch, fl, hold) ? nb | sid0 << 4 | tn << 11 | ch << 14 | fl << 15 | (u32)hold << 21 | 1u << 23 : 0u;
    }
  hipLaunchKernelGGL(ag_gemm_silu, dim3(GRID), dim3(NTHREAD), 0, stream, p);
  hipError_t err = hipGetLastError();
  if (err != hipSuccess) fprintf(stderr, "launch failed: %s\n", hipGetErrorString(err));
}
```

```cpp
#include <hip/hip_runtime.h>
#include <rccl/rccl.h>
#include <cstdio>
#include <cstdint>
#include <cstdlib>
#include <algorithm>

typedef unsigned u32;
typedef unsigned long long u64;
typedef unsigned u32x4 __attribute__((ext_vector_type(4)));
typedef unsigned u32x8 __attribute__((ext_vector_type(8)));
typedef unsigned short u16x2 __attribute__((ext_vector_type(2)));
typedef __bf16 bf16x2 __attribute__((ext_vector_type(2)));
typedef __bf16 bf16x4 __attribute__((ext_vector_type(4)));
typedef __bf16 bf16x8 __attribute__((ext_vector_type(8)));
typedef float f32x2 __attribute__((ext_vector_type(2)));
typedef float f32x4 __attribute__((ext_vector_type(4)));
typedef float f32x16 __attribute__((ext_vector_type(16)));
#define LAS __attribute__((address_space(3)))

#define P_NA 11
#define P_NCWG 4
#define P_WIN 3
#define P_WIN0 4
#define P_WIN1 5
#define P_TAPER 8
#define P_DIST 3
#define P_BD 7
#define P_ALOOK 3
#define P_LPRIO 2
#define P_UROT(t) (5 * (t))
#define P_STG 3
#define P_SIGMA 0.1
#define P_OVL 0.0
#define P_OVL2 0.0
#define P_LEAD 0.7
#define P_PP 15
#define P_PARK 10
#define P_RLY 2
#define P_S0 16
#define P_S1 16
#define P_S2 4
#define P_UT 100
#define P_REL 2

constexpr int NR = 8;
constexpr int KDIM = 4096, NLOC = 512;
constexpr int NSTEP = 32;
constexpr int NA = P_NA, NB = 16 - NA;
constexpr int N1 = 3, N3 = 4, N2 = NA - N1 - N3;
constexpr int S_1 = NA, S_2 = S_1 + 3 * N1, S_INTER = S_2 + 3 * N2, S_3 = 64 - 3 * N3, S_WIN = 64, S_RLY = 80, S_OWN2 = 81;
constexpr int RB = N1 + N2 - 1, RT0 = NSTEP - 4 * P_RLY;
constexpr int NS_PAR = S_OWN2 + NB, S_IMG = 2 * NS_PAR, S_PARK = S_IMG + 32, NSTREAM = S_PARK + 4;
constexpr size_t SLOT = 8192;
constexpr size_t META_OFF = (size_t)NSTREAM * NSTEP * SLOT;
constexpr size_t DONE_OFF = META_OFF + (size_t)NSTREAM * NSTEP * 32;
constexpr size_t REL_OFF = DONE_OFF + 1024;
constexpr size_t CNT_OFF = REL_OFF + 1024;
constexpr size_t TMO_OFF = CNT_OFF + 256 * 128;
constexpr size_t WS_NEED = TMO_OFF + 128;
constexpr int N_INTRA = NA * NSTEP - 4 * P_RLY, N_INTER = NB * NSTEP + 4 * NSTEP + 4 * P_RLY;
constexpr int N_ORD = N_INTRA > N_INTER ? N_INTRA : N_INTER;
constexpr int RT_MED = 13;
constexpr int NGEMM = RT_MED * 16 + 16;
constexpr int NFLAG = 2 * RT_MED + 12 + P_PARK;
constexpr int NCWG = P_NCWG;
constexpr int GRID = NGEMM + 8 * NCWG;
static_assert(P_PARK == 10 && P_BD <= 7 && P_RLY <= 8 && N_ORD % 2 == 0 && NB > 4 && NB <= 8 && S_INTER % 4 == 0, "tile map");
static_assert(GRID <= 256, "one workgroup per CU, all resident");
constexpr int NTHREAD = 768;
constexpr int SCR_OFF = 0, EPI_OFF = 32768;
constexpr int HDR_OFF = 131072;
constexpr int CLAIM_OFF = HDR_OFF + 4096 + 64;
constexpr int LDS_BYTES = CLAIM_OFF + 64;
constexpr u64 DEADLINE_TICKS = 80000000ull;
constexpr int ERS = 272;

#define P_PHYS 0x75416320u
__host__ __device__ inline u32 phys_rank(u32 l) { return (P_PHYS >> (4 * l)) & 7u; }
__host__ __device__ inline u32 peer_of_xcd(u32 rank, u32 x) { return (((rank ^ x) & 4u) | ((rank + x) & 3u)) & 7u; }

struct Params {
  const char* x;
  const char* w;
  char* myws;
  int rank, pad;
  char* ws[NR];
  char* out[NR];
  u32 row[NS_PAR + 4];
  u32 tile[NGEMM][2];
  alignas(4) unsigned short ord[2][N_ORD];
};
static_assert(sizeof(Params) <= 4096, "kernel argument");
typedef const __attribute__((address_space(4))) Params* KP;

#define WG_BARRIER() do { asm volatile("" ::: "memory"); __builtin_amdgcn_s_waitcnt(0xC07F); \
    asm volatile("s_barrier" ::: "memory"); } while (0)
#define ST16(voff, data, sbase, imm) \
  asm volatile("global_store_dwordx4 %0, %1, %2 offset:" #imm " sc0 sc1\n\ts_nop 1" :: "v"(voff), "v"(data), "s"(sbase) : "memory")
#define ST4(voff, data, sbase) \
  asm volatile("global_store_dword %0, %1, %2 sc0 sc1" :: "v"(voff), "v"(data), "s"(sbase) : "memory")
#define ST_SIX(voff, s, b) do { ST16(voff, s[0], b, -4096); ST16(voff, s[1], b, -3072); ST16(voff, s[2], b, -2048); \
    ST16(voff, s[3], b, -1024); ST16(voff, s[4], b, 0); ST16(voff, s[5], b, 1024); } while (0)

__device__ __forceinline__ u32* counter(char* myws) { return (u32*)(myws + CNT_OFF + blockIdx.x * 128); }
__device__ __forceinline__ u32 take_role(char* myws, u32 r, u32 call) {
#pragma nounroll
  for (int i = 0; i < GRID; ++i, r = (r + 1) % GRID) {
    u32 old;
    asm volatile("s_mov_b64 exec, 1\n\tglobal_atomic_umax %0, %1, %2, %3 sc0\n\ts_mov_b64 exec, -1\n\ts_waitcnt vmcnt(0)"
                 : "=&v"(old) : "v"(r * 128), "v"(call), "s"(myws + CNT_OFF + 64) : "memory");
    if ((u32)__builtin_amdgcn_readfirstlane(old) < call) break;
  }
  return r;
}
__device__ __forceinline__ void claim(char* smem, char* myws, int wv, u32& bid, u32& call) {
  LAS u32* const cl = (LAS u32*)(smem + CLAIM_OFF);
  if (wv == 0) {
    const u32 c = __builtin_amdgcn_readfirstlane(__hip_atomic_load(counter(myws), __ATOMIC_RELAXED, __HIP_MEMORY_SCOPE_AGENT)) + 1u;
    cl[1] = take_role(myws, bid, c);
    cl[0] = c;
    if (threadIdx.x == 0) __hip_atomic_store(counter(myws), c, __ATOMIC_RELAXED, __HIP_MEMORY_SCOPE_AGENT);
  }
  WG_BARRIER();
  call = __builtin_amdgcn_readfirstlane(cl[0]);
  bid = __builtin_amdgcn_readfirstlane(cl[1]);
}
__device__ __forceinline__ u32 call_tag(u32 call) { return (call % 2047u + 1u) << 21; }
__device__ __forceinline__ u32 hdr_word(u32 data, u32 call, int lane) { return lane == 7 ? call : data | call_tag(call); }
__device__ __forceinline__ bool hdr_there(u32 word, u32 call, int lane) {
  return __all((lane & 7) == 7 ? word >= call : (word ^ call_tag(call)) < (1u << 21));
}

__device__ __forceinline__ bool spin_fail(char* myws, u32 call, u64 t0) {
  u32* tmo = (u32*)(myws + TMO_OFF);
  u32 t = __builtin_amdgcn_readfirstlane(__hip_atomic_load(tmo, __ATOMIC_RELAXED, __HIP_MEMORY_SCOPE_AGENT));
  if (t == call || __builtin_amdgcn_s_memrealtime() - t0 > DEADLINE_TICKS) {
    __hip_atomic_store(tmo, call, __ATOMIC_RELAXED, __HIP_MEMORY_SCOPE_AGENT);
    return true;
  }
  return false;
}

__device__ __forceinline__ size_t meta_off(u32 unit) { return META_OFF + (size_t)unit * 32; }
__device__ __forceinline__ u32 dest_stream(u32 sl, u32 rank, u32 dest, u32 par) {
  const u32 j = ((rank - dest) & 3) - 1;
  const u32 s = sl >= 16 ? S_WIN + (rank & 3) * 4 + (sl - 16) : sl >= (u32)NA ? S_INTER + (rank & 3) * NB + (sl - NA)
              : sl >= (u32)(N1 + N2) ? S_3 + j * N3 + (sl - N1 - N2) : sl >= (u32)N1 ? S_2 + j * N2 + (sl - N1) : S_1 + j * N1 + sl;
  return par * NS_PAR + s;
}

inline u32 global_row(u32 rank, u32 sid) {
  const u32 g0 = rank & 4u, o0 = g0 ^ 4u;
  u32 src, bi;
  if (sid < (u32)S_1) { src = rank; bi = sid; }
  else if (sid < (u32)S_2) { const u32 q = sid - S_1; src = g0 + ((rank + 1 + q / N1) & 3); bi = q % N1; }
  else if (sid < (u32)S_INTER) { const u32 q = sid - S_2; src = g0 + ((rank + 1 + q / N2) & 3); bi = N1 + q % N2; }
  else if (sid < (u32)S_3) { const u32 q = sid - S_INTER; src = o0 + q / NB; bi = NA + q % NB; }
  else if (sid >= (u32)S_OWN2) { src = rank; bi = NA + sid - S_OWN2; }
  else { const u32 q = sid - S_3; src = g0 + ((rank + 1 + q / N3) & 3); bi = N1 + N2 + q % N3; }
  return (phys_rank(src & 7u) * 512 + bi * 32) & 4095u;
}

__device__ __forceinline__ void load_raw(KP p, u32x4 (&d)[8], int lane, u32 sl, u32 q, u32 t) {
  const bool isw = sl >= 16;
  const u32 rs = isw ? NLOC * 2 : KDIM * 2;
  const u32 vo = (lane >> 4) * rs + (lane & 15) * 16;
  const u64 aw = (u64)p->w + (u64)((t * 128 + (sl & 3) * 32) * (NLOC * 2) + q * 256), ax = (u64)p->x + (u64)(sl * 32 * (KDIM * 2) + t * 256);
  const u64 b0 = isw ? aw : ax;
#pragma unroll
  for (int j = 0; j < 8; ++j)
    asm volatile("global_load_dwordx4 %0, %1, %2" : "=&v"(d[j]) : "v"(vo), "s"(b0 + (u64)(j * 4 * rs)) : "memory");
}
#define WAIT_RAW(d) asm volatile("s_waitcnt vmcnt(0)" : "+v"(d[0]), "+v"(d[1]), "+v"(d[2]), "+v"(d[3]), "+v"(d[4]), "+v"(d[5]), "+v"(d[6]), "+v"(d[7]) :: "memory")
__device__ __forceinline__ void to_fragments(u32x4 (&d)[8], bool isw, char* scr, int lane) {
  LAS char* const sc = (LAS char*)scr;
  auto off = [](u32 row, u32 ch) { return 256u * row + 16u * (ch ^ (((row & 3) << 2) | ((row >> 2) & 3))); };
#pragma unroll
  for (int j = 0; j < 8; ++j) *(LAS u32x4*)(sc + off(4 * j + (lane >> 4), lane & 15)) = d[j];
  asm volatile("" ::: "memory");
  if (isw) {
    const u32 r0 = 8 * (lane >> 5) + ((lane & 15) >> 2), c0 = 2 * ((lane >> 4) & 1) + ((lane & 3) >> 1);
#pragma unroll
    for (int j = 0; j < 8; ++j) {
      const bf16x4 lo = __builtin_amdgcn_ds_read_tr16_b64_v4bf16((LAS bf16x4*)(sc + off(16 * (j & 1) + r0, 4 * (j >> 1) + c0) + 8 * (lane & 1)));
      const bf16x4 hi = __builtin_amdgcn_ds_read_tr16_b64_v4bf16((LAS bf16x4*)(sc + off(16 * (j & 1) + r0 + 4, 4 * (j >> 1) + c0) + 8 * (lane & 1)));
      d[j] = __builtin_bit_cast(u32x4, __builtin_shufflevector(lo, hi, 0, 1, 2, 3, 4, 5, 6, 7));
    }
  } else {
#pragma unroll
    for (int j = 0; j < 8; ++j) d[j] = *(const LAS u32x4*)(sc + off(lane & 31, 2 * j + (lane >> 5)));
  }
  asm volatile("" ::: "memory");
}
__device__ __forceinline__ void low_slabs(const u32x4 (&d)[8], u32x4 (&s)[8]) {
#pragma unroll
  for (int i = 0; i < 16; ++i) s[i >> 2][i & 3] = __builtin_amdgcn_perm(d[i >> 1][2 * (i & 1) + 1], d[i >> 1][2 * (i & 1)], 0x06040200u);
}
__device__ __forceinline__ bool encode(const u32x4 (&d)[8], u32x4 (&s)[8], u32& meta, int lane) {
  u32 H[16], T[16];
  u16x2 m1 = {0, 0}, m2 = {0, 0}, m3 = {0, 0}, m4 = {0, 0};
#pragma unroll
  for (int i = 0; i < 16; ++i) {
    const u32 a = d[i >> 1][2 * (i & 1)], b = d[i >> 1][2 * (i & 1) + 1];
    H[i] = __builtin_amdgcn_perm(b, a, 0x07050301u);
  }
#pragma unroll
  for (int i = 0; i < 16; i += 2) {
    T[i] = H[i] | 0x80808080u;
    T[i + 1] = H[i + 1] | 0x80808080u;
    m1 = __builtin_elementwise_max(m1, __builtin_bit_cast(u16x2, T[i]));
    m2 = __builtin_elementwise_max(m2, __builtin_bit_cast(u16x2, T[i] << 8));
    m3 = __builtin_elementwise_max(m3, __builtin_bit_cast(u16x2, T[i + 1]));
    m4 = __builtin_elementwise_max(m4, __builtin_bit_cast(u16x2, T[i + 1] << 8));
  }
  m1 = __builtin_elementwise_max(__builtin_elementwise_max(m1, m2), __builtin_elementwise_max(m3, m4));
  u32 mx = (u32)(m1.x > m1.y ? m1.x : m1.y);
  mx = max(mx, (u32)__builtin_amdgcn_update_dpp(0, mx, 0x111, 0xf, 0xf, true));
  mx = max(mx, (u32)__builtin_amdgcn_update_dpp(0, mx, 0x112, 0xf, 0xf, true));
  mx = max(mx, (u32)__builtin_amdgcn_update_dpp(0, mx, 0x114, 0xf, 0xf, true));
  mx = max(mx, (u32)__builtin_amdgcn_update_dpp(0, mx, 0x118, 0xf, 0xf, true));
  const u32 top = max(max((u32)__builtin_amdgcn_readlane(mx, 15), (u32)__builtin_amdgcn_readlane(mx, 31)),
                      max((u32)__builtin_amdgcn_readlane(mx, 47), (u32)__builtin_amdgcn_readlane(mx, 63)));
  const u32 h7 = (top >> 8) & 0x7f;
  const u32 base = __builtin_amdgcn_readfirstlane(h7 > 7 ? h7 - 7 : 0);
  const u32 base4 = base * 0x01010101u;
  u32 ok = 0xffffffffu;
  u32 N[8];
#pragma unroll
  for (int j = 0; j < 8; ++j) {
    const u32 ta = T[2 * j] - base4, tb = T[2 * j + 1] - base4;
    ok &= ta & tb;
    const u32 ca = (ta & 0x07070707u) | (H[2 * j] & 0x80808080u);
    const u32 cb = (tb & 0x07070707u) | (H[2 * j + 1] & 0x80808080u);
    N[j] = ca | __builtin_amdgcn_alignbit(cb, cb, 28);
  }
  meta = 0;
  int c0 = 0, c1 = 0;
  if (__any((~ok & 0x80808080u) != 0)) {
    u32 M0 = 0, M1 = 0;
#pragma unroll
    for (int i = 0; i < 8; ++i) {
      M0 |= ~((T[i] - base4) >> i) & (0x80808080u >> i);
      M1 |= ~((T[8 + i] - base4) >> i) & (0x80808080u >> i);
    }
    u64 bal = __ballot((M0 | M1) != 0);
    while (bal != 0 && c0 + c1 <= 6) {
      const int ln = __builtin_ctzll(bal);
      bal &= bal - 1;
      u64 mm = (u64)(u32)__builtin_amdgcn_readlane(M0, ln) | (u64)(u32)__builtin_amdgcn_readlane(M1, ln) << 32;
      while (mm != 0) {
        const u32 bit = __builtin_ctzll(mm), hf = bit >> 5, by = (bit >> 3) & 3, i = hf * 8 + 7 - (bit & 7);
        mm &= mm - 1;
        c0 += hf ^ 1; c1 += hf;
        u32 hw = 0;
#pragma unroll
        for (int z = 0; z < 16; ++z) hw |= H[z] & (u32)__builtin_amdgcn_sbfe(1 << i, z, 1);
        const u32 hb = ((u32)__builtin_amdgcn_readlane(hw, ln) >> (8 * by)) & 0x7f;
        const u32 ent = (u32)ln | (i & 7) << 6 | by << 9 | ((((hb - base) & 7) + base) ^ hb) << 11;
        if (lane == (hf ? 7 - c1 : c0)) meta = ent;
      }
    }
  }
  const bool raw = c0 + c1 > 6;
  if (lane == 0) meta = base | (raw ? 0x400u : (u32)(c0 << 11 | c1 << 14));
  s[4] = raw ? u32x4{H[0], H[1], H[2], H[3]} : u32x4{N[0], N[1], N[2], N[3]};
  s[5] = raw ? u32x4{H[4], H[5], H[6], H[7]} : u32x4{N[4], N[5], N[6], N[7]};
  s[6] = u32x4{H[8], H[9], H[10], H[11]};
  s[7] = u32x4{H[12], H[13], H[14], H[15]};
#pragma unroll
  for (int j = 0; j < 8; ++j) asm volatile("" : "+v"(s[j]));
  return raw;
}

template <int N> struct Pay { u32x4 v[N]; };
__device__ __forceinline__ void pay_load(Pay<6>& P, u32 l16, const char* sb) {
  asm volatile("global_load_dwordx4 %0, %6, %7 offset:-4096 sc0 sc1\n\t"
               "global_load_dwordx4 %1, %6, %7 offset:-3072 sc0 sc1\n\t"
               "global_load_dwordx4 %2, %6, %7 offset:-2048 sc0 sc1\n\t"
               "global_load_dwordx4 %3, %6, %7 offset:-1024 sc0 sc1\n\t"
               "global_load_dwordx4 %4, %6, %7 offset:0 sc0 sc1\n\t"
               "global_load_dwordx4 %5, %6, %7 offset:1024 sc0 sc1"
               : "=&v"(P.v[0]), "=&v"(P.v[1]), "=&v"(P.v[2]), "=&v"(P.v[3]), "=&v"(P.v[4]), "=&v"(P.v[5]) : "v"(l16), "s"(sb) : "memory");
}
__device__ __forceinline__ void pay_load(Pay<3>& P, u32 vlo, u32 vcode, const char* sb) {
  asm volatile("global_load_dwordx4 %2, %4, %5 offset:0 sc0 sc1\n\t"
               "global_load_dwordx4 %0, %3, %5 offset:-4096 sc0 sc1\n\t"
               "global_load_dwordx4 %1, %3, %5 offset:-3072 sc0 sc1"
               : "=&v"(P.v[0]), "=&v"(P.v[1]), "=&v"(P.v[2]) : "v"(vlo), "v"(vcode), "s"(sb) : "memory");
}
template <int C> __device__ __forceinline__ void pay_wait(Pay<6>& P) {
  asm volatile("s_waitcnt vmcnt(%6)" : "+v"(P.v[0]), "+v"(P.v[1]), "+v"(P.v[2]), "+v"(P.v[3]), "+v"(P.v[4]), "+v"(P.v[5]) : "n"(C) : "memory");
}
template <int C> __device__ __forceinline__ void pay_wait(Pay<3>& P) {
  asm volatile("s_waitcnt vmcnt(%3)" : "+v"(P.v[0]), "+v"(P.v[1]), "+v"(P.v[2]) : "n"(C) : "memory");
}
__device__ __forceinline__ u32x8 load_two(const char* b, u32 l16) {
  u32x4 x, y;
  asm volatile("global_load_dwordx4 %0, %2, %3 sc0 sc1\n\t"
               "global_load_dwordx4 %1, %2, %3 offset:1024 sc0 sc1\n\ts_waitcnt vmcnt(0)"
               : "=&v"(x), "=&v"(y) : "v"(l16), "s"(b) : "memory");
  return __builtin_shufflevector(x, y, 0, 1, 2, 3, 4, 5, 6, 7);
}

__device__ __forceinline__ void load_slot(u32x4 (&d)[8], u32 l16, const char* sb) {
  asm volatile("global_load_dwordx4 %0, %8, %9 offset:-4096 sc0 sc1\n\tglobal_load_dwordx4 %1, %8, %9 offset:-3072 sc0 sc1\n\t"
               "global_load_dwordx4 %2, %8, %9 offset:-2048 sc0 sc1\n\tglobal_load_dwordx4 %3, %8, %9 offset:-1024 sc0 sc1\n\t"
               "global_load_dwordx4 %4, %8, %9 offset:0 sc0 sc1\n\tglobal_load_dwordx4 %5, %8, %9 offset:1024 sc0 sc1\n\t"
               "global_load_dwordx4 %6, %8, %9 offset:2048 sc0 sc1\n\tglobal_load_dwordx4 %7, %8, %9 offset:3072 sc0 sc1\n\ts_waitcnt vmcnt(0)"
               : "=&v"(d[0]), "=&v"(d[1]), "=&v"(d[2]), "=&v"(d[3]), "=&v"(d[4]), "=&v"(d[5]), "=&v"(d[6]), "=&v"(d[7])
               : "v"(l16), "s"(sb) : "memory");
}

template <bool ask>
__device__ __forceinline__ bool sender(KP p, char* smem, char* myws, int wv, int lane, u32& bid, u32& call) {
  const u32 rank = p->rank;
  const u32 cb = bid - NGEMM;
  const u32 link = cb & 7, cw = cb >> 3;
  const u32 l16 = lane * 16;
  if (link != 0) {
    const u32 dest = peer_of_xcd(rank, link);
    const bool intra = link < 4;
    const int n = ((intra ? N_INTRA : N_INTER) - (int)cw + NCWG - 1) / NCWG;
    auto code_of = [=](int k) {
      const u32 e = cw + (u32)(k < n ? k : 0) * NCWG;
      const u32 word = ((const __attribute__((address_space(4))) u32*)p->ord[intra ? 0 : 1])[e >> 1];
      return (word >> (16 * (e & 1))) & 0xffffu;
    };
    LAS int* const done = (LAS int*)(smem + 12 * 8192);
    u32x4 d[8];
    auto fetch = [&](u32 c) {
      const bool rly = c >= 20 * 32;
      load_raw(p, d, lane, rly ? RB : c >> 5, dest & 3, rly ? RT0 + 4 * (c & 7) + (dest & 3) : c & 31);
    };
    const u32 e0 = cw + wv * NCWG;
    u32 code = intra ? (e0 % N1) << 5 | e0 / N1 : (16 + (e0 & 3)) << 5 | e0 >> 2;
    LAS u32* const got = (LAS u32*)(smem + CLAIM_OFF + 16);
    if (wv >= 4) {
      if (threadIdx.x == 256) { *done = 0; *got = 0; }
      WG_BARRIER();
      if (wv >= 8) __builtin_amdgcn_s_sleep(4);
    }
    fetch(code);
    u32 ticket = 0;
    if (ask) asm volatile("global_load_dword %0, %1, %2 sc1" : "=&v"(ticket) : "v"(0u), "s"(counter(myws)) : "memory");
    if (wv < 4) WG_BARRIER();
    bool lost = false;
    char* const rws = p->ws[dest];
    if (wv < 4) __builtin_amdgcn_s_setprio(2); else if (wv < 8) __builtin_amdgcn_s_setprio(1);
    int k = wv;
    for (;; k += 12) {
      const u32 sl = code >> 5;
      u32 ud = sl < 20 ? dest_stream(sl, rank, dest, 0) * NSTEP + (code & 31) : sl == 20 ? S_RLY * NSTEP + (rank & 3) * 8 + (code & 7)
             : (S_2 + (sl - 21) * N2 + RB - N1) * NSTEP + RT0 + 4 * (code & 7) + (rank & 3);
      asm volatile("" : "+s"(ud));
      WAIT_RAW(d);
      if (ask && k < 24) {
        if (k < 12) {
          asm volatile("" : "+v"(ticket));
          call = __builtin_amdgcn_readfirstlane(ticket) + 1u;
          if (wv == 11) __hip_atomic_store(got, take_role(myws, bid, call) + 1u, __ATOMIC_RELAXED, __HIP_MEMORY_SCOPE_WORKGROUP);
        } else {
          u32 g;
          while ((g = __builtin_amdgcn_readfirstlane(__hip_atomic_load(got, __ATOMIC_RELAXED, __HIP_MEMORY_SCOPE_WORKGROUP))) == 0) __builtin_amdgcn_s_sleep(1);
          if (g - 1 != bid) { bid = g - 1; lost = true; break; }
        }
      }
      ud += (call & 1) * (NS_PAR * NSTEP);
      char* const b = rws + (size_t)ud * SLOT + 4096;
      u32x4 s[8];
      u32 meta;
      bool raw;
      if (P_RLY > 0 && sl > 20) {
        const u32 us = ((call & 1) * NS_PAR + S_RLY) * NSTEP + ((dest + sl - 20) & 3) * 8 + (code & 7);
        const u64 t0 = __builtin_amdgcn_s_memrealtime();
        const u32* const hd = (const u32*)(myws + meta_off(us)) + (lane & 7);
        for (u32 it = 1;; ++it) {
          meta = __hip_atomic_load(hd, __ATOMIC_RELAXED, __HIP_MEMORY_SCOPE_SYSTEM);
          if (hdr_there(meta, call, lane)) break;
          if ((it & 31) == 0 && spin_fail(myws, call, t0)) break;
          __builtin_amdgcn_s_sleep(2);
        }
        load_slot(s, l16, myws + (size_t)us * SLOT + 4096);
        raw = __builtin_amdgcn_readfirstlane(meta) & 0x400;
      } else {
        to_fragments(d, sl >= 16 && sl < 20, smem + wv * 8192, lane);
        low_slabs(d, s);
        if (k < P_WIN0) { ST16(l16, s[0], b, -4096); ST16(l16, s[1], b, -3072); ST16(l16, s[2], b, -2048); ST16(l16, s[3], b, -1024); }
        raw = encode(d, s, meta, lane);
        meta = hdr_word(meta, call, lane);
      }
      __builtin_amdgcn_s_setprio(0);
      const int need = k < P_WIN0 ? 0 : k + 1 - max(P_WIN, P_WIN1 - (k - P_WIN0) / P_TAPER);
      while (__builtin_amdgcn_readfirstlane(__hip_atomic_load(done, __ATOMIC_RELAXED, __HIP_MEMORY_SCOPE_WORKGROUP)) < need) __builtin_amdgcn_s_sleep(1);
      if (k >= P_WIN0) { ST16(l16, s[0], b, -4096); ST16(l16, s[1], b, -3072); ST16(l16, s[2], b, -2048); ST16(l16, s[3], b, -1024); }
      ST16(l16, s[4], b, 0); ST16(l16, s[5], b, 1024);
      if (raw) { ST16(l16, s[6], b, 2048); ST16(l16, s[7], b, 3072); }
      code = code_of(k + 12);
      asm volatile("" : "+s"(code));
      asm volatile("s_waitcnt vmcnt(0)" ::: "memory");
      char* const m = rws + meta_off(ud);
      if (lane < 8) ST4(lane * 4, meta, m);
      if (lane == 0) __hip_atomic_fetch_add(done, 1, __ATOMIC_RELAXED, __HIP_MEMORY_SCOPE_WORKGROUP);
      if (k + 12 >= n) { k += 12; break; }
      fetch(code);
      WAIT_RAW(d);
    }
    if (lost) WG_BARRIER();
    if (ask && threadIdx.x == 0) __hip_atomic_store(counter(myws), call, __ATOMIC_RELAXED, __HIP_MEMORY_SCOPE_AGENT);
    if (lost) { __builtin_amdgcn_s_setprio(0); return false; }
    const u32 y = cw + (u32)(k - n) * NCWG;
    if (!intra && y < 2u * P_PARK) {
      const u32 us = (S_PARK + (dest & 3)) * NSTEP + y;
      const u64 t0 = __builtin_amdgcn_s_memrealtime();
      const u32* const hd = (const u32*)(myws + meta_off(us));
      for (u32 it = 1; (u32)__builtin_amdgcn_readfirstlane(__hip_atomic_load(hd, __ATOMIC_RELAXED, __HIP_MEMORY_SCOPE_SYSTEM)) < call; ++it) {
        if ((it & 31) == 0 && spin_fail(myws, call, t0)) break;
        __builtin_amdgcn_s_sleep(4);
      }
      load_slot(d, l16, myws + (size_t)us * SLOT + 4096);
      if (__hip_atomic_load((u32*)(myws + TMO_OFF), __ATOMIC_RELAXED, __HIP_MEMORY_SCOPE_AGENT) == call) d[0] = u32x4{0x7fc07fc0u, 0x7fc07fc0u, 0x7fc07fc0u, 0x7fc07fc0u};
      const u32 tile = y >> 1, sid = (tile >> 1) * 4 + (y & 1) * 2;
      char* const ob = p->out[dest] + (rank & 3) * 256 + (tile & 1) * 128;
      const u32 vo = (lane >> 3) * (NLOC * 2) + (lane & 7) * 16;
      char* r0 = ob + (size_t)p->row[sid] * (NLOC * 2);
      char* r1 = ob + (size_t)p->row[sid + 1] * (NLOC * 2);
      asm volatile("" : "+s"(r0), "+s"(r1));
      while (__builtin_amdgcn_readfirstlane(__hip_atomic_load(done, __ATOMIC_RELAXED, __HIP_MEMORY_SCOPE_WORKGROUP)) < k - (P_WIN - 1)) __builtin_amdgcn_s_sleep(1);
#pragma unroll
      for (int g = 0; g < 8; ++g) {
        char* const b = (g < 4 ? r0 : r1) + (g & 3) * 8 * (NLOC * 2);
        ST16(vo, d[g], b, 0);
      }
      if (y + P_REL * NCWG >= 2u * P_PARK && y + P_REL * NCWG < 2u * P_PARK + NCWG && lane == 0)
        __hip_atomic_store((u32*)(myws + REL_OFF + link * 128) + cw, call, __ATOMIC_RELAXED, __HIP_MEMORY_SCOPE_AGENT);
      asm volatile("s_waitcnt vmcnt(0)" ::: "memory");
      char* const fl = rws + DONE_OFF + ((rank & 3) * 64 + ((y & 1) ? 2 * RT_MED + 12 + tile : tile)) * 4;
      if (lane == 0) { ST4(0u, call, fl); __hip_atomic_fetch_add(done, 1, __ATOMIC_RELAXED, __HIP_MEMORY_SCOPE_WORKGROUP); }
    }
    return true;
  }
  if (ask) {
    const u32 asked = bid;
    claim(smem, myws, wv, bid, call);
    if (bid != asked) return false;
  }
  if (cb == 0 && wv == 0) {
    const u64 t0 = __builtin_amdgcn_s_memrealtime();
    while (__builtin_amdgcn_s_memrealtime() - t0 < N_INTRA * P_UT / 10) __builtin_amdgcn_s_sleep(16);
    const char* const f = myws + DONE_OFF;
    const int f0 = (4 * lane) & 63;
    for (u32 n = 1;; ++n) {
      u32x4 v;
      asm volatile("global_load_dwordx4 %0, %1, %2 sc0 sc1\n\ts_waitcnt vmcnt(0)" : "=&v"(v) : "v"(l16), "s"(f) : "memory");
      if (__all((f0 >= NFLAG || v[0] >= call) && (f0 + 1 >= NFLAG || v[1] >= call) && (f0 + 2 >= NFLAG || v[2] >= call) && (f0 + 3 >= NFLAG || v[3] >= call))) break;
      if ((n & 31) == 0 && spin_fail(myws, call, t0)) {
        if (lane == 0) *(unsigned short*)(p->out[rank] + (size_t)phys_rank(rank) * 512 * (NLOC * 2)) = 0x7fc0;
        break;
      }
    }
  }
  return true;
}

__device__ __forceinline__ u32x4 join(u32 l0, u32 l1, u32 h0, u32 h1) {
  return u32x4{__builtin_amdgcn_perm(h0, l0, 0x05010400u), __builtin_amdgcn_perm(h0, l0, 0x07030602u),
               __builtin_amdgcn_perm(h1, l1, 0x05010400u), __builtin_amdgcn_perm(h1, l1, 0x07030602u)};
}
__device__ __forceinline__ void fix_word(u32 (&h)[8], u32 e) {
  asm volatile("s_lshl_b64 exec, 1, %8\n\ts_getpc_b64 vcc\n\ts_add_u32 vcc_lo, vcc_lo, %9\n\ts_addc_u32 vcc_hi, vcc_hi, 0\n\ts_setpc_b64 vcc\n\t"
               "v_xor_b32_e32 %0, %10, %0\n\ts_branch 1f\n\tv_xor_b32_e32 %1, %10, %1\n\ts_branch 1f\n\t"
               "v_xor_b32_e32 %2, %10, %2\n\ts_branch 1f\n\tv_xor_b32_e32 %3, %10, %3\n\ts_branch 1f\n\t"
               "v_xor_b32_e32 %4, %10, %4\n\ts_branch 1f\n\tv_xor_b32_e32 %5, %10, %5\n\ts_branch 1f\n\t"
               "v_xor_b32_e32 %6, %10, %6\n\ts_branch 1f\n\tv_xor_b32_e32 %7, %10, %7\n"
               "1:\n\ts_mov_b64 exec, -1"
               : "+v"(h[0]), "+v"(h[1]), "+v"(h[2]), "+v"(h[3]), "+v"(h[4]), "+v"(h[5]), "+v"(h[6]), "+v"(h[7])
               : "s"(e), "s"(((e >> 3) & 56) + 12), "s"(((e >> 11) & 0x7f) << ((e >> 6) & 24)) : "vcc", "scc");
}

struct Stream {
  const char* sb; const char* mb; char* myws;
  u32 l16, tag, call; u64 t0;
  u32x4 W; int rdy; bool eager;
  __device__ __forceinline__ void init(char* ws, u32 call_, u32 sid, int lane) {
    myws = ws; call = call_; tag = call_tag(call_);
    sb = ws + (size_t)sid * NSTEP * SLOT + 4096;
    mb = ws + meta_off(sid * NSTEP);
    l16 = lane * 16;
    t0 = __builtin_amdgcn_s_memrealtime();
    W = u32x4{0u, 0u, 0u, 0u}; rdy = 0; eager = false;
  }
  __device__ __forceinline__ void poll(bool wide = true) {
    const u64 mask = (wide ? 0xffffull : 0xffull) << (rdy < 28 ? (2 * rdy) & 56 : 56);
    asm volatile("s_mov_b64 exec, %3\n\tglobal_load_dwordx4 %0, %1, %2 sc0 sc1\n\ts_mov_b64 exec, -1"
                 : "+v"(W) : "v"(l16), "s"(mb), "s"(mask) : "memory");
  }
  __device__ __forceinline__ void look() {
    asm volatile("" : "+v"(W));
    const u32 x = (W[0] ^ tag) | (W[1] ^ tag) | (W[2] ^ tag);
    const u64 odd = 0xaaaaaaaaaaaaaaaaull;
    const u64 bad = __ballot(x >= (1u << 21)) | (__ballot((W[3] ^ tag) >= (1u << 21)) & ~odd) | (__ballot(W[3] < call) & odd);
    rdy = bad ? __builtin_ctzll(bad) >> 1 : NSTEP;
  }
  __device__ __forceinline__ void spin(int t) {
    for (u32 n = 1; rdy <= t; ++n) {
      const int was = rdy;
      poll(n == 1);
      asm volatile("s_waitcnt vmcnt(0)" ::: "memory");
      look();
      if (rdy == was) {
        if (eager) __builtin_amdgcn_s_sleep(P_S2);
        else if (rdy == 0) __builtin_amdgcn_s_sleep(P_S0);
        else __builtin_amdgcn_s_sleep(P_S1);
      }
      if ((n & 31) == 0 && spin_fail(myws, call, t0)) rdy = NSTEP;
    }
  }
};

__device__ __forceinline__ void repair(u32 (&h)[8], u32 hd, u32 hdr, u32 hf, const char* sb, u32 l16) {
#pragma nounroll
  for (u32 n = (hdr >> (11 + 3 * hf)) & 7, k = hf ? 6 : 1; n != 0; --n, k += hf ? -1 : 1) fix_word(h, __builtin_amdgcn_readlane(hd, k));
  if (__builtin_expect(hdr & 0x400u, 0)) {
    const u32x8 g = load_two(sb + hf * 2048, l16);
#pragma unroll
    for (int j = 0; j < 8; ++j) asm volatile("v_mov_b32 %0, %1" : "+v"(h[j]) : "v"(g[j]));
  }
}

__device__ __forceinline__ void unpack_unit(char* myws, int lane, u32 call, u32 r, int t, u64 tk) {
  Stream S;
  S.init(myws, call, (call & 1) * NS_PAR + S_WIN + r, lane);
  const u32 l4 = lane * 4;
  const u32 uo = (S_IMG + 16 + r) * NSTEP;
  {
    if (r < 16) while (__builtin_amdgcn_s_memrealtime() - tk < (u32)(20 + 4 * t) * P_UT / 10) __builtin_amdgcn_s_sleep(4);
    u32 hd;
    for (u32 n = 1;; ++n) {
      hd = __hip_atomic_load((const u32*)(S.mb + t * 32) + (lane & 7), __ATOMIC_RELAXED, __HIP_MEMORY_SCOPE_SYSTEM);
      if (hdr_there(hd, call, lane)) break;
      if ((n & 31) == 0 && spin_fail(myws, call, S.t0)) break;
      __builtin_amdgcn_s_sleep(4);
    }
    Pay<6> P;
    pay_load(P, S.l16, S.sb + (size_t)t * SLOT);
    pay_wait<0>(P);
    const u32 hdr = __builtin_amdgcn_readfirstlane(hd), base4 = (hdr & 0x7f) * 0x01010101u;
    u32x4 o[8];
#pragma unroll
    for (int hf = 0; hf < 2; ++hf) {
      u32 g[8];
#pragma unroll
      for (int j = 0; j < 4; ++j) {
        const u32 n = P.v[4 + hf][j];
        g[2 * j] = (n & 0x87878787u) + base4;
        g[2 * j + 1] = (__builtin_amdgcn_alignbit(n, n, 4) & 0x87878787u) + base4;
      }
      if (hdr & (0x400u | 0x3800u << (3 * hf))) repair(g, hd, hdr, hf, S.sb + (size_t)t * SLOT, S.l16);
#pragma unroll
      for (int j = 0; j < 4; ++j)
        o[4 * hf + j] = join(P.v[2 * hf + (j >> 1)][2 * (j & 1)], P.v[2 * hf + (j >> 1)][2 * (j & 1) + 1], g[2 * j], g[2 * j + 1]);
    }
    char* const ob = myws + (size_t)(uo + t) * SLOT + 4096;
    ST_SIX(S.l16, o, ob); ST16(S.l16, o[6], ob, 2048); ST16(S.l16, o[7], ob, 3072);
    asm volatile("s_waitcnt vmcnt(0)" ::: "memory");
    char* const mo = myws + meta_off((uo + t));
    if (lane < 8) ST4(l4, hdr_word(0, call, lane), mo);
  }
}

__device__ __forceinline__ float silu(float y) {
  const float z = __builtin_amdgcn_fmed3f(y, -60.0f, 60.0f);
  const float e = __builtin_amdgcn_exp2f(z * -1.44269504088896341f);
  return y * __builtin_amdgcn_rcpf(1.0f + e);
}

__device__ __forceinline__ void gemm_tile(KP p, char* smem, char* myws, int wv, int lane, u32 bid, u32 call, u32 nb, u32 sid0, u32 tn, u32 ch, u32 fl, int hold, u64 tk, bool next, u32*& owed) {
  const u32 rank = p->rank;
  if (wv >= 8) __builtin_amdgcn_s_setprio(P_LPRIO);
  const u32 o0 = (rank & 4u) ^ 4u;
  const bool remote = tn >= 4;
  const u32 dst = (remote ? o0 + (tn - 4) : rank) & 7u;
  char* const ob = p->out[dst] + (remote ? (rank & 3) : tn) * 256 + ch * 128;
  u32* flag = (u32*)(p->ws[dst] + DONE_OFF) + (rank & 3) * 64 + fl;
  auto grow = [&](u32 rg) { return p->row[sid0 + (rg >> 2)] + (rg & 3) * 8; };
  char* b0 = ob + (size_t)grow(wv & 7) * (NLOC * 2);
  char* b1 = ob + (size_t)grow((wv & 7) + 8) * (NLOC * 2);
  if (hold == 2) {
    const u32 us = (S_PARK + (tn & 3)) * NSTEP + 2 * fl;
    b0 = myws + (size_t)us * SLOT + (wv & 7) * 1024; b1 = b0 + SLOT;
    flag = (u32*)(myws + meta_off(us));
  }
  asm volatile("" : "+s"(b0), "+s"(b1), "+s"(flag));
  if (nb == 0) {
    if (threadIdx.x == 0) ST4(0u, call, flag);
    return;
  }
  {
    const u32 first = sid0 < (u32)S_2 || sid0 >= (u32)S_OWN2 ? 0 : sid0 < (u32)S_INTER ? N1 * NSTEP : sid0 < (u32)S_3 ? 4 * NSTEP : N_INTRA - N3 * NSTEP;
    while (__builtin_amdgcn_s_memrealtime() - tk < first * P_UT / 10) __builtin_amdgcn_s_sleep(8);
  }
  Stream S;
  const u32 m = wv >> 1, kh = wv & 1;
  u32 tmo = 0;
  auto read_tmo = [&]() { asm volatile("global_load_dword %0, %1, %2 sc1" : "=&v"(tmo) : "v"(0u), "s"(myws + TMO_OFF) : "memory"); };
  if (wv >= 8) {
    const u32 un = wv - 8;
    S.init(myws, call, S_IMG + 4 * tn + un, lane);
    const u32 lds0 = (u32)(size_t)(LAS char*)smem + un * 4096;
    const char* const g0 = S.sb - 4096 + ch * 4096;
    Stream A;
    A.init(myws, call, (call & 1) * NS_PAR + sid0 + un, lane);
    A.eager = sid0 >= (u32)S_3 && sid0 < (u32)S_OWN2;
    asm volatile("global_load_dwordx4 %0, %2, %3 sc0 sc1\n\tglobal_load_dwordx4 %1, %2, %4 sc0 sc1\n\ts_waitcnt vmcnt(0)"
                 : "+v"(A.W), "+v"(S.W) : "v"(S.l16), "s"(A.mb), "s"(S.mb) : "memory");
    int is = 0, seen = ((nb >> un) & 1) ? 0 : NSTEP;
    for (int t = -1; t < NSTEP; ++t) {
      if (seen < NSTEP && seen <= t + P_DIST + P_ALOOK) {
        A.look();
        A.spin(min(t + P_DIST, NSTEP - 1));
        if (A.rdy != seen) { *(LAS u32x4*)((LAS char*)smem + HDR_OFF + un * 1024 + A.l16) = A.W; seen = A.rdy; }
        asm volatile("" ::: "memory");
        if (seen < NSTEP) A.poll();
      }
      const int lim = min(min(NSTEP, is + 2), t + P_BD);
      if (S.rdy < min(NSTEP, lim + 2)) { S.look(); S.spin(t); if (S.rdy < NSTEP) S.poll(); }
      for (; is < min(lim, S.rdy); ++is) {
        const char* g = g0 + (size_t)is * SLOT;
        const u32 l = lds0 + (is & 7) * 16384;
        u32 keep;
        asm volatile("s_mov_b32 %0, m0\n\ts_mov_b32 m0, %3\n\ts_nop 0\n\t"
                     "global_load_lds_dwordx4 %1, %2 sc0 sc1\n\tglobal_load_lds_dwordx4 %1, %2 offset:1024 sc0 sc1\n\t"
                     "global_load_lds_dwordx4 %1, %2 offset:2048 sc0 sc1\n\tglobal_load_lds_dwordx4 %1, %2 offset:3072 sc0 sc1\n\t"
                     "s_mov_b32 m0, %0" : "=&s"(keep) : "v"(S.l16), "s"(g), "s"(l) : "memory");
      }
      const int ahead = is - 1 - t;
      if (ahead >= 6 || t < 0) asm volatile("s_waitcnt vmcnt(24)" ::: "memory");
      else if (ahead >= 4) { if (ahead == 5) asm volatile("s_waitcnt vmcnt(20)" ::: "memory"); else asm volatile("s_waitcnt vmcnt(16)" ::: "memory"); }
      else if (ahead >= 2) { if (ahead == 3) asm volatile("s_waitcnt vmcnt(12)" ::: "memory"); else asm volatile("s_waitcnt vmcnt(8)" ::: "memory"); }
      else if (ahead == 1) asm volatile("s_waitcnt vmcnt(4)" ::: "memory");
      else asm volatile("s_waitcnt vmcnt(0)" ::: "memory");
      WG_BARRIER();
    }
    asm volatile("s_waitcnt vmcnt(0)" : "+v"(S.W), "+v"(A.W) :: "memory");
    __builtin_amdgcn_s_setprio(0);
    if (hold == 1 && wv == 8) {
      const u32* const r = (const u32*)(myws + REL_OFF + (bid & 7) * 128) + (lane < NCWG ? lane : 0);
      for (u32 n = 1; !__all(__hip_atomic_load(r, __ATOMIC_RELAXED, __HIP_MEMORY_SCOPE_AGENT) >= call); ++n) {
        if ((n & 31) == 0 && spin_fail(myws, call, S.t0)) break;
        __builtin_amdgcn_s_sleep(4);
      }
    }
  } else if (!((nb >> m) & 1)) {
    asm volatile("s_waitcnt vmcnt(0)" ::: "memory");
    for (int t = -1; t < NSTEP; ++t) WG_BARRIER();
    read_tmo();
    asm volatile("s_waitcnt vmcnt(0)" : "+v"(tmo) :: "memory");
  } else {
    const char* const sb = myws + (size_t)((call & 1) * NS_PAR + sid0 + m) * NSTEP * SLOT + 4096;
    const u32 l16 = lane * 16, vlo = l16 + kh * 2048, vcode = l16 + kh * 1024;
    u32 ho = HDR_OFF + m * 1024 + (lane & 7) * 4, hd;
    const u32 rmask = 0x400u | 0x3800u << (3 * kh);
    f32x16 acc[2] = {};
    u32 bb = kh * 8192 + lane * 16;
    bf16x8 wf[4];
    u32x4 X[2][4];
    Pay<3> P[4];
    u32 h[8];
    u32 hdr = 0, base4 = 0;
    auto rd = [&](int stage, int f) {
      wf[f & 3] = *(const LAS bf16x8*)((LAS char*)smem + bb + stage * 16384 + (f >> 2) * 4096 + ((f >> 1) & 1) * 1024 + (f & 1) * 2048);
    };
    auto piece = [&](int k, int t, int ts, int waitn, bool fetch) {
      Pay<3>& Q = P[ts];
      if (k == 0) {
        if (waitn == 3) pay_wait<9>(Q); else if (waitn == 2) pay_wait<6>(Q); else if (waitn == 1) pay_wait<3>(Q); else pay_wait<0>(Q);
        hdr = __builtin_amdgcn_readfirstlane(hd);
        base4 = (hdr & 0x7f) * 0x01010101u;
      } else if (k < 6) {
#pragma unroll
        for (int j = 0; j < 4; ++j) {
          const int i = k < 4 ? 2 * j + (k & 1) : 4 * (k - 4) + j;
          if (k == 1) h[i] = __builtin_amdgcn_alignbit(Q.v[2][j], Q.v[2][j], 4);
          else if (k == 2) h[i] = Q.v[2][j] & 0x87878787u;
          else if (k == 3) h[i] &= 0x87878787u;
          else h[i] += base4;
          asm volatile("" : "+v"(h[i]));
        }
      } else if (k == 6) {
        if (hdr & rmask) repair(h, hd, hdr, kh, sb + (size_t)t * SLOT, l16);
      } else if (k == 7) {
      } else if (k < 12) {
        const int c = k - 8;
        X[ts & 1][c] = join(Q.v[c >> 1][2 * (c & 1)], Q.v[c >> 1][2 * (c & 1) + 1], h[2 * c], h[2 * c + 1]);
        asm volatile("" : "+v"(X[ts & 1][c]));
      } else if (k == 12) {
        ho += 32;
        hd = *(const LAS u32*)((LAS char*)smem + ho);
      } else if (k == 13 && fetch) {
        const int tn4 = t + P_DIST < NSTEP ? t + P_DIST : NSTEP - 1;
        pay_load(P[(ts + P_DIST) & 3], vlo, vcode, sb + (size_t)tn4 * SLOT);
      }
    };
    auto step = [&](int t, int ts, int waitn, bool fetch, bool more) {
#pragma unroll
      for (int q = 0; q < 8; ++q) {
        if (q == 4 && more) { WG_BARRIER(); if (ts == 3) bb ^= 65536u; }
        else if (q % 2 == 0 && (more || q < 6)) __builtin_amdgcn_s_waitcnt(0xC27F);
        __builtin_amdgcn_sched_barrier(0);
        acc[q & 1] = __builtin_amdgcn_mfma_f32_32x32x16_bf16(wf[q & 3], __builtin_bit_cast(bf16x8, X[ts & 1][q >> 1]), acc[q & 1], 0, 0, 0);
        __builtin_amdgcn_sched_barrier(0);
        if (q < 4) rd(ts, q + 4);
        else if (more) rd((ts + 1) & 3, q - 4);
        if (more) {
          piece(2 * q, t + 1, (ts + 1) & 3, waitn, fetch);
          piece(2 * q + 1, t + 1, (ts + 1) & 3, waitn, fetch);
        }
      }
    };
    asm volatile("s_waitcnt vmcnt(0)" ::: "memory");
    WG_BARRIER();
    if (owed && threadIdx.x < 2) ST4(threadIdx.x * 32, call, owed);
#pragma unroll
    for (int d = 0; d < P_DIST; ++d) pay_load(P[d], vlo, vcode, sb + (size_t)d * SLOT);
    hd = *(const LAS u32*)((LAS char*)smem + ho);
#pragma unroll
    for (int k = 0; k < 13; ++k) piece(k, 0, 0, P_DIST - 1, true);
    WG_BARRIER();
    piece(13, 0, 0, P_DIST - 1, true);
#pragma unroll
    for (int f = 0; f < 4; ++f) { rd(0, f); __builtin_amdgcn_sched_barrier(0); }
    for (int tb = 0; tb < NSTEP - 4; tb += 4) {
#pragma unroll
      for (int ts = 0; ts < 4; ++ts) step(tb + ts, ts, P_DIST - 1, true, true);
    }
    step(NSTEP - 4, 0, P_DIST - 1, false, true);
    step(NSTEP - 3, 1, P_DIST - 2, false, true);
    step(NSTEP - 2, 2, P_DIST - 3, false, true);
    read_tmo();
    step(NSTEP - 1, 3, 0, false, false);
    char* const xq = smem + EPI_OFF + (kh * 128 + m * 32 + (lane & 31)) * ERS + (lane >> 5) * 16;
#pragma unroll
    for (int n = 0; n < 2; ++n)
#pragma unroll
      for (int g = 0; g < 4; ++g)
        *(f32x4*)(xq + n * 128 + g * 32) = f32x4{acc[n][4 * g], acc[n][4 * g + 1], acc[n][4 * g + 2], acc[n][4 * g + 3]};
    if (P_DIST == 4) pay_wait<0>(P[0]);
    asm volatile("s_waitcnt vmcnt(0)" : "+v"(tmo) :: "memory");
  }
  WG_BARRIER();
  if (wv < 8) {
    const u32 vo = hold == 2 ? lane * 16 : (lane >> 3) * (NLOC * 2) + (lane & 7) * 16;
    auto get = [&](u32 rg) {
      const char* const q = smem + EPI_OFF + (rg * 8 + (lane >> 3)) * ERS + (lane & 7) * 32;
      const f32x4 a = *(const f32x4*)q + *(const f32x4*)(q + 128 * ERS), c = *(const f32x4*)(q + 16) + *(const f32x4*)(q + 128 * ERS + 16);
      auto pk = [](float lo, float hi) { const f32x2 v = {silu(lo), silu(hi)}; return __builtin_bit_cast(u32, __builtin_convertvector(v, bf16x2)); };
      u32x4 v = {pk(a[0], a[1]), pk(a[2], a[3]), pk(c[0], c[1]), pk(c[2], c[3])};
      if (tmo == call) v = u32x4{0x7fc07fc0u, 0x7fc07fc0u, 0x7fc07fc0u, 0x7fc07fc0u};
      return v;
    };
    if ((nb >> (wv >> 2)) & 1) { const u32x4 v0 = get(wv); ST16(vo, v0, b0, 0); }
    if ((nb >> (wv >> 2)) & 4) { const u32x4 v1 = get(wv + 8); ST16(vo, v1, b1, 0); }
  }
  asm volatile("ag_code_mark:");
  owed = nullptr;
  if (next) { if (remote) owed = flag; }
  else if (remote) {
    asm volatile("s_waitcnt vmcnt(0)" ::: "memory");
    WG_BARRIER();
    if (threadIdx.x == 0) ST4(0u, call, flag);
  }
}

inline bool tile_of(u32 rank, u32 b, int k, u32& nb, u32& sid0, u32& tn, u32& ch, u32& fl, int& hold) {
  const u32 peer = peer_of_xcd(rank, b & 7), i = b >> 3;
  const bool home = !((peer ^ rank) & 4), spare = i >= 2 * RT_MED;
  hold = 0;
  constexpr u32 I0 = S_INTER / 2;
  const bool extra = home && i >= I0 && i < I0 + (NB + 1) / 2 * 2;
  if (extra && k == 0) {
    const u32 j = i - I0;
    nb = (j | 1) < (u32)NB ? 3 : 1; tn = peer & 3; sid0 = S_OWN2 + (j >> 1) * 2; ch = j & 1; fl = 0;
    return true;
  }
  if (k == (extra ? 1 : 0) && !spare) {
    nb = 15; tn = (peer & 3) + (home ? 0 : 4); sid0 = (i >> 1) * 4; ch = i & 1; fl = i;
    hold = home || sid0 >= (u32)S_INTER ? 0 : i < (u32)P_PARK ? 2 : i >= 10 ? 1 : 0;
    if (!home)
      for (u32 m = 0; m < 4; ++m)
        if (sid0 + m >= (u32)S_INTER && (sid0 + m - S_INTER) / NB == (peer & 3)) nb &= ~(1u << m);
    return true;
  }
  const u32 j = spare ? i - 2 * RT_MED + 10 : i;
  nb = 3; tn = (peer & 3) + (home ? 4 : 0); sid0 = RT_MED * 4 + (j >> 1) * 2; ch = j & 1; fl = 2 * RT_MED + j;
  return spare ? k == 0 : i < 10;
}

constexpr u32 NHOME = 32 * NSTEP;
__host__ __device__ inline u32 pack_index(u32 b, u32 wv) {
  constexpr u32 NP1 = 80;
  return b < NP1 ? (wv < 2 ? b + NP1 * wv : NHOME) : 2 * NP1 + (b - NP1) + (NGEMM - NP1) * wv;
}

constexpr u32 UB0 = RT_MED * 16, UNW = 16;
__host__ __device__ inline u32 unpack_index(u32 b, u32 wv) {
  return b >= UB0 && b < UB0 + UNW ? (11 - wv) * UNW + (b - UB0) : 512u;
}

__global__ void __launch_bounds__(NTHREAD, 3) ag_gemm_silu(const Params) {
  KP p = (KP)__builtin_amdgcn_kernarg_segment_ptr();
  __shared__ __attribute__((aligned(16))) char smem[LDS_BYTES];
  const int lane = threadIdx.x & 63;
  const int wv = __builtin_amdgcn_readfirstlane(threadIdx.x >> 6);
  char* const myws = p->myws;
  u64 pc; u32 len;
  asm volatile("s_getpc_b64 %0\nag_code_begin:\n\ts_mov_b32 %1, ag_code_mark-ag_code_begin" : "=s"(pc), "=s"(len));
  const u64 tk = __builtin_amdgcn_s_memrealtime();
  u32 bid = (((blockIdx.x & ~7u) | ((__builtin_amdgcn_s_getreg((3 << 11) | 20)) & 7u)) + NGEMM) % GRID, call = 0;
  if (bid >= (u32)NGEMM) { if (sender<true>(p, smem, myws, wv, lane, bid, call)) return; }
  else {
    if (wv >= 10) {
      const u32 o = ((bid >> 3) * 2 + (wv - 10)) * 1024 + lane * 16;
      u32x4 junk;
      if (o + 16 <= len && len < 65536u) asm volatile("global_load_dwordx4 %0, %1, %2\n\ts_waitcnt vmcnt(0)" : "=&v"(junk) : "v"(o), "s"(pc) : "memory");
    }
    claim(smem, myws, wv, bid, call);
  }
  bid = __builtin_amdgcn_readfirstlane(bid); call = __builtin_amdgcn_readfirstlane(call);
  if (__builtin_expect(bid >= (u32)NGEMM, 0)) { sender<false>(p, smem, myws, wv, lane, bid, call); return; }
  const u32 u = pack_index(bid, wv);
  if (u < NHOME) {
    const u32 t = u >> 5, xc = u & 7, sub = (u >> 3) & 3;
    const u32 c = xc < 4 ? sub * 4 + xc : 16 + (peer_of_xcd(p->rank, xc) & 3) * 4 + sub;
    const bool isw = c >= 16;
    const u32 cw = c - 16, l16 = lane * 16;
    while (__builtin_amdgcn_s_memrealtime() - tk < t * P_PP) __builtin_amdgcn_s_sleep(2);
    u32x4 d[8], s[8];
    load_raw(p, d, lane, isw ? 16 + (cw & 3) : c, (cw >> 2) & 3, t);
    WAIT_RAW(d);
    u32 meta = 0;
    bool eight = true;
    to_fragments(d, isw, smem + SCR_OFF + wv * 8192, lane);
    if (isw) {
#pragma unroll
      for (int j = 0; j < 8; ++j) s[j] = d[j];
    } else { low_slabs(d, s); eight = encode(d, s, meta, lane); }
    const u32 ud = (isw ? S_IMG + cw : (call & 1) * NS_PAR + (c < (u32)NA ? c : S_OWN2 - NA + c)) * NSTEP + t;
    char* const b = myws + (size_t)ud * SLOT + 4096;
    ST_SIX(l16, s, b);
    if (eight) { ST16(l16, s[6], b, 2048); ST16(l16, s[7], b, 3072); }
    asm volatile("s_waitcnt vmcnt(0)" ::: "memory");
    char* const m = myws + meta_off(ud);
    if (lane < 8) ST4(lane * 4, hdr_word(meta, call, lane), m);
  }
#pragma nounroll
  for (u32 v = unpack_index(bid, wv); v < 512; v += 12 * UNW) unpack_unit(myws, lane, call, (v + P_UROT(v >> 4)) & 15, v >> 4, tk);
  WG_BARRIER();
  u32* owed = nullptr;
#pragma nounroll
  for (int k = 0; k < 2; ++k) {
    const u32 td = p->tile[bid][k];
    if (!(td >> 23)) break;
    gemm_tile(p, smem, myws, wv, lane, bid, call, td & 15, (td >> 4) & 127, (td >> 11) & 7, (td >> 14) & 1, (td >> 15) & 63, (td >> 21) & 3, tk, k == 0 && p->tile[bid][1] != 0, owed);
  }
}

static void build_orders(Params& p) {
  struct E { double tau; unsigned short code; };
  E e[N_ORD];
  double sigma = P_SIGMA, ovl = P_OVL, ovl2 = P_OVL2, lead = P_LEAD, f1 = 14.5, f2 = 6, stg = P_STG;
  for (int kind = 0; kind < 2; ++kind) {
    int n = 0;
    for (int t = 0; t < NSTEP; ++t) {
      if (kind == 0) {
        for (int b = 0; b < NA; ++b) if (b != RB || t < RT0) e[n++] = {t + (b < N1 ? 0 : b < N1 + N2 ? NSTEP - ovl : 2 * NSTEP - ovl - ovl2 + (b >= N1 + N2 + 2 ? stg : 0)), (unsigned short)(b << 5 | t)};
      } else {
        for (int i = 0; i < 4; ++i) e[n++] = {t * sigma - lead, (unsigned short)((16 + i) << 5 | t)};
        for (int b = NA; b < 16; ++b) e[n++] = {t + 0.5, (unsigned short)(b << 5 | t)};
      }
    }
    if (kind == 1)
      for (int j = 0; j < P_RLY; ++j) {
        e[n++] = {f1 * sigma - lead, (unsigned short)(20 << 5 | j)};
        for (int i = 0; i < 3; ++i) e[n++] = {f2 + j, (unsigned short)((21 + i) << 5 | j)};
      }
    std::stable_sort(e, e + n, [](const E& a, const E& b) { return a.tau < b.tau; });
    for (int i = 0; i < N_ORD; ++i) p.ord[kind][i] = i < n ? e[i].code : 0;
  }
}

extern "C" void kernel_launch(int rank, int nranks, ncclComm_t comm, hipStream_t stream,
                              void* const* d_in, const int* in_sizes, int n_in,
                              void* d_out, int out_size, const int* in_dtypes, int out_dtype,
                              void* const* d_out_all, void* const* d_ws_all, size_t ws_bytes) {
  (void)comm; (void)in_dtypes; (void)out_dtype; (void)d_out;
  if (nranks != NR || (unsigned)rank >= (unsigned)NR || n_in != 2 || ws_bytes < WS_NEED ||
      in_sizes[0] != 512 * KDIM || in_sizes[1] != KDIM * NLOC || out_size != 4096 * NLOC) {
    fprintf(stderr, "unexpected problem shape\n");
    return;
  }
  Params p = {};
  p.x = (const char*)d_in[0];
  p.w = (const char*)d_in[1];
  for (int i = 0; i < NR; ++i) {
    p.out[i] = (char*)d_out_all[phys_rank(i)]; p.ws[i] = (char*)d_ws_all[phys_rank(i)];
    if ((int)phys_rank(i) == rank) p.rank = i;
  }
  p.myws = (char*)d_ws_all[rank];
  build_orders(p);
  for (u32 e = 0; e < 12 * NCWG; ++e)
    if (p.ord[0][e] != ((e % N1) << 5 | e / N1) || p.ord[1][e] != ((16 + (e & 3)) << 5 | e >> 2)) { fprintf(stderr, "send orders\n"); return; }
  for (int i = 0; i < NS_PAR + 4; ++i) p.row[i] = i < 64 || (i >= S_OWN2 && i < NS_PAR) ? global_row(p.rank, i) : 0;
  for (u32 b = 0; b < (u32)NGEMM; ++b)
    for (int k = 0; k < 2; ++k) {
      u32 nb, sid0, tn, ch, fl; int hold;
      p.tile[b][k] = tile_of(p.rank, b, k, nb, sid0, tn, ch, fl, hold) ? nb | sid0 << 4 | tn << 11 | ch << 14 | fl << 15 | (u32)hold << 21 | 1u << 23 : 0u;
    }
  hipLaunchKernelGGL(ag_gemm_silu, dim3(GRID), dim3(NTHREAD), 0, stream, p);
  hipError_t err = hipGetLastError();
  if (err != hipSuccess) fprintf(stderr, "launch failed: %s\n", hipGetErrorString(err));
}
```

```cpp
#include <hip/hip_runtime.h>
#include <rccl/rccl.h>
#include <cstdio>
#include <cstdint>
#include <cstdlib>
#include <algorithm>

typedef unsigned u32;
typedef unsigned long long u64;
typedef unsigned u32x4 __attribute__((ext_vector_type(4)));
typedef unsigned u32x8 __attribute__((ext_vector_type(8)));
typedef unsigned short u16x2 __attribute__((ext_vector_type(2)));
typedef __bf16 bf16x2 __attribute__((ext_vector_type(2)));
typedef __bf16 bf16x4 __attribute__((ext_vector_type(4)));
typedef __bf16 bf16x8 __attribute__((ext_vector_type(8)));
typedef float f32x2 __attribute__((ext_vector_type(2)));
typedef float f32x4 __attribute__((ext_vector_type(4)));
typedef float f32x16 __attribute__((ext_vector_type(16)));
#define LAS __attribute__((address_space(3)))

#define P_NA 11
#define P_NCWG 4
#define P_WIN 3
#define P_WIN0 4
#define P_WIN1 5
#define P_TAPER 8
#define P_DIST 2
#define P_BD 7
#define P_ALOOK 3
#define P_LPRIO 2
#define P_UROT(t) (5 * (t))
#define P_STG 0
#define P_SIGMA 0.1
#define P_OVL 0.0
#define P_OVL2 0.0
#define P_LEAD 0.7
#define P_PP 15
#define P_PARK 10
#define P_RLY 2
#define P_S0 16
#define P_S1 16
#define P_S2 4
#define P_UT 100
#define P_REL 4

constexpr int NR = 8;
constexpr int KDIM = 4096, NLOC = 512;
constexpr int NSTEP = 32;
constexpr int NA = P_NA, NB = 16 - NA;
constexpr int N1 = 3, N3 = 4, N2 = NA - N1 - N3;
constexpr int S_1 = NA, S_2 = S_1 + 3 * N1, S_INTER = S_2 + 3 * N2, S_3 = 64 - 3 * N3, S_WIN = 64, S_RLY = 80, S_OWN2 = 81;
constexpr int RB = N1 + N2 - 1, RT0 = NSTEP - 4 * P_RLY;
constexpr int NS_PAR = S_OWN2 + NB, S_IMG = 2 * NS_PAR, S_PARK = S_IMG + 32, NSTREAM = S_PARK + 4;
constexpr size_t SLOT = 8192;
constexpr size_t META_OFF = (size_t)NSTREAM * NSTEP * SLOT;
constexpr size_t DONE_OFF = META_OFF + (size_t)NSTREAM * NSTEP * 32;
constexpr size_t REL_OFF = DONE_OFF + 1024;
constexpr size_t CNT_OFF = REL_OFF + 1024;
constexpr size_t TMO_OFF = CNT_OFF + 256 * 128;
constexpr size_t WS_NEED = TMO_OFF + 128;
constexpr int N_INTRA = NA * NSTEP - 4 * P_RLY, N_INTER = NB * NSTEP + 4 * NSTEP + 4 * P_RLY;
constexpr int N_ORD = N_INTRA > N_INTER ? N_INTRA : N_INTER;
constexpr int RT_MED = 13;
constexpr int NGEMM = RT_MED * 16 + 16;
constexpr int NFLAG = 2 * RT_MED + 12 + P_PARK;
constexpr int NCWG = P_NCWG;
constexpr int GRID = NGEMM + 8 * NCWG;
static_assert(P_DIST == 2 && P_PARK == 10 && P_BD <= 7 && P_RLY <= 8 && N_ORD % 2 == 0 && NB > 4 && NB <= 8 && S_INTER % 4 == 0, "tile map");
static_assert(GRID <= 256, "one workgroup per CU, all resident");
constexpr int NTHREAD = 768;
constexpr int SCR_OFF = 0, EPI_OFF = 32768;
constexpr int HDR_OFF = 131072;
constexpr int CLAIM_OFF = HDR_OFF + 4096 + 64;
constexpr int LDS_BYTES = CLAIM_OFF + 64;
constexpr u64 DEADLINE_TICKS = 80000000ull;
constexpr int ERS = 272;

#define P_PHYS 0x75416320u
__host__ __device__ inline u32 phys_rank(u32 l) { return (P_PHYS >> (4 * l)) & 7u; }
__host__ __device__ inline u32 peer_of_xcd(u32 rank, u32 x) { return (((rank ^ x) & 4u) | ((rank + x) & 3u)) & 7u; }

struct Params {
  const char* x;
  const char* w;
  char* myws;
  int rank, pad;
  char* ws[NR];
  char* out[NR];
  u32 row[NS_PAR + 4];
  u32 tile[NGEMM][2];
  alignas(4) unsigned short ord[2][N_ORD];
};
static_assert(sizeof(Params) <= 4096, "kernel argument");
typedef const __attribute__((address_space(4))) Params* KP;

#define WG_BARRIER() do { asm volatile("" ::: "memory"); __builtin_amdgcn_s_waitcnt(0xC07F); \
    asm volatile("s_barrier" ::: "memory"); } while (0)
#define ST16(voff, data, sbase, imm) \
  asm volatile("global_store_dwordx4 %0, %1, %2 offset:" #imm " sc0 sc1\n\ts_nop 1" :: "v"(voff), "v"(data), "s"(sbase) : "memory")
#define ST4(voff, data, sbase) \
  asm volatile("global_store_dword %0, %1, %2 sc0 sc1" :: "v"(voff), "v"(data), "s"(sbase) : "memory")
#define ST_SIX(voff, s, b) do { ST16(voff, s[0], b, -4096); ST16(voff, s[1], b, -3072); ST16(voff, s[2], b, -2048); \
    ST16(voff, s[3], b, -1024); ST16(voff, s[4], b, 0); ST16(voff, s[5], b, 1024); } while (0)

__device__ __forceinline__ u32* counter(char* myws) { return (u32*)(myws + CNT_OFF + blockIdx.x * 128); }
__device__ __forceinline__ u32 take_role(char* myws, u32 r, u32 call) {
#pragma nounroll
  for (int i = 0; i < GRID; ++i, r = (r + 1) % GRID) {
    u32 old;
    asm volatile("s_mov_b64 exec, 1\n\tglobal_atomic_umax %0, %1, %2, %3 sc0\n\ts_mov_b64 exec, -1\n\ts_waitcnt vmcnt(0)"
                 : "=&v"(old) : "v"(r * 128), "v"(call), "s"(myws + CNT_OFF + 64) : "memory");
    if ((u32)__builtin_amdgcn_readfirstlane(old) < call) break;
  }
  return r;
}
__device__ __forceinline__ void claim(char* smem, char* myws, int wv, u32& bid, u32& call) {
  LAS u32* const cl = (LAS u32*)(smem + CLAIM_OFF);
  if (wv == 0) {
    const u32 c = __builtin_amdgcn_readfirstlane(__hip_atomic_load(counter(myws), __ATOMIC_RELAXED, __HIP_MEMORY_SCOPE_AGENT)) + 1u;
    cl[1] = take_role(myws, bid, c);
    cl[0] = c;
    if (threadIdx.x == 0) __hip_atomic_store(counter(myws), c, __ATOMIC_RELAXED, __HIP_MEMORY_SCOPE_AGENT);
  }
  WG_BARRIER();
  call = __builtin_amdgcn_readfirstlane(cl[0]);
  bid = __builtin_amdgcn_readfirstlane(cl[1]);
}
__device__ __forceinline__ u32 call_tag(u32 call) { return (call % 2047u + 1u) << 21; }
__device__ __forceinline__ u32 hdr_word(u32 data, u32 call, int lane) { return lane == 7 ? call : data | call_tag(call); }
__device__ __forceinline__ bool hdr_there(u32 word, u32 call, int lane) {
  return __all((lane & 7) == 7 ? word >= call : (word ^ call_tag(call)) < (1u << 21));
}

__device__ __forceinline__ bool spin_fail(char* myws, u32 call, u64 t0) {
  u32* tmo = (u32*)(myws + TMO_OFF);
  u32 t = __builtin_amdgcn_readfirstlane(__hip_atomic_load(tmo, __ATOMIC_RELAXED, __HIP_MEMORY_SCOPE_AGENT));
  if (t == call || __builtin_amdgcn_s_memrealtime() - t0 > DEADLINE_TICKS) {
    __hip_atomic_store(tmo, call, __ATOMIC_RELAXED, __HIP_MEMORY_SCOPE_AGENT);
    return true;
  }
  return false;
}

__device__ __forceinline__ size_t meta_off(u32 unit) { return META_OFF + (size_t)unit * 32; }
__device__ __forceinline__ u32 dest_stream(u32 sl, u32 rank, u32 dest, u32 par) {
  const u32 j = ((rank - dest) & 3) - 1;
  const u32 s = sl >= 16 ? S_WIN + (rank & 3) * 4 + (sl - 16) : sl >= (u32)NA ? S_INTER + (rank & 3) * NB + (sl - NA)
              : sl >= (u32)(N1 + N2) ? S_3 + j * N3 + (sl - N1 - N2) : sl >= (u32)N1 ? S_2 + j * N2 + (sl - N1) : S_1 + j * N1 + sl;
  return par * NS_PAR + s;
}

inline u32 global_row(u32 rank, u32 sid) {
  const u32 g0 = rank & 4u, o0 = g0 ^ 4u;
  u32 src, bi;
  if (sid < (u32)S_1) { src = rank; bi = sid; }
  else if (sid < (u32)S_2) { const u32 q = sid - S_1; src = g0 + ((rank + 1 + q / N1) & 3); bi = q % N1; }
  else if (sid < (u32)S_INTER) { const u32 q = sid - S_2; src = g0 + ((rank + 1 + q / N2) & 3); bi = N1 + q % N2; }
  else if (sid < (u32)S_3) { const u32 q = sid - S_INTER; src = o0 + q / NB; bi = NA + q % NB; }
  else if (sid >= (u32)S_OWN2) { src = rank; bi = NA + sid - S_OWN2; }
  else { const u32 q = sid - S_3; src = g0 + ((rank + 1 + q / N3) & 3); bi = N1 + N2 + q % N3; }
  return (phys_rank(src & 7u) * 512 + bi * 32) & 4095u;
}

__device__ __forceinline__ void load_raw(KP p, u32x4 (&d)[8], int lane, u32 sl, u32 q, u32 t) {
  const bool isw = sl >= 16;
  const u32 rs = isw ? NLOC * 2 : KDIM * 2;
  const u32 vo = (lane >> 4) * rs + (lane & 15) * 16;
  const u64 aw = (u64)p->w + (u64)((t * 128 + (sl & 3) * 32) * (NLOC * 2) + q * 256), ax = (u64)p->x + (u64)(sl * 32 * (KDIM * 2) + t * 256);
  const u64 b0 = isw ? aw : ax;
#pragma unroll
  for (int j = 0; j < 8; ++j)
    asm volatile("global_load_dwordx4 %0, %1, %2" : "=&v"(d[j]) : "v"(vo), "s"(b0 + (u64)(j * 4 * rs)) : "memory");
}
#define WAIT_RAW(d) asm volatile("s_waitcnt vmcnt(0)" : "+v"(d[0]), "+v"(d[1]), "+v"(d[2]), "+v"(d[3]), "+v"(d[4]), "+v"(d[5]), "+v"(d[6]), "+v"(d[7]) :: "memory")
__device__ __forceinline__ void to_fragments(u32x4 (&d)[8], bool isw, char* scr, int lane) {
  LAS char* const sc = (LAS char*)scr;
  auto off = [](u32 row, u32 ch) { return 256u * row + 16u * (ch ^ (((row & 3) << 2) | ((row >> 2) & 3))); };
#pragma unroll
  for (int j = 0; j < 8; ++j) *(LAS u32x4*)(sc + off(4 * j + (lane >> 4), lane & 15)) = d[j];
  asm volatile("" ::: "memory");
  if (isw) {
    const u32 r0 = 8 * (lane >> 5) + ((lane & 15) >> 2), c0 = 2 * ((lane >> 4) & 1) + ((lane & 3) >> 1);
#pragma unroll
    for (int j = 0; j < 8; ++j) {
      const bf16x4 lo = __builtin_amdgcn_ds_read_tr16_b64_v4bf16((LAS bf16x4*)(sc + off(16 * (j & 1) + r0, 4 * (j >> 1) + c0) + 8 * (lane & 1)));
      const bf16x4 hi = __builtin_amdgcn_ds_read_tr16_b64_v4bf16((LAS bf16x4*)(sc + off(16 * (j & 1) + r0 + 4, 4 * (j >> 1) + c0) + 8 * (lane & 1)));
      d[j] = __builtin_bit_cast(u32x4, __builtin_shufflevector(lo, hi, 0, 1, 2, 3, 4, 5, 6, 7));
    }
  } else {
#pragma unroll
    for (int j = 0; j < 8; ++j) d[j] = *(const LAS u32x4*)(sc + off(lane & 31, 2 * j + (lane >> 5)));
  }
  asm volatile("" ::: "memory");
}
__device__ __forceinline__ void low_slabs(const u32x4 (&d)[8], u32x4 (&s)[8]) {
#pragma unroll
  for (int i = 0; i < 16; ++i) s[i >> 2][i & 3] = __builtin_amdgcn_perm(d[i >> 1][2 * (i & 1) + 1], d[i >> 1][2 * (i & 1)], 0x06040200u);
}
__device__ __forceinline__ bool encode(const u32x4 (&d)[8], u32x4 (&s)[8], u32& meta, int lane) {
  u32 H[16], T[16];
  u16x2 m1 = {0, 0}, m2 = {0, 0}, m3 = {0, 0}, m4 = {0, 0};
#pragma unroll
  for (int i = 0; i < 16; ++i) {
    const u32 a = d[i >> 1][2 * (i & 1)], b = d[i >> 1][2 * (i & 1) + 1];
    H[i] = __builtin_amdgcn_perm(b, a, 0x07050301u);
  }
#pragma unroll
  for (int i = 0; i < 16; i += 2) {
    T[i] = H[i] | 0x80808080u;
    T[i + 1] = H[i + 1] | 0x80808080u;
    m1 = __builtin_elementwise_max(m1, __builtin_bit_cast(u16x2, T[i]));
    m2 = __builtin_elementwise_max(m2, __builtin_bit_cast(u16x2, T[i] << 8));
    m3 = __builtin_elementwise_max(m3, __builtin_bit_cast(u16x2, T[i + 1]));
    m4 = __builtin_elementwise_max(m4, __builtin_bit_cast(u16x2, T[i + 1] << 8));
  }
  m1 = __builtin_elementwise_max(__builtin_elementwise_max(m1, m2), __builtin_elementwise_max(m3, m4));
  u32 mx = (u32)(m1.x > m1.y ? m1.x : m1.y);
  mx = max(mx, (u32)__builtin_amdgcn_update_dpp(0, mx, 0x111, 0xf, 0xf, true));
  mx = max(mx, (u32)__builtin_amdgcn_update_dpp(0, mx, 0x112, 0xf, 0xf, true));
  mx = max(mx, (u32)__builtin_amdgcn_update_dpp(0, mx, 0x114, 0xf, 0xf, true));
  mx = max(mx, (u32)__builtin_amdgcn_update_dpp(0, mx, 0x118, 0xf, 0xf, true));
  const u32 top = max(max((u32)__builtin_amdgcn_readlane(mx, 15), (u32)__builtin_amdgcn_readlane(mx, 31)),
                      max((u32)__builtin_amdgcn_readlane(mx, 47), (u32)__builtin_amdgcn_readlane(mx, 63)));
  const u32 h7 = (top >> 8) & 0x7f;
  const u32 base = __builtin_amdgcn_readfirstlane(h7 > 7 ? h7 - 7 : 0);
  const u32 base4 = base * 0x01010101u;
  u32 ok = 0xffffffffu;
  u32 N[8];
#pragma unroll
  for (int j = 0; j < 8; ++j) {
    const u32 ta = T[2 * j] - base4, tb = T[2 * j + 1] - base4;
    ok &= ta & tb;
    const u32 ca = (ta & 0x07070707u) | (H[2 * j] & 0x80808080u);
    const u32 cb = (tb & 0x07070707u) | (H[2 * j + 1] & 0x80808080u);
    N[j] = ca | __builtin_amdgcn_alignbit(cb, cb, 28);
  }
  meta = 0;
  int c0 = 0, c1 = 0;
  if (__any((~ok & 0x80808080u) != 0)) {
    u32 M0 = 0, M1 = 0;
#pragma unroll
    for (int i = 0; i < 8; ++i) {
      M0 |= ~((T[i] - base4) >> i) & (0x80808080u >> i);
      M1 |= ~((T[8 + i] - base4) >> i) & (0x80808080u >> i);
    }
    u64 bal = __ballot((M0 | M1) != 0);
    while (bal != 0 && c0 + c1 <= 6) {
      const int ln = __builtin_ctzll(bal);
      bal &= bal - 1;
      u64 mm = (u64)(u32)__builtin_amdgcn_readlane(M0, ln) | (u64)(u32)__builtin_amdgcn_readlane(M1, ln) << 32;
      while (mm != 0) {
        const u32 bit = __builtin_ctzll(mm), hf = bit >> 5, by = (bit >> 3) & 3, i = hf * 8 + 7 - (bit & 7);
        mm &= mm - 1;
        c0 += hf ^ 1; c1 += hf;
        u32 hw = 0;
#pragma unroll
        for (int z = 0; z < 16; ++z) hw |= H[z] & (u32)__builtin_amdgcn_sbfe(1 << i, z, 1);
        const u32 hb = ((u32)__builtin_amdgcn_readlane(hw, ln) >> (8 * by)) & 0x7f;
        const u32 ent = (u32)ln | (i & 7) << 6 | by << 9 | ((((hb - base) & 7) + base) ^ hb) << 11;
        if (lane == (hf ? 7 - c1 : c0)) meta = ent;
      }
    }
  }
  const bool raw = c0 + c1 > 6;
  if (lane == 0) meta = base | (raw ? 0x400u : (u32)(c0 << 11 | c1 << 14));
  s[4] = raw ? u32x4{H[0], H[1], H[2], H[3]} : u32x4{N[0], N[1], N[2], N[3]};
  s[5] = raw ? u32x4{H[4], H[5], H[6], H[7]} : u32x4{N[4], N[5], N[6], N[7]};
  s[6] = u32x4{H[8], H[9], H[10], H[11]};
  s[7] = u32x4{H[12], H[13], H[14], H[15]};
#pragma unroll
  for (int j = 0; j < 8; ++j) asm volatile("" : "+v"(s[j]));
  return raw;
}

template <int N> struct Pay { u32x4 v[N]; };
__device__ __forceinline__ void pay_load(Pay<6>& P, u32 l16, const char* sb) {
  asm volatile("global_load_dwordx4 %0, %6, %7 offset:-4096 sc0 sc1\n\t"
               "global_load_dwordx4 %1, %6, %7 offset:-3072 sc0 sc1\n\t"
               "global_load_dwordx4 %2, %6, %7 offset:-2048 sc0 sc1\n\t"
               "global_load_dwordx4 %3, %6, %7 offset:-1024 sc0 sc1\n\t"
               "global_load_dwordx4 %4, %6, %7 offset:0 sc0 sc1\n\t"
               "global_load_dwordx4 %5, %6, %7 offset:1024 sc0 sc1"
               : "=&v"(P.v[0]), "=&v"(P.v[1]), "=&v"(P.v[2]), "=&v"(P.v[3]), "=&v"(P.v[4]), "=&v"(P.v[5]) : "v"(l16), "s"(sb) : "memory");
}
__device__ __forceinline__ void pay_load(Pay<3>& P, u32 vlo, u32 vcode, const char* sb) {
  asm volatile("global_load_dwordx4 %2, %4, %5 offset:0 sc0 sc1\n\t"
               "global_load_dwordx4 %0, %3, %5 offset:-4096 sc0 sc1\n\t"
               "global_load_dwordx4 %1, %3, %5 offset:-3072 sc0 sc1"
               : "=&v"(P.v[0]), "=&v"(P.v[1]), "=&v"(P.v[2]) : "v"(vlo), "v"(vcode), "s"(sb) : "memory");
}
template <int C> __device__ __forceinline__ void pay_wait(Pay<6>& P) {
  asm volatile("s_waitcnt vmcnt(%6)" : "+v"(P.v[0]), "+v"(P.v[1]), "+v"(P.v[2]), "+v"(P.v[3]), "+v"(P.v[4]), "+v"(P.v[5]) : "n"(C) : "memory");
}
template <int C> __device__ __forceinline__ void pay_wait(Pay<3>& P) {
  asm volatile("s_waitcnt vmcnt(%3)" : "+v"(P.v[0]), "+v"(P.v[1]), "+v"(P.v[2]) : "n"(C) : "memory");
}
__device__ __forceinline__ u32x8 load_two(const char* b, u32 l16) {
  u32x4 x, y;
  asm volatile("global_load_dwordx4 %0, %2, %3 sc0 sc1\n\t"
               "global_load_dwordx4 %1, %2, %3 offset:1024 sc0 sc1\n\ts_waitcnt vmcnt(0)"
               : "=&v"(x), "=&v"(y) : "v"(l16), "s"(b) : "memory");
  return __builtin_shufflevector(x, y, 0, 1, 2, 3, 4, 5, 6, 7);
}

__device__ __forceinline__ void load_slot(u32x4 (&d)[8], u32 l16, const char* sb) {
  asm volatile("global_load_dwordx4 %0, %8, %9 offset:-4096 sc0 sc1\n\tglobal_load_dwordx4 %1, %8, %9 offset:-3072 sc0 sc1\n\t"
               "global_load_dwordx4 %2, %8, %9 offset:-2048 sc0 sc1\n\tglobal_load_dwordx4 %3, %8, %9 offset:-1024 sc0 sc1\n\t"
               "global_load_dwordx4 %4, %8, %9 offset:0 sc0 sc1\n\tglobal_load_dwordx4 %5, %8, %9 offset:1024 sc0 sc1\n\t"
               "global_load_dwordx4 %6, %8, %9 offset:2048 sc0 sc1\n\tglobal_load_dwordx4 %7, %8, %9 offset:3072 sc0 sc1\n\ts_waitcnt vmcnt(0)"
               : "=&v"(d[0]), "=&v"(d[1]), "=&v"(d[2]), "=&v"(d[3]), "=&v"(d[4]), "=&v"(d[5]), "=&v"(d[6]), "=&v"(d[7])
               : "v"(l16), "s"(sb) : "memory");
}

template <bool ask>
__device__ __forceinline__ bool sender(KP p, char* smem, char* myws, int wv, int lane, u32& bid, u32& call) {
  const u32 rank = p->rank;
  const u32 cb = bid - NGEMM;
  const u32 link = cb & 7, cw = cb >> 3;
  const u32 l16 = lane * 16;
  if (link != 0) {
    const u32 dest = peer_of_xcd(rank, link);
    const bool intra = link < 4;
    const int n = ((intra ? N_INTRA : N_INTER) - (int)cw + NCWG - 1) / NCWG;
    auto code_of = [=](int k) {
      const u32 e = cw + (u32)(k < n ? k : 0) * NCWG;
      const u32 word = ((const __attribute__((address_space(4))) u32*)p->ord[intra ? 0 : 1])[e >> 1];
      return (word >> (16 * (e & 1))) & 0xffffu;
    };
    LAS int* const done = (LAS int*)(smem + 12 * 8192);
    u32x4 d[8];
    auto fetch = [&](u32 c) {
      const bool rly = c >= 20 * 32;
      load_raw(p, d, lane, rly ? RB : c >> 5, dest & 3, rly ? RT0 + 4 * (c & 7) + (dest & 3) : c & 31);
    };
    const u32 e0 = cw + wv * NCWG;
    u32 code = intra ? (e0 % N1) << 5 | e0 / N1 : (16 + (e0 & 3)) << 5 | e0 >> 2;
    LAS u32* const got = (LAS u32*)(smem + CLAIM_OFF + 16);
    if (wv >= 4) {
      if (threadIdx.x == 256) { *done = 0; *got = 0; }
      WG_BARRIER();
      if (wv >= 8) __builtin_amdgcn_s_sleep(4);
    }
    fetch(code);
    u32 ticket = 0;
    if (ask) asm volatile("global_load_dword %0, %1, %2 sc1" : "=&v"(ticket) : "v"(0u), "s"(counter(myws)) : "memory");
    if (wv < 4) WG_BARRIER();
    bool lost = false;
    char* const rws = p->ws[dest];
    if (wv < 4) __builtin_amdgcn_s_setprio(2); else if (wv < 8) __builtin_amdgcn_s_setprio(1);
    int k = wv;
    for (;; k += 12) {
      const u32 sl = code >> 5;
      u32 ud = sl < 20 ? dest_stream(sl, rank, dest, 0) * NSTEP + (code & 31) : sl == 20 ? S_RLY * NSTEP + (rank & 3) * 8 + (code & 7)
             : (S_2 + (sl - 21) * N2 + RB - N1) * NSTEP + RT0 + 4 * (code & 7) + (rank & 3);
      asm volatile("" : "+s"(ud));
      WAIT_RAW(d);
      if (ask && k < 24) {
        if (k < 12) {
          asm volatile("" : "+v"(ticket));
          call = __builtin_amdgcn_readfirstlane(ticket) + 1u;
          if (wv == 11) __hip_atomic_store(got, take_role(myws, bid, call) + 1u, __ATOMIC_RELAXED, __HIP_MEMORY_SCOPE_WORKGROUP);
        } else {
          u32 g;
          while ((g = __builtin_amdgcn_readfirstlane(__hip_atomic_load(got, __ATOMIC_RELAXED, __HIP_MEMORY_SCOPE_WORKGROUP))) == 0) __builtin_amdgcn_s_sleep(1);
          if (g - 1 != bid) { bid = g - 1; lost = true; break; }
        }
      }
      ud += (call & 1) * (NS_PAR * NSTEP);
      char* const b = rws + (size_t)ud * SLOT + 4096;
      u32x4 s[8];
      u32 meta;
      bool raw;
      if (P_RLY > 0 && sl > 20) {
        const u32 us = ((call & 1) * NS_PAR + S_RLY) * NSTEP + ((dest + sl - 20) & 3) * 8 + (code & 7);
        const u64 t0 = __builtin_amdgcn_s_memrealtime();
        const u32* const hd = (const u32*)(myws + meta_off(us)) + (lane & 7);
        for (u32 it = 1;; ++it) {
          meta = __hip_atomic_load(hd, __ATOMIC_RELAXED, __HIP_MEMORY_SCOPE_SYSTEM);
          if (hdr_there(meta, call, lane)) break;
          if ((it & 31) == 0 && spin_fail(myws, call, t0)) break;
          __builtin_amdgcn_s_sleep(2);
        }
        load_slot(s, l16, myws + (size_t)us * SLOT + 4096);
        raw = __builtin_amdgcn_readfirstlane(meta) & 0x400;
      } else {
        to_fragments(d, sl >= 16 && sl < 20, smem + wv * 8192, lane);
        low_slabs(d, s);
        if (k < P_WIN0) { ST16(l16, s[0], b, -4096); ST16(l16, s[1], b, -3072); ST16(l16, s[2], b, -2048); ST16(l16, s[3], b, -1024); }
        raw = encode(d, s, meta, lane);
        meta = hdr_word(meta, call, lane);
      }
      __builtin_amdgcn_s_setprio(0);
      const int need = k < P_WIN0 ? 0 : k + 1 - max(P_WIN, P_WIN1 - (k - P_WIN0) / P_TAPER);
      while (__builtin_amdgcn_readfirstlane(__hip_atomic_load(done, __ATOMIC_RELAXED, __HIP_MEMORY_SCOPE_WORKGROUP)) < need) __builtin_amdgcn_s_sleep(1);
      if (k >= P_WIN0) { ST16(l16, s[0], b, -4096); ST16(l16, s[1], b, -3072); ST16(l16, s[2], b, -2048); ST16(l16, s[3], b, -1024); }
      ST16(l16, s[4], b, 0); ST16(l16, s[5], b, 1024);
      if (raw) { ST16(l16, s[6], b, 2048); ST16(l16, s[7], b, 3072); }
      code = code_of(k + 12);
      asm volatile("" : "+s"(code));
      asm volatile("s_waitcnt vmcnt(0)" ::: "memory");
      char* const m = rws + meta_off(ud);
      if (lane < 8) ST4(lane * 4, meta, m);
      if (lane == 0) __hip_atomic_fetch_add(done, 1, __ATOMIC_RELAXED, __HIP_MEMORY_SCOPE_WORKGROUP);
      if (k + 12 >= n) { k += 12; break; }
      fetch(code);
      WAIT_RAW(d);
    }
    if (lost) WG_BARRIER();
    if (ask && threadIdx.x == 0) __hip_atomic_store(counter(myws), call, __ATOMIC_RELAXED, __HIP_MEMORY_SCOPE_AGENT);
    if (lost) { __builtin_amdgcn_s_setprio(0); return false; }
    const u32 y = cw + (u32)(k - n) * NCWG;
    if (!intra && y < 2u * P_PARK) {
      const u32 us = (S_PARK + (dest & 3)) * NSTEP + y;
      const u64 t0 = __builtin_amdgcn_s_memrealtime();
      const u32* const hd = (const u32*)(myws + meta_off(us));
      for (u32 it = 1; (u32)__builtin_amdgcn_readfirstlane(__hip_atomic_load(hd, __ATOMIC_RELAXED, __HIP_MEMORY_SCOPE_SYSTEM)) < call; ++it) {
        if ((it & 31) == 0 && spin_fail(myws, call, t0)) break;
        __builtin_amdgcn_s_sleep(4);
      }
      load_slot(d, l16, myws + (size_t)us * SLOT + 4096);
      if (__hip_atomic_load((u32*)(myws + TMO_OFF), __ATOMIC_RELAXED, __HIP_MEMORY_SCOPE_AGENT) == call) d[0] = u32x4{0x7fc07fc0u, 0x7fc07fc0u, 0x7fc07fc0u, 0x7fc07fc0u};
      const u32 tile = y >> 1, sid = (tile >> 1) * 4 + (y & 1) * 2;
      char* const ob = p->out[dest] + (rank & 3) * 256 + (tile & 1) * 128;
      const u32 vo = (lane >> 3) * (NLOC * 2) + (lane & 7) * 16;
      char* r0 = ob + (size_t)p->row[sid] * (NLOC * 2);
      char* r1 = ob + (size_t)p->row[sid + 1] * (NLOC * 2);
      asm volatile("" : "+s"(r0), "+s"(r1));
      while (__builtin_amdgcn_readfirstlane(__hip_atomic_load(done, __ATOMIC_RELAXED, __HIP_MEMORY_SCOPE_WORKGROUP)) < k - (P_WIN - 1)) __builtin_amdgcn_s_sleep(1);
#pragma unroll
      for (int g = 0; g < 8; ++g) {
        char* const b = (g < 4 ? r0 : r1) + (g & 3) * 8 * (NLOC * 2);
        ST16(vo, d[g], b, 0);
      }
      if (y + P_REL * NCWG >= 2u * P_PARK && y + P_REL * NCWG < 2u * P_PARK + NCWG && lane == 0)
        __hip_atomic_store((u32*)(myws + REL_OFF + link * 128) + cw, call, __ATOMIC_RELAXED, __HIP_MEMORY_SCOPE_AGENT);
      asm volatile("s_waitcnt vmcnt(0)" ::: "memory");
      char* const fl = rws + DONE_OFF + ((rank & 3) * 64 + ((y & 1) ? 2 * RT_MED + 12 + tile : tile)) * 4;
      if (lane == 0) { ST4(0u, call, fl); __hip_atomic_fetch_add(done, 1, __ATOMIC_RELAXED, __HIP_MEMORY_SCOPE_WORKGROUP); }
    }
    return true;
  }
  if (ask) {
    const u32 asked = bid;
    claim(smem, myws, wv, bid, call);
    if (bid != asked) return false;
  }
  if (cb == 0 && wv == 0) {
    const u64 t0 = __builtin_amdgcn_s_memrealtime();
    while (__builtin_amdgcn_s_memrealtime() - t0 < N_INTRA * P_UT / 10) __builtin_amdgcn_s_sleep(16);
    const char* const f = myws + DONE_OFF;
    const int f0 = (4 * lane) & 63;
    for (u32 n = 1;; ++n) {
      u32x4 v;
      asm volatile("global_load_dwordx4 %0, %1, %2 sc0 sc1\n\ts_waitcnt vmcnt(0)" : "=&v"(v) : "v"(l16), "s"(f) : "memory");
      if (__all((f0 >= NFLAG || v[0] >= call) && (f0 + 1 >= NFLAG || v[1] >= call) && (f0 + 2 >= NFLAG || v[2] >= call) && (f0 + 3 >= NFLAG || v[3] >= call))) break;
      if ((n & 31) == 0 && spin_fail(myws, call, t0)) {
        if (lane == 0) *(unsigned short*)(p->out[rank] + (size_t)phys_rank(rank) * 512 * (NLOC * 2)) = 0x7fc0;
        break;
      }
    }
  }
  return true;
}

__device__ __forceinline__ u32x4 join(u32 l0, u32 l1, u32 h0, u32 h1) {
  return u32x4{__builtin_amdgcn_perm(h0, l0, 0x05010400u), __builtin_amdgcn_perm(h0, l0, 0x07030602u),
               __builtin_amdgcn_perm(h1, l1, 0x05010400u), __builtin_amdgcn_perm(h1, l1, 0x07030602u)};
}
__device__ __forceinline__ void fix_word(u32 (&h)[8], u32 e) {
  asm volatile("s_lshl_b64 exec, 1, %8\n\ts_getpc_b64 vcc\n\ts_add_u32 vcc_lo, vcc_lo, %9\n\ts_addc_u32 vcc_hi, vcc_hi, 0\n\ts_setpc_b64 vcc\n\t"
               "v_xor_b32_e32 %0, %10, %0\n\ts_branch 1f\n\tv_xor_b32_e32 %1, %10, %1\n\ts_branch 1f\n\t"
               "v_xor_b32_e32 %2, %10, %2\n\ts_branch 1f\n\tv_xor_b32_e32 %3, %10, %3\n\ts_branch 1f\n\t"
               "v_xor_b32_e32 %4, %10, %4\n\ts_branch 1f\n\tv_xor_b32_e32 %5, %10, %5\n\ts_branch 1f\n\t"
               "v_xor_b32_e32 %6, %10, %6\n\ts_branch 1f\n\tv_xor_b32_e32 %7, %10, %7\n"
               "1:\n\ts_mov_b64 exec, -1"
               : "+v"(h[0]), "+v"(h[1]), "+v"(h[2]), "+v"(h[3]), "+v"(h[4]), "+v"(h[5]), "+v"(h[6]), "+v"(h[7])
               : "s"(e), "s"(((e >> 3) & 56) + 12), "s"(((e >> 11) & 0x7f) << ((e >> 6) & 24)) : "vcc", "scc");
}

struct Stream {
  const char* sb; const char* mb; char* myws;
  u32 l16, tag, call; u64 t0;
  u32x4 W; int rdy; bool eager;
  __device__ __forceinline__ void init(char* ws, u32 call_, u32 sid, int lane) {
    myws = ws; call = call_; tag = call_tag(call_);
    sb = ws + (size_t)sid * NSTEP * SLOT + 4096;
    mb = ws + meta_off(sid * NSTEP);
    l16 = lane * 16;
    t0 = __builtin_amdgcn_s_memrealtime();
    W = u32x4{0u, 0u, 0u, 0u}; rdy = 0; eager = false;
  }
  __device__ __forceinline__ void poll(bool wide = true) {
    const u64 mask = (wide ? 0xffffull : 0xffull) << (rdy < 28 ? (2 * rdy) & 56 : 56);
    asm volatile("s_mov_b64 exec, %3\n\tglobal_load_dwordx4 %0, %1, %2 sc0 sc1\n\ts_mov_b64 exec, -1"
                 : "+v"(W) : "v"(l16), "s"(mb), "s"(mask) : "memory");
  }
  __device__ __forceinline__ void look() {
    asm volatile("" : "+v"(W));
    const u32 x = (W[0] ^ tag) | (W[1] ^ tag) | (W[2] ^ tag);
    const u64 odd = 0xaaaaaaaaaaaaaaaaull;
    const u64 bad = __ballot(x >= (1u << 21)) | (__ballot((W[3] ^ tag) >= (1u << 21)) & ~odd) | (__ballot(W[3] < call) & odd);
    rdy = bad ? __builtin_ctzll(bad) >> 1 : NSTEP;
  }
  __device__ __forceinline__ void spin(int t) {
    for (u32 n = 1; rdy <= t; ++n) {
      const int was = rdy;
      poll(n == 1);
      asm volatile("s_waitcnt vmcnt(0)" ::: "memory");
      look();
      if (rdy == was) {
        if (eager) __builtin_amdgcn_s_sleep(P_S2);
        else if (rdy == 0) __builtin_amdgcn_s_sleep(P_S0);
        else __builtin_amdgcn_s_sleep(P_S1);
      }
      if ((n & 31) == 0 && spin_fail(myws, call, t0)) rdy = NSTEP;
    }
  }
};

__device__ __forceinline__ void repair(u32 (&h)[8], u32 hd, u32 hdr, u32 hf, const char* sb, u32 l16) {
#pragma nounroll
  for (u32 n = (hdr >> (11 + 3 * hf)) & 7, k = hf ? 6 : 1; n != 0; --n, k += hf ? -1 : 1) fix_word(h, __builtin_amdgcn_readlane(hd, k));
  if (__builtin_expect(hdr & 0x400u, 0)) {
    const u32x8 g = load_two(sb + hf * 2048, l16);
#pragma unroll
    for (int j = 0; j < 8; ++j) asm volatile("v_mov_b32 %0, %1" : "+v"(h[j]) : "v"(g[j]));
  }
}

__device__ __forceinline__ void unpack_unit(char* myws, int lane, u32 call, u32 r, int t, u64 tk) {
  Stream S;
  S.init(myws, call, (call & 1) * NS_PAR + S_WIN + r, lane);
  const u32 l4 = lane * 4;
  const u32 uo = (S_IMG + 16 + r) * NSTEP;
  {
    if (r < 16) while (__builtin_amdgcn_s_memrealtime() - tk < (u32)(20 + 4 * t) * P_UT / 10) __builtin_amdgcn_s_sleep(4);
    u32 hd;
    for (u32 n = 1;; ++n) {
      hd = __hip_atomic_load((const u32*)(S.mb + t * 32) + (lane & 7), __ATOMIC_RELAXED, __HIP_MEMORY_SCOPE_SYSTEM);
      if (hdr_there(hd, call, lane)) break;
      if ((n & 31) == 0 && spin_fail(myws, call, S.t0)) break;
      __builtin_amdgcn_s_sleep(4);
    }
    Pay<6> P;
    pay_load(P, S.l16, S.sb + (size_t)t * SLOT);
    pay_wait<0>(P);
    const u32 hdr = __builtin_amdgcn_readfirstlane(hd), base4 = (hdr & 0x7f) * 0x01010101u;
    u32x4 o[8];
#pragma unroll
    for (int hf = 0; hf < 2; ++hf) {
      u32 g[8];
#pragma unroll
      for (int j = 0; j < 4; ++j) {
        const u32 n = P.v[4 + hf][j];
        g[2 * j] = (n & 0x87878787u) + base4;
        g[2 * j + 1] = (__builtin_amdgcn_alignbit(n, n, 4) & 0x87878787u) + base4;
      }
      if (hdr & (0x400u | 0x3800u << (3 * hf))) repair(g, hd, hdr, hf, S.sb + (size_t)t * SLOT, S.l16);
#pragma unroll
      for (int j = 0; j < 4; ++j)
        o[4 * hf + j] = join(P.v[2 * hf + (j >> 1)][2 * (j & 1)], P.v[2 * hf + (j >> 1)][2 * (j & 1) + 1], g[2 * j], g[2 * j + 1]);
    }
    char* const ob = myws + (size_t)(uo + t) * SLOT + 4096;
    ST_SIX(S.l16, o, ob); ST16(S.l16, o[6], ob, 2048); ST16(S.l16, o[7], ob, 3072);
    asm volatile("s_waitcnt vmcnt(0)" ::: "memory");
    char* const mo = myws + meta_off((uo + t));
    if (lane < 8) ST4(l4, hdr_word(0, call, lane), mo);
  }
}

__device__ __forceinline__ float silu(float y) {
  const float z = __builtin_amdgcn_fmed3f(y, -60.0f, 60.0f);
  const float e = __builtin_amdgcn_exp2f(z * -1.44269504088896341f);
  return y * __builtin_amdgcn_rcpf(1.0f + e);
}

__device__ __forceinline__ void gemm_tile(KP p, char* smem, char* myws, int wv, int lane, u32 bid, u32 call, u32 nb, u32 sid0, u32 tn, u32 ch, u32 fl, int hold, u64 tk, bool next, u32*& owed) {
  const u32 rank = p->rank;
  if (wv >= 8) __builtin_amdgcn_s_setprio(P_LPRIO);
  const u32 o0 = (rank & 4u) ^ 4u;
  const bool remote = tn >= 4;
  const u32 dst = (remote ? o0 + (tn - 4) : rank) & 7u;
  char* const ob = p->out[dst] + (remote ? (rank & 3) : tn) * 256 + ch * 128;
  u32* flag = (u32*)(p->ws[dst] + DONE_OFF) + (rank & 3) * 64 + fl;
  auto grow = [&](u32 rg) { return p->row[sid0 + (rg >> 2)] + (rg & 3) * 8; };
  char* b0 = ob + (size_t)grow(wv & 7) * (NLOC * 2);
  char* b1 = ob + (size_t)grow((wv & 7) + 8) * (NLOC * 2);
  if (hold == 2) {
    const u32 us = (S_PARK + (tn & 3)) * NSTEP + 2 * fl;
    b0 = myws + (size_t)us * SLOT + (wv & 7) * 1024; b1 = b0 + SLOT;
    flag = (u32*)(myws + meta_off(us));
  }
  asm volatile("" : "+s"(b0), "+s"(b1), "+s"(flag));
  if (nb == 0) {
    if (threadIdx.x == 0) ST4(0u, call, flag);
    return;
  }
  {
    const u32 first = sid0 < (u32)S_2 || sid0 >= (u32)S_OWN2 ? 0 : sid0 < (u32)S_INTER ? N1 * NSTEP : sid0 < (u32)S_3 ? 4 * NSTEP : N_INTRA - N3 * NSTEP;
    while (__builtin_amdgcn_s_memrealtime() - tk < first * P_UT / 10) __builtin_amdgcn_s_sleep(8);
  }
  Stream S;
  const u32 m = wv >> 1, kh = wv & 1;
  u32 tmo = 0;
  auto read_tmo = [&]() { asm volatile("global_load_dword %0, %1, %2 sc1" : "=&v"(tmo) : "v"(0u), "s"(myws + TMO_OFF) : "memory"); };
  if (wv >= 8) {
    const u32 un = wv - 8;
    S.init(myws, call, S_IMG + 4 * tn + un, lane);
    const u32 lds0 = (u32)(size_t)(LAS char*)smem + un * 4096;
    const char* const g0 = S.sb - 4096 + ch * 4096;
    Stream A;
    A.init(myws, call, (call & 1) * NS_PAR + sid0 + un, lane);
    A.eager = sid0 >= (u32)S_3 && sid0 < (u32)S_OWN2;
    asm volatile("global_load_dwordx4 %0, %2, %3 sc0 sc1\n\tglobal_load_dwordx4 %1, %2, %4 sc0 sc1\n\ts_waitcnt vmcnt(0)"
                 : "+v"(A.W), "+v"(S.W) : "v"(S.l16), "s"(A.mb), "s"(S.mb) : "memory");
    int is = 0, seen = ((nb >> un) & 1) ? 0 : NSTEP;
    for (int t = -1; t < NSTEP; ++t) {
      if (seen < NSTEP && seen <= t + P_DIST + P_ALOOK) {
        A.look();
        A.spin(min(t + P_DIST, NSTEP - 1));
        if (A.rdy != seen) { *(LAS u32x4*)((LAS char*)smem + HDR_OFF + un * 1024 + A.l16) = A.W; seen = A.rdy; }
        asm volatile("" ::: "memory");
        if (seen < NSTEP) A.poll();
      }
      const int lim = min(min(NSTEP, is + 2), t + P_BD);
      if (S.rdy < min(NSTEP, lim + 2)) { S.look(); S.spin(t); if (S.rdy < NSTEP) S.poll(); }
      for (; is < min(lim, S.rdy); ++is) {
        const char* g = g0 + (size_t)is * SLOT;
        const u32 l = lds0 + (is & 7) * 16384;
        u32 keep;
        asm volatile("s_mov_b32 %0, m0\n\ts_mov_b32 m0, %3\n\ts_nop 0\n\t"
                     "global_load_lds_dwordx4 %1, %2 sc0 sc1\n\tglobal_load_lds_dwordx4 %1, %2 offset:1024 sc0 sc1\n\t"
                     "global_load_lds_dwordx4 %1, %2 offset:2048 sc0 sc1\n\tglobal_load_lds_dwordx4 %1, %2 offset:3072 sc0 sc1\n\t"
                     "s_mov_b32 m0, %0" : "=&s"(keep) : "v"(S.l16), "s"(g), "s"(l) : "memory");
      }
      const int ahead = is - 1 - t;
      if (ahead >= 6 || t < 0) asm volatile("s_waitcnt vmcnt(24)" ::: "memory");
      else if (ahead >= 4) { if (ahead == 5) asm volatile("s_waitcnt vmcnt(20)" ::: "memory"); else asm volatile("s_waitcnt vmcnt(16)" ::: "memory"); }
      else if (ahead >= 2) { if (ahead == 3) asm volatile("s_waitcnt vmcnt(12)" ::: "memory"); else asm volatile("s_waitcnt vmcnt(8)" ::: "memory"); }
      else if (ahead == 1) asm volatile("s_waitcnt vmcnt(4)" ::: "memory");
      else asm volatile("s_waitcnt vmcnt(0)" ::: "memory");
      WG_BARRIER();
    }
    asm volatile("s_waitcnt vmcnt(0)" : "+v"(S.W), "+v"(A.W) :: "memory");
    __builtin_amdgcn_s_setprio(0);
    if (hold == 1 && wv == 8) {
      const u32* const r = (const u32*)(myws + REL_OFF + (bid & 7) * 128) + (lane < NCWG ? lane : 0);
      for (u32 n = 1; !__all(__hip_atomic_load(r, __ATOMIC_RELAXED, __HIP_MEMORY_SCOPE_AGENT) >= call); ++n) {
        if ((n & 31) == 0 && spin_fail(myws, call, S.t0)) break;
        __builtin_amdgcn_s_sleep(4);
      }
    }
  } else if (!((nb >> m) & 1)) {
    asm volatile("s_waitcnt vmcnt(0)" ::: "memory");
    for (int t = -1; t < NSTEP; ++t) WG_BARRIER();
    read_tmo();
    asm volatile("s_waitcnt vmcnt(0)" : "+v"(tmo) :: "memory");
  } else {
    const char* const sb = myws + (size_t)((call & 1) * NS_PAR + sid0 + m) * NSTEP * SLOT + 4096;
    const u32 l16 = lane * 16, vlo = l16 + kh * 2048, vcode = l16 + kh * 1024;
    u32 ho = HDR_OFF + m * 1024 + (lane & 7) * 4, hd;
    const u32 rmask = 0x400u | 0x3800u << (3 * kh);
    f32x16 acc[2] = {};
    u32 bb = kh * 8192 + lane * 16;
    bf16x8 wf[4];
    u32x4 X[2][4];
    Pay<3> P[4];
    u32 h[8];
    u32 hdr = 0, base4 = 0;
    auto rd = [&](int stage, int f) {
      wf[f & 3] = *(const LAS bf16x8*)((LAS char*)smem + bb + stage * 16384 + (f >> 2) * 4096 + ((f >> 1) & 1) * 1024 + (f & 1) * 2048);
    };
    auto piece = [&](int k, int t, int ts, int waitn, bool fetch) {
      Pay<3>& Q = P[ts];
      if (k == 0) {
        if (waitn == 1) pay_wait<3>(Q); else pay_wait<0>(Q);
        hdr = __builtin_amdgcn_readfirstlane(hd);
        base4 = (hdr & 0x7f) * 0x01010101u;
      } else if (k < 6) {
#pragma unroll
        for (int j = 0; j < 4; ++j) {
          const int i = k < 4 ? 2 * j + (k & 1) : 4 * (k - 4) + j;
          if (k == 1) h[i] = __builtin_amdgcn_alignbit(Q.v[2][j], Q.v[2][j], 4);
          else if (k == 2) h[i] = Q.v[2][j] & 0x87878787u;
          else if (k == 3) h[i] &= 0x87878787u;
          else h[i] += base4;
          asm volatile("" : "+v"(h[i]));
        }
      } else if (k == 6) {
        if (hdr & rmask) repair(h, hd, hdr, kh, sb + (size_t)t * SLOT, l16);
      } else if (k == 7) {
      } else if (k < 12) {
        const int c = k - 8;
        if (k == 8 && fetch) pay_load(P[(ts + P_DIST) & 3], vlo, vcode, sb + (size_t)(t + P_DIST) * SLOT);
        X[ts & 1][c] = join(Q.v[c >> 1][2 * (c & 1)], Q.v[c >> 1][2 * (c & 1) + 1], h[2 * c], h[2 * c + 1]);
        asm volatile("" : "+v"(X[ts & 1][c]));
      } else if (k == 12) {
        ho += 32;
        hd = *(const LAS u32*)((LAS char*)smem + ho);
      }
    };
    auto step = [&](int t, int ts, int waitn, bool fetch, bool more) {
#pragma unroll
      for (int q = 0; q < 8; ++q) {
        if (q == 4 && more) { WG_BARRIER(); if (ts == 3) bb ^= 65536u; }
        else if (q % 2 == 0 && (more || q < 6)) __builtin_amdgcn_s_waitcnt(0xC27F);
        __builtin_amdgcn_sched_barrier(0);
        acc[q & 1] = __builtin_amdgcn_mfma_f32_32x32x16_bf16(wf[q & 3], __builtin_bit_cast(bf16x8, X[ts & 1][q >> 1]), acc[q & 1], 0, 0, 0);
        __builtin_amdgcn_sched_barrier(0);
        if (q < 4) rd(ts, q + 4);
        else if (more) rd((ts + 1) & 3, q - 4);
        if (more) {
          piece(2 * q, t + 1, (ts + 1) & 3, waitn, fetch);
          piece(2 * q + 1, t + 1, (ts + 1) & 3, waitn, fetch);
        }
      }
    };
    asm volatile("s_waitcnt vmcnt(0)" ::: "memory");
    WG_BARRIER();
    if (owed && threadIdx.x < 2) ST4(threadIdx.x * 32, call, owed);
#pragma unroll
    for (int d = 0; d < P_DIST; ++d) pay_load(P[d], vlo, vcode, sb + (size_t)d * SLOT);
    hd = *(const LAS u32*)((LAS char*)smem + ho);
#pragma unroll
    for (int k = 0; k < 13; ++k) piece(k, 0, 0, 1, false);
    WG_BARRIER();
    pay_load(P[P_DIST], vlo, vcode, sb + (size_t)P_DIST * SLOT);
#pragma unroll
    for (int f = 0; f < 4; ++f) { rd(0, f); __builtin_amdgcn_sched_barrier(0); }
    for (int tb = 0; tb < NSTEP - 4; tb += 4) {
#pragma unroll
      for (int ts = 0; ts < 4; ++ts) step(tb + ts, ts, 1, true, true);
    }
    step(NSTEP - 4, 0, 1, true, true);
    step(NSTEP - 3, 1, 1, false, true);
    step(NSTEP - 2, 2, 0, false, true);
    read_tmo();
    step(NSTEP - 1, 3, 0, false, false);
    char* const xq = smem + EPI_OFF + (kh * 128 + m * 32 + (lane & 31)) * ERS + (lane >> 5) * 16;
#pragma unroll
    for (int n = 0; n < 2; ++n)
#pragma unroll
      for (int g = 0; g < 4; ++g)
        *(f32x4*)(xq + n * 128 + g * 32) = f32x4{acc[n][4 * g], acc[n][4 * g + 1], acc[n][4 * g + 2], acc[n][4 * g + 3]};
    asm volatile("s_waitcnt vmcnt(0)" : "+v"(tmo) :: "memory");
  }
  WG_BARRIER();
  if (wv < 8) {
    const u32 vo = hold == 2 ? lane * 16 : (lane >> 3) * (NLOC * 2) + (lane & 7) * 16;
    auto get = [&](u32 rg) {
      const char* const q = smem + EPI_OFF + (rg * 8 + (lane >> 3)) * ERS + (lane & 7) * 32;
      const f32x4 a = *(const f32x4*)q + *(const f32x4*)(q + 128 * ERS), c = *(const f32x4*)(q + 16) + *(const f32x4*)(q + 128 * ERS + 16);
      auto pk = [](float lo, float hi) { const f32x2 v = {silu(lo), silu(hi)}; return __builtin_bit_cast(u32, __builtin_convertvector(v, bf16x2)); };
      u32x4 v = {pk(a[0], a[1]), pk(a[2], a[3]), pk(c[0], c[1]), pk(c[2], c[3])};
      if (tmo == call) v = u32x4{0x7fc07fc0u, 0x7fc07fc0u, 0x7fc07fc0u, 0x7fc07fc0u};
      return v;
    };
    if ((nb >> (wv >> 2)) & 1) { const u32x4 v0 = get(wv); ST16(vo, v0, b0, 0); }
    if ((nb >> (wv >> 2)) & 4) { const u32x4 v1 = get(wv + 8); ST16(vo, v1, b1, 0); }
  }
  asm volatile("ag_code_mark:");
  owed = nullptr;
  if (next) { if (remote) owed = flag; }
  else if (remote) {
    asm volatile("s_waitcnt vmcnt(0)" ::: "memory");
    WG_BARRIER();
    if (threadIdx.x == 0) ST4(0u, call, flag);
  }
}

inline bool tile_of(u32 rank, u32 b, int k, u32& nb, u32& sid0, u32& tn, u32& ch, u32& fl, int& hold) {
  const u32 peer = peer_of_xcd(rank, b & 7), i = b >> 3;
  const bool home = !((peer ^ rank) & 4), spare = i >= 2 * RT_MED;
  hold = 0;
  constexpr u32 I0 = S_INTER / 2;
  const bool extra = home && i >= I0 && i < I0 + (NB + 1) / 2 * 2;
  if (extra && k == 0) {
    const u32 j = i - I0;
    nb = (j | 1) < (u32)NB ? 3 : 1; tn = peer & 3; sid0 = S_OWN2 + (j >> 1) * 2; ch = j & 1; fl = 0;
    return true;
  }
  if (k == (extra ? 1 : 0) && !spare) {
    nb = 15; tn = (peer & 3) + (home ? 0 : 4); sid0 = (i >> 1) * 4; ch = i & 1; fl = i;
    hold = home || sid0 >= (u32)S_INTER ? 0 : i < (u32)P_PARK ? 2 : i >= 10 ? 1 : 0;
    if (!home)
      for (u32 m = 0; m < 4; ++m)
        if (sid0 + m >= (u32)S_INTER && (sid0 + m - S_INTER) / NB == (peer & 3)) nb &= ~(1u << m);
    return true;
  }
  const u32 j = spare ? i - 2 * RT_MED + 10 : i;
  nb = 3; tn = (peer & 3) + (home ? 4 : 0); sid0 = RT_MED * 4 + (j >> 1) * 2; ch = j & 1; fl = 2 * RT_MED + j;
  return spare ? k == 0 : i < 10;
}

constexpr u32 NHOME = 32 * NSTEP;
__host__ __device__ inline u32 pack_index(u32 b, u32 wv) {
  constexpr u32 NP1 = 80;
  return b < NP1 ? (wv < 2 ? b + NP1 * wv : NHOME) : 2 * NP1 + (b - NP1) + (NGEMM - NP1) * wv;
}

constexpr u32 UB0 = RT_MED * 16, UNW = 16;
__host__ __device__ inline u32 unpack_index(u32 b, u32 wv) {
  return b >= UB0 && b < UB0 + UNW ? (11 - wv) * UNW + (b - UB0) : 512u;
}

__global__ void __launch_bounds__(NTHREAD, 3) ag_gemm_silu(const Params) {
  KP p = (KP)__builtin_amdgcn_kernarg_segment_ptr();
  __shared__ __attribute__((aligned(16))) char smem[LDS_BYTES];
  const int lane = threadIdx.x & 63;
  const int wv = __builtin_amdgcn_readfirstlane(threadIdx.x >> 6);
  char* const myws = p->myws;
  u64 pc; u32 len;
  asm volatile("s_getpc_b64 %0\nag_code_begin:\n\ts_mov_b32 %1, ag_code_mark-ag_code_begin" : "=s"(pc), "=s"(len));
  const u64 tk = __builtin_amdgcn_s_memrealtime();
  u32 bid = (((blockIdx.x & ~7u) | ((__builtin_amdgcn_s_getreg((3 << 11) | 20)) & 7u)) + NGEMM) % GRID, call = 0;
  if (bid >= (u32)NGEMM) { if (sender<true>(p, smem, myws, wv, lane, bid, call)) return; }
  else {
    if (wv >= 10) {
      const u32 o = ((bid >> 3) * 2 + (wv - 10)) * 1024 + lane * 16;
      u32x4 junk;
      if (o + 16 <= len && len < 65536u) asm volatile("global_load_dwordx4 %0, %1, %2\n\ts_waitcnt vmcnt(0)" : "=&v"(junk) : "v"(o), "s"(pc) : "memory");
    }
    claim(smem, myws, wv, bid, call);
  }
  bid = __builtin_amdgcn_readfirstlane(bid); call = __builtin_amdgcn_readfirstlane(call);
  if (__builtin_expect(bid >= (u32)NGEMM, 0)) { sender<false>(p, smem, myws, wv, lane, bid, call); return; }
  const u32 u = pack_index(bid, wv);
  if (u < NHOME) {
    const u32 t = u >> 5, xc = u & 7, sub = (u >> 3) & 3;
    const u32 c = xc < 4 ? sub * 4 + xc : 16 + (peer_of_xcd(p->rank, xc) & 3) * 4 + sub;
    const bool isw = c >= 16;
    const u32 cw = c - 16, l16 = lane * 16;
    while (__builtin_amdgcn_s_memrealtime() - tk < t * P_PP) __builtin_amdgcn_s_sleep(2);
    u32x4 d[8], s[8];
    load_raw(p, d, lane, isw ? 16 + (cw & 3) : c, (cw >> 2) & 3, t);
    WAIT_RAW(d);
    u32 meta = 0;
    bool eight = true;
    to_fragments(d, isw, smem + SCR_OFF + wv * 8192, lane);
    if (isw) {
#pragma unroll
      for (int j = 0; j < 8; ++j) s[j] = d[j];
    } else { low_slabs(d, s); eight = encode(d, s, meta, lane); }
    const u32 ud = (isw ? S_IMG + cw : (call & 1) * NS_PAR + (c < (u32)NA ? c : S_OWN2 - NA + c)) * NSTEP + t;
    char* const b = myws + (size_t)ud * SLOT + 4096;
    ST_SIX(l16, s, b);
    if (eight) { ST16(l16, s[6], b, 2048); ST16(l16, s[7], b, 3072); }
    asm volatile("s_waitcnt vmcnt(0)" ::: "memory");
    char* const m = myws + meta_off(ud);
    if (lane < 8) ST4(lane * 4, hdr_word(meta, call, lane), m);
  }
#pragma nounroll
  for (u32 v = unpack_index(bid, wv); v < 512; v += 12 * UNW) unpack_unit(myws, lane, call, (v + P_UROT(v >> 4)) & 15, v >> 4, tk);
  WG_BARRIER();
  u32* owed = nullptr;
#pragma nounroll
  for (int k = 0; k < 2; ++k) {
    const u32 td = p->tile[bid][k];
    if (!(td >> 23)) break;
    gemm_tile(p, smem, myws, wv, lane, bid, call, td & 15, (td >> 4) & 127, (td >> 11) & 7, (td >> 14) & 1, (td >> 15) & 63, (td >> 21) & 3, tk, k == 0 && p->tile[bid][1] != 0, owed);
  }
}

static void build_orders(Params& p) {
  struct E { double tau; unsigned short code; };
  E e[N_ORD];
  double sigma = P_SIGMA, ovl = P_OVL, ovl2 = P_OVL2, lead = P_LEAD, f1 = 14.5, f2 = 6, stg = P_STG;
  for (int kind = 0; kind < 2; ++kind) {
    int n = 0;
    for (int t = 0; t < NSTEP; ++t) {
      if (kind == 0) {
        for (int b = 0; b < NA; ++b) if (b != RB || t < RT0) e[n++] = {t + (b < N1 ? 0 : b < N1 + N2 ? NSTEP - ovl : 2 * NSTEP - ovl - ovl2 + (b >= N1 + N2 + 2 ? stg : 0)), (unsigned short)(b << 5 | t)};
      } else {
        for (int i = 0; i < 4; ++i) e[n++] = {t * sigma - lead, (unsigned short)((16 + i) << 5 | t)};
        for (int b = NA; b < 16; ++b) e[n++] = {t + 0.5, (unsigned short)(b << 5 | t)};
      }
    }
    if (kind == 1)
      for (int j = 0; j < P_RLY; ++j) {
        e[n++] = {f1 * sigma - lead, (unsigned short)(20 << 5 | j)};
        for (int i = 0; i < 3; ++i) e[n++] = {f2 + j, (unsigned short)((21 + i) << 5 | j)};
      }
    std::stable_sort(e, e + n, [](const E& a, const E& b) { return a.tau < b.tau; });
    for (int i = 0; i < N_ORD; ++i) p.ord[kind][i] = i < n ? e[i].code : 0;
  }
}

extern "C" void kernel_launch(int rank, int nranks, ncclComm_t comm, hipStream_t stream,
                              void* const* d_in, const int* in_sizes, int n_in,
                              void* d_out, int out_size, const int* in_dtypes, int out_dtype,
                              void* const* d_out_all, void* const* d_ws_all, size_t ws_bytes) {
  (void)comm; (void)in_dtypes; (void)out_dtype; (void)d_out;
  if (nranks != NR || (unsigned)rank >= (unsigned)NR || n_in != 2 || ws_bytes < WS_NEED ||
      in_sizes[0] != 512 * KDIM || in_sizes[1] != KDIM * NLOC || out_size != 4096 * NLOC) {
    fprintf(stderr, "unexpected problem shape\n");
    return;
  }
  Params p = {};
  p.x = (const char*)d_in[0];
  p.w = (const char*)d_in[1];
  for (int i = 0; i < NR; ++i) {
    p.out[i] = (char*)d_out_all[phys_rank(i)]; p.ws[i] = (char*)d_ws_all[phys_rank(i)];
    if ((int)phys_rank(i) == rank) p.rank = i;
  }
  p.myws = (char*)d_ws_all[rank];
  build_orders(p);
  for (u32 e = 0; e < 12 * NCWG; ++e)
    if (p.ord[0][e] != ((e % N1) << 5 | e / N1) || p.ord[1][e] != ((16 + (e & 3)) << 5 | e >> 2)) { fprintf(stderr, "send orders\n"); return; }
  for (int i = 0; i < NS_PAR + 4; ++i) p.row[i] = i < 64 || (i >= S_OWN2 && i < NS_PAR) ? global_row(p.rank, i) : 0;
  for (u32 b = 0; b < (u32)NGEMM; ++b)
    for (int k = 0; k < 2; ++k) {
      u32 nb, sid0, tn, ch, fl; int hold;
      p.tile[b][k] = tile_of(p.rank, b, k, nb, sid0, tn, ch, fl, hold) ? nb | sid0 << 4 | tn << 11 | ch << 14 | fl << 15 | (u32)hold << 21 | 1u << 23 : 0u;
    }
  hipLaunchKernelGGL(ag_gemm_silu, dim3(GRID), dim3(NTHREAD), 0, stream, p);
  hipError_t err = hipGetLastError();
  if (err != hipSuccess) fprintf(stderr, "launch failed: %s\n", hipGetErrorString(err));
}
```
